```python
import numpy as np

kernel_name = "gpu_gemm_rs_m4096_k4096_n16384_bf16_gelu"


ABSMAX_RTOL = 0.008

COMPUTE_DTYPE = "bfloat16"
EPILOGUE = "gelu"
IN_QUANT = None
OUT_QUANT = None
IN_DTYPES = {"x": "bfloat16", "w_mat": "bfloat16"}
WORLD_SIZE = 8

def _to_bf16(x):
    u = np.ascontiguousarray(x, dtype=np.float32).view(np.uint32)
    return ((u + 0x7FFF + ((u >> 16) & 1)) & 0xFFFF0000).view(np.float32)

def setup_inputs():

    x = _to_bf16(np.random.randn(4096, 4096).astype(np.float32))
    w_mat = _to_bf16((np.random.randn(4096, 16384) / np.sqrt(4096)).astype(np.float32))
    return {"x": x, "w_mat": w_mat}

def _epilogue(y):
    c = 0.7978845608028654
    return (0.5 * y * (1.0 + np.tanh(c * (y + 0.044715 * y ** 3)))).astype(np.float32)

def reference(x, w_mat):
    y = (x @ w_mat).astype(np.float32)
    return _epilogue(y)
```

```cpp
#include <hip/hip_runtime.h>
#include <rccl/rccl.h>
#include <cstdio>
#include <cstdint>

#define HIP_CHECK(expr) do { hipError_t _e = (expr); if (_e != hipSuccess) { \
    fprintf(stderr, "HIP error %s at %s:%d\n", hipGetErrorString(_e), __FILE__, __LINE__); \
    return; } } while (0)

namespace {

#define LAS __attribute__((address_space(3)))
typedef short bf16x8 __attribute__((ext_vector_type(8)));
typedef float f32x4 __attribute__((ext_vector_type(4)));
typedef unsigned u32x4 __attribute__((ext_vector_type(4)));
typedef unsigned long long u64;

constexpr int NR = 8;
constexpr int GM = 4096, GK = 4096, GN = 16384;
constexpr int KS = GK / NR;
constexpr int BLK = 32768;
constexpr int HTB = 16384;
#define SEND_G 8
constexpr int NSEND = 7 * SEND_G;
constexpr int NCOMM = 128;
constexpr u64 DEADLINE = 50000000ull;

constexpr size_t OFF_CNT = 0;
constexpr size_t OFF_TMO = 32768;
constexpr size_t OFF_FX = 36864;
constexpr size_t OFF_FW = 40960;
constexpr size_t OFF_FD = 45056;
constexpr size_t OFF_SENT = 53248;
constexpr size_t OFF_FP = 65536;
constexpr size_t OFF_STAMP = 262144;
constexpr size_t OFF_XA = 1u << 20;
constexpr size_t XA_PAR = (size_t)64 * 16 * BLK;
constexpr size_t OFF_WB = OFF_XA + 2 * XA_PAR;
constexpr size_t WB_PAR = (size_t)64 * 8 * BLK;
constexpr size_t OFF_WD = OFF_WB + 2 * WB_PAR;
constexpr size_t OFF_PB = OFF_WD + (size_t)64 * 8 * BLK;
constexpr size_t WS_NEED = OFF_PB + (size_t)128 * 131072;

struct Params {
    const unsigned char* x;
    const unsigned char* w;
    unsigned char* out;
    unsigned char* ws[NR];
    int rank, pad;
};

__device__ __forceinline__ void st16_sys(unsigned char* p, u32x4 v) {
    asm volatile("global_store_dwordx4 %0, %1, off sc0 sc1\n\ts_nop 1" :: "v"(p), "v"(v) : "memory");
}
__device__ __forceinline__ void st_flag(unsigned char* p, unsigned v) {
    __hip_atomic_store((unsigned*)p, v, __ATOMIC_RELAXED, __HIP_MEMORY_SCOPE_SYSTEM);
}
__device__ __forceinline__ unsigned ld_flag(const unsigned char* p) {
    return __hip_atomic_load((const unsigned*)p, __ATOMIC_RELAXED, __HIP_MEMORY_SCOPE_SYSTEM);
}
__device__ __forceinline__ unsigned lane_id() {
    return __builtin_amdgcn_mbcnt_hi(~0u, __builtin_amdgcn_mbcnt_lo(~0u, 0u));
}
__device__ __forceinline__ unsigned cvt_pk_bf16(float lo, float hi) {
    unsigned r; asm volatile("v_cvt_pk_bf16_f32 %0, %1, %2" : "=v"(r) : "v"(lo), "v"(hi)); return r;
}
__device__ __forceinline__ float gelu(float y) {
    const float k1 = -2.0f * 0.7978845608028654f * 1.4426950408889634f;
    const float k3 = k1 * 0.044715f;
    const float e = __builtin_amdgcn_exp2f(y * (k1 + k3 * y * y));
    return y * __builtin_amdgcn_rcpf(1.0f + e);
}

__device__ __forceinline__ void wait_word(const unsigned char* p, unsigned want, unsigned char* tmo, unsigned call) {
    const u64 t0 = __builtin_amdgcn_s_memrealtime();
    while ((unsigned)__builtin_amdgcn_readfirstlane(ld_flag(p)) < want) {
        if ((unsigned)__builtin_amdgcn_readfirstlane(__hip_atomic_load((const unsigned*)tmo, __ATOMIC_RELAXED, __HIP_MEMORY_SCOPE_AGENT)) == call
            || __builtin_amdgcn_s_memrealtime() - t0 > DEADLINE) {
            __hip_atomic_store((unsigned*)tmo, call, __ATOMIC_RELAXED, __HIP_MEMORY_SCOPE_AGENT);
            break;
        }
        __builtin_amdgcn_s_sleep(2);
    }
}

__device__ __forceinline__ void gate(u64& ready, int idx, const unsigned char* fa, const unsigned char* fb,
                                     unsigned char* tmo, unsigned call) {
    if ((ready >> idx) & 1ull) return;
    const u64 t0 = __builtin_amdgcn_s_memrealtime();
    for (;;) {
        unsigned a, b, t; const unsigned off = lane_id() * 4u, zero = 0u;
        asm volatile("global_load_dword %0, %3, %5 sc0 sc1\n\t"
                     "global_load_dword %1, %3, %6 sc0 sc1\n\t"
                     "global_load_dword %2, %4, %7 sc1\n\t"
                     "s_waitcnt vmcnt(0)"
                     : "=&v"(a), "=&v"(b), "=&v"(t) : "v"(off), "v"(zero), "s"(fa), "s"(fb), "s"(tmo) : "memory");
        ready = __ballot(a >= call && b >= call);
        if ((ready >> idx) & 1ull) break;
        if ((unsigned)__builtin_amdgcn_readfirstlane(t) == call || __builtin_amdgcn_s_memrealtime() - t0 > DEADLINE) {
            __hip_atomic_store((unsigned*)tmo, call, __ATOMIC_RELAXED, __HIP_MEMORY_SCOPE_AGENT);
            ready = ~0ull;
            break;
        }
        __builtin_amdgcn_s_sleep(1);
    }
}

#define STAMP(i) do {} while (0)

__device__ __forceinline__ void decode_item(int n, int self, int dest, int& isw, int& kc, int& tile) {
    if (dest == self) {
        kc = n / 24; const int r = n - kc * 24;
        isw = r < 8; tile = isw ? r : r - 8;
        return;
    }
    const int m = n < 15 ? n : n + 7;
    const int s = m / 22; int r = m - s * 22;
    if (s == 8) r += 15;
    if (r < 8) { isw = 1; kc = s; tile = r; return; }
    isw = 0;
    const int grp = r >= 15, e = grp ? r - 15 : r - 8;
    kc = s - grp;
    const int d = (self - dest - 1) & 7;
    const int pos = e + (e >= d);
    tile = 2 * ((dest + 1 + pos) & 7) + grp;
}

__global__ __launch_bounds__(512, 2) void fused_gemm_rs(const Params P) {
    __shared__ __attribute__((aligned(16))) unsigned char lds_raw[8 * HTB];
    LAS unsigned char* lds = (LAS unsigned char*)lds_raw;
    const int tid = threadIdx.x, wid = __builtin_amdgcn_readfirstlane(tid >> 6), lane = tid & 63;
    const int b = blockIdx.x, self = P.rank;
    unsigned char* const ws = P.ws[self];
    unsigned char* const tmo = ws + OFF_TMO;
    STAMP(0);
    const unsigned call = (unsigned)__builtin_amdgcn_readfirstlane(
        __hip_atomic_load((const unsigned*)(ws + OFF_CNT + b * 128), __ATOMIC_RELAXED, __HIP_MEMORY_SCOPE_AGENT)) + 1u;
    const size_t par = call & 1u;

    const bool is_comm = b < NCOMM;
    if (is_comm) {
        const bool sender = b < NSEND;
        const int dest = sender ? ((self + 1 + b % 7) & 7) : self;
        const int first = sender ? b / 7 : b - NSEND, stride = sender ? SEND_G : NCOMM - NSEND;
        const int npairs = sender ? 88 : 96;
        unsigned char* const dws = P.ws[dest];
        const int q = wid & 3, s16 = lane & 15, o4 = lane >> 4;
        u32x4 cur[8], nxt[8];
        auto load_item = [&](int n, u32x4 (&r)[8]) {
            int isw, kc, tile; decode_item(n, self, dest, isw, kc, tile);
            if (isw) {
                const unsigned char* src = P.w + (size_t)(kc * 64 + ((q & 1) * 4 + o4) * 8) * (GN * 2)
                                               + (size_t)((dest * 8 + tile) * 256 + (q >> 1) * 128 + s16 * 8) * 2;
#pragma unroll
                for (int i = 0; i < 8; ++i) r[i] = *(const u32x4*)(src + (size_t)i * (GN * 2));
            } else {
                const unsigned char* src = P.x + (size_t)(tile * 256 + q * 64 + s16) * (KS * 2) + (kc * 64 + o4 * 8) * 2;
#pragma unroll
                for (int i = 0; i < 8; ++i) r[i] = *(const u32x4*)(src + (i >> 1) * 16 * (KS * 2) + (i & 1) * 64);
            }
        };
        int pr = first;
        load_item(2 * pr + (wid >> 2), cur);
#pragma unroll
        for (int i = 0; i < 8; ++i) asm volatile("" : "+v"(cur[i]));
        for (; pr < npairs; pr += stride) {
            const int n = 2 * pr + (wid >> 2);
            int isw, kc, tile; decode_item(n, self, dest, isw, kc, tile);
            if (pr + stride < npairs) load_item(n + 2 * stride, nxt);
            const int idx = self * 8 + kc;
            if (isw) {
                u32x4 t[8];
#pragma unroll
                for (int j = 0; j < 8; ++j)
#pragma unroll
                    for (int qd = 0; qd < 4; ++qd)
                        t[j][qd] = __builtin_amdgcn_perm(cur[2 * qd + 1][j >> 1], cur[2 * qd][j >> 1], (j & 1) ? 0x07060302u : 0x05040100u);
                const unsigned off = (q >> 1) * 8192 + (q & 1) * 1024 + lane * 16;
                unsigned char* d0 = dws + OFF_WB + par * WB_PAR + (size_t)(idx * 8 + tile) * BLK + off;
#pragma unroll
                for (int j = 0; j < 8; ++j) st16_sys(d0 + (j >> 2) * HTB + (j & 3) * 2048, t[j]);
                if (sender) {
                    unsigned char* d1 = ws + OFF_WD + (size_t)((dest * 8 + tile) * 8 + kc) * BLK + off;
#pragma unroll
                    for (int j = 0; j < 8; ++j) st16_sys(d1 + (j >> 2) * HTB + (j & 3) * 2048, t[j]);
                }
            } else {
                unsigned char* d0 = dws + OFF_XA + par * XA_PAR + (size_t)(idx * 16 + tile) * BLK + q * 8192 + lane * 16;
#pragma unroll
                for (int i = 0; i < 8; ++i) st16_sys(d0 + i * 1024, cur[i]);
            }
            asm volatile("s_waitcnt vmcnt(0)" ::: "memory");
#pragma unroll
            for (int i = 0; i < 8; ++i) { cur[i] = nxt[i]; asm volatile("" : "+v"(cur[i])); }
            __syncthreads();
            if ((tid & 255) == 0) {
                st_flag(dws + (isw ? OFF_FW : OFF_FX) + (tile * 64 + idx) * 4, call);
                if (isw && sender) st_flag(ws + OFF_FD + ((dest * 8 + tile) * 8 + kc) * 4, call);
            }
        }
        if (sender && tid == 0)
            __hip_atomic_fetch_add((unsigned*)(ws + OFF_SENT + dest * 128), 1u, __ATOMIC_RELAXED, __HIP_MEMORY_SCOPE_AGENT);
        STAMP(1);
        if (b < 16) {
            if (tid == 0) __hip_atomic_store((unsigned*)(ws + OFF_CNT + b * 128), call, __ATOMIC_RELAXED, __HIP_MEMORY_SCOPE_AGENT);
            STAMP(5);
            return;
        }
    }

    int mt, gnt, skip, nkt;
    const unsigned char *abase, *fa; size_t astride;
    if (is_comm) {
        const int d = b - 16, e = d >> 1;
        mt = 2 * self + (d & 1); gnt = ((self + 1 + (e >> 3)) & 7) * 8 + (e & 7);
        skip = self; nkt = 8;
        abase = ws + OFF_WD + (size_t)(gnt * 8 - self * 8) * BLK; astride = BLK;
        fa = ws + OFF_FD + (gnt * 8 - self * 8) * 4;
    } else {
        const int xcd = b & 7, sl = (b >> 3) - 16;
        mt = 4 * (xcd >> 1) + (sl & 3); const int nt = 4 * (xcd & 1) + (sl >> 2);
        gnt = self * 8 + nt;
        skip = mt >> 1; nkt = skip == self ? 64 : 56;
        abase = ws + OFF_WB + par * WB_PAR + (size_t)nt * BLK; astride = 8 * (size_t)BLK;
        fa = ws + OFF_FW + nt * 256;
    }
    const unsigned char* const bbase = ws + OFF_XA + par * XA_PAR + (size_t)mt * BLK;
    const unsigned char* const fb = ws + OFF_FX + mt * 256;
    const int wr = wid >> 2, wc = wid & 3;
    const unsigned voff = tid * 16;
    const unsigned ldsw = wid * 1024;
    const int aoff = wr * 8192 + lane * 16, boff = wc * 4096 + lane * 16;
#define SA(bf, h) (((bf) * 2 + (h)) * HTB)
#define SB(bf, h) ((4 + (bf) * 2 + (h)) * HTB)
#define STAGE(bufoff, gbase) do { _Pragma("unroll") for (int _i = 0; _i < 2; ++_i) \
        __builtin_amdgcn_global_load_lds((const unsigned*)((gbase) + _i * 8192 + voff), (LAS unsigned*)(lds + (bufoff) + ldsw + _i * 8192), 16, 0, 17); } while (0)
#define LDA(dst, bf, h) do { _Pragma("unroll") for (int m = 0; m < 4; ++m) _Pragma("unroll") for (int k = 0; k < 2; ++k) \
        dst[m][k] = *(const LAS bf16x8*)(lds + SA(bf, h) + aoff + m * 2048 + k * 1024); } while (0)
#define LDB(dst, bf, h) do { _Pragma("unroll") for (int n = 0; n < 2; ++n) _Pragma("unroll") for (int k = 0; k < 2; ++k) \
        dst[n][k] = *(const LAS bf16x8*)(lds + SB(bf, h) + boff + n * 2048 + k * 1024); } while (0)
#define MMA(ai, bj, At, Bt) do { __builtin_amdgcn_s_setprio(1); _Pragma("unroll") for (int m = 0; m < 4; ++m) _Pragma("unroll") for (int n = 0; n < 2; ++n) _Pragma("unroll") for (int k = 0; k < 2; ++k) \
        acc[ai][bj][m][n] = __builtin_amdgcn_mfma_f32_16x16x32_bf16(Bt[n][k], At[m][k], acc[ai][bj][m][n], 0, 0, 0); __builtin_amdgcn_s_setprio(0); } while (0)
#define WAIT_V(n) asm volatile("s_waitcnt vmcnt(" #n ")" ::: "memory")
#define WAIT_L(n) asm volatile("s_waitcnt lgkmcnt(" #n ")" ::: "memory")
#define BAR __builtin_amdgcn_s_barrier()
#define SCHED __builtin_amdgcn_sched_barrier(0)
    f32x4 acc[2][2][4][2];
#pragma unroll
    for (int a = 0; a < 2; ++a)
#pragma unroll
        for (int c = 0; c < 2; ++c)
#pragma unroll
            for (int m = 0; m < 4; ++m)
#pragma unroll
                for (int n = 0; n < 2; ++n) acc[a][c][m][n] = (f32x4){0.f, 0.f, 0.f, 0.f};
    bf16x8 At[4][2], B0[2][2], B1[2][2];
    u64 ready = 0;
    int so = 0, skc = 0;
    auto next_idx = [&]() {
        const int idx = (((self + so) & 7) << 3) + skc;
        if (so == 0) { if (++skc == 8) { skc = 0; so = 1; } else return idx; }
        else if (++so == 8) { so = 1; ++skc; }
        if (((self + so) & 7) == skip) { if (++so == 8) { so = 1; ++skc; } }
        return idx;
    };
    const int i0 = next_idx(), i1 = next_idx();
    const unsigned char* const a0 = abase + (size_t)i0 * astride; const unsigned char* const b0 = bbase + (size_t)i0 * (16 * (size_t)BLK);
    const unsigned char* const a1f = abase + (size_t)i1 * astride; const unsigned char* const b1f = bbase + (size_t)i1 * (16 * (size_t)BLK);
    gate(ready, i0, fa, fb, tmo, call);
    STAMP(2);
    STAGE(SB(0, 0), b0); STAGE(SA(0, 0), a0); STAGE(SB(0, 1), b0 + HTB); STAGE(SA(0, 1), a0 + HTB);
    gate(ready, i1, fa, fb, tmo, call);
    if (wr == 1) BAR;
    WAIT_V(4); BAR;
    STAGE(SB(1, 0), b1f); STAGE(SA(1, 0), a1f); STAGE(SB(1, 1), b1f + HTB);
    WAIT_V(6); BAR;
    const unsigned char* a1 = a1f;
    for (int t = 0; t < nkt; t += 2) {
        const bool last = (t == nkt - 2);
        const int i2 = last ? i0 : next_idx(), i3 = last ? i1 : next_idx();
        const unsigned char* a2 = abase + (size_t)i2 * astride; const unsigned char* b2 = bbase + (size_t)i2 * (16 * (size_t)BLK);
        const unsigned char* a3 = abase + (size_t)i3 * astride; const unsigned char* b3 = bbase + (size_t)i3 * (16 * (size_t)BLK);
        LDB(B0, 0, 0); SCHED; LDA(At, 0, 0); STAGE(SA(1, 1), a1 + HTB);
        WAIT_L(8); BAR; WAIT_L(0); MMA(0, 0, At, B0); BAR; SCHED;
        gate(ready, i2, fa, fb, tmo, call);
        LDB(B1, 0, 1); STAGE(SB(0, 0), b2);
        BAR; WAIT_L(0); MMA(0, 1, At, B1); BAR;
        LDA(At, 0, 1); STAGE(SA(0, 0), a2);
        BAR; WAIT_L(0); MMA(1, 0, At, B0); BAR; SCHED;
        STAGE(SB(0, 1), b2 + HTB);
        WAIT_V(6); BAR; MMA(1, 1, At, B1); BAR;
        LDB(B0, 1, 0); SCHED; LDA(At, 1, 0); STAGE(SA(0, 1), a2 + HTB);
        WAIT_L(8); BAR; WAIT_L(0); MMA(0, 0, At, B0); BAR; SCHED;
        gate(ready, i3, fa, fb, tmo, call);
        LDB(B1, 1, 1); STAGE(SB(1, 0), b3);
        BAR; WAIT_L(0); MMA(0, 1, At, B1); BAR;
        LDA(At, 1, 1); STAGE(SA(1, 0), a3);
        BAR; WAIT_L(0); MMA(1, 0, At, B0); BAR; SCHED;
        STAGE(SB(1, 1), b3 + HTB);
        WAIT_V(6); BAR; MMA(1, 1, At, B1); BAR;
        a1 = a3;
    }
    WAIT_V(0);
    if (wr == 0) BAR;
    STAMP(3);

    const int lz = lane_id(), fr = lz & 15, fq = lz >> 4;
    const int owner = mt >> 1;
    const int ptile = (mt & 1) * 64 + gnt;
    if (!is_comm && owner != self) {
        wait_word(ws + OFF_SENT + owner * 128, call * SEND_G, tmo, call);
        wait_word(fa + owner * 32, call, tmo, call);
        const bool bad = (unsigned)__builtin_amdgcn_readfirstlane(__hip_atomic_load((const unsigned*)tmo, __ATOMIC_RELAXED, __HIP_MEMORY_SCOPE_AGENT)) == call;
        STAMP(4);
        unsigned char* dst = P.ws[owner] + OFF_PB + (size_t)ptile * 131072 + wid * 16384 + lz * 16;
#pragma unroll
        for (int bj = 0; bj < 2; ++bj)
#pragma unroll
            for (int n = 0; n < 2; ++n)
#pragma unroll
                for (int r = 0; r < 4; ++r) {
                    u32x4 v;
                    v.x = cvt_pk_bf16(acc[0][bj][0][n][r], acc[0][bj][1][n][r]); v.y = cvt_pk_bf16(acc[0][bj][2][n][r], acc[0][bj][3][n][r]);
                    v.z = cvt_pk_bf16(acc[1][bj][0][n][r], acc[1][bj][1][n][r]); v.w = cvt_pk_bf16(acc[1][bj][2][n][r], acc[1][bj][3][n][r]);
                    if (bad) v = (u32x4){0x7fc07fc0u, 0x7fc07fc0u, 0x7fc07fc0u, 0x7fc07fc0u};
                    st16_sys(dst + ((bj * 2 + n) * 4 + r) * 1024, v);
                }
        asm volatile("s_waitcnt vmcnt(0)" ::: "memory");
        if (lz == 0) st_flag(P.ws[owner] + OFF_FP + (ptile * 8 + wid) * 128, call);
    } else {
        if (is_comm) {
            wait_word(ws + OFF_FP + (ptile * 8 + wid) * 128, call, tmo, call);
            STAMP(4);
            const unsigned char* src = ws + OFF_PB + (size_t)ptile * 131072 + wid * 16384;
            const unsigned lo = lz * 16;
#pragma unroll
            for (int bj = 0; bj < 2; ++bj) {
                u32x4 p[8];
                asm volatile("global_load_dwordx4 %0, %8, %9 sc0 sc1\n\t"
                             "global_load_dwordx4 %1, %8, %9 offset:1024 sc0 sc1\n\t"
                             "global_load_dwordx4 %2, %8, %9 offset:2048 sc0 sc1\n\t"
                             "global_load_dwordx4 %3, %8, %9 offset:3072 sc0 sc1\n\t"
                             "global_load_dwordx4 %4, %8, %10 sc0 sc1\n\t"
                             "global_load_dwordx4 %5, %8, %10 offset:1024 sc0 sc1\n\t"
                             "global_load_dwordx4 %6, %8, %10 offset:2048 sc0 sc1\n\t"
                             "global_load_dwordx4 %7, %8, %10 offset:3072 sc0 sc1\n\t"
                             "s_waitcnt vmcnt(0)"
                             : "=&v"(p[0]), "=&v"(p[1]), "=&v"(p[2]), "=&v"(p[3]), "=&v"(p[4]), "=&v"(p[5]), "=&v"(p[6]), "=&v"(p[7])
                             : "v"(lo), "s"(src + bj * 8192), "s"(src + bj * 8192 + 4096) : "memory");
#pragma unroll
                for (int n = 0; n < 2; ++n)
#pragma unroll
                    for (int r = 0; r < 4; ++r)
#pragma unroll
                        for (int j = 0; j < 8; ++j) {
                            const unsigned u = p[n * 4 + r][j >> 1];
                            acc[j >> 2][bj][j & 3][n][r] += __uint_as_float((j & 1) ? (u & 0xffff0000u) : (u << 16));
                        }
            }
        }
        const bool bad = (unsigned)__builtin_amdgcn_readfirstlane(__hip_atomic_load((const unsigned*)tmo, __ATOMIC_RELAXED, __HIP_MEMORY_SCOPE_AGENT)) == call;
        unsigned char* dst = P.out + (size_t)((mt & 1) * 256 + wc * 32 + fq * 4) * (GN * 2) + (size_t)(gnt * 256 + wr * 128 + fr * 8) * 2;
#pragma unroll
        for (int bj = 0; bj < 2; ++bj)
#pragma unroll
            for (int n = 0; n < 2; ++n)
#pragma unroll
                for (int r = 0; r < 4; ++r) {
                    u32x4 v;
                    v.x = cvt_pk_bf16(gelu(acc[0][bj][0][n][r]), gelu(acc[0][bj][1][n][r])); v.y = cvt_pk_bf16(gelu(acc[0][bj][2][n][r]), gelu(acc[0][bj][3][n][r]));
                    v.z = cvt_pk_bf16(gelu(acc[1][bj][0][n][r]), gelu(acc[1][bj][1][n][r])); v.w = cvt_pk_bf16(gelu(acc[1][bj][2][n][r]), gelu(acc[1][bj][3][n][r]));
                    if (bad) v = (u32x4){0x7fc07fc0u, 0x7fc07fc0u, 0x7fc07fc0u, 0x7fc07fc0u};
                    *(u32x4*)(dst + (size_t)(bj * 128 + n * 16 + r) * (GN * 2)) = v;
                }
    }
    if (tid == 0) __hip_atomic_store((unsigned*)(ws + OFF_CNT + b * 128), call, __ATOMIC_RELAXED, __HIP_MEMORY_SCOPE_AGENT);
    STAMP(5);
}

}

extern "C" void kernel_launch(int rank, int nranks, ncclComm_t comm, hipStream_t stream,
                              void* const* d_in, const int* in_sizes, int n_in,
                              void* d_out, int out_size, const int* in_dtypes, int out_dtype,
                              void* const* d_out_all, void* const* d_ws_all, size_t ws_bytes) {
    (void)comm; (void)in_dtypes; (void)out_dtype; (void)d_out_all;
    if (nranks != NR || (unsigned)rank >= (unsigned)NR || n_in < 2 || ws_bytes < WS_NEED
        || in_sizes[0] != GM * KS || in_sizes[1] != KS * GN || out_size != (GM / NR) * GN) {
        fprintf(stderr, "kernel_launch: unexpected problem shape\n");
        return;
    }
    Params p;
    p.x = (const unsigned char*)d_in[0];
    p.w = (const unsigned char*)d_in[1];
    p.out = (unsigned char*)d_out;
    for (int i = 0; i < NR; ++i) p.ws[i] = (unsigned char*)d_ws_all[i];
    p.rank = rank; p.pad = 0;
    hipLaunchKernelGGL(fused_gemm_rs, dim3(256), dim3(512), 0, stream, p);
    HIP_CHECK(hipGetLastError());
}
```

```cpp
#include <hip/hip_runtime.h>
#include <rccl/rccl.h>
#include <cstdio>
#include <cstdint>

#define HIP_CHECK(expr) do { hipError_t _e = (expr); if (_e != hipSuccess) { \
    fprintf(stderr, "HIP error %s at %s:%d\n", hipGetErrorString(_e), __FILE__, __LINE__); \
    return; } } while (0)

namespace {

#define LAS __attribute__((address_space(3)))
typedef short bf16x8 __attribute__((ext_vector_type(8)));
typedef float f32x4 __attribute__((ext_vector_type(4)));
typedef unsigned u32x4 __attribute__((ext_vector_type(4)));
typedef unsigned long long u64;

constexpr int NR = 8;
constexpr int GM = 4096, GK = 4096, GN = 16384;
constexpr int KS = GK / NR;
constexpr int BLK = 32768;
constexpr int HTB = 16384;
#define SEND_G 6
#define OUT_MODE 0
#define TMAP 0
#define DIAG_GATE 0
#define CH_AT 2
#define XSTR2(x) #x
#define XSTR(x) XSTR2(x)
constexpr int NSEND = 7 * SEND_G;
constexpr int NCOMM = 112;
constexpr u64 DEADLINE = 50000000ull;

constexpr int NTASK = 704;
constexpr int UW = (NCOMM - NSEND) * 8 / 7;

constexpr size_t OFF_CNT = 0;
constexpr size_t OFF_TMO = 32768;
constexpr size_t OFF_TICK = 33792;
constexpr size_t OFF_CRED = 34816;
constexpr size_t OFF_BDONE = 35840;
constexpr size_t OFF_CH = 393216;
constexpr size_t OFF_FX = 65536;
constexpr size_t OFF_FW = 81920;
constexpr size_t OFF_FD = 98304;
constexpr size_t OFF_HIN = 131072;
constexpr size_t OFF_HP = 327680;
constexpr size_t OFF_STAMP = 524288;
constexpr size_t OFF_XA = 1u << 20;
constexpr size_t OFF_WB = OFF_XA + (size_t)64 * 16 * BLK;
constexpr size_t OFF_WD = OFF_WB + (size_t)64 * 8 * BLK;
constexpr size_t OFF_RP = OFF_WD + (size_t)64 * 8 * BLK;
constexpr size_t OFF_RE = OFF_RP + (size_t)14 * NTASK * 6144;
constexpr size_t OFF_PP = OFF_RE + (size_t)14 * NTASK * 2048;
constexpr size_t OFF_PE = OFF_PP + (size_t)2048 * 6144;
constexpr size_t WS_NEED = OFF_PE + (size_t)2048 * 2048;
static_assert(OFF_HIN + 14 * NTASK * 16 <= OFF_HP && OFF_HP + 2048 * 16 <= OFF_CH && OFF_CH + 512 * 128 <= OFF_STAMP, "flag region");
static_assert(WS_NEED <= (size_t)161 * 1048576, "workspace");

struct Params {
    const unsigned char* x;
    const unsigned char* w;
    unsigned char* out;
    unsigned char* ws[NR];
    int rank, pad;
};

__device__ __forceinline__ void st16_sys(unsigned char* p, u32x4 v) {
    asm volatile("global_store_dwordx4 %0, %1, off sc0 sc1\n\ts_nop 1" :: "v"(p), "v"(v) : "memory");
}
__device__ __forceinline__ void st_flag(unsigned char* p, unsigned v) {
    __hip_atomic_store((unsigned*)p, v, __ATOMIC_RELAXED, __HIP_MEMORY_SCOPE_SYSTEM);
}
__device__ __forceinline__ void st_flag8(unsigned char* p, unsigned v) {
    __hip_atomic_store(p, (unsigned char)v, __ATOMIC_RELAXED, __HIP_MEMORY_SCOPE_SYSTEM);
}
__device__ __forceinline__ unsigned ld_flag(const unsigned char* p) {
    return __hip_atomic_load((const unsigned*)p, __ATOMIC_RELAXED, __HIP_MEMORY_SCOPE_SYSTEM);
}
__device__ __forceinline__ unsigned lane_id() {
    return __builtin_amdgcn_mbcnt_hi(~0u, __builtin_amdgcn_mbcnt_lo(~0u, 0u));
}
__device__ __forceinline__ unsigned cvt_pk_bf16(float lo, float hi) {
    unsigned r; asm volatile("v_cvt_pk_bf16_f32 %0, %1, %2" : "=v"(r) : "v"(lo), "v"(hi)); return r;
}
typedef float f32x2 __attribute__((ext_vector_type(2)));
__device__ __forceinline__ f32x2 gelu2(f32x2 y) {
    const float k1 = -2.0f * 0.7978845608028654f * 1.4426950408889634f;
    const float k3 = k1 * 0.044715f;
    const f32x2 z = y * ((y * y) * k3 + k1);
    const f32x2 d = (f32x2){__builtin_amdgcn_exp2f(z.x), __builtin_amdgcn_exp2f(z.y)} + 1.0f;
    return y * (f32x2){__builtin_amdgcn_rcpf(d.x), __builtin_amdgcn_rcpf(d.y)};
}

__device__ __forceinline__ int pick(u64& ready, u64& todo, bool idle, const unsigned char* fa, const unsigned char* fb,
                                    unsigned char* tmo, unsigned call, unsigned tag4) {
    if ((ready & todo) == 0ull) {
        if (!idle) return 64;
        const u64 t0 = __builtin_amdgcn_s_memrealtime();
        for (;;) {
            unsigned a, b, t; const unsigned off = lane_id() * 4u, zero = 0u;
            asm volatile("global_load_dword %0, %3, %5 sc0 sc1\n\t"
                         "global_load_dword %1, %3, %6 sc0 sc1\n\t"
                         "global_load_dword %2, %4, %7 sc1\n\t"
                         "s_waitcnt vmcnt(0)"
                         : "=&v"(a), "=&v"(b), "=&v"(t) : "v"(off), "v"(zero), "s"(fa), "s"(fb), "s"(tmo) : "memory");
            ready = __ballot(a == tag4 && b == tag4);
            if ((ready & todo) != 0ull) break;
            if ((unsigned)__builtin_amdgcn_readfirstlane(t) == call || __builtin_amdgcn_s_memrealtime() - t0 > DEADLINE) {
                __hip_atomic_store((unsigned*)tmo, call, __ATOMIC_RELAXED, __HIP_MEMORY_SCOPE_AGENT);
                ready = ~0ull;
                break;
            }
            __builtin_amdgcn_s_sleep(1);
        }
    }
    const int idx = __builtin_ctzll(ready & todo);
    todo &= ~(1ull << idx);
    return idx;
}

typedef unsigned short u16x2 __attribute__((ext_vector_type(2)));
__device__ __forceinline__ unsigned bfi(unsigned m, unsigned a, unsigned b) { return (a & m) | (b & ~m); }

struct Packed { u32x4 lo[4], nb[2], es[2]; u64 mask; unsigned base; };

__device__ __forceinline__ void pack(const u32x4 (&d)[8], Packed& o) {
    u16x2 mx = {0, 0};
#pragma unroll
    for (int i = 0; i < 32; ++i) mx = __builtin_elementwise_max(mx, __builtin_bit_cast(u16x2, d[i >> 2][i & 3] & 0x7fff7fffu));
    unsigned bm = (unsigned)(mx.x > mx.y ? mx.x : mx.y) >> 8;
    bm = max(bm, (unsigned)__builtin_amdgcn_mov_dpp((int)bm, 0xB1, 0xf, 0xf, true));
    bm = max(bm, (unsigned)__builtin_amdgcn_mov_dpp((int)bm, 0x4E, 0xf, 0xf, true));
    bm = max(bm, (unsigned)__builtin_amdgcn_mov_dpp((int)bm, 0x124, 0xf, 0xf, true));
    bm = max(bm, (unsigned)__builtin_amdgcn_mov_dpp((int)bm, 0x128, 0xf, 0xf, true));
    o.base = max(max((unsigned)__builtin_amdgcn_readlane(bm, 0), (unsigned)__builtin_amdgcn_readlane(bm, 16)),
                 max((unsigned)__builtin_amdgcn_readlane(bm, 32), (unsigned)__builtin_amdgcn_readlane(bm, 48)));
    const unsigned base4 = o.base * 0x01010101u;
    unsigned any = 0;
#pragma unroll
    for (int i = 0; i < 8; ++i) {
        const unsigned h0 = __builtin_amdgcn_perm(d[i].y, d[i].x, 0x07050301u), h1 = __builtin_amdgcn_perm(d[i].w, d[i].z, 0x07050301u);
        o.lo[i >> 1][(i & 1) * 2] = __builtin_amdgcn_perm(d[i].y, d[i].x, 0x06040200u);
        o.lo[i >> 1][(i & 1) * 2 + 1] = __builtin_amdgcn_perm(d[i].w, d[i].z, 0x06040200u);
        const unsigned d0 = base4 - (h0 & 0x7f7f7f7fu), d1 = base4 - (h1 & 0x7f7f7f7fu);
        o.nb[i >> 2][i & 3] = bfi(0x0f0f0f0fu, bfi(0x07070707u, d0, h0 >> 4), bfi(0x80808080u, h1, d1 << 4));
        const unsigned e = bfi(0x0f0f0f0fu, d0 >> 3, d1 << 1);
        o.es[i >> 2][i & 3] = e; any |= e;
    }
    o.mask = __ballot(any != 0u);
}
__device__ __forceinline__ void send(const Packed& o, unsigned char* pl, unsigned char* es, unsigned lane) {
    if ((o.mask >> lane) & 1ull) { st16_sys(es + lane * 32, o.es[0]); st16_sys(es + lane * 32 + 16, o.es[1]); }
    unsigned char* p = pl + lane * 16;
#pragma unroll
    for (int i = 0; i < 4; ++i) st16_sys(p + i * 1024, o.lo[i]);
    st16_sys(p + 4096, o.nb[0]); st16_sys(p + 5120, o.nb[1]);
}
__device__ __forceinline__ u32x4 header(const Packed& o, unsigned call) {
    return (u32x4){(unsigned)o.mask, call, (unsigned)(o.mask >> 32), (call << 7) | o.base};
}

__device__ __forceinline__ unsigned recv_unpack(u32x4 (&d)[8], const unsigned char* pl, const unsigned char* es, const unsigned char* hd,
                                            unsigned char* tmo, unsigned call, unsigned lane) {
    unsigned base, mlo, mhi;
    {
        const u64 t0 = __builtin_amdgcn_s_memrealtime();
        for (;;) {
            u32x4 h; unsigned t; const unsigned zero = 0u;
            asm volatile("global_load_dwordx4 %0, %2, %3 sc0 sc1\n\t"
                         "global_load_dword %1, %2, %4 sc1\n\t"
                         "s_waitcnt vmcnt(0)" : "=&v"(h), "=&v"(t) : "v"(zero), "s"(hd), "s"(tmo) : "memory");
            const unsigned t1 = __builtin_amdgcn_readfirstlane(h.y), t3 = __builtin_amdgcn_readfirstlane(h.w);
            mlo = __builtin_amdgcn_readfirstlane(h.x); mhi = __builtin_amdgcn_readfirstlane(h.z); base = t3 & 127u;
            if (t1 == call && (t3 >> 7) == (call & 0x1ffffffu)) break;
            if ((unsigned)__builtin_amdgcn_readfirstlane(t) == call || __builtin_amdgcn_s_memrealtime() - t0 > DEADLINE) {
                __hip_atomic_store((unsigned*)tmo, call, __ATOMIC_RELAXED, __HIP_MEMORY_SCOPE_AGENT);
                break;
            }
            __builtin_amdgcn_s_sleep(1);
        }
    }
    u32x4 lo[4], nb[2], e[2] = {{0u, 0u, 0u, 0u}, {0u, 0u, 0u, 0u}};
    const unsigned off = lane * 16;
    const u64 mask = ((u64)mhi << 32) | mlo;
    u64 keep; unsigned tw = 0u;
    asm volatile("s_mov_b64 %8, exec\n\t"
                 "s_mov_b64 exec, %15\n\t"
                 "global_load_dwordx4 %6, %11, %14 sc0 sc1\n\t"
                 "global_load_dwordx4 %7, %11, %14 offset:16 sc0 sc1\n\t"
                 "s_mov_b64 exec, %8\n\t"
                 "global_load_dwordx4 %0, %10, %12 sc0 sc1\n\t"
                 "global_load_dwordx4 %1, %10, %12 offset:1024 sc0 sc1\n\t"
                 "global_load_dwordx4 %2, %10, %12 offset:2048 sc0 sc1\n\t"
                 "global_load_dwordx4 %3, %10, %12 offset:3072 sc0 sc1\n\t"
                 "global_load_dwordx4 %4, %10, %13 sc0 sc1\n\t"
                 "global_load_dwordx4 %5, %10, %13 offset:1024 sc0 sc1\n\t"
                 "global_load_dword %9, %9, %16 sc1\n\t"
                 "s_waitcnt vmcnt(0)"
                 : "=&v"(lo[0]), "=&v"(lo[1]), "=&v"(lo[2]), "=&v"(lo[3]), "=&v"(nb[0]), "=&v"(nb[1]), "+v"(e[0]), "+v"(e[1]), "=&s"(keep), "+v"(tw)
                 : "v"(off), "v"(off * 2), "s"(pl), "s"(pl + 4096), "s"(es), "s"(mask), "s"(tmo) : "memory");
    const unsigned base4 = base * 0x01010101u;
#pragma unroll
    for (int i = 0; i < 8; ++i) {
        const unsigned n = nb[i >> 2][i & 3], ee = e[i >> 2][i & 3];
        const unsigned d0 = (n & 0x07070707u) | ((ee & 0x0f0f0f0fu) << 3), d1 = ((n >> 4) & 0x07070707u) | ((ee & 0xf0f0f0f0u) >> 1);
        const unsigned h0 = (base4 - d0) | ((n << 4) & 0x80808080u), h1 = (base4 - d1) | (n & 0x80808080u);
        const unsigned l0 = lo[i >> 1][(i & 1) * 2], l1 = lo[i >> 1][(i & 1) * 2 + 1];
        d[i].x = __builtin_amdgcn_perm(h0, l0, 0x05010400u); d[i].y = __builtin_amdgcn_perm(h0, l0, 0x07030602u);
        d[i].z = __builtin_amdgcn_perm(h1, l1, 0x05010400u); d[i].w = __builtin_amdgcn_perm(h1, l1, 0x07030602u);
    }
    return __builtin_amdgcn_readfirstlane(tw);
}

#define STAMP(i) do {} while (0)

#define ORD_LAG 2.3
#define ORD_C0 7.0
#define ORD_CS 0.32
#define ORD_W 0.3
struct alignas(4) Order { unsigned char v[176]; };
constexpr Order make_order() {
    double key[176] = {}; unsigned char id[176] = {}; int n = 0;
    for (int kc = 0; kc < 8; ++kc) {
        for (int e = 0; e < 8; ++e) { key[n] = kc - ORD_W + (e + 0.4) / 8; id[n++] = (unsigned char)(0 << 6 | kc << 3 | e); }
        for (int e = 0; e < 6; ++e) { key[n] = kc + (e + 0.5) / 6; id[n++] = (unsigned char)(1 << 6 | kc << 3 | e); }
        for (int e = 0; e < 6; ++e) { key[n] = kc + ORD_LAG + (e + 0.6) / 6; id[n++] = (unsigned char)(2 << 6 | kc << 3 | e); }
        for (int e = 0; e < 2; ++e) { key[n] = ORD_C0 + (2 * kc + e) * ORD_CS; id[n++] = (unsigned char)(3 << 6 | kc << 3 | e); }
    }
    Order o = {};
    for (int i = 0; i < 176; ++i) { int r = 0; for (int j = 0; j < 176; ++j) r += (key[j] < key[i]) || (key[j] == key[i] && j < i); o.v[r] = id[i]; }
    return o;
}
constexpr Order ORDER = make_order();
constexpr unsigned order_word(int i) { return ORDER.v[4 * i] | ORDER.v[4 * i + 1] << 8 | ORDER.v[4 * i + 2] << 16 | (unsigned)ORDER.v[4 * i + 3] << 24; }
template <int I> __device__ __forceinline__ unsigned order_vgpr(unsigned v) {
    if constexpr (I == 44) return v;
    else {
        unsigned t;
        asm("s_mov_b32 %1, %2\n\tv_writelane_b32 %0, %1, %3" : "+v"(v), "=&s"(t) : "i"(order_word(I)), "i"(I));
        return order_vgpr<I + 1>(v);
    }
}

__device__ __forceinline__ void decode_item(unsigned ordv, int n, int src, int dest, int& isw, int& kc, int& tile) {
    if (dest == src) {
        kc = n / 24; const int r = n - kc * 24;
        isw = r < 8; tile = isw ? r : r - 8;
        return;
    }
    const unsigned v = ((unsigned)__builtin_amdgcn_readlane(ordv, n >> 2) >> ((n & 3) * 8)) & 255u;
    const int kind = v >> 6, e = v & 7;
    kc = (v >> 3) & 7; isw = kind == 0;
    const int d = (dest - src - 1) & 7;
    tile = kind == 0 ? ((e + src) & 7) : kind == 3 ? 2 * dest + e : 2 * ((src + 1 + e + (e >= d)) & 7) + (kind - 1);
}

__global__ __launch_bounds__(512, 2) void fused_gemm_rs(const Params P) {
    __shared__ __attribute__((aligned(16))) unsigned char lds_raw[8 * HTB + 64];
    LAS unsigned char* lds = (LAS unsigned char*)lds_raw;
    const int tid = threadIdx.x, wid = __builtin_amdgcn_readfirstlane(tid >> 6), lane = tid & 63;
    const int b = blockIdx.x, self = P.rank;
    unsigned char* const ws = P.ws[self];
    unsigned char* const tmo = ws + OFF_TMO;
    STAMP(0);
    const bool is_comm = b < NCOMM, is_send = b < NSEND;
    const unsigned ordv = order_vgpr<0>(0u);
    const unsigned lw = (lane >> 4) * 8 * (GN * 2) + (lane & 15) * 16, lx = (lane & 15) * (KS * 2) + (lane >> 4) * 16;
    auto load_task = [&](int task, int dest, u32x4 (&r)[8]) {
        int isw, kc, tile; decode_item(ordv, task >> 2, self, dest, isw, kc, tile); const int q = task & 3;
        const unsigned char* src = isw ? P.w + ((kc * 64 + (q & 1) * 32) * (GN * 2) + ((dest * 8 + tile) * 256 + (q >> 1) * 128) * 2)
                                       : P.x + ((tile * 256 + q * 64) * (KS * 2) + kc * 128);
        const unsigned lo = isw ? lw : lx, s1 = isw ? GN * 2 : 64, s2 = isw ? GN * 4 : 16 * KS * 2;
#pragma unroll
        for (int i = 0; i < 8; ++i) r[i] = *(const u32x4*)(src + ((i >> 1) * s2 + (i & 1) * s1) + lo);
    };
    u32x4 cur[8];
    const int link = b % 7, mdest = is_send ? (self + 1 + link) & 7 : self;
    const int task0 = is_send ? (b / 7) * 8 + wid : (b - NSEND) * 8 + wid;
    if (is_comm) load_task(task0, mdest, cur);
    const unsigned call = (unsigned)__builtin_amdgcn_readfirstlane(
        __hip_atomic_load((const unsigned*)(ws + OFF_CNT + b * 128), __ATOMIC_RELAXED, __HIP_MEMORY_SCOPE_AGENT)) + 1u;
    const size_t par = call & 1u;
    const unsigned tag4 = (call & 255u) * 0x01010101u;
    if (is_comm) {
        auto transpose8 = [&](u32x4 (&c)[8]) {
            u32x4 t[8];
#pragma unroll
            for (int j = 0; j < 8; ++j)
#pragma unroll
                for (int qd = 0; qd < 4; ++qd)
                    t[j][qd] = __builtin_amdgcn_perm(c[2 * qd + 1][j >> 1], c[2 * qd][j >> 1], (j & 1) ? 0x07060302u : 0x05040100u);
#pragma unroll
            for (int j = 0; j < 8; ++j) c[j] = t[j];
        };
        auto store_image = [&](unsigned char* blk, int isw, int q, const u32x4 (&c)[8]) {
            if (isw) {
                unsigned char* d0 = blk + (q >> 1) * 8192 + (q & 1) * 1024 + lane * 16;
#pragma unroll
                for (int j = 0; j < 8; ++j) st16_sys(d0 + (j >> 2) * HTB + (j & 3) * 2048, c[j]);
            } else {
                unsigned char* d0 = blk + q * 8192 + lane * 16;
#pragma unroll
                for (int i = 0; i < 8; ++i) st16_sys(d0 + i * 1024, c[i]);
            }
        };
        if (is_send) {
            const int dest = mdest;
            unsigned char* const dws = P.ws[dest];
            const size_t slot = (par * 7 + (6 - link)) * NTASK;
            unsigned* const tick = (unsigned*)(ws + OFF_TICK + dest * 128);
            const unsigned tbase = (call - 1u) * NTASK - SEND_G * 8;
            auto draw = [&](unsigned& nn) {
                u64 keep; const unsigned zero = 0u; nn = 1u;
                asm volatile("s_mov_b64 %1, exec\n\ts_mov_b64 exec, 1\n\t"
                             "global_atomic_add %0, %2, %0, %3 sc0\n\t"
                             "s_mov_b64 exec, %1" : "+v"(nn), "=&s"(keep) : "v"(zero), "s"(tick) : "memory");
            };
            int task = task0;
            unsigned nn;
#pragma unroll
            for (int i = 0; i < 8; ++i) asm volatile("" : "+v"(cur[i]));
            draw(nn);
            if (lane == 0) st_flag(ws + OFF_CNT + b * 128 + 64 + wid * 4, call);
            for (;;) {
                int isw, kc, tile; decode_item(ordv, task >> 2, self, dest, isw, kc, tile); const int q = task & 3;
                if (isw) {
                    transpose8(cur);
                    store_image(ws + OFF_WD + (size_t)((dest * 8 + tile) * 8 + kc) * BLK, 1, q, cur);
                }
                Packed pk; pack(cur, pk);
                send(pk, dws + OFF_RP + (slot + task) * 6144, dws + OFF_RE + (slot + task) * 2048, lane);
                const u32x4 hd = header(pk, call);
                if (isw) asm volatile("s_waitcnt vmcnt(15)" ::: "memory"); else asm volatile("s_waitcnt vmcnt(7)" ::: "memory");
                asm volatile("" : "+v"(nn));
                const int ntask = (int)(__builtin_amdgcn_readfirstlane(nn) - tbase);
                const bool more = ntask < NTASK;
                load_task(more ? ntask : task, dest, cur);
                asm volatile("s_waitcnt vmcnt(0)" ::: "memory");
#pragma unroll
                for (int i = 0; i < 8; ++i) asm volatile("" : "+v"(cur[i]));
                if (isw && lane == 0) st_flag8(ws + OFF_FD + ((dest * 8 + tile) * 8 + kc) * 4 + q, call);
                if (more) draw(nn);
                if (lane == 0) {
                    st16_sys(dws + OFF_HIN + (slot + task) * 16, hd);
                }
                if (!more) break;
                task = ntask;
            }
            if (lane == 0) __hip_atomic_fetch_add((unsigned*)(ws + OFF_CRED + dest * 128), 1u, __ATOMIC_RELAXED, __HIP_MEMORY_SCOPE_AGENT);
        } else {
            const int u = task0;
            for (int task = u; task < 768; task += (NCOMM - NSEND) * 8) {
                if (task != u) load_task(task, self, cur);
#pragma unroll
                for (int i = 0; i < 8; ++i) asm volatile("" : "+v"(cur[i]));
                int isw, kc, tile; decode_item(ordv, task >> 2, self, self, isw, kc, tile); const int q = task & 3, idx = self * 8 + kc;
                if (isw) transpose8(cur);
                store_image(isw ? ws + OFF_WB + (size_t)(idx * 8 + tile) * BLK : ws + OFF_XA + (size_t)(idx * 16 + tile) * BLK, isw, q, cur);
                asm volatile("s_waitcnt vmcnt(0)" ::: "memory");
                if (lane == 0) st_flag8(ws + (isw ? OFF_FW : OFF_FX) + (tile * 64 + idx) * 4 + q, call);
            }
            STAMP(6);
            const int sl = u % 7, src = (self + 1 + sl) & 7;
            const size_t slot = (par * 7 + sl) * NTASK;
            if (u < UW * 7)
            for (int task = u / 7; task < NTASK; task += UW) {
                recv_unpack(cur, ws + OFF_RP + (slot + task) * 6144, ws + OFF_RE + (slot + task) * 2048, ws + OFF_HIN + (slot + task) * 16, tmo, call, lane);
                int isw, kc, tile; decode_item(ordv, task >> 2, src, self, isw, kc, tile); const int q = task & 3, idx = src * 8 + kc;
                store_image(isw ? ws + OFF_WB + (size_t)(idx * 8 + tile) * BLK : ws + OFF_XA + (size_t)(idx * 16 + tile) * BLK, isw, q, cur);
                asm volatile("s_waitcnt vmcnt(0)" ::: "memory");
                if (lane == 0) st_flag8(ws + (isw ? OFF_FW : OFF_FX) + (tile * 64 + idx) * 4 + q, call);
            }
        }
        STAMP(1);
        __syncthreads();
    }

    int mt, gnt, nkt; u64 todo;
    const unsigned char *abase, *fa; size_t astride;
    const bool is_main = !is_comm && b < 240;
    if (is_comm) {
        mt = 2 * self + ((b >> 3) & 1); gnt = ((self + 1 + (b >> 4)) & 7) * 8 + (b & 7);
        nkt = 8; todo = 0xffull << (self * 8);
        abase = ws + OFF_WD + (size_t)(gnt * 8 - self * 8) * BLK; astride = BLK;
        fa = ws + OFF_FD + (gnt * 8 - self * 8) * 4;
    } else {
        const int xcd = b & 7, sl = (b >> 3) - 14;
        mt = is_main ? 4 * (xcd >> 1) + (sl & 3) : 2 * self + (sl & 1);
        const int nt = 4 * (xcd & 1) + (is_main ? sl >> 2 : xcd >> 1);
        gnt = self * 8 + nt;
        const bool own = (mt >> 1) == self;
        nkt = own ? 32 : 56;
        todo = !own ? ~(0xffull << ((mt >> 1) * 8)) : is_main ? 0x5555555555555555ull : 0xaaaaaaaaaaaaaaaaull;
        abase = ws + OFF_WB + (size_t)nt * BLK; astride = 8 * (size_t)BLK;
        fa = ws + OFF_FW + nt * 256;
    }
    const unsigned char* const bbase = ws + OFF_XA + (size_t)mt * BLK;
    const unsigned char* const fb = ws + OFF_FX + mt * 256;
    const int wr = wid >> 2, wc = wid & 3;
    const unsigned voff = tid * 16;
    const unsigned ldsw = wid * 1024;
    const int aoff = wr * 8192 + lane * 16;
    int boff = 4 * HTB + wc * 4096 + lane * 16;
    asm volatile("" : "+v"(boff));
#define SA(bf, h) (((bf) * 2 + (h)) * HTB)
#define SB(bf, h) ((4 + (bf) * 2 + (h)) * HTB)
#define STAGE(bufoff, gbase) do { _Pragma("unroll") for (int _i = 0; _i < 2; ++_i) \
        __builtin_amdgcn_global_load_lds((const unsigned*)((gbase) + _i * 8192 + voff), (LAS unsigned*)(lds + (bufoff) + ldsw + _i * 8192), 16, 0, 17); } while (0)
#define LDA(dst, bf, h) do { _Pragma("unroll") for (int m = 0; m < 4; ++m) _Pragma("unroll") for (int k = 0; k < 2; ++k) \
        dst[m][k] = *(const LAS bf16x8*)(lds + SA(bf, h) + aoff + m * 2048 + k * 1024); } while (0)
#define LDB(dst, bf, h) do { _Pragma("unroll") for (int n = 0; n < 2; ++n) _Pragma("unroll") for (int k = 0; k < 2; ++k) \
        dst[n][k] = *(const LAS bf16x8*)(lds + boff + (SB(bf, h) - 4 * HTB + n * 2048 + k * 1024)); } while (0)
#define MMA(ai, bj, At, Bt) do { __builtin_amdgcn_s_setprio(1); _Pragma("unroll") for (int m = 0; m < 4; ++m) _Pragma("unroll") for (int n = 0; n < 2; ++n) _Pragma("unroll") for (int k = 0; k < 2; ++k) \
        acc[ai][bj][m][n] = __builtin_amdgcn_mfma_f32_16x16x32_bf16(Bt[n][k], At[m][k], acc[ai][bj][m][n], 0, 0, 0); __builtin_amdgcn_s_setprio(0); } while (0)
#define WAIT_V(n) asm volatile("s_waitcnt vmcnt(" #n ")" ::: "memory")
#define WAIT_L0 do { asm volatile("" ::: "memory"); __builtin_amdgcn_s_waitcnt(0xC07F); asm volatile("" ::: "memory"); } while (0)
#define BAR __builtin_amdgcn_s_barrier()
#define SCHED __builtin_amdgcn_sched_barrier(0)
    f32x4 acc[2][2][4][2];
#pragma unroll
    for (int a = 0; a < 2; ++a)
#pragma unroll
        for (int c = 0; c < 2; ++c)
#pragma unroll
            for (int m = 0; m < 4; ++m)
#pragma unroll
                for (int n = 0; n < 2; ++n) acc[a][c][m][n] = (f32x4){0.f, 0.f, 0.f, 0.f};
    bf16x8 At[4][2], B0[2][2], B1[2][2];
    u64 ready = 0;
    volatile LAS unsigned* const pickw = (volatile LAS unsigned*)(lds + 8 * HTB);
    STAMP(2);
    if (wr == 1) BAR;
    const unsigned char* a1 = abase;
    unsigned sna = ~tag4, snb = ~tag4;
    const unsigned foff = lane * 4;
    const unsigned pickoff = (unsigned)(size_t)pickw;
    bool vcur = false, vnxt = false;
    int left = nkt;
#define KTILE(bf) do { \
        LDB(B0, bf, 0); LDB(B1, bf, 1); LDA(At, bf, 0); \
        if (vnxt) { STAGE(SA(1 - bf, 1), a1 + HTB); WAIT_V(8); } else WAIT_V(0); \
        asm volatile("" : "+v"(sna), "+v"(snb)); \
        if (wid == 0) { ready |= __ballot(sna == tag4 && snb == tag4); pickw[bf] = pick(ready, todo, !vcur && !vnxt, fa, fb, tmo, call, tag4); \
            asm volatile("global_load_dword %0, %2, %3 sc0 sc1\n\tglobal_load_dword %1, %2, %4 sc0 sc1" \
                         : "+v"(sna), "+v"(snb) : "v"(foff), "s"(fa), "s"(fb) : "memory"); } \
        WAIT_L0; BAR; SCHED; if (vcur) { MMA(0, 0, At, B0); MMA(0, 1, At, B1); } BAR; SCHED; \
        unsigned v2; asm volatile("ds_read_b32 %0, %1 offset:%2" : "=v"(v2) : "v"(pickoff), "i"(4 * (bf)) : "memory"); SCHED; \
        LDA(At, bf, 1); SCHED; \
        asm volatile("s_waitcnt lgkmcnt(8)" : "+v"(v2) :: "memory"); \
        const int i2 = __builtin_amdgcn_readfirstlane(v2); \
        const bool vnn = i2 < 64; \
        const unsigned char* a2 = abase + (size_t)i2 * astride; const unsigned char* b2 = bbase + (size_t)i2 * (16 * (size_t)BLK); \
        if (vnn) { STAGE(SB(bf, 0), b2); STAGE(SB(bf, 1), b2 + HTB); STAGE(SA(bf, 0), a2); if (wid == 0) WAIT_V(10); else WAIT_V(8); } \
        else { if (wid == 0) WAIT_V(4); else WAIT_V(2); } \
        WAIT_L0; BAR; SCHED; if (vcur) { MMA(1, 0, At, B0); MMA(1, 1, At, B1); } BAR; SCHED; \
        left -= vcur; KSTAMP(); vcur = vnxt; vnxt = vnn; a1 = a2; } while (0)
#define KSTAMP() do {} while (0)
    for (;;) {
        KTILE(0);
        if (left == 0) break;
        KTILE(1);
        if (left == 0) break;
    }
    WAIT_V(0);
    if (wr == 0) BAR;
    STAMP(3);

    const int lz = lane_id(), fr = lz & 15, fq = lz >> 4;
    const int owner = mt >> 1;
    const int ptile = (mt & 1) * 64 + gnt;
    const size_t pt = (size_t)(ptile * 8 + wid) * 2;
    if (is_main) {
        if (tid == 0 && (mt & 1) && owner != self) __hip_atomic_fetch_add((unsigned*)(ws + OFF_BDONE), 1u, __ATOMIC_RELAXED, __HIP_MEMORY_SCOPE_AGENT);
        auto pack_half = [&](int bj, Packed& pk) {
            u32x4 v[8];
#pragma unroll
            for (int n = 0; n < 2; ++n)
#pragma unroll
                for (int r = 0; r < 4; ++r) {
                    u32x4& o = v[n * 4 + r];
                    o.x = cvt_pk_bf16(acc[0][bj][0][n][r], acc[0][bj][1][n][r]); o.y = cvt_pk_bf16(acc[0][bj][2][n][r], acc[0][bj][3][n][r]);
                    o.z = cvt_pk_bf16(acc[1][bj][0][n][r], acc[1][bj][1][n][r]); o.w = cvt_pk_bf16(acc[1][bj][2][n][r], acc[1][bj][3][n][r]);
                }
            pack(v, pk);
        };
        auto poison = [&](Packed& pk) {
#pragma unroll
            for (int i = 0; i < 4; ++i) pk.lo[i] = (u32x4){0xc0c0c0c0u, 0xc0c0c0c0u, 0xc0c0c0c0u, 0xc0c0c0c0u};
            pk.nb[0] = pk.nb[1] = (u32x4){0u, 0u, 0u, 0u}; pk.mask = 0ull; pk.base = 127u;
        };
        Packed pk0, pk1;
        pack_half(0, pk0);
        bool bad = false;
        unsigned char* const chain = ws + OFF_CH + ((owner * 8 + (gnt & 7)) * 8 + wid) * 128;
        {
            const u64 t0 = __builtin_amdgcn_s_memrealtime();
            const unsigned char* gp = (mt & 1) ? chain : ws + OFF_CRED + owner * 128;
            const unsigned want = owner == self ? 0u : (mt & 1) ? call : (call - 1u) * (SEND_G * 8) + 1u + (wid * 8 + (gnt & 7)) * SEND_G / 8;
            for (;;) {
                unsigned g, f, t;
                asm volatile("global_load_dword %0, %3, off sc0 sc1\n\t"
                             "global_load_dword %1, %4, off sc0 sc1\n\t"
                             "global_load_dword %2, %5, off sc1\n\t"
                             "s_waitcnt vmcnt(0)"
                             : "=&v"(g), "=&v"(f), "=&v"(t) : "v"(gp), "v"(fa + owner * 32), "v"(tmo) : "memory");
                if ((unsigned)__builtin_amdgcn_readfirstlane(t) == call || __builtin_amdgcn_s_memrealtime() - t0 > DEADLINE) {
                    __hip_atomic_store((unsigned*)tmo, call, __ATOMIC_RELAXED, __HIP_MEMORY_SCOPE_AGENT);
                    bad = true;
                    break;
                }
                if ((unsigned)__builtin_amdgcn_readfirstlane(g) >= want && (unsigned)__builtin_amdgcn_readfirstlane(f) == tag4) break;
                __builtin_amdgcn_s_sleep(4);
            }
        }
        if (bad) poison(pk0);
        STAMP(4);
        unsigned char* const ows = P.ws[owner];
        send(pk0, ows + OFF_PP + pt * 6144, ows + OFF_PE + pt * 2048, lz);
        pack_half(1, pk1);
        if (bad) poison(pk1);
        asm volatile("s_waitcnt vmcnt(0)" ::: "memory");
        if (lz == 0) st16_sys(ows + OFF_HP + pt * 16, header(pk0, call));
        send(pk1, ows + OFF_PP + (pt + 1) * 6144, ows + OFF_PE + (pt + 1) * 2048, lz);
        asm volatile("s_waitcnt vmcnt(" XSTR(CH_AT) ")" ::: "memory");
        if (lz == 0 && !(mt & 1)) st_flag(chain, call);
        asm volatile("s_waitcnt vmcnt(0)" ::: "memory");
        if (lz == 0) st16_sys(ows + OFF_HP + (pt + 1) * 16, header(pk1, call));
    } else {
        unsigned char* dst = P.out + (size_t)((mt & 1) * 256 + wc * 32 + fq * 4) * (GN * 2) + (size_t)(gnt * 256 + wr * 128 + fr * 8) * 2;
#pragma unroll
        for (int bj = 0; bj < 2; ++bj) {
            u32x4 p[8];
            const bool bad = recv_unpack(p, ws + OFF_PP + (pt + bj) * 6144, ws + OFF_PE + (pt + bj) * 2048, ws + OFF_HP + (pt + bj) * 16, tmo, call, lz) == call;
            if (bj == 1) STAMP(4);
#pragma unroll
            for (int n = 0; n < 2; ++n)
#pragma unroll
                for (int r = 0; r < 4; ++r) {
                    u32x4 v;
#pragma unroll
                    for (int j = 0; j < 4; ++j) {
                        const unsigned u = p[n * 4 + r][j];
                        const f32x2 y = (f32x2){acc[j >> 1][bj][(j & 1) * 2][n][r], acc[j >> 1][bj][(j & 1) * 2 + 1][n][r]}
                                      + (f32x2){__uint_as_float(u << 16), __uint_as_float(u & 0xffff0000u)};
                        const f32x2 g = gelu2(y);
                        v[j] = cvt_pk_bf16(g.x, g.y);
                    }
                    if (bad) v = (u32x4){0x7fc07fc0u, 0x7fc07fc0u, 0x7fc07fc0u, 0x7fc07fc0u};
                    *(u32x4*)(dst + (size_t)(bj * 128 + n * 16 + r) * (GN * 2)) = v;
                }
        }
    }
    if (tid == 0) __hip_atomic_store((unsigned*)(ws + OFF_CNT + b * 128), call, __ATOMIC_RELAXED, __HIP_MEMORY_SCOPE_AGENT);
    STAMP(5);
}

}

extern "C" void kernel_launch(int rank, int nranks, ncclComm_t comm, hipStream_t stream,
                              void* const* d_in, const int* in_sizes, int n_in,
                              void* d_out, int out_size, const int* in_dtypes, int out_dtype,
                              void* const* d_out_all, void* const* d_ws_all, size_t ws_bytes) {
    (void)comm; (void)in_dtypes; (void)out_dtype; (void)d_out_all;
    if (nranks != NR || (unsigned)rank >= (unsigned)NR || n_in < 2 || ws_bytes < WS_NEED
        || in_sizes[0] != GM * KS || in_sizes[1] != KS * GN || out_size != (GM / NR) * GN) {
        fprintf(stderr, "kernel_launch: unexpected problem shape\n");
        return;
    }
    Params p;
    p.x = (const unsigned char*)d_in[0];
    p.w = (const unsigned char*)d_in[1];
    p.out = (unsigned char*)d_out;
    for (int i = 0; i < NR; ++i) p.ws[i] = (unsigned char*)d_ws_all[i];
    p.rank = rank; p.pad = 0;
    hipLaunchKernelGGL(fused_gemm_rs, dim3(256), dim3(512), 0, stream, p);
    HIP_CHECK(hipGetLastError());
}
```

```cpp
#include <hip/hip_runtime.h>
#include <rccl/rccl.h>
#include <cstdio>
#include <cstdint>

#define HIP_CHECK(expr) do { hipError_t _e = (expr); if (_e != hipSuccess) { \
    fprintf(stderr, "HIP error %s at %s:%d\n", hipGetErrorString(_e), __FILE__, __LINE__); \
    return; } } while (0)

namespace {

#define LAS __attribute__((address_space(3)))
typedef short bf16x8 __attribute__((ext_vector_type(8)));
typedef float f32x4 __attribute__((ext_vector_type(4)));
typedef unsigned u32x4 __attribute__((ext_vector_type(4)));
typedef unsigned long long u64;

constexpr int NR = 8;
constexpr int GM = 4096, GK = 4096, GN = 16384;
constexpr int KS = GK / NR;
constexpr int BLK = 32768;
constexpr int HTB = 16384;
#define NPULL 35
#define OUT_MODE 1
#define LINK_PER_WAVE 0
#define NCH 2
#define CH_REL 0
#define TMAP 0
constexpr int PW = NPULL * 8 / 7;
constexpr int NCOMM = 112;
constexpr int NPACKW = (NCOMM - NPULL) * 8;
constexpr u64 DEADLINE = 50000000ull;
constexpr int NTASK = 704;

constexpr size_t OFF_CNT = 0;
constexpr size_t OFF_TMO = 32768;
constexpr size_t OFF_TICK = 33792;
constexpr size_t OFF_CRED = 34816;
constexpr size_t OFF_FX = 65536;
constexpr size_t OFF_FW = 81920;
constexpr size_t OFF_FD = 98304;
constexpr size_t OFF_HIN = 131072;
constexpr size_t OFF_HP = 327680;
constexpr size_t OFF_CH = 393216;
constexpr size_t OFF_STAMP = 524288;
constexpr size_t OFF_XA = 1u << 20;
constexpr size_t OFF_WB = OFF_XA + (size_t)64 * 16 * BLK;
constexpr size_t OFF_WD = OFF_WB + (size_t)64 * 8 * BLK;
constexpr size_t OFF_RP = OFF_WD + (size_t)64 * 8 * BLK;
constexpr size_t OFF_RE = OFF_RP + (size_t)5120 * 6144;
constexpr size_t OFF_PP = OFF_RE + (size_t)5120 * 2048;
constexpr size_t OFF_PE = OFF_PP + (size_t)2048 * 6144;
constexpr size_t WS_NEED = OFF_PE + (size_t)2048 * 2048;
static_assert(OFF_HIN + 16 * 768 * 16 <= OFF_HP && OFF_HP + 2048 * 16 <= OFF_CH && OFF_CH + 1024 * 128 <= OFF_STAMP, "flag region");
static_assert(WS_NEED <= (size_t)161 * 1048576, "workspace");

struct Params {
    const unsigned char* x;
    const unsigned char* w;
    unsigned char* out;
    unsigned char* ws[NR];
    int rank, pad;
};

__device__ __forceinline__ void st16_sys(unsigned char* p, u32x4 v) {
    asm volatile("global_store_dwordx4 %0, %1, off sc0 sc1\n\ts_nop 1" :: "v"(p), "v"(v) : "memory");
}
__device__ __forceinline__ void st_flag(unsigned char* p, unsigned v) {
    __hip_atomic_store((unsigned*)p, v, __ATOMIC_RELAXED, __HIP_MEMORY_SCOPE_SYSTEM);
}
__device__ __forceinline__ void st_flag8(unsigned char* p, unsigned v) {
    __hip_atomic_store(p, (unsigned char)v, __ATOMIC_RELAXED, __HIP_MEMORY_SCOPE_SYSTEM);
}
__device__ __forceinline__ unsigned ld_flag(const unsigned char* p) {
    return __hip_atomic_load((const unsigned*)p, __ATOMIC_RELAXED, __HIP_MEMORY_SCOPE_SYSTEM);
}
__device__ __forceinline__ unsigned lane_id() {
    return __builtin_amdgcn_mbcnt_hi(~0u, __builtin_amdgcn_mbcnt_lo(~0u, 0u));
}
__device__ __forceinline__ unsigned cvt_pk_bf16(float lo, float hi) {
    unsigned r; asm volatile("v_cvt_pk_bf16_f32 %0, %1, %2" : "=v"(r) : "v"(lo), "v"(hi)); return r;
}
typedef float f32x2 __attribute__((ext_vector_type(2)));
__device__ __forceinline__ f32x2 gelu2(f32x2 y) {
    const float k1 = -2.0f * 0.7978845608028654f * 1.4426950408889634f;
    const float k3 = k1 * 0.044715f;
    const f32x2 z = y * ((y * y) * k3 + k1);
    const f32x2 d = (f32x2){__builtin_amdgcn_exp2f(z.x), __builtin_amdgcn_exp2f(z.y)} + 1.0f;
    return y * (f32x2){__builtin_amdgcn_rcpf(d.x), __builtin_amdgcn_rcpf(d.y)};
}

template <bool EQ>
__device__ __forceinline__ bool wait_word(const unsigned char* p, unsigned want, unsigned char* tmo, unsigned call) {
    const u64 t0 = __builtin_amdgcn_s_memrealtime();
    for (;;) {
        unsigned g, t;
        asm volatile("global_load_dword %0, %2, off sc0 sc1\n\t"
                     "global_load_dword %1, %3, off sc1\n\t"
                     "s_waitcnt vmcnt(0)" : "=&v"(g), "=&v"(t) : "v"(p), "v"(tmo) : "memory");
        if ((unsigned)__builtin_amdgcn_readfirstlane(t) == call || __builtin_amdgcn_s_memrealtime() - t0 > DEADLINE) {
            __hip_atomic_store((unsigned*)tmo, call, __ATOMIC_RELAXED, __HIP_MEMORY_SCOPE_AGENT);
            return true;
        }
        const unsigned v = __builtin_amdgcn_readfirstlane(g);
        if (EQ ? v == want : v >= want) return false;
        __builtin_amdgcn_s_sleep(4);
    }
}

__device__ __forceinline__ int pick(u64& ready, u64& todo, bool idle, const unsigned char* fa, const unsigned char* fb,
                                    unsigned char* tmo, unsigned call, unsigned tag4) {
    if ((ready & todo) == 0ull) {
        if (!idle) return 64;
        const u64 t0 = __builtin_amdgcn_s_memrealtime();
        for (;;) {
            unsigned a, b, t; const unsigned off = lane_id() * 4u, zero = 0u;
            asm volatile("global_load_dword %0, %3, %5 sc0 sc1\n\t"
                         "global_load_dword %1, %3, %6 sc0 sc1\n\t"
                         "global_load_dword %2, %4, %7 sc1\n\t"
                         "s_waitcnt vmcnt(0)"
                         : "=&v"(a), "=&v"(b), "=&v"(t) : "v"(off), "v"(zero), "s"(fa), "s"(fb), "s"(tmo) : "memory");
            ready = __ballot(a == tag4 && b == tag4);
            if ((ready & todo) != 0ull) break;
            if ((unsigned)__builtin_amdgcn_readfirstlane(t) == call || __builtin_amdgcn_s_memrealtime() - t0 > DEADLINE) {
                __hip_atomic_store((unsigned*)tmo, call, __ATOMIC_RELAXED, __HIP_MEMORY_SCOPE_AGENT);
                ready = ~0ull;
                break;
            }
            __builtin_amdgcn_s_sleep(1);
        }
    }
    const int idx = __builtin_ctzll(ready & todo);
    todo &= ~(1ull << idx);
    return idx;
}

typedef unsigned short u16x2 __attribute__((ext_vector_type(2)));
__device__ __forceinline__ unsigned bfi(unsigned m, unsigned a, unsigned b) { return (a & m) | (b & ~m); }

struct Packed { u32x4 lo[4], nb[2], es[2]; u64 mask; unsigned base; };

__device__ __forceinline__ void pack(const u32x4 (&d)[8], Packed& o) {
    u16x2 mx = {0, 0};
#pragma unroll
    for (int i = 0; i < 32; ++i) mx = __builtin_elementwise_max(mx, __builtin_bit_cast(u16x2, d[i >> 2][i & 3] & 0x7fff7fffu));
    unsigned bm = (unsigned)(mx.x > mx.y ? mx.x : mx.y) >> 8;
    bm = max(bm, (unsigned)__builtin_amdgcn_mov_dpp((int)bm, 0xB1, 0xf, 0xf, true));
    bm = max(bm, (unsigned)__builtin_amdgcn_mov_dpp((int)bm, 0x4E, 0xf, 0xf, true));
    bm = max(bm, (unsigned)__builtin_amdgcn_mov_dpp((int)bm, 0x124, 0xf, 0xf, true));
    bm = max(bm, (unsigned)__builtin_amdgcn_mov_dpp((int)bm, 0x128, 0xf, 0xf, true));
    o.base = max(max((unsigned)__builtin_amdgcn_readlane(bm, 0), (unsigned)__builtin_amdgcn_readlane(bm, 16)),
                 max((unsigned)__builtin_amdgcn_readlane(bm, 32), (unsigned)__builtin_amdgcn_readlane(bm, 48)));
    const unsigned base4 = o.base * 0x01010101u;
    unsigned any = 0;
#pragma unroll
    for (int i = 0; i < 8; ++i) {
        const unsigned h0 = __builtin_amdgcn_perm(d[i].y, d[i].x, 0x07050301u), h1 = __builtin_amdgcn_perm(d[i].w, d[i].z, 0x07050301u);
        o.lo[i >> 1][(i & 1) * 2] = __builtin_amdgcn_perm(d[i].y, d[i].x, 0x06040200u);
        o.lo[i >> 1][(i & 1) * 2 + 1] = __builtin_amdgcn_perm(d[i].w, d[i].z, 0x06040200u);
        const unsigned d0 = base4 - (h0 & 0x7f7f7f7fu), d1 = base4 - (h1 & 0x7f7f7f7fu);
        o.nb[i >> 2][i & 3] = bfi(0x0f0f0f0fu, bfi(0x07070707u, d0, h0 >> 4), bfi(0x80808080u, h1, d1 << 4));
        const unsigned e = bfi(0x0f0f0f0fu, d0 >> 3, d1 << 1);
        o.es[i >> 2][i & 3] = e; any |= e;
    }
    o.mask = __ballot(any != 0u);
}
__device__ __forceinline__ void send(const Packed& o, unsigned char* pl, unsigned char* es, unsigned lane) {
    if ((o.mask >> lane) & 1ull) { st16_sys(es + lane * 32, o.es[0]); st16_sys(es + lane * 32 + 16, o.es[1]); }
    unsigned char* p = pl + lane * 16;
#pragma unroll
    for (int i = 0; i < 4; ++i) st16_sys(p + i * 1024, o.lo[i]);
    st16_sys(p + 4096, o.nb[0]); st16_sys(p + 5120, o.nb[1]);
}
__device__ __forceinline__ u32x4 header(const Packed& o, unsigned call) {
    return (u32x4){(unsigned)o.mask, call, (unsigned)(o.mask >> 32), (call << 7) | o.base};
}

__device__ __forceinline__ unsigned recv_unpack(u32x4 (&d)[8], const unsigned char* pl, const unsigned char* es, const unsigned char* hd,
                                            unsigned char* tmo, unsigned call, unsigned lane) {
    unsigned base, mlo, mhi;
    {
        const u64 t0 = __builtin_amdgcn_s_memrealtime();
        for (;;) {
            u32x4 h; unsigned t; const unsigned zero = 0u;
            asm volatile("global_load_dwordx4 %0, %2, %3 sc0 sc1\n\t"
                         "global_load_dword %1, %2, %4 sc1\n\t"
                         "s_waitcnt vmcnt(0)" : "=&v"(h), "=&v"(t) : "v"(zero), "s"(hd), "s"(tmo) : "memory");
            const unsigned t1 = __builtin_amdgcn_readfirstlane(h.y), t3 = __builtin_amdgcn_readfirstlane(h.w);
            mlo = __builtin_amdgcn_readfirstlane(h.x); mhi = __builtin_amdgcn_readfirstlane(h.z); base = t3 & 127u;
            if (t1 == call && (t3 >> 7) == (call & 0x1ffffffu)) break;
            if ((unsigned)__builtin_amdgcn_readfirstlane(t) == call || __builtin_amdgcn_s_memrealtime() - t0 > DEADLINE) {
                __hip_atomic_store((unsigned*)tmo, call, __ATOMIC_RELAXED, __HIP_MEMORY_SCOPE_AGENT);
                break;
            }
            __builtin_amdgcn_s_sleep(1);
        }
    }
    u32x4 lo[4], nb[2], e[2] = {{0u, 0u, 0u, 0u}, {0u, 0u, 0u, 0u}};
    const unsigned off = lane * 16;
    const u64 mask = ((u64)mhi << 32) | mlo;
    u64 keep; unsigned tw = 0u;
    asm volatile("s_mov_b64 %8, exec\n\t"
                 "s_mov_b64 exec, %15\n\t"
                 "global_load_dwordx4 %6, %11, %14 sc0 sc1\n\t"
                 "global_load_dwordx4 %7, %11, %14 offset:16 sc0 sc1\n\t"
                 "s_mov_b64 exec, %8\n\t"
                 "global_load_dwordx4 %0, %10, %12 sc0 sc1\n\t"
                 "global_load_dwordx4 %1, %10, %12 offset:1024 sc0 sc1\n\t"
                 "global_load_dwordx4 %2, %10, %12 offset:2048 sc0 sc1\n\t"
                 "global_load_dwordx4 %3, %10, %12 offset:3072 sc0 sc1\n\t"
                 "global_load_dwordx4 %4, %10, %13 sc0 sc1\n\t"
                 "global_load_dwordx4 %5, %10, %13 offset:1024 sc0 sc1\n\t"
                 "global_load_dword %9, %9, %16 sc1\n\t"
                 "s_waitcnt vmcnt(0)"
                 : "=&v"(lo[0]), "=&v"(lo[1]), "=&v"(lo[2]), "=&v"(lo[3]), "=&v"(nb[0]), "=&v"(nb[1]), "+v"(e[0]), "+v"(e[1]), "=&s"(keep), "+v"(tw)
                 : "v"(off), "v"(off * 2), "s"(pl), "s"(pl + 4096), "s"(es), "s"(mask), "s"(tmo) : "memory");
    const unsigned base4 = base * 0x01010101u;
#pragma unroll
    for (int i = 0; i < 8; ++i) {
        const unsigned n = nb[i >> 2][i & 3], ee = e[i >> 2][i & 3];
        const unsigned d0 = (n & 0x07070707u) | ((ee & 0x0f0f0f0fu) << 3), d1 = ((n >> 4) & 0x07070707u) | ((ee & 0xf0f0f0f0u) >> 1);
        const unsigned h0 = (base4 - d0) | ((n << 4) & 0x80808080u), h1 = (base4 - d1) | (n & 0x80808080u);
        const unsigned l0 = lo[i >> 1][(i & 1) * 2], l1 = lo[i >> 1][(i & 1) * 2 + 1];
        d[i].x = __builtin_amdgcn_perm(h0, l0, 0x05010400u); d[i].y = __builtin_amdgcn_perm(h0, l0, 0x07030602u);
        d[i].z = __builtin_amdgcn_perm(h1, l1, 0x05010400u); d[i].w = __builtin_amdgcn_perm(h1, l1, 0x07030602u);
    }
    return __builtin_amdgcn_readfirstlane(tw);
}

#define STAMP(i) do {} while (0)

#define ORD_LAG 2.3
#define ORD_C0 6.5
#define ORD_CS 0.32
#define ORD_W 0.3
struct alignas(4) Order { unsigned char v[176]; };
constexpr Order make_order() {
    double key[176] = {}; unsigned char id[176] = {}; int n = 0;
    for (int kc = 0; kc < 8; ++kc) {
        for (int e = 0; e < 8; ++e) { key[n] = kc - ORD_W + (e + 0.4) / 8; id[n++] = (unsigned char)(0 << 6 | kc << 3 | e); }
        for (int e = 0; e < 6; ++e) { key[n] = kc + (e + 0.5) / 6; id[n++] = (unsigned char)(1 << 6 | kc << 3 | e); }
        for (int e = 0; e < 6; ++e) { key[n] = kc + ORD_LAG + (e + 0.6) / 6; id[n++] = (unsigned char)(2 << 6 | kc << 3 | e); }
        for (int e = 0; e < 2; ++e) { key[n] = ORD_C0 + (2 * kc + e) * ORD_CS; id[n++] = (unsigned char)(3 << 6 | kc << 3 | e); }
    }
    Order o = {};
    for (int i = 0; i < 176; ++i) { int r = 0; for (int j = 0; j < 176; ++j) r += (key[j] < key[i]) || (key[j] == key[i] && j < i); o.v[r] = id[i]; }
    return o;
}
constexpr Order ORDER = make_order();
constexpr unsigned order_word(int i) { return ORDER.v[4 * i] | ORDER.v[4 * i + 1] << 8 | ORDER.v[4 * i + 2] << 16 | (unsigned)ORDER.v[4 * i + 3] << 24; }
template <int I> __device__ __forceinline__ unsigned order_vgpr(unsigned v) {
    if constexpr (I == 44) return v;
    else {
        unsigned t;
        asm("s_mov_b32 %1, %2\n\tv_writelane_b32 %0, %1, %3" : "+v"(v), "=&s"(t) : "i"(order_word(I)), "i"(I));
        return order_vgpr<I + 1>(v);
    }
}

__device__ __forceinline__ void decode_item(unsigned ordv, int n, int src, int dest, int& isw, int& kc, int& tile) {
    const unsigned v = ((unsigned)__builtin_amdgcn_readlane(ordv, n >> 2) >> ((n & 3) * 8)) & 255u;
    const int kind = v >> 6, e = v & 7;
    kc = (v >> 3) & 7; isw = kind == 0;
    const int d = (dest - src - 1) & 7;
    tile = kind == 0 ? ((e + src) & 7) : kind == 3 ? 2 * dest + e : 2 * ((src + 1 + e + (e >= d)) & 7) + (kind - 1);
}

__global__ __launch_bounds__(512, 2) void fused_gemm_rs(const Params P) {
    __shared__ __attribute__((aligned(16))) unsigned char lds_raw[8 * HTB + 64];
    LAS unsigned char* lds = (LAS unsigned char*)lds_raw;
    const int tid = threadIdx.x, wid = __builtin_amdgcn_readfirstlane(tid >> 6), lane = tid & 63;
    const int b = blockIdx.x, self = P.rank;
    unsigned char* const ws = P.ws[self];
    unsigned char* const tmo = ws + OFF_TMO;
    STAMP(0);
    const bool is_comm = b < NCOMM, is_pull = b < NPULL;
    auto pack_decode = [&](int t, int& isw, int& kc, int& tile) {
        kc = t / 320; const int blk = (t - kc * 320) >> 2, m = blk - 8;
        isw = blk >= 8 && blk < 72;
        tile = isw ? ((self + 1 + m) & 7) * 8 + (((m >> 3) + self) & 7) : (blk & 7) * 2 + (blk >= 8);
    };
    const unsigned lw = (lane >> 4) * 8 * (GN * 2) + (lane & 15) * 16, lx = (lane & 15) * (KS * 2) + (lane >> 4) * 16;
    auto load_unit = [&](int isw, int kc, int tile, int q, u32x4 (&r)[8]) {
        const unsigned char* src = isw ? P.w + ((kc * 64 + (q & 1) * 32) * (GN * 2) + (tile * 256 + (q >> 1) * 128) * 2)
                                       : P.x + ((tile * 256 + q * 64) * (KS * 2) + kc * 128);
        const unsigned lo = isw ? lw : lx, s1 = isw ? GN * 2 : 64, s2 = isw ? GN * 4 : 16 * KS * 2;
#pragma unroll
        for (int i = 0; i < 8; ++i) r[i] = *(const u32x4*)(src + ((i >> 1) * s2 + (i & 1) * s1) + lo);
    };
    auto load_task = [&](int t, u32x4 (&r)[8]) { int isw, kc, tile; pack_decode(t, isw, kc, tile); load_unit(isw, kc, tile, t & 3, r); };
    u32x4 cur[8];
    const int task0 = is_pull ? (b / 7) * 8 + wid : wid * (NCOMM - NPULL) + (b - NPULL);
    if (is_comm && !is_pull) load_task(task0, cur);
    const unsigned call = (unsigned)__builtin_amdgcn_readfirstlane(
        __hip_atomic_load((const unsigned*)(ws + OFF_CNT + b * 128), __ATOMIC_RELAXED, __HIP_MEMORY_SCOPE_AGENT)) + 1u;
    const size_t par = call & 1u;
    const unsigned tag4 = (call & 255u) * 0x01010101u;
    if (is_comm) {
        auto store_image = [&](unsigned char* blk, int isw, int q, const u32x4 (&c)[8]) {
            unsigned char* d0 = blk + (isw ? (q >> 1) * 8192 + (q & 1) * 1024 : q * 8192) + lane * 16;
            const unsigned s1 = isw ? 2048 : 1024, s4 = isw ? HTB : 4096;
#pragma unroll
            for (int j = 0; j < 8; ++j) st16_sys(d0 + ((j >> 2) * s4 + (j & 3) * s1), c[j]);
        };
        if (!is_pull) {
            unsigned char* hdst = P.ws[0];
#pragma unroll
            for (int r = 1; r < NR; ++r) hdst = lane == r ? P.ws[r] : hdst;
            hdst += OFF_HIN + (par * 8 + self) * (768 * 16);
#pragma unroll
            for (int i = 0; i < 8; ++i) asm volatile("" : "+v"(cur[i]));
            auto transpose8 = [&](u32x4 (&c)[8]) {
                u32x4 tr[8];
#pragma unroll
                for (int j = 0; j < 8; ++j)
#pragma unroll
                    for (int qd = 0; qd < 4; ++qd)
                        tr[j][qd] = __builtin_amdgcn_perm(c[2 * qd + 1][j >> 1], c[2 * qd][j >> 1], (j & 1) ? 0x07060302u : 0x05040100u);
#pragma unroll
                for (int j = 0; j < 8; ++j) c[j] = tr[j];
            };
            for (int t = task0;;) {
                const int tn = t + NPACKW; const bool more = tn < 2560;
                u32x4 nxt[8];
                load_task(more ? tn : t, nxt);
                int isw, kc, tile; pack_decode(t, isw, kc, tile); const int q = t & 3, idx = self * 8 + kc;
                if (isw) transpose8(cur);
                const bool own = (isw ? tile >> 3 : tile >> 1) == self;
                const int unit = (isw ? tile * 8 + kc : 512 + kc * 16 + tile) * 4 + q;
                u32x4 hd = {0u, 0u, 0u, 0u};
                if (!own) {
                    Packed pk; pack(cur, pk);
                    send(pk, ws + OFF_RP + (par * 2560 + unit) * 6144, ws + OFF_RE + (par * 2560 + unit) * 2048, lane);
                    hd = header(pk, call);
                }
                if (!isw || own)
                    store_image(isw ? ws + OFF_WB + (size_t)(idx * 8 + (tile & 7)) * BLK : ws + OFF_XA + (size_t)(idx * 16 + tile) * BLK, isw, q, cur);
                asm volatile("s_waitcnt vmcnt(0)" ::: "memory");
#pragma unroll
                for (int i = 0; i < 8; ++i) { cur[i] = nxt[i]; asm volatile("" : "+v"(cur[i])); }
                if (!own && lane < NR && lane != self && (!isw || lane == (tile >> 3)))
                    st16_sys(hdst + ((isw ? kc * 8 + (tile & 7) : 64 + kc * 16 + tile) * 4 + q) * 16, hd);
                if ((!isw || own) && lane == 0) st_flag8(ws + (isw ? OFF_FW + ((tile & 7) * 64 + idx) * 4 : OFF_FX + (tile * 64 + idx) * 4) + q, call);
                if (!more) break;
                t = tn;
            }
            STAMP(6);
            if (b - NPULL < 56) {
                const int g = ((self + 1 + (b - NPULL) % 7) & 7) * 8 + 7 - (b - NPULL) / 7;
                for (int q = 0; q < 4; ++q) {
                    load_unit(1, wid, g, q, cur);
                    transpose8(cur);
                    store_image(ws + OFF_WD + (size_t)(g * 8 + wid) * BLK, 1, q, cur);
                }
                asm volatile("s_waitcnt vmcnt(0)" ::: "memory");
                if (lane == 0) st_flag(ws + OFF_FD + (g * 8 + wid) * 4, tag4);
            }
        } else {
            const unsigned ordv = order_vgpr<0>(0u);
            const int link = b % 7, src = (self + 1 + link) & 7;
            const unsigned char* const sws = P.ws[src];
            unsigned* const tick = (unsigned*)(ws + OFF_TICK + src * 128);
            const unsigned tbase = (call - 1u) * NTASK - PW;
            if (task0 < PW)
            for (int task = task0;;) {
                unsigned nn = 1u;
                {
                    u64 keep; const unsigned zero = 0u;
                    asm volatile("s_mov_b64 %1, exec\n\ts_mov_b64 exec, 1\n\t"
                                 "global_atomic_add %0, %2, %0, %3 sc0\n\t"
                                 "s_mov_b64 exec, %1" : "+v"(nn), "=&s"(keep) : "v"(zero), "s"(tick) : "memory");
                }
                int isw, kc, tile; decode_item(ordv, task >> 2, src, self, isw, kc, tile); const int q = task & 3, idx = src * 8 + kc;
                const int unit = (isw ? (self * 8 + tile) * 8 + kc : 512 + kc * 16 + tile) * 4 + q;
                recv_unpack(cur, sws + OFF_RP + (par * 2560 + unit) * 6144, sws + OFF_RE + (par * 2560 + unit) * 2048,
                            ws + OFF_HIN + (par * 8 + src) * (768 * 16) + ((isw ? kc * 8 + tile : 64 + kc * 16 + tile) * 4 + q) * 16, tmo, call, lane);
                asm volatile("" : "+v"(nn));
                const int ntask = (int)(__builtin_amdgcn_readfirstlane(nn) - tbase);
                if (ntask >= NTASK && lane == 0) __hip_atomic_fetch_add((unsigned*)(ws + OFF_CRED + src * 128), 1u, __ATOMIC_RELAXED, __HIP_MEMORY_SCOPE_AGENT);
                store_image(isw ? ws + OFF_WB + (size_t)(idx * 8 + tile) * BLK : ws + OFF_XA + (size_t)(idx * 16 + tile) * BLK, isw, q, cur);
                asm volatile("s_waitcnt vmcnt(0)" ::: "memory");
                if (lane == 0) st_flag8(ws + (isw ? OFF_FW : OFF_FX) + (tile * 64 + idx) * 4 + q, call);
                if (ntask >= NTASK) break;
                task = ntask;
            }
        }
        STAMP(1);
        __syncthreads();
    }

    int mt, gnt, nkt; u64 todo;
    const unsigned char *abase, *fa; size_t astride;
    const bool is_main = !is_comm && b < 240;
    if (is_comm) {
        const int j = b - NPULL, odd = is_pull || j >= 56, e = is_pull ? b : j < 56 ? j : b - 56;
        mt = 2 * self + odd; gnt = ((self + 1 + e % 7) & 7) * 8 + 7 - e / 7;
        nkt = 8; todo = 0xffull << (self * 8);
        abase = ws + OFF_WD + (size_t)(gnt * 8 - self * 8) * BLK; astride = BLK;
        fa = ws + OFF_FD + (gnt * 8 - self * 8) * 4;
    } else {
        const int xcd = b & 7, sl = (b >> 3) - 14;
        mt = is_main ? 4 * (xcd >> 1) + (sl & 3) : 2 * self + (sl & 1);
        const int nt = 4 * (xcd & 1) + (is_main ? sl >> 2 : xcd >> 1);
        gnt = self * 8 + nt;
        const bool own = (mt >> 1) == self;
        nkt = own ? 32 : 56;
        todo = !own ? ~(0xffull << ((mt >> 1) * 8)) : is_main ? 0x5555555555555555ull : 0xaaaaaaaaaaaaaaaaull;
        abase = ws + OFF_WB + (size_t)nt * BLK; astride = 8 * (size_t)BLK;
        fa = ws + OFF_FW + nt * 256;
    }
    const unsigned char* const bbase = ws + OFF_XA + (size_t)mt * BLK;
    const unsigned char* const fb = ws + OFF_FX + mt * 256;
    const int wr = wid >> 2, wc = wid & 3;
    const unsigned voff = tid * 16;
    const unsigned ldsw = wid * 1024;
    const int aoff = wr * 8192 + lane * 16;
    int boff = 4 * HTB + wc * 4096 + lane * 16;
    asm volatile("" : "+v"(boff));
#define SA(bf, h) (((bf) * 2 + (h)) * HTB)
#define SB(bf, h) ((4 + (bf) * 2 + (h)) * HTB)
#define STAGE(bufoff, gbase) do { _Pragma("unroll") for (int _i = 0; _i < 2; ++_i) \
        __builtin_amdgcn_global_load_lds((const unsigned*)((gbase) + _i * 8192 + voff), (LAS unsigned*)(lds + (bufoff) + ldsw + _i * 8192), 16, 0, 17); } while (0)
#define LDA(dst, bf, h) do { _Pragma("unroll") for (int m = 0; m < 4; ++m) _Pragma("unroll") for (int k = 0; k < 2; ++k) \
        dst[m][k] = *(const LAS bf16x8*)(lds + SA(bf, h) + aoff + m * 2048 + k * 1024); } while (0)
#define LDB(dst, bf, h) do { _Pragma("unroll") for (int n = 0; n < 2; ++n) _Pragma("unroll") for (int k = 0; k < 2; ++k) \
        dst[n][k] = *(const LAS bf16x8*)(lds + boff + (SB(bf, h) - 4 * HTB + n * 2048 + k * 1024)); } while (0)
#define MMA(ai, bj, At, Bt) do { __builtin_amdgcn_s_setprio(1); _Pragma("unroll") for (int m = 0; m < 4; ++m) _Pragma("unroll") for (int n = 0; n < 2; ++n) _Pragma("unroll") for (int k = 0; k < 2; ++k) \
        acc[ai][bj][m][n] = __builtin_amdgcn_mfma_f32_16x16x32_bf16(Bt[n][k], At[m][k], acc[ai][bj][m][n], 0, 0, 0); __builtin_amdgcn_s_setprio(0); } while (0)
#define WAIT_V(n) asm volatile("s_waitcnt vmcnt(" #n ")" ::: "memory")
#define WAIT_L0 do { asm volatile("" ::: "memory"); __builtin_amdgcn_s_waitcnt(0xC07F); asm volatile("" ::: "memory"); } while (0)
#define BAR __builtin_amdgcn_s_barrier()
#define SCHED __builtin_amdgcn_sched_barrier(0)
    f32x4 acc[2][2][4][2];
#pragma unroll
    for (int a = 0; a < 2; ++a)
#pragma unroll
        for (int c = 0; c < 2; ++c)
#pragma unroll
            for (int m = 0; m < 4; ++m)
#pragma unroll
                for (int n = 0; n < 2; ++n) acc[a][c][m][n] = (f32x4){0.f, 0.f, 0.f, 0.f};
    bf16x8 At[4][2], B0[2][2], B1[2][2];
    u64 ready = 0;
    volatile LAS unsigned* const pickw = (volatile LAS unsigned*)(lds + 8 * HTB);
    STAMP(2);
    if (wr == 1) BAR;
    const unsigned char* a1 = abase;
    unsigned sna = ~tag4, snb = ~tag4;
    const unsigned foff = lane * 4;
    const unsigned pickoff = (unsigned)(size_t)pickw;
    bool vcur = false, vnxt = false;
    int left = nkt;
#define KTILE(bf) do { \
        LDB(B0, bf, 0); LDB(B1, bf, 1); LDA(At, bf, 0); \
        if (vnxt) { STAGE(SA(1 - bf, 1), a1 + HTB); WAIT_V(8); } else WAIT_V(0); \
        asm volatile("" : "+v"(sna), "+v"(snb)); \
        if (wid == 0) { ready |= __ballot(sna == tag4 && snb == tag4); pickw[bf] = pick(ready, todo, !vcur && !vnxt, fa, fb, tmo, call, tag4); \
            asm volatile("global_load_dword %0, %2, %3 sc0 sc1\n\tglobal_load_dword %1, %2, %4 sc0 sc1" \
                         : "+v"(sna), "+v"(snb) : "v"(foff), "s"(fa), "s"(fb) : "memory"); } \
        WAIT_L0; BAR; SCHED; if (vcur) { MMA(0, 0, At, B0); MMA(0, 1, At, B1); } BAR; SCHED; \
        unsigned v2; asm volatile("ds_read_b32 %0, %1 offset:%2" : "=v"(v2) : "v"(pickoff), "i"(4 * (bf)) : "memory"); SCHED; \
        LDA(At, bf, 1); SCHED; \
        asm volatile("s_waitcnt lgkmcnt(8)" : "+v"(v2) :: "memory"); \
        const int i2 = __builtin_amdgcn_readfirstlane(v2); \
        const bool vnn = i2 < 64; \
        const unsigned char* a2 = abase + (size_t)i2 * astride; const unsigned char* b2 = bbase + (size_t)i2 * (16 * (size_t)BLK); \
        if (vnn) { STAGE(SB(bf, 0), b2); STAGE(SB(bf, 1), b2 + HTB); STAGE(SA(bf, 0), a2); if (wid == 0) WAIT_V(10); else WAIT_V(8); } \
        else { if (wid == 0) WAIT_V(4); else WAIT_V(2); } \
        WAIT_L0; BAR; SCHED; if (vcur) { MMA(1, 0, At, B0); MMA(1, 1, At, B1); } BAR; SCHED; \
        left -= vcur; KSTAMP(); vcur = vnxt; vnxt = vnn; a1 = a2; } while (0)
#define KSTAMP() do {} while (0)
    for (;;) {
        KTILE(0);
        if (left == 0) break;
        KTILE(1);
        if (left == 0) break;
    }
    WAIT_V(0);
    if (wr == 0) BAR;
    STAMP(3);

    const int lz = lane_id(), fr = lz & 15, fq = lz >> 4;
    const int owner = mt >> 1;
    const int ptile = (mt & 1) * 64 + gnt;
    const size_t pt = (size_t)(ptile * 8 + wid) * 2;
    const size_t ps = (size_t)((mt * 8 + (gnt & 7)) * 8 + wid) * 2;
    if (is_main) {
        auto pack_half = [&](int bj, Packed& pk) {
            u32x4 v[8];
#pragma unroll
            for (int n = 0; n < 2; ++n)
#pragma unroll
                for (int r = 0; r < 4; ++r) {
                    u32x4& o = v[n * 4 + r];
                    o.x = cvt_pk_bf16(acc[0][bj][0][n][r], acc[0][bj][1][n][r]); o.y = cvt_pk_bf16(acc[0][bj][2][n][r], acc[0][bj][3][n][r]);
                    o.z = cvt_pk_bf16(acc[1][bj][0][n][r], acc[1][bj][1][n][r]); o.w = cvt_pk_bf16(acc[1][bj][2][n][r], acc[1][bj][3][n][r]);
                }
            pack(v, pk);
        };
        auto poison = [&](Packed& pk) {
#pragma unroll
            for (int i = 0; i < 4; ++i) pk.lo[i] = (u32x4){0xc0c0c0c0u, 0xc0c0c0c0u, 0xc0c0c0c0u, 0xc0c0c0c0u};
            pk.nb[0] = pk.nb[1] = (u32x4){0u, 0u, 0u, 0u}; pk.mask = 0ull; pk.base = 127u;
        };
        Packed pk0, pk1;
        pack_half(0, pk0);
        bool bad = false;
        bad = wait_word<true>(fa + owner * 32, tag4, tmo, call);
        if (bad) poison(pk0);
        STAMP(4);
        unsigned char* const ows = P.ws[owner];
        send(pk0, ws + OFF_PP + ps * 6144, ws + OFF_PE + ps * 2048, lz);
        pack_half(1, pk1);
        if (bad) poison(pk1);
        asm volatile("s_waitcnt vmcnt(0)" ::: "memory");
        if (lz == 0) st16_sys(ows + OFF_HP + pt * 16, header(pk0, call));
        send(pk1, ws + OFF_PP + (ps + 1) * 6144, ws + OFF_PE + (ps + 1) * 2048, lz);
        asm volatile("s_waitcnt vmcnt(0)" ::: "memory");
        if (lz == 0) st16_sys(ows + OFF_HP + (pt + 1) * 16, header(pk1, call));
    } else {
        const int src = gnt >> 3;
        const unsigned char* const sws = P.ws[src];
        const int cpos = (mt & 1) * 8 + wid;
        unsigned char* const chain = ws + OFF_CH + (gnt * 16 + cpos) * 128;
        if (src != self)
            wait_word<false>(cpos >= NCH ? chain - NCH * 128 : ws + OFF_CRED + src * 128,
                             cpos >= NCH ? call : (call - 1u) * PW + 1u + (wid * 8 + (gnt & 7)) * PW / (8 * NCH), tmo, call);
        unsigned char* dst = P.out + (size_t)((mt & 1) * 256 + wc * 32 + fq * 4) * (GN * 2) + (size_t)(gnt * 256 + wr * 128 + fr * 8) * 2;
#pragma unroll
        for (int bj = 0; bj < 2; ++bj) {
            u32x4 p[8];
            const bool bad = recv_unpack(p, sws + OFF_PP + (ps + bj) * 6144, sws + OFF_PE + (ps + bj) * 2048, ws + OFF_HP + (pt + bj) * 16, tmo, call, lz) == call;
            if (bj == CH_REL && lz == 0) st_flag(chain, call);
            if (bj == 1) STAMP(4);
#pragma unroll
            for (int n = 0; n < 2; ++n)
#pragma unroll
                for (int r = 0; r < 4; ++r) {
                    u32x4 v;
#pragma unroll
                    for (int j = 0; j < 4; ++j) {
                        const unsigned u = p[n * 4 + r][j];
                        const f32x2 y = (f32x2){acc[j >> 1][bj][(j & 1) * 2][n][r], acc[j >> 1][bj][(j & 1) * 2 + 1][n][r]}
                                      + (f32x2){__uint_as_float(u << 16), __uint_as_float(u & 0xffff0000u)};
                        const f32x2 g = gelu2(y);
                        v[j] = cvt_pk_bf16(g.x, g.y);
                    }
                    if (bad) v = (u32x4){0x7fc07fc0u, 0x7fc07fc0u, 0x7fc07fc0u, 0x7fc07fc0u};
                    st16_sys(dst + (size_t)(bj * 128 + n * 16 + r) * (GN * 2), v);
                }
        }
    }
    if (tid == 0) __hip_atomic_store((unsigned*)(ws + OFF_CNT + b * 128), call, __ATOMIC_RELAXED, __HIP_MEMORY_SCOPE_AGENT);
    STAMP(5);
}

}

extern "C" void kernel_launch(int rank, int nranks, ncclComm_t comm, hipStream_t stream,
                              void* const* d_in, const int* in_sizes, int n_in,
                              void* d_out, int out_size, const int* in_dtypes, int out_dtype,
                              void* const* d_out_all, void* const* d_ws_all, size_t ws_bytes) {
    (void)comm; (void)in_dtypes; (void)out_dtype; (void)d_out_all;
    if (nranks != NR || (unsigned)rank >= (unsigned)NR || n_in < 2 || ws_bytes < WS_NEED
        || in_sizes[0] != GM * KS || in_sizes[1] != KS * GN || out_size != (GM / NR) * GN) {
        fprintf(stderr, "kernel_launch: unexpected problem shape\n");
        return;
    }
    Params p;
    p.x = (const unsigned char*)d_in[0];
    p.w = (const unsigned char*)d_in[1];
    p.out = (unsigned char*)d_out;
    for (int i = 0; i < NR; ++i) p.ws[i] = (unsigned char*)d_ws_all[i];
    p.rank = rank; p.pad = 0;
    hipLaunchKernelGGL(fused_gemm_rs, dim3(256), dim3(512), 0, stream, p);
    HIP_CHECK(hipGetLastError());
}
```

```cpp
#include <hip/hip_runtime.h>
#include <rccl/rccl.h>
#include <cstdio>
#include <cstdint>

#define HIP_CHECK(expr) do { hipError_t _e = (expr); if (_e != hipSuccess) { \
    fprintf(stderr, "HIP error %s at %s:%d\n", hipGetErrorString(_e), __FILE__, __LINE__); \
    return; } } while (0)

namespace {

#define LAS __attribute__((address_space(3)))
typedef short bf16x8 __attribute__((ext_vector_type(8)));
typedef float f32x4 __attribute__((ext_vector_type(4)));
typedef unsigned u32x4 __attribute__((ext_vector_type(4)));
typedef unsigned long long u64;

constexpr int NR = 8;
constexpr int GM = 4096, GK = 4096, GN = 16384;
constexpr int KS = GK / NR;
constexpr int BLK = 32768;
constexpr int HTB = 16384;
#define NPULL 56
#define PW 22
#define OUT_MODE 1
#define XSTR2(x) #x
#define XSTR(x) XSTR2(x)
#define FAKE_MASK 0xffffff
#define NCH 2
#define PULL_DELAY 200
#define MID_Q 2
#define PACK_STAG 0
#define TMAP 0

constexpr int NCOMM = 112;
constexpr u64 DEADLINE = 50000000ull;
constexpr int NTASK = 704;

constexpr size_t OFF_CNT = 0;
constexpr size_t OFF_TMO = 32768;
constexpr size_t OFF_TICK = 33792;
constexpr size_t OFF_CRED = 34816;
constexpr size_t OFF_FX = 65536;
constexpr size_t OFF_FW = 81920;
constexpr size_t OFF_FD = 98304;
constexpr size_t OFF_HIN = 131072;
constexpr size_t OFF_HP = 327680;
constexpr size_t OFF_CH = 393216;
constexpr size_t OFF_STAMP = 524288;
constexpr size_t OFF_XA = 1u << 20;
constexpr size_t OFF_WB = OFF_XA + (size_t)64 * 16 * BLK;
constexpr size_t OFF_WD = OFF_WB + (size_t)64 * 8 * BLK;
constexpr size_t OFF_RP = OFF_WD + (size_t)64 * 8 * BLK;
constexpr size_t OFF_PP = OFF_RP + (size_t)5120 * 10240;
constexpr size_t WS_NEED = OFF_PP + (size_t)2048 * 10240;
static_assert(OFF_HIN + 16 * 768 * 16 <= OFF_HP && OFF_HP + 2048 * 16 <= OFF_CH && OFF_CH + 1024 * 128 <= OFF_STAMP, "flag region");
static_assert(WS_NEED <= (size_t)161 * 1048576, "workspace");

struct Params {
    const unsigned char* x;
    const unsigned char* w;
    unsigned char* out;
    unsigned char* ws[NR];
    int rank, pad;
};

__device__ __forceinline__ void st16_sys(unsigned char* p, u32x4 v) {
    asm volatile("global_store_dwordx4 %0, %1, off sc0 sc1\n\ts_nop 1" :: "v"(p), "v"(v) : "memory");
}
__device__ __forceinline__ void st_flag(unsigned char* p, unsigned v) {
    __hip_atomic_store((unsigned*)p, v, __ATOMIC_RELAXED, __HIP_MEMORY_SCOPE_SYSTEM);
}
__device__ __forceinline__ void st_flag8(unsigned char* p, unsigned v) {
    __hip_atomic_store(p, (unsigned char)v, __ATOMIC_RELAXED, __HIP_MEMORY_SCOPE_SYSTEM);
}
__device__ __forceinline__ unsigned ld_flag(const unsigned char* p) {
    return __hip_atomic_load((const unsigned*)p, __ATOMIC_RELAXED, __HIP_MEMORY_SCOPE_SYSTEM);
}
__device__ __forceinline__ unsigned lane_id() {
    return __builtin_amdgcn_mbcnt_hi(~0u, __builtin_amdgcn_mbcnt_lo(~0u, 0u));
}
__device__ __forceinline__ unsigned cvt_pk_bf16(float lo, float hi) {
    unsigned r; asm volatile("v_cvt_pk_bf16_f32 %0, %1, %2" : "=v"(r) : "v"(lo), "v"(hi)); return r;
}
typedef float f32x2 __attribute__((ext_vector_type(2)));
__device__ __forceinline__ f32x2 gelu2(f32x2 y) {
    const float k1 = -2.0f * 0.7978845608028654f * 1.4426950408889634f;
    const float k3 = k1 * 0.044715f;
    const f32x2 z = y * ((y * y) * k3 + k1);
    const f32x2 d = (f32x2){__builtin_amdgcn_exp2f(z.x), __builtin_amdgcn_exp2f(z.y)} + 1.0f;
    return y * (f32x2){__builtin_amdgcn_rcpf(d.x), __builtin_amdgcn_rcpf(d.y)};
}

template <bool EQ>
__device__ __forceinline__ bool wait_word(const unsigned char* p, unsigned want, unsigned char* tmo, unsigned call) {
    const u64 t0 = __builtin_amdgcn_s_memrealtime();
    for (;;) {
        unsigned g, t;
        asm volatile("global_load_dword %0, %2, off sc0 sc1\n\t"
                     "global_load_dword %1, %3, off sc1\n\t"
                     "s_waitcnt vmcnt(0)" : "=&v"(g), "=&v"(t) : "v"(p), "v"(tmo) : "memory");
        if ((unsigned)__builtin_amdgcn_readfirstlane(t) == call || __builtin_amdgcn_s_memrealtime() - t0 > DEADLINE) {
            __hip_atomic_store((unsigned*)tmo, call, __ATOMIC_RELAXED, __HIP_MEMORY_SCOPE_AGENT);
            return true;
        }
        const unsigned v = __builtin_amdgcn_readfirstlane(g);
        if (EQ ? v == want : v >= want) return false;
        __builtin_amdgcn_s_sleep(2);
    }
}

__device__ __forceinline__ int pick(u64& ready, u64& todo, bool idle, const unsigned char* fa, const unsigned char* fb,
                                    unsigned char* tmo, unsigned call, unsigned tag4) {
    if ((ready & todo) == 0ull) {
        if (!idle) return 64;
        const u64 t0 = __builtin_amdgcn_s_memrealtime();
        for (;;) {
            unsigned a, b, t; const unsigned off = lane_id() * 4u, zero = 0u;
            asm volatile("global_load_dword %0, %3, %5 sc0 sc1\n\t"
                         "global_load_dword %1, %3, %6 sc0 sc1\n\t"
                         "global_load_dword %2, %4, %7 sc1\n\t"
                         "s_waitcnt vmcnt(0)"
                         : "=&v"(a), "=&v"(b), "=&v"(t) : "v"(off), "v"(zero), "s"(fa), "s"(fb), "s"(tmo) : "memory");
            ready = __ballot(a == tag4 && b == tag4);
            if ((ready & todo) != 0ull) break;
            if ((unsigned)__builtin_amdgcn_readfirstlane(t) == call || __builtin_amdgcn_s_memrealtime() - t0 > DEADLINE) {
                __hip_atomic_store((unsigned*)tmo, call, __ATOMIC_RELAXED, __HIP_MEMORY_SCOPE_AGENT);
                ready = ~0ull;
                break;
            }
            __builtin_amdgcn_s_sleep(1);
        }
    }
    const int idx = __builtin_ctzll(ready & todo);
    todo &= ~(1ull << idx);
    return idx;
}

__device__ __forceinline__ unsigned bfi(unsigned m, unsigned a, unsigned b) { return (a & m) | (b & ~m); }
__device__ __forceinline__ unsigned perm(unsigned hi, unsigned lo, unsigned sel) { return __builtin_amdgcn_perm(hi, lo, sel); }
#define SDWA2(k) " dst_sel:DWORD dst_unused:UNUSED_PAD src0_sel:BYTE_" #k " src1_sel:BYTE_" #k "\n\t"
#define SDWA1(k) " dst_sel:DWORD dst_unused:UNUSED_PAD src0_sel:DWORD src1_sel:BYTE_" #k "\n\t"
__device__ __forceinline__ unsigned spread_bits(unsigned f, unsigned s) {
    unsigned r, a, b;
    asm("v_lshlrev_b32_sdwa %0, %4, %3" SDWA2(0) "v_lshlrev_b32_sdwa %1, %4, %3" SDWA2(1) "v_lshlrev_b32_sdwa %2, %4, %3" SDWA2(2)
        "v_or3_b32 %0, %0, %1, %2\n\t"
        "v_lshlrev_b32_sdwa %1, %4, %3" SDWA2(3)
        "v_or_b32 %0, %0, %1" : "=&v"(r), "=&v"(a), "=&v"(b) : "v"(f), "v"(s));
    return r;
}
__device__ __forceinline__ unsigned take_codes(unsigned& w, unsigned f) {
    unsigned ln, t, l;
    asm("v_xor_b32_sdwa %2, %0, %4" SDWA1(0) "v_ffbl_b32 %1, %2\n\tv_lshrrev_b32 %0, %1, %0\n\t"
        "v_xor_b32_sdwa %2, %0, %4" SDWA1(1) "v_ffbl_b32 %3, %2\n\tv_lshrrev_b32 %0, %3, %0\n\t"
        "v_xor_b32_sdwa %2, %0, %4" SDWA1(2) "v_lshl_or_b32 %1, %3, 8, %1\n\tv_ffbl_b32 %3, %2\n\tv_lshrrev_b32 %0, %3, %0\n\t"
        "v_xor_b32_sdwa %2, %0, %4" SDWA1(3) "v_lshl_or_b32 %1, %3, 16, %1\n\tv_ffbl_b32 %3, %2\n\tv_lshrrev_b32 %0, %3, %0\n\t"
        "v_lshl_or_b32 %1, %3, 24, %1" : "+v"(w), "=&v"(ln), "=&v"(t), "=&v"(l) : "v"(f));
    return ln;
}
__device__ __forceinline__ unsigned wave_scan(unsigned v) {
    v += (unsigned)__builtin_amdgcn_update_dpp(0, (int)v, 0x111, 0xf, 0xf, false);
    v += (unsigned)__builtin_amdgcn_update_dpp(0, (int)v, 0x112, 0xf, 0xf, false);
    v += (unsigned)__builtin_amdgcn_update_dpp(0, (int)v, 0x114, 0xf, 0xf, false);
    v += (unsigned)__builtin_amdgcn_update_dpp(0, (int)v, 0x118, 0xf, 0xf, false);
    v += (unsigned)__builtin_amdgcn_update_dpp(0, (int)v, 0x142, 0xa, 0xf, false);
    v += (unsigned)__builtin_amdgcn_update_dpp(0, (int)v, 0x143, 0xc, 0xf, false);
    return v;
}
__device__ __forceinline__ unsigned rank_in(u64 mask) {
    return __builtin_amdgcn_mbcnt_hi((unsigned)(mask >> 32), __builtin_amdgcn_mbcnt_lo((unsigned)mask, 0u));
}
constexpr int UNIT = 10240;
constexpr int SCR = 8192;
#define LDS_ORDER() asm volatile("" ::: "memory")

struct Packed { u32x4 sm[4], l1, st; u64 mask; unsigned meta; };

__device__ __forceinline__ void pack(const u32x4 (&d)[8], Packed& o, LAS unsigned char* scr, unsigned lane) {
    *(LAS u32x4*)(scr + lane * 16) = (u32x4){0u, 0u, 0u, 0u};
    unsigned sum = 0; u32x4 er[4];
#pragma unroll
    for (int g = 0; g < 16; ++g) {
        const unsigned d0 = d[g >> 1][(g & 1) * 2], d1 = d[g >> 1][(g & 1) * 2 + 1];
        const unsigned lb = perm(d1, d0, 0x06040200u), hb = perm(d1, d0, 0x07050301u);
        o.sm[g >> 2][g & 3] = bfi(0x7f7f7f7fu, lb, hb);
        const unsigned e = bfi(0x01010101u, lb >> 7, hb << 1);
        er[g >> 2][g & 3] = e;
        sum = __builtin_amdgcn_sad_u8(e, 0u, sum);
    }
    const unsigned r2 = (((unsigned)__builtin_amdgcn_readlane(wave_scan(sum), 63) + 16036u) >> 12) & 255u;
    const unsigned kk = (r2 | 0x80u) * 0x01010101u, cc = (~r2 & 0x80u) * 0x01010101u;
    unsigned bad = 0, longest = 0, total = 0, piece[8], plen[8];
#pragma unroll
    for (int q = 0; q < 4; ++q) {
        unsigned l1 = 0, acc = 0, base = 0;
#pragma unroll
        for (int gg = 0; gg < 4; ++gg) {
            const unsigned e = er[q][gg];
            const unsigned u = (kk - (e & 0x7f7f7f7fu)) ^ (e & 0x80808080u) ^ cc;
            bad |= u;
            const unsigned sel = u | ((u << 1) & 0x10101010u);
            const unsigned c = perm(0x03030302u, 0x01000303u, sel);
            l1 |= (c & 0x03030303u) << (2 * gg);
            const unsigned len4 = (perm(0xfffefefeu, 0xfffcf8dau, sel) ^ 0xfefefefeu) + u;
            const unsigned f = c & (c >> 1) & 0x01010101u;
            unsigned ps = len4 + (len4 << 8); asm("" : "+v"(ps)); ps += ps << 16; ps -= 0x100c0804u;
            const unsigned bits = spread_bits(f, ps);
            if ((gg & 1) == 0) { acc = bits; base = ps >> 24; }
            else {
                acc |= bits << (base & 31u); base += ps >> 24;
                piece[q * 2 + (gg >> 1)] = acc >> 1; plen[q * 2 + (gg >> 1)] = base;
                longest = max(longest, base); total += base;
            }
        }
        o.l1[q] = l1;
    }
    const bool raw = (bad & 0xe0e0e0e0u) != 0u || longest > 31u || total > 112u;
    o.mask = __ballot(raw);
    if (raw) total = 0;
    const unsigned incl = wave_scan(total);
    const unsigned rawoff = (((unsigned)__builtin_amdgcn_readlane(incl, 63) + 512u + 127u) >> 7) << 4;
    o.meta = r2 | ((rawoff + 64u * (unsigned)__builtin_popcountll(o.mask) + 127u) >> 7) << 8;
    LDS_ORDER();
    scr[lane] = (unsigned char)total;
    if (!raw) {
        unsigned off = incl - total + 512u;
#pragma unroll
        for (int i = 0; i < 8; ++i) {
            const u64 w = (u64)piece[i] << (off & 31u);
            LAS unsigned* p = (LAS unsigned*)(scr + ((off >> 5) << 2));
            __hip_atomic_fetch_or(p, (unsigned)w, __ATOMIC_RELAXED, __HIP_MEMORY_SCOPE_WORKGROUP);
            __hip_atomic_fetch_or(p + 1, (unsigned)(w >> 32), __ATOMIC_RELAXED, __HIP_MEMORY_SCOPE_WORKGROUP);
            off += plen[i];
        }
    } else {
        LAS u32x4* p = (LAS u32x4*)(scr + rawoff + rank_in(o.mask) * 64u);
#pragma unroll
        for (int i = 0; i < 4; ++i) p[i] = er[i];
    }
    LDS_ORDER();
    o.st = *(const LAS u32x4*)(scr + lane * 16);
    LDS_ORDER();
}
__device__ __forceinline__ void send(const Packed& o, unsigned char* pl, const LAS unsigned char* scr, unsigned lane) {
    unsigned char* p = pl + lane * 16;
    for (unsigned k = 1; k * 8u < (o.meta >> 8); ++k) st16_sys(p + 5120 + k * 1024, *(const LAS u32x4*)(scr + k * 1024 + lane * 16));
#pragma unroll
    for (int i = 0; i < 4; ++i) st16_sys(p + i * 1024, o.sm[i]);
    st16_sys(p + 4096, o.l1);
    if (lane < (o.meta >> 8) * 8u) st16_sys(p + 5120, o.st);
}
__device__ __forceinline__ u32x4 header(const Packed& o, unsigned call) {
    return (u32x4){(unsigned)o.mask, call, (unsigned)(o.mask >> 32), (call << 14) | o.meta};
}
__device__ __forceinline__ bool header_ok(u32x4 h, unsigned call, u64& mask, unsigned& meta) {
    const unsigned t1 = __builtin_amdgcn_readfirstlane(h.y), t3 = __builtin_amdgcn_readfirstlane(h.w);
    mask = ((u64)(unsigned)__builtin_amdgcn_readfirstlane(h.z) << 32) | (unsigned)__builtin_amdgcn_readfirstlane(h.x);
    meta = t3 & 0x3fffu;
    return t1 == call && (t3 >> 14) == (call & 0x3ffffu);
}
__device__ __forceinline__ void wait_header(const unsigned char* hd, unsigned char* tmo, unsigned call, u64& mask, unsigned& meta) {
    const u64 t0 = __builtin_amdgcn_s_memrealtime();
    for (;;) {
        u32x4 h; unsigned t;
        asm volatile("global_load_dwordx4 %0, %2, off sc0 sc1\n\t"
                     "global_load_dword %1, %3, off sc1\n\t"
                     "s_waitcnt vmcnt(0)" : "=&v"(h), "=&v"(t) : "v"(hd), "v"(tmo) : "memory");
        if (header_ok(h, call, mask, meta)) break;
        if ((unsigned)__builtin_amdgcn_readfirstlane(t) == call || __builtin_amdgcn_s_memrealtime() - t0 > DEADLINE) {
            __hip_atomic_store((unsigned*)tmo, call, __ATOMIC_RELAXED, __HIP_MEMORY_SCOPE_AGENT);
            mask = 0ull; meta = 0u;
            break;
        }
        __builtin_amdgcn_s_sleep(1);
    }
}
__device__ __forceinline__ void fetch(Packed& o, const unsigned char* pl, unsigned lane) {
    const unsigned n = min(max(o.meta >> 8, 1u), 8u) * 8u;
    const u64 smask = n >= 64u ? ~0ull : (1ull << n) - 1ull;
    u64 keep;
    asm volatile("s_mov_b64 %6, exec\n\t"
                 "global_load_dwordx4 %0, %7, %8 sc0 sc1\n\t"
                 "global_load_dwordx4 %1, %7, %8 offset:1024 sc0 sc1\n\t"
                 "global_load_dwordx4 %2, %7, %8 offset:2048 sc0 sc1\n\t"
                 "global_load_dwordx4 %3, %7, %8 offset:3072 sc0 sc1\n\t"
                 "global_load_dwordx4 %4, %7, %9 sc0 sc1\n\t"
                 "s_mov_b64 exec, %10\n\t"
                 "global_load_dwordx4 %5, %7, %9 offset:1024 sc0 sc1\n\t"
                 "s_mov_b64 exec, %6"
                 : "=&v"(o.sm[0]), "=&v"(o.sm[1]), "=&v"(o.sm[2]), "=&v"(o.sm[3]), "=&v"(o.l1), "=&v"(o.st), "=&s"(keep)
                 : "v"(lane * 16u), "s"(pl), "s"(pl + 4096), "s"(smask) : "memory");
}
#define FETCH_WAIT(o, n) asm volatile("s_waitcnt vmcnt(" #n ")" : "+v"(o.sm[0]), "+v"(o.sm[1]), "+v"(o.sm[2]), "+v"(o.sm[3]), "+v"(o.l1), "+v"(o.st) :: "memory")
template <class Mid>
__device__ __forceinline__ void unpack(const Packed& o, u32x4 (&d)[8], const unsigned char* pl, LAS unsigned char* scr, unsigned lane, Mid mid) {
    *(LAS u32x4*)(scr + lane * 16) = o.st;
    if ((o.meta >> 8) > 8u) {
        u32x4 m[4];
        asm volatile("global_load_dwordx4 %0, %4, off sc0 sc1\n\t"
                     "global_load_dwordx4 %1, %4, off offset:1024 sc0 sc1\n\t"
                     "global_load_dwordx4 %2, %4, off offset:2048 sc0 sc1\n\t"
                     "global_load_dwordx4 %3, %4, off offset:3072 sc0 sc1\n\t"
                     "s_waitcnt vmcnt(0)" : "=&v"(m[0]), "=&v"(m[1]), "=&v"(m[2]), "=&v"(m[3]) : "v"(pl + 6144 + lane * 16) : "memory");
#pragma unroll
        for (int k = 0; k < 4; ++k) *(LAS u32x4*)(scr + (k + 1) * 1024 + lane * 16) = m[k];
    }
    LDS_ORDER();
    const unsigned len = scr[lane] & 127u;
    const unsigned incl = wave_scan(len), off = incl - len + 512u;
    const LAS unsigned* p = (const LAS unsigned*)(scr + ((off >> 5) << 2));
    const unsigned w0 = p[0], w1 = p[1], w2 = p[2], w3 = p[3], w4 = p[4];
    unsigned s0 = __builtin_amdgcn_alignbit(w1, w0, off), s1 = __builtin_amdgcn_alignbit(w2, w1, off);
    unsigned s2 = __builtin_amdgcn_alignbit(w3, w2, off), s3 = __builtin_amdgcn_alignbit(w4, w3, off);
    const unsigned r7 = (o.meta + 5u) & 255u;
    const unsigned kk = (r7 | 0x80u) * 0x01010101u, cc = (~r7 & 0x80u) * 0x01010101u;
    u32x4 er[4];
#pragma unroll
    for (int q = 0; q < 4; ++q) {
        if (q == MID_Q) { asm volatile("" : "+v"(s0)); mid(); asm volatile("" : "+v"(s0)); }
        const unsigned l1 = o.l1[q], x = l1 & (l1 >> 1);
        unsigned w = 0, used = 0;
#pragma unroll
        for (int gg = 0; gg < 4; ++gg) {
            if ((gg & 1) == 0) { w = (s0 << 1) | 1u; used = 0; }
            const unsigned f = (x >> (2 * gg)) & 0x01010101u;
            const unsigned ln = take_codes(w, f);
            used = __builtin_amdgcn_sad_u8(ln, 0u, used);
            const unsigned y = (perm(0xfa00fc00u, 0xfc00fd00u, (ln << 1) | 0x01010101u) ^ 0xfafafafau) + ln + ((l1 >> (2 * gg)) & 0x03030303u);
            er[q][gg] = (kk - y) ^ cc;
            if (gg & 1) {
                s0 = __builtin_amdgcn_alignbit(s1, s0, used); s1 = __builtin_amdgcn_alignbit(s2, s1, used);
                s2 = __builtin_amdgcn_alignbit(s3, s2, used); s3 >>= used & 31u;
            }
        }
    }
    if (o.mask != 0ull) {
        const unsigned rawoff = (((unsigned)__builtin_amdgcn_readlane(incl, 63) + 512u + 127u) >> 7) << 4;
        if ((o.mask >> lane) & 1ull) {
            const LAS u32x4* r = (const LAS u32x4*)(scr + rawoff + rank_in(o.mask) * 64u);
#pragma unroll
            for (int i = 0; i < 4; ++i) er[i] = r[i];
        }
    }
    LDS_ORDER();
#pragma unroll
    for (int g = 0; g < 16; ++g) {
        const unsigned e = er[g >> 2][g & 3], sm = o.sm[g >> 2][g & 3];
        const unsigned hb = bfi(0x80808080u, sm, e >> 1), lb = bfi(0x7f7f7f7fu, sm, e << 7);
        d[g >> 1][(g & 1) * 2] = perm(hb, lb, 0x05010400u); d[g >> 1][(g & 1) * 2 + 1] = perm(hb, lb, 0x07030602u);
    }
}

#define STAMP(i) do {} while (0)

#define ORD_LAG 2.3
#define ORD_C0 6.5
#define ORD_CS 0.32
#define ORD_W 0.3
struct alignas(4) Order { unsigned char v[176]; };
constexpr Order make_order() {
    double key[176] = {}; unsigned char id[176] = {}; int n = 0;
    for (int kc = 0; kc < 8; ++kc) {
        for (int e = 0; e < 8; ++e) { key[n] = kc - ORD_W + (e + 0.4) / 8; id[n++] = (unsigned char)(0 << 6 | kc << 3 | e); }
        for (int e = 0; e < 6; ++e) { key[n] = kc + (e + 0.5) / 6; id[n++] = (unsigned char)(1 << 6 | kc << 3 | e); }
        for (int e = 0; e < 6; ++e) { key[n] = kc + ORD_LAG + (e + 0.6) / 6; id[n++] = (unsigned char)(2 << 6 | kc << 3 | e); }
        for (int e = 0; e < 2; ++e) { key[n] = ORD_C0 + (2 * kc + e) * ORD_CS; id[n++] = (unsigned char)(3 << 6 | kc << 3 | e); }
    }
    Order o = {};
    for (int i = 0; i < 176; ++i) { int r = 0; for (int j = 0; j < 176; ++j) r += (key[j] < key[i]) || (key[j] == key[i] && j < i); o.v[r] = id[i]; }
    return o;
}
constexpr Order ORDER = make_order();
constexpr unsigned order_word(int i) { return ORDER.v[4 * i] | ORDER.v[4 * i + 1] << 8 | ORDER.v[4 * i + 2] << 16 | (unsigned)ORDER.v[4 * i + 3] << 24; }
template <int I> __device__ __forceinline__ unsigned order_vgpr(unsigned v) {
    if constexpr (I == 44) return v;
    else {
        unsigned t;
        asm("s_mov_b32 %1, %2\n\tv_writelane_b32 %0, %1, %3" : "+v"(v), "=&s"(t) : "i"(order_word(I)), "i"(I));
        return order_vgpr<I + 1>(v);
    }
}

__device__ __forceinline__ void decode_item(unsigned ordv, int n, int src, int dest, int& isw, int& kc, int& tile) {
    const unsigned v = ((unsigned)__builtin_amdgcn_readlane(ordv, n >> 2) >> ((n & 3) * 8)) & 255u;
    const int kind = v >> 6, e = v & 7;
    kc = (v >> 3) & 7; isw = kind == 0;
    const int d = (dest - src - 1) & 7;
    tile = kind == 0 ? ((e + src) & 7) : kind == 3 ? 2 * dest + e : 2 * ((src + 1 + e + (e >= d)) & 7) + (kind - 1);
}

__global__ __launch_bounds__(512, 2) void fused_gemm_rs(const Params P) {
    __shared__ __attribute__((aligned(16))) unsigned char lds_raw[8 * HTB + 128];
    LAS unsigned char* lds = (LAS unsigned char*)lds_raw;
    const int tid = threadIdx.x, wid = __builtin_amdgcn_readfirstlane(tid >> 6), lane = tid & 63;
    const int b = blockIdx.x, self = P.rank;
    unsigned char* const ws = P.ws[self];
    unsigned char* const tmo = ws + OFF_TMO;
    LAS unsigned char* const scr = lds + wid * SCR;
    LAS unsigned* const tok = (LAS unsigned*)(lds + 8 * HTB + 64);
    if (tid < 16) tok[tid] = 0u;
    STAMP(0);
    const u64 tstart = __builtin_amdgcn_s_memrealtime();
    const bool is_comm = b < NCOMM, is_pull = b < NPULL;
    const bool is_main = !is_comm && b < 240;
    int mt, gnt;
    if (is_comm) {
        const int j = b - NPULL, odd = is_pull || j >= 56, e = is_pull ? b : j < 56 ? j : b - 56;
        mt = 2 * self + odd; gnt = ((self + 1 + e % 7) & 7) * 8 + 7 - e / 7;
    } else {
        const int xcd = b & 7, sl = (b >> 3) - 14;
        mt = is_main ? 4 * (xcd >> 1) + (sl & 3) : 2 * self + (sl & 1);
        gnt = self * 8 + 4 * (xcd & 1) + (is_main ? sl >> 2 : xcd >> 1);
    }
    const bool urgent = is_main && !(mt & 1) && (mt >> 1) != self;
    auto pack_decode = [&](int t, int& isw, int& kc, int& tile) {
        const int blk = t >> 2;
        if (blk < 72) {
            const int m = blk - 8; kc = 0; isw = blk >= 8;
            tile = isw ? ((self + 1 + m) & 7) * 8 + (((m >> 3) + self) & 7) : blk * 2;
        } else if (blk < 184) {
            const int m = blk - 72; kc = 1 + (m >> 4); isw = (m >> 3) & 1;
            tile = isw ? self * 8 + (m & 7) : (m & 7) * 2;
        } else {
            const int m = blk - 184, r = m & 63, w = r - 8; isw = r >= 8; kc = (m >> 6) + isw;
            tile = isw ? ((self + 1 + w % 7) & 7) * 8 + ((w / 7 + self) & 7) : r * 2 + 1;
        }
    };
    const unsigned lw = (lane >> 4) * 8 * (GN * 2) + (lane & 15) * 16, lx = (lane & 15) * (KS * 2) + (lane >> 4) * 16;
    auto load_unit = [&](int isw, int kc, int tile, int q, u32x4 (&r)[8]) {
        const unsigned char* src = isw ? P.w + ((kc * 64 + (q & 1) * 32) * (GN * 2) + (tile * 256 + (q >> 1) * 128) * 2)
                                       : P.x + ((tile * 256 + q * 64) * (KS * 2) + kc * 128);
        const unsigned lo = isw ? lw : lx, s1 = isw ? GN * 2 : 64, s2 = isw ? GN * 4 : 16 * KS * 2;
#pragma unroll
        for (int i = 0; i < 8; ++i) r[i] = *(const u32x4*)(src + ((i >> 1) * s2 + (i & 1) * s1) + lo);
    };
    auto load_task = [&](int t, u32x4 (&r)[8]) { int isw, kc, tile; pack_decode(t, isw, kc, tile); load_unit(isw, kc, tile, t & 3, r); };
    u32x4 cur[8];
    const int rank2 = is_comm ? b - NPULL : NCOMM - NPULL + (!is_main ? 72 + b - 240 : (mt & 1) ? (mt >> 1) * 8 + (gnt & 7) : 64 + (gnt & 7));
    const int task0 = is_pull ? wid * (NPULL / 7) + b / 7 : wid * (200 - NPULL) + rank2;
    if (!is_pull && !urgent) load_task(task0, cur);
    const unsigned call = (unsigned)__builtin_amdgcn_readfirstlane(
        __hip_atomic_load((const unsigned*)(ws + OFF_CNT + b * 128), __ATOMIC_RELAXED, __HIP_MEMORY_SCOPE_AGENT)) + 1u;
    const size_t par = call & 1u;
    const unsigned tag4 = (call & 255u) * 0x01010101u;
    if (!urgent) {
        auto store_image = [&](unsigned char* blk, int isw, int q, const u32x4 (&c)[8]) {
            unsigned char* d0 = blk + (isw ? (q >> 1) * 8192 + (q & 1) * 1024 : q * 8192) + lane * 16;
            const unsigned s1 = isw ? 2048 : 1024, s4 = isw ? HTB : 4096;
#pragma unroll
            for (int j = 0; j < 8; ++j) st16_sys(d0 + ((j >> 2) * s4 + (j & 3) * s1), c[j]);
        };
        if (!is_pull) {
            unsigned char* hdst = P.ws[0];
#pragma unroll
            for (int r = 1; r < NR; ++r) hdst = lane == r ? P.ws[r] : hdst;
            hdst += OFF_HIN + (par * 8 + self) * (768 * 16);
#pragma unroll
            for (int i = 0; i < 8; ++i) asm volatile("" : "+v"(cur[i]));
            auto transpose8 = [&](u32x4 (&c)[8]) {
                u32x4 tr[8];
#pragma unroll
                for (int j = 0; j < 8; ++j)
#pragma unroll
                    for (int qd = 0; qd < 4; ++qd)
                        tr[j][qd] = __builtin_amdgcn_perm(c[2 * qd + 1][j >> 1], c[2 * qd][j >> 1], (j & 1) ? 0x07060302u : 0x05040100u);
#pragma unroll
                for (int j = 0; j < 8; ++j) c[j] = tr[j];
            };
            for (int t = task0;;) {
                const int tn = t + (200 - NPULL) * 8; const bool more = tn < 2560;
                u32x4 nxt[8];
                load_task(more ? tn : t, nxt);
                int isw, kc, tile; pack_decode(t, isw, kc, tile); const int q = t & 3, idx = self * 8 + kc;
                if (isw) transpose8(cur);
                const bool own = (isw ? tile >> 3 : tile >> 1) == self;
                const int unit = (isw ? tile * 8 + kc : 512 + kc * 16 + tile) * 4 + q;
                u32x4 hd = {0u, 0u, 0u, 0u};
                if (!own) {
                    Packed pk; pack(cur, pk, scr, lane);
                    send(pk, ws + OFF_RP + (par * 2560 + unit) * UNIT, scr, lane);
                    hd = header(pk, call);
                }
                if (!isw || own)
                    store_image(isw ? ws + OFF_WB + (size_t)(idx * 8 + (tile & 7)) * BLK : ws + OFF_XA + (size_t)(idx * 16 + tile) * BLK, isw, q, cur);
                asm volatile("s_waitcnt vmcnt(0)" ::: "memory");
#pragma unroll
                for (int i = 0; i < 8; ++i) { cur[i] = nxt[i]; asm volatile("" : "+v"(cur[i])); }
                if (!own && lane < NR && lane != self && (!isw || lane == (tile >> 3)))
                    st16_sys(hdst + ((isw ? kc * 8 + (tile & 7) : 64 + kc * 16 + tile) * 4 + q) * 16, hd);
                if ((!isw || own) && lane == 0) st_flag8(ws + (isw ? OFF_FW + ((tile & 7) * 64 + idx) * 4 : OFF_FX + (tile * 64 + idx) * 4) + q, call);
                if (!more) break;
                t = tn;
            }
            STAMP(6);
            if (b - NPULL < 56) {
                const int g = gnt;
                for (int q = 0; q < 4; ++q) {
                    load_unit(1, wid, g, q, cur);
                    transpose8(cur);
                    store_image(ws + OFF_WD + (size_t)(g * 8 + wid) * BLK, 1, q, cur);
                }
                asm volatile("s_waitcnt vmcnt(0)" ::: "memory");
                if (lane == 0) st_flag(ws + OFF_FD + (g * 8 + wid) * 4, tag4);
            }
        } else {
            const unsigned ordv = order_vgpr<0>(0u);
            const int link = b % 7, src = (self + 1 + link) & 7;
            const unsigned char* const sws = P.ws[src];
            unsigned* const tick = (unsigned*)(ws + OFF_TICK + src * 128);
            const unsigned tbase = (call - 1u) * (NTASK - PW) - 2 * PW;
            const unsigned char* const hbase = ws + OFF_HIN + (par * 8 + src) * (768 * 16);
            auto locate = [&](int t, const unsigned char*& pl, const unsigned char*& hd) {
                int isw, kc, tile; decode_item(ordv, t >> 2, src, self, isw, kc, tile); const int q = t & 3;
                pl = sws + OFF_RP + (par * 2560 + (isw ? (self * 8 + tile) * 8 + kc : 512 + kc * 16 + tile) * 4 + q) * UNIT;
                hd = hbase + ((isw ? kc * 8 + tile : 64 + kc * 16 + tile) * 4 + q) * 16;
            };
            auto ahead = [&](const unsigned char* hd, bool draw, u32x4& h, unsigned& nn) {
                u64 keep; const unsigned zero = 0u; nn = 1u;
                asm volatile("global_load_dwordx4 %0, %3, %4 sc0 sc1\n\t"
                             "s_mov_b64 %2, exec\n\ts_mov_b32 exec_lo, %6\n\ts_mov_b32 exec_hi, 0\n\t"
                             "global_atomic_add %1, %3, %1, %5 sc0\n\t"
                             "s_mov_b64 exec, %2" : "=&v"(h), "+v"(nn), "=&s"(keep)
                             : "v"(zero), "s"(hd), "s"(tick), "s"(__builtin_amdgcn_readfirstlane(draw ? 1 : 0)) : "memory");
            };
            if (task0 < PW) {
                int task = task0, t1 = task0 + PW;
                const unsigned char *pl0, *pl1, *hd0, *hd1;
                Packed pa, pb; u32x4 h; unsigned nn;
                locate(task, pl0, hd0); locate(t1, pl1, hd1);
                while (__builtin_amdgcn_s_memrealtime() - tstart < PULL_DELAY) __builtin_amdgcn_s_sleep(2);
                wait_header(hd0, tmo, call, pa.mask, pa.meta);
                fetch(pa, pl0, lane);
                ahead(hd1, true, h, nn);
                FETCH_WAIT(pa, 0);
                asm volatile("" : "+v"(h), "+v"(nn));
                int t2 = (int)(__builtin_amdgcn_readfirstlane(nn) - tbase);
                unsigned char* fprev = ws + OFF_CNT + b * 128 + 64 + wid;
                for (;;) {
                    const bool v1 = t1 < NTASK, v2 = t2 < NTASK;
                    if (!header_ok(h, call, pb.mask, pb.meta) && v1) wait_header(hd1, tmo, call, pb.mask, pb.meta);
                    const unsigned char *pl2, *hd2;
                    locate(v2 ? t2 : 0, pl2, hd2);
                    ahead(hd2, v2, h, nn);
                    if (!v1 && lane == 0) __hip_atomic_fetch_add((unsigned*)(ws + OFF_CRED + src * 128), 1u, __ATOMIC_RELAXED, __HIP_MEMORY_SCOPE_AGENT);
                    fetch(pb, v1 ? pl1 : ws + OFF_RP, lane);
                    unpack(pa, cur, pl0, scr, lane, [&]() {
                        asm volatile("s_waitcnt vmcnt(6)" ::: "memory");
                        if (lane == 0) st_flag8(fprev, call);
                    });
                    int isw, kc, tile; decode_item(ordv, task >> 2, src, self, isw, kc, tile); const int q = task & 3, idx = src * 8 + kc;
                    fprev = ws + (isw ? OFF_FW : OFF_FX) + (tile * 64 + idx) * 4 + q;
                    store_image(isw ? ws + OFF_WB + (size_t)(idx * 8 + tile) * BLK : ws + OFF_XA + (size_t)(idx * 16 + tile) * BLK, isw, q, cur);
                    FETCH_WAIT(pb, 8);
                    asm volatile("" : "+v"(h), "+v"(nn));
                    if (!v1) break;
                    pa = pb; task = t1; pl0 = pl1; t1 = t2; pl1 = pl2; hd1 = hd2;
                    t2 = v2 ? (int)(__builtin_amdgcn_readfirstlane(nn) - tbase) : NTASK;
                }
                asm volatile("s_waitcnt vmcnt(0)" ::: "memory");
                if (lane == 0) st_flag8(fprev, call);
            }
        }
        STAMP(1);
        __syncthreads();
    }

    int nkt; u64 todo;
    const unsigned char *abase, *fa; size_t astride;
    if (is_comm) {
        nkt = 8; todo = 0xffull << (self * 8);
        abase = ws + OFF_WD + (size_t)(gnt * 8 - self * 8) * BLK; astride = BLK;
        fa = ws + OFF_FD + (gnt * 8 - self * 8) * 4;
    } else {
        const bool own = (mt >> 1) == self;
        nkt = own ? 32 : 56;
        todo = !own ? ~(0xffull << ((mt >> 1) * 8)) : is_main ? 0x5555555555555555ull : 0xaaaaaaaaaaaaaaaaull;
        abase = ws + OFF_WB + (size_t)(gnt & 7) * BLK; astride = 8 * (size_t)BLK;
        fa = ws + OFF_FW + (gnt & 7) * 256;
    }
    const unsigned char* const bbase = ws + OFF_XA + (size_t)mt * BLK;
    const unsigned char* const fb = ws + OFF_FX + mt * 256;
    const int wr = wid >> 2, wc = wid & 3;
    const unsigned voff = tid * 16;
    const unsigned ldsw = wid * 1024;
    const int aoff = wr * 8192 + lane * 16;
    int boff = 4 * HTB + wc * 4096 + lane * 16;
    asm volatile("" : "+v"(boff));
#define SA(bf, h) (((bf) * 2 + (h)) * HTB)
#define SB(bf, h) ((4 + (bf) * 2 + (h)) * HTB)
#define STAGE(bufoff, gbase) do { _Pragma("unroll") for (int _i = 0; _i < 2; ++_i) \
        __builtin_amdgcn_global_load_lds((const unsigned*)((gbase) + _i * 8192 + voff), (LAS unsigned*)(lds + (bufoff) + ldsw + _i * 8192), 16, 0, 17); } while (0)
#define LDA(dst, bf, h) do { _Pragma("unroll") for (int m = 0; m < 4; ++m) _Pragma("unroll") for (int k = 0; k < 2; ++k) \
        dst[m][k] = *(const LAS bf16x8*)(lds + SA(bf, h) + aoff + m * 2048 + k * 1024); } while (0)
#define LDB(dst, bf, h) do { _Pragma("unroll") for (int n = 0; n < 2; ++n) _Pragma("unroll") for (int k = 0; k < 2; ++k) \
        dst[n][k] = *(const LAS bf16x8*)(lds + boff + (SB(bf, h) - 4 * HTB + n * 2048 + k * 1024)); } while (0)
#define MMA(ai, bj, At, Bt) do { __builtin_amdgcn_s_setprio(1); _Pragma("unroll") for (int m = 0; m < 4; ++m) _Pragma("unroll") for (int n = 0; n < 2; ++n) _Pragma("unroll") for (int k = 0; k < 2; ++k) \
        acc[ai][bj][m][n] = __builtin_amdgcn_mfma_f32_16x16x32_bf16(Bt[n][k], At[m][k], acc[ai][bj][m][n], 0, 0, 0); __builtin_amdgcn_s_setprio(0); } while (0)
#define WAIT_V(n) asm volatile("s_waitcnt vmcnt(" #n ")" ::: "memory")
#define WAIT_L0 do { asm volatile("" ::: "memory"); __builtin_amdgcn_s_waitcnt(0xC07F); asm volatile("" ::: "memory"); } while (0)
#define BAR __builtin_amdgcn_s_barrier()
#define SCHED __builtin_amdgcn_sched_barrier(0)
    f32x4 acc[2][2][4][2];
#pragma unroll
    for (int a = 0; a < 2; ++a)
#pragma unroll
        for (int c = 0; c < 2; ++c)
#pragma unroll
            for (int m = 0; m < 4; ++m)
#pragma unroll
                for (int n = 0; n < 2; ++n) acc[a][c][m][n] = (f32x4){0.f, 0.f, 0.f, 0.f};
    bf16x8 At[4][2], B0[2][2], B1[2][2];
    u64 ready = 0;
    volatile LAS unsigned* const pickw = (volatile LAS unsigned*)(lds + 8 * HTB);
    STAMP(2);
    if (wr == 1) BAR;
    const unsigned char* a1 = abase;
    unsigned sna = ~tag4, snb = ~tag4;
    const unsigned foff = lane * 4;
    const unsigned pickoff = (unsigned)(size_t)pickw;
    bool vcur = false, vnxt = false;
    int left = nkt;
#define KTILE(bf) do { \
        LDB(B0, bf, 0); LDB(B1, bf, 1); LDA(At, bf, 0); \
        if (vnxt) { STAGE(SA(1 - bf, 1), a1 + HTB); WAIT_V(8); } else WAIT_V(0); \
        asm volatile("" : "+v"(sna), "+v"(snb)); \
        if (wid == 0) { ready |= __ballot(sna == tag4 && snb == tag4); pickw[bf] = pick(ready, todo, !vcur && !vnxt, fa, fb, tmo, call, tag4); \
            asm volatile("global_load_dword %0, %2, %3 sc0 sc1\n\tglobal_load_dword %1, %2, %4 sc0 sc1" \
                         : "+v"(sna), "+v"(snb) : "v"(foff), "s"(fa), "s"(fb) : "memory"); } \
        WAIT_L0; BAR; SCHED; if (vcur) { MMA(0, 0, At, B0); MMA(0, 1, At, B1); } BAR; SCHED; \
        unsigned v2; asm volatile("ds_read_b32 %0, %1 offset:%2" : "=v"(v2) : "v"(pickoff), "i"(4 * (bf)) : "memory"); SCHED; \
        LDA(At, bf, 1); SCHED; \
        asm volatile("s_waitcnt lgkmcnt(8)" : "+v"(v2) :: "memory"); \
        const int i2 = __builtin_amdgcn_readfirstlane(v2); \
        const bool vnn = i2 < 64; \
        const unsigned char* a2 = abase + (size_t)i2 * astride; const unsigned char* b2 = bbase + (size_t)i2 * (16 * (size_t)BLK); \
        if (vnn) { STAGE(SB(bf, 0), b2); STAGE(SB(bf, 1), b2 + HTB); STAGE(SA(bf, 0), a2); if (wid == 0) WAIT_V(10); else WAIT_V(8); } \
        else { if (wid == 0) WAIT_V(4); else WAIT_V(2); } \
        WAIT_L0; BAR; SCHED; if (vcur) { MMA(1, 0, At, B0); MMA(1, 1, At, B1); } BAR; SCHED; \
        left -= vcur; KSTAMP(); vcur = vnxt; vnxt = vnn; a1 = a2; } while (0)
#define KSTAMP() do {} while (0)
    for (;;) {
        KTILE(0);
        if (left == 0) break;
        KTILE(1);
        if (left == 0) break;
    }
    WAIT_V(0);
    if (wr == 0) BAR;
    STAMP(3);

    const int lz = lane_id(), fr = lz & 15, fq = lz >> 4;
    const int owner = mt >> 1;
    const int ptile = (mt & 1) * 64 + gnt;
    const size_t pt = (size_t)(ptile * 8 + wid) * 2;
    const size_t ps = (size_t)((mt * 8 + (gnt & 7)) * 8 + wid) * 2;
    if (is_main) {
        auto pack_half = [&](int bj, Packed& pk) {
            u32x4 v[8];
#pragma unroll
            for (int n = 0; n < 2; ++n)
#pragma unroll
                for (int r = 0; r < 4; ++r) {
                    u32x4& o = v[n * 4 + r];
                    o.x = cvt_pk_bf16(acc[0][bj][0][n][r], acc[0][bj][1][n][r]); o.y = cvt_pk_bf16(acc[0][bj][2][n][r], acc[0][bj][3][n][r]);
                    o.z = cvt_pk_bf16(acc[1][bj][0][n][r], acc[1][bj][1][n][r]); o.w = cvt_pk_bf16(acc[1][bj][2][n][r], acc[1][bj][3][n][r]);
                }
            pack(v, pk, scr, lz);
        };
        auto poison = [&](Packed& pk) {
#pragma unroll
            for (int i = 0; i < 4; ++i) pk.sm[i] = (u32x4){0x40404040u, 0x40404040u, 0x40404040u, 0x40404040u};
            pk.l1 = pk.st = (u32x4){0u, 0u, 0u, 0u}; pk.mask = 0ull; pk.meta = 0x101u;
        };
        Packed pk0, pk1;
        unsigned og, ot;
        asm volatile("global_load_dword %0, %2, off sc0 sc1\n\tglobal_load_dword %1, %3, off sc1" : "=&v"(og), "=&v"(ot) : "v"(fa + owner * 32), "v"(tmo) : "memory");
        if (wid < 4) __builtin_amdgcn_s_setprio(2);
        pack_half(0, pk0);
        __builtin_amdgcn_s_setprio(0);
        asm volatile("s_waitcnt vmcnt(0)" : "+v"(og), "+v"(ot) :: "memory");
        bool bad = (unsigned)__builtin_amdgcn_readfirstlane(ot) == call;
        if ((unsigned)__builtin_amdgcn_readfirstlane(og) != tag4) bad = wait_word<true>(fa + owner * 32, tag4, tmo, call);
        if (bad) poison(pk0);
        STAMP(4);
        unsigned char* const ows = P.ws[owner];
        send(pk0, ws + OFF_PP + ps * UNIT, scr, lz);
        asm volatile("s_waitcnt vmcnt(0)" ::: "memory");
        if (lz == 0) st16_sys(ows + OFF_HP + pt * 16, header(pk0, call));
        pack_half(1, pk1);
        if (bad) poison(pk1);
        send(pk1, ws + OFF_PP + (ps + 1) * UNIT, scr, lz);
        asm volatile("s_waitcnt vmcnt(0)" ::: "memory");
        if (lz == 0) st16_sys(ows + OFF_HP + (pt + 1) * 16, header(pk1, call));
    } else {
        const int src = gnt >> 3;
        const unsigned char* const sws = P.ws[src];
        unsigned char* dst = P.out + (size_t)((mt & 1) * 256 + wc * 32 + fq * 4) * (GN * 2) + (size_t)(gnt * 256 + wr * 128 + fr * 8) * 2;
#pragma unroll
        for (int bj = 0; bj < 2; ++bj) {
            u32x4 p[8];
            Packed pk;
            wait_header(ws + OFF_HP + (pt + bj) * 16, tmo, call, pk.mask, pk.meta);
            const int pos = bj * 8 + wid;
            if (src != self) {
                if (pos >= NCH) {
                    const u64 t0 = __builtin_amdgcn_s_memrealtime();
                    while (__hip_atomic_load(&tok[pos - NCH], __ATOMIC_ACQUIRE, __HIP_MEMORY_SCOPE_WORKGROUP) == 0u
                           && __builtin_amdgcn_s_memrealtime() - t0 < 2 * DEADLINE) __builtin_amdgcn_s_sleep(1);
                } else {
                    wait_word<false>((mt & 1) ? ws + OFF_CH + (gnt * 8 + pos) * 128 : ws + OFF_CRED + src * 128,
                                     (mt & 1) ? call : (call - 1u) * PW + 1u + (pos * 8 + (gnt & 7)) * PW / (8 * NCH), tmo, call);
                }
            }
            fetch(pk, sws + OFF_PP + (ps + bj) * UNIT, lz);
            unsigned tw;
            asm volatile("global_load_dword %0, %1, off sc1" : "=&v"(tw) : "v"(tmo) : "memory");
            FETCH_WAIT(pk, 0);
            asm volatile("" : "+v"(tw));
            const bool bad = (unsigned)__builtin_amdgcn_readfirstlane(tw) == call;
            if (lz == 0) {
                __hip_atomic_store(&tok[pos], 1u, __ATOMIC_RELEASE, __HIP_MEMORY_SCOPE_WORKGROUP);
                if (pos >= 16 - NCH && !(mt & 1)) st_flag(ws + OFF_CH + (gnt * 8 + pos - (16 - NCH)) * 128, call);
            }
            unpack(pk, p, sws + OFF_PP + (ps + bj) * UNIT, scr, lz, []() {});
            if (bj == 1) STAMP(4);
#pragma unroll
            for (int n = 0; n < 2; ++n)
#pragma unroll
                for (int r = 0; r < 4; ++r) {
                    u32x4 v;
#pragma unroll
                    for (int j = 0; j < 4; ++j) {
                        const unsigned u = p[n * 4 + r][j];
                        const f32x2 y = (f32x2){acc[j >> 1][bj][(j & 1) * 2][n][r], acc[j >> 1][bj][(j & 1) * 2 + 1][n][r]}
                                      + (f32x2){__uint_as_float(u << 16), __uint_as_float(u & 0xffff0000u)};
                        const f32x2 g = gelu2(y);
                        v[j] = cvt_pk_bf16(g.x, g.y);
                    }
                    if (bad) v = (u32x4){0x7fc07fc0u, 0x7fc07fc0u, 0x7fc07fc0u, 0x7fc07fc0u};
                    st16_sys(dst + (size_t)(bj * 128 + n * 16 + r) * (GN * 2), v);
                }
        }
    }
    if (tid == 0) __hip_atomic_store((unsigned*)(ws + OFF_CNT + b * 128), call, __ATOMIC_RELAXED, __HIP_MEMORY_SCOPE_AGENT);
    STAMP(5);
}

}

extern "C" void kernel_launch(int rank, int nranks, ncclComm_t comm, hipStream_t stream,
                              void* const* d_in, const int* in_sizes, int n_in,
                              void* d_out, int out_size, const int* in_dtypes, int out_dtype,
                              void* const* d_out_all, void* const* d_ws_all, size_t ws_bytes) {
    (void)comm; (void)in_dtypes; (void)out_dtype; (void)d_out_all;
    if (nranks != NR || (unsigned)rank >= (unsigned)NR || n_in < 2 || ws_bytes < WS_NEED
        || in_sizes[0] != GM * KS || in_sizes[1] != KS * GN || out_size != (GM / NR) * GN) {
        fprintf(stderr, "kernel_launch: unexpected problem shape\n");
        return;
    }
    Params p;
    p.x = (const unsigned char*)d_in[0];
    p.w = (const unsigned char*)d_in[1];
    p.out = (unsigned char*)d_out;
    for (int i = 0; i < NR; ++i) p.ws[i] = (unsigned char*)d_ws_all[i];
    p.rank = rank; p.pad = 0;
    hipLaunchKernelGGL(fused_gemm_rs, dim3(256), dim3(512), 0, stream, p);
    HIP_CHECK(hipGetLastError());
}
```

```cpp
#include <hip/hip_runtime.h>
#include <rccl/rccl.h>
#include <cstdio>
#include <cstdint>

#define HIP_CHECK(expr) do { hipError_t _e = (expr); if (_e != hipSuccess) { \
    fprintf(stderr, "HIP error %s at %s:%d\n", hipGetErrorString(_e), __FILE__, __LINE__); \
    return; } } while (0)

namespace {

#define LAS __attribute__((address_space(3)))
typedef short bf16x8 __attribute__((ext_vector_type(8)));
typedef float f32x4 __attribute__((ext_vector_type(4)));
typedef unsigned u32x4 __attribute__((ext_vector_type(4)));
typedef unsigned long long u64;

constexpr int NR = 8;
constexpr int GM = 4096, GK = 4096, GN = 16384;
constexpr int KS = GK / NR;
constexpr int BLK = 32768;
constexpr int HTB = 16384;
#define PWV 7
#define WIN 16
#define PULL_DELAY 200
#define HEAD_W 2
#define PACK_DELAY 150
#define PK_ALL 2
#define NTX 1
#define SNAP_AT 2
#define MID_Q 2
constexpr int NPULL = 56, PW = 8 * PWV;
constexpr u64 DEADLINE = 50000000ull;
constexpr int NTASK = 704;
constexpr int UNIT = 9344;

constexpr size_t OFF_CNT = 0;
constexpr size_t OFF_TMO = 32768;
constexpr size_t OFF_TICK = 33792;
constexpr size_t OFF_FX = 65536;
constexpr size_t OFF_FW = 81920;
constexpr size_t OFF_FD = 98304;
constexpr size_t OFF_FP = 106496;
constexpr size_t OFF_HIN = 131072;
constexpr size_t OFF_HP = 327680;
constexpr size_t OFF_STAMP = 524288;
constexpr size_t OFF_XA = 1u << 20;
constexpr size_t OFF_WB = OFF_XA + (size_t)64 * 16 * BLK;
constexpr size_t OFF_WD = OFF_WB + (size_t)64 * 8 * BLK;
constexpr size_t OFF_RP = OFF_WD + (size_t)64 * 8 * BLK;
constexpr size_t OFF_PP = OFF_RP + (size_t)5120 * UNIT;
constexpr size_t OFF_PD = OFF_PP + (size_t)2048 * UNIT;
constexpr size_t WS_NEED = OFF_PD + (size_t)128 * 131072 + 4096;
static_assert(OFF_HIN + 16 * 768 * 16 <= OFF_HP && OFF_HP + 2048 * 16 <= OFF_STAMP, "flag region");
static_assert(WS_NEED <= (size_t)161 * 1048576, "workspace");

struct Params {
    const unsigned char* x;
    const unsigned char* w;
    unsigned char* out;
    unsigned char* ws[NR];
    int rank, pad;
};

__device__ __forceinline__ void st16_sys(unsigned char* p, u32x4 v) {
    asm volatile("global_store_dwordx4 %0, %1, off sc0 sc1\n\ts_nop 1" :: "v"(p), "v"(v) : "memory");
}
__device__ __forceinline__ void st_flag(unsigned char* p, unsigned v) {
    __hip_atomic_store((unsigned*)p, v, __ATOMIC_RELAXED, __HIP_MEMORY_SCOPE_SYSTEM);
}
__device__ __forceinline__ void st_flag8(unsigned char* p, unsigned v) {
    __hip_atomic_store(p, (unsigned char)v, __ATOMIC_RELAXED, __HIP_MEMORY_SCOPE_SYSTEM);
}
__device__ __forceinline__ unsigned ld_flag(const unsigned char* p) {
    return __hip_atomic_load((const unsigned*)p, __ATOMIC_RELAXED, __HIP_MEMORY_SCOPE_SYSTEM);
}
__device__ __forceinline__ unsigned lane_id() {
    return __builtin_amdgcn_mbcnt_hi(~0u, __builtin_amdgcn_mbcnt_lo(~0u, 0u));
}
__device__ __forceinline__ unsigned cvt_pk_bf16(float lo, float hi) {
    unsigned r; asm volatile("v_cvt_pk_bf16_f32 %0, %1, %2" : "=v"(r) : "v"(lo), "v"(hi)); return r;
}
typedef float f32x2 __attribute__((ext_vector_type(2)));
__device__ __forceinline__ f32x2 gelu2(f32x2 y) {
    const float k1 = -2.0f * 0.7978845608028654f * 1.4426950408889634f;
    const float k3 = k1 * 0.044715f;
    const f32x2 z = y * ((y * y) * k3 + k1);
    const f32x2 d = (f32x2){__builtin_amdgcn_exp2f(z.x), __builtin_amdgcn_exp2f(z.y)} + 1.0f;
    return y * (f32x2){__builtin_amdgcn_rcpf(d.x), __builtin_amdgcn_rcpf(d.y)};
}

template <bool EQ>
__device__ __forceinline__ bool wait_word(const unsigned char* p, unsigned want, unsigned char* tmo, unsigned call) {
    const u64 t0 = __builtin_amdgcn_s_memrealtime();
    for (;;) {
        unsigned g, t;
        asm volatile("global_load_dword %0, %2, off sc0 sc1\n\t"
                     "global_load_dword %1, %3, off sc1\n\t"
                     "s_waitcnt vmcnt(0)" : "=&v"(g), "=&v"(t) : "v"(p), "v"(tmo) : "memory");
        if ((unsigned)__builtin_amdgcn_readfirstlane(t) == call || __builtin_amdgcn_s_memrealtime() - t0 > DEADLINE) {
            __hip_atomic_store((unsigned*)tmo, call, __ATOMIC_RELAXED, __HIP_MEMORY_SCOPE_AGENT);
            return true;
        }
        const unsigned v = __builtin_amdgcn_readfirstlane(g);
        if (EQ ? v == want : v >= want) return false;
        __builtin_amdgcn_s_sleep(2);
    }
}

__device__ __forceinline__ u64 now() {
    u64 t; asm volatile("s_memrealtime %0\n\ts_waitcnt lgkmcnt(0)" : "=s"(t) :: "memory"); return t;
}
__device__ __forceinline__ u64 poll_ready(u64 todo, const unsigned char* fa, const unsigned char* fb,
                                          unsigned char* tmo, unsigned call, unsigned tag4, unsigned xm) {
    const u64 t0 = now();
    for (;;) {
        unsigned a, b, t; const unsigned off = lane_id() * 4u, zero = 0u;
        asm volatile("global_load_dword %0, %3, %5 sc0 sc1\n\t"
                     "global_load_dword %1, %3, %6 sc0 sc1\n\t"
                     "global_load_dword %2, %4, %7 sc1\n\t"
                     "s_waitcnt vmcnt(0)"
                     : "=&v"(a), "=&v"(b), "=&v"(t) : "v"(off), "v"(zero), "s"(fa), "s"(fb), "s"(tmo) : "memory");
        const u64 ready = __ballot(a == tag4 && ((b ^ tag4) & xm) == 0u);
        if ((ready & todo) != 0ull) return ready;
        if ((unsigned)__builtin_amdgcn_readfirstlane(t) == call || now() - t0 > DEADLINE) {
            __hip_atomic_store((unsigned*)tmo, call, __ATOMIC_RELAXED, __HIP_MEMORY_SCOPE_AGENT);
            return ~0ull;
        }
        __builtin_amdgcn_s_sleep(1);
    }
}

__device__ __forceinline__ unsigned bfi(unsigned m, unsigned a, unsigned b) { return (a & m) | (b & ~m); }
__device__ __forceinline__ unsigned perm(unsigned hi, unsigned lo, unsigned sel) { return __builtin_amdgcn_perm(hi, lo, sel); }
#define SDWA2(k) " dst_sel:DWORD dst_unused:UNUSED_PAD src0_sel:BYTE_" #k " src1_sel:BYTE_" #k "\n\t"
#define SDWA1(k) " dst_sel:DWORD dst_unused:UNUSED_PAD src0_sel:DWORD src1_sel:BYTE_" #k "\n\t"
__device__ __forceinline__ unsigned spread_bits(unsigned f, unsigned s) {
    unsigned r, a, b;
    asm("v_lshlrev_b32_sdwa %0, %4, %3" SDWA2(0) "v_lshlrev_b32_sdwa %1, %4, %3" SDWA2(1) "v_lshlrev_b32_sdwa %2, %4, %3" SDWA2(2)
        "v_or3_b32 %0, %0, %1, %2\n\t"
        "v_lshlrev_b32_sdwa %1, %4, %3" SDWA2(3)
        "v_or_b32 %0, %0, %1" : "=&v"(r), "=&v"(a), "=&v"(b) : "v"(f), "v"(s));
    return r;
}
__device__ __forceinline__ unsigned take_codes(unsigned& w, unsigned f) {
    unsigned ln, t, l;
    asm("v_xor_b32_sdwa %2, %0, %4" SDWA1(0) "v_ffbl_b32 %1, %2\n\tv_lshrrev_b32 %0, %1, %0\n\t"
        "v_xor_b32_sdwa %2, %0, %4" SDWA1(1) "v_ffbl_b32 %3, %2\n\tv_lshrrev_b32 %0, %3, %0\n\t"
        "v_xor_b32_sdwa %2, %0, %4" SDWA1(2) "v_lshl_or_b32 %1, %3, 8, %1\n\tv_ffbl_b32 %3, %2\n\tv_lshrrev_b32 %0, %3, %0\n\t"
        "v_xor_b32_sdwa %2, %0, %4" SDWA1(3) "v_lshl_or_b32 %1, %3, 16, %1\n\tv_ffbl_b32 %3, %2\n\tv_lshrrev_b32 %0, %3, %0\n\t"
        "v_lshl_or_b32 %1, %3, 24, %1" : "+v"(w), "=&v"(ln), "=&v"(t), "=&v"(l) : "v"(f));
    return ln;
}
__device__ __forceinline__ unsigned wave_scan(unsigned v) {
    v += (unsigned)__builtin_amdgcn_update_dpp(0, (int)v, 0x111, 0xf, 0xf, false);
    v += (unsigned)__builtin_amdgcn_update_dpp(0, (int)v, 0x112, 0xf, 0xf, false);
    v += (unsigned)__builtin_amdgcn_update_dpp(0, (int)v, 0x114, 0xf, 0xf, false);
    v += (unsigned)__builtin_amdgcn_update_dpp(0, (int)v, 0x118, 0xf, 0xf, false);
    v += (unsigned)__builtin_amdgcn_update_dpp(0, (int)v, 0x142, 0xa, 0xf, false);
    v += (unsigned)__builtin_amdgcn_update_dpp(0, (int)v, 0x143, 0xc, 0xf, false);
    return v;
}
__device__ __forceinline__ unsigned rank_in(u64 mask) {
    return __builtin_amdgcn_mbcnt_hi((unsigned)(mask >> 32), __builtin_amdgcn_mbcnt_lo((unsigned)mask, 0u));
}
constexpr int SCR = 8192;
#define LDS_ORDER() asm volatile("" ::: "memory")

struct Packed { u32x4 sm[4], l1, st; u64 mask; unsigned meta; };

__device__ __forceinline__ void pack(const u32x4 (&d)[8], Packed& o, LAS unsigned char* scr, unsigned lane) {
    *(LAS u32x4*)(scr + lane * 16) = (u32x4){0u, 0u, 0u, 0u};
    unsigned sum = 0; u32x4 er[4];
#pragma unroll
    for (int g = 0; g < 16; ++g) {
        const unsigned d0 = d[g >> 1][(g & 1) * 2], d1 = d[g >> 1][(g & 1) * 2 + 1];
        const unsigned lb = perm(d1, d0, 0x06040200u), hb = perm(d1, d0, 0x07050301u);
        o.sm[g >> 2][g & 3] = bfi(0x7f7f7f7fu, lb, hb);
        const unsigned e = bfi(0x01010101u, lb >> 7, hb << 1);
        er[g >> 2][g & 3] = e;
        sum = __builtin_amdgcn_sad_u8(e, 0u, sum);
    }
    const unsigned r2 = (((unsigned)__builtin_amdgcn_readlane(wave_scan(sum), 63) + 16036u) >> 12) & 255u;
    const unsigned kk = (r2 | 0x80u) * 0x01010101u, cc = (~r2 & 0x80u) * 0x01010101u;
    unsigned bad = 0, longest = 0, total = 0, piece[8], plen[8];
#pragma unroll
    for (int q = 0; q < 4; ++q) {
        unsigned l1 = 0, acc = 0, base = 0;
#pragma unroll
        for (int gg = 0; gg < 4; ++gg) {
            const unsigned e = er[q][gg];
            const unsigned u = (kk - (e & 0x7f7f7f7fu)) ^ (e & 0x80808080u) ^ cc;
            bad |= u;
            const unsigned sel = u | ((u << 1) & 0x10101010u);
            const unsigned c = perm(0x03030302u, 0x01000303u, sel);
            l1 |= (c & 0x03030303u) << (2 * gg);
            const unsigned len4 = (perm(0xfffefefeu, 0xfffcf8dau, sel) ^ 0xfefefefeu) + u;
            const unsigned f = c & (c >> 1) & 0x01010101u;
            unsigned ps = len4 + (len4 << 8); asm("" : "+v"(ps)); ps += ps << 16; ps -= 0x100c0804u;
            const unsigned bits = spread_bits(f, ps);
            if ((gg & 1) == 0) { acc = bits; base = ps >> 24; }
            else {
                acc |= bits << (base & 31u); base += ps >> 24;
                piece[q * 2 + (gg >> 1)] = acc >> 1; plen[q * 2 + (gg >> 1)] = base;
                longest = max(longest, base); total += base;
            }
        }
        o.l1[q] = l1;
    }
    const bool raw = (bad & 0xe0e0e0e0u) != 0u || longest > 31u || total > 112u;
    o.mask = __ballot(raw);
    if (raw) total = 0;
    const unsigned incl = wave_scan(total);
    const unsigned rawoff = (((unsigned)__builtin_amdgcn_readlane(incl, 63) + 512u + 127u) >> 7) << 4;
    o.meta = r2 | ((rawoff + 64u * (unsigned)__builtin_popcountll(o.mask) + 127u) >> 7) << 8;
    LDS_ORDER();
    scr[lane] = (unsigned char)total;
    if (!raw) {
        unsigned off = incl - total + 512u;
#pragma unroll
        for (int i = 0; i < 8; ++i) {
            const u64 w = (u64)piece[i] << (off & 31u);
            LAS unsigned* p = (LAS unsigned*)(scr + ((off >> 5) << 2));
            __hip_atomic_fetch_or(p, (unsigned)w, __ATOMIC_RELAXED, __HIP_MEMORY_SCOPE_WORKGROUP);
            __hip_atomic_fetch_or(p + 1, (unsigned)(w >> 32), __ATOMIC_RELAXED, __HIP_MEMORY_SCOPE_WORKGROUP);
            off += plen[i];
        }
    } else {
        LAS u32x4* p = (LAS u32x4*)(scr + rawoff + rank_in(o.mask) * 64u);
#pragma unroll
        for (int i = 0; i < 4; ++i) p[i] = er[i];
    }
    LDS_ORDER();
    o.st = *(const LAS u32x4*)(scr + lane * 16);
    LDS_ORDER();
}
__device__ __forceinline__ void send(const Packed& o, unsigned char* pl, const LAS unsigned char* scr, unsigned lane) {
    unsigned char* p = pl + lane * 16;
    for (unsigned k = 1; k * 8u < (o.meta >> 8); ++k)
        if (k * 64u + lane < (o.meta >> 8) * 8u) st16_sys(p + 5120 + k * 1024, *(const LAS u32x4*)(scr + k * 1024 + lane * 16));
#pragma unroll
    for (int i = 0; i < 4; ++i) st16_sys(p + i * 1024, o.sm[i]);
    st16_sys(p + 4096, o.l1);
    if (lane < (o.meta >> 8) * 8u) st16_sys(p + 5120, o.st);
}
__device__ __forceinline__ u32x4 header(const Packed& o, unsigned call) {
    return (u32x4){(unsigned)o.mask, call, (unsigned)(o.mask >> 32), (call << 14) | o.meta};
}
__device__ __forceinline__ bool header_ok(u32x4 h, unsigned call, u64& mask, unsigned& meta) {
    const unsigned t1 = __builtin_amdgcn_readfirstlane(h.y), t3 = __builtin_amdgcn_readfirstlane(h.w);
    mask = ((u64)(unsigned)__builtin_amdgcn_readfirstlane(h.z) << 32) | (unsigned)__builtin_amdgcn_readfirstlane(h.x);
    meta = t3 & 0x3fffu;
    return t1 == call && (t3 >> 14) == (call & 0x3ffffu);
}
__device__ __forceinline__ void wait_header(const unsigned char* hd, unsigned char* tmo, unsigned call, u64& mask, unsigned& meta) {
    const u64 t0 = __builtin_amdgcn_s_memrealtime();
    for (;;) {
        u32x4 h; unsigned t;
        asm volatile("global_load_dwordx4 %0, %2, off sc0 sc1\n\t"
                     "global_load_dword %1, %3, off sc1\n\t"
                     "s_waitcnt vmcnt(0)" : "=&v"(h), "=&v"(t) : "v"(hd), "v"(tmo) : "memory");
        if (header_ok(h, call, mask, meta)) break;
        if ((unsigned)__builtin_amdgcn_readfirstlane(t) == call || __builtin_amdgcn_s_memrealtime() - t0 > DEADLINE) {
            __hip_atomic_store((unsigned*)tmo, call, __ATOMIC_RELAXED, __HIP_MEMORY_SCOPE_AGENT);
            mask = 0ull; meta = 0u;
            break;
        }
        __builtin_amdgcn_s_sleep(1);
    }
}
__device__ __forceinline__ void fetch(Packed& o, const unsigned char* pl, unsigned lane) {
    const unsigned n = min(max(o.meta >> 8, 1u), 8u) * 8u;
    const u64 smask = n >= 64u ? ~0ull : (1ull << n) - 1ull;
    u64 keep;
    asm volatile("s_mov_b64 %6, exec\n\t"
                 "global_load_dwordx4 %0, %7, %8 sc0 sc1\n\t"
                 "global_load_dwordx4 %1, %7, %8 offset:1024 sc0 sc1\n\t"
                 "global_load_dwordx4 %2, %7, %8 offset:2048 sc0 sc1\n\t"
                 "global_load_dwordx4 %3, %7, %8 offset:3072 sc0 sc1\n\t"
                 "global_load_dwordx4 %4, %7, %9 sc0 sc1\n\t"
                 "s_mov_b64 exec, %10\n\t"
                 "global_load_dwordx4 %5, %7, %9 offset:1024 sc0 sc1\n\t"
                 "s_mov_b64 exec, %6"
                 : "=&v"(o.sm[0]), "=&v"(o.sm[1]), "=&v"(o.sm[2]), "=&v"(o.sm[3]), "=&v"(o.l1), "=&v"(o.st), "=&s"(keep)
                 : "v"(lane * 16u), "s"(pl), "s"(pl + 4096), "s"(smask) : "memory");
}
#define FETCH_WAIT(o, n) asm volatile("s_waitcnt vmcnt(" #n ")" : "+v"(o.sm[0]), "+v"(o.sm[1]), "+v"(o.sm[2]), "+v"(o.sm[3]), "+v"(o.l1), "+v"(o.st) :: "memory")
template <class Mid>
__device__ __forceinline__ void unpack(const Packed& o, u32x4 (&d)[8], const unsigned char* pl, LAS unsigned char* scr, unsigned lane, Mid mid) {
    *(LAS u32x4*)(scr + lane * 16) = o.st;
    if ((o.meta >> 8) > 8u) {
        u32x4 m[4];
        asm volatile("global_load_dwordx4 %0, %4, off sc0 sc1\n\t"
                     "global_load_dwordx4 %1, %4, off offset:1024 sc0 sc1\n\t"
                     "global_load_dwordx4 %2, %4, off offset:2048 sc0 sc1\n\t"
                     "global_load_dwordx4 %3, %4, off offset:3072 sc0 sc1\n\t"
                     "s_waitcnt vmcnt(0)" : "=&v"(m[0]), "=&v"(m[1]), "=&v"(m[2]), "=&v"(m[3]) : "v"(pl + 6144 + lane * 16) : "memory");
#pragma unroll
        for (int k = 0; k < 4; ++k) *(LAS u32x4*)(scr + (k + 1) * 1024 + lane * 16) = m[k];
    }
    LDS_ORDER();
    const unsigned len = scr[lane] & 127u;
    const unsigned incl = wave_scan(len), off = incl - len + 512u;
    const LAS unsigned* p = (const LAS unsigned*)(scr + ((off >> 5) << 2));
    const unsigned w0 = p[0], w1 = p[1], w2 = p[2], w3 = p[3], w4 = p[4];
    unsigned s0 = __builtin_amdgcn_alignbit(w1, w0, off), s1 = __builtin_amdgcn_alignbit(w2, w1, off);
    unsigned s2 = __builtin_amdgcn_alignbit(w3, w2, off), s3 = __builtin_amdgcn_alignbit(w4, w3, off);
    const unsigned r7 = (o.meta + 5u) & 255u;
    const unsigned kk = (r7 | 0x80u) * 0x01010101u, cc = (~r7 & 0x80u) * 0x01010101u;
    u32x4 er[4];
#pragma unroll
    for (int q = 0; q < 4; ++q) {
        if (q == MID_Q) { asm volatile("" : "+v"(s0)); mid(); asm volatile("" : "+v"(s0)); }
        const unsigned l1 = o.l1[q], x = l1 & (l1 >> 1);
        unsigned w = 0, used = 0;
#pragma unroll
        for (int gg = 0; gg < 4; ++gg) {
            if ((gg & 1) == 0) { w = (s0 << 1) | 1u; used = 0; }
            const unsigned f = (x >> (2 * gg)) & 0x01010101u;
            const unsigned ln = take_codes(w, f);
            used = __builtin_amdgcn_sad_u8(ln, 0u, used);
            const unsigned y = (perm(0xfa00fc00u, 0xfc00fd00u, (ln << 1) | 0x01010101u) ^ 0xfafafafau) + ln + ((l1 >> (2 * gg)) & 0x03030303u);
            er[q][gg] = (kk - y) ^ cc;
            if (gg & 1) {
                s0 = __builtin_amdgcn_alignbit(s1, s0, used); s1 = __builtin_amdgcn_alignbit(s2, s1, used);
                s2 = __builtin_amdgcn_alignbit(s3, s2, used); s3 >>= used & 31u;
            }
        }
    }
    if (o.mask != 0ull) {
        const unsigned rawoff = (((unsigned)__builtin_amdgcn_readlane(incl, 63) + 512u + 127u) >> 7) << 4;
        if ((o.mask >> lane) & 1ull) {
            const LAS u32x4* r = (const LAS u32x4*)(scr + rawoff + rank_in(o.mask) * 64u);
#pragma unroll
            for (int i = 0; i < 4; ++i) er[i] = r[i];
        }
    }
    LDS_ORDER();
#pragma unroll
    for (int g = 0; g < 16; ++g) {
        const unsigned e = er[g >> 2][g & 3], sm = o.sm[g >> 2][g & 3];
        const unsigned hb = bfi(0x80808080u, sm, e >> 1), lb = bfi(0x7f7f7f7fu, sm, e << 7);
        d[g >> 1][(g & 1) * 2] = perm(hb, lb, 0x05010400u); d[g >> 1][(g & 1) * 2 + 1] = perm(hb, lb, 0x07030602u);
    }
}

__device__ __forceinline__ void load_parked(u32x4 (&dg)[8], unsigned& tw, const unsigned char* dp, const unsigned char* tmo, unsigned lane) {
    asm volatile("global_load_dwordx4 %0, %9, %10 sc0 sc1\n\t"
                 "global_load_dwordx4 %1, %9, %10 offset:1024 sc0 sc1\n\t"
                 "global_load_dwordx4 %2, %9, %10 offset:2048 sc0 sc1\n\t"
                 "global_load_dwordx4 %3, %9, %10 offset:3072 sc0 sc1\n\t"
                 "global_load_dwordx4 %4, %9, %11 sc0 sc1\n\t"
                 "global_load_dwordx4 %5, %9, %11 offset:1024 sc0 sc1\n\t"
                 "global_load_dwordx4 %6, %9, %11 offset:2048 sc0 sc1\n\t"
                 "global_load_dwordx4 %7, %9, %11 offset:3072 sc0 sc1\n\t"
                 "global_load_dword %8, %8, %12 sc1"
                 : "=&v"(dg[0]), "=&v"(dg[1]), "=&v"(dg[2]), "=&v"(dg[3]), "=&v"(dg[4]), "=&v"(dg[5]), "=&v"(dg[6]), "=&v"(dg[7]), "+v"(tw)
                 : "v"(lane * 16u), "s"(dp), "s"(dp + 4096), "s"(tmo) : "memory");
}
#define PARKED_WAIT(dg, tw) asm volatile("s_waitcnt vmcnt(0)" : "+v"(dg[0]), "+v"(dg[1]), "+v"(dg[2]), "+v"(dg[3]), "+v"(dg[4]), "+v"(dg[5]), \
                                         "+v"(dg[6]), "+v"(dg[7]), "+v"(tw) :: "memory")

#define STAMP(i) do {} while (0)

#define ORD_A1 1.15
#define ORD_B0 0.4
#define ORD_BS 1.2
#define ORD_C0 6.0
#define ORD_CS 0.35
#define ORD_W 0.3
struct Order { unsigned short v[354]; };
constexpr Order make_order() {
    double key[352] = {}; unsigned short id[352] = {}; int n = 0;
    for (int kc = 0; kc < 8; ++kc)
        for (int h = 0; h < 2; ++h) {
            for (int e = 0; e < 8; ++e) { key[n] = kc - ORD_W + (e + 0.4) / 8 + h * 1e-3; id[n++] = (unsigned short)(h << 8 | 0 << 6 | kc << 3 | e); }
            for (int e = 0; e < 6; ++e) { key[n] = (kc + (e + 0.5) / 6) * (h ? ORD_A1 : 1.0); id[n++] = (unsigned short)(h << 8 | 1 << 6 | kc << 3 | e); }
            for (int e = 0; e < 6; ++e) { key[n] = ORD_B0 + (kc + (e + 0.6) / 6) * ORD_BS + h * 1e-3; id[n++] = (unsigned short)(h << 8 | 2 << 6 | kc << 3 | e); }
            for (int e = 0; e < 2; ++e) { key[n] = ORD_C0 + (2 * kc + e + 0.5 * h) * ORD_CS; id[n++] = (unsigned short)(h << 8 | 3 << 6 | kc << 3 | e); }
        }
    Order o = {};
    for (int i = 0; i < 352; ++i) { int r = 0; for (int j = 0; j < 352; ++j) r += (key[j] < key[i]) || (key[j] == key[i] && j < i); o.v[r] = id[i]; }
    return o;
}
constexpr Order ORDER = make_order();
constexpr unsigned order_word(int i) { return ORDER.v[3 * i] | ORDER.v[3 * i + 1] << 10 | (unsigned)ORDER.v[3 * i + 2] << 20; }
template <int I, int N> __device__ __forceinline__ unsigned order_vgpr(unsigned v) {
    if constexpr (I == N) return v;
    else {
        unsigned t;
        asm("s_mov_b32 %1, %2\n\tv_writelane_b32 %0, %1, %3" : "+v"(v), "=&s"(t) : "i"(order_word(I)), "i"(I & 63));
        return order_vgpr<I + 1, N>(v);
    }
}
__device__ __forceinline__ void decode_unit(unsigned ord0, unsigned ord1, int t, int src, int dest, int& isw, int& kc, int& tile, int& q) {
    const int n = t >> 1, dw = n / 3;
    const unsigned w0 = __builtin_amdgcn_readlane(ord0, dw & 63), w1 = __builtin_amdgcn_readlane(ord1, dw & 63);
    const unsigned v = (dw < 64 ? w0 : w1) >> ((n - dw * 3) * 10);
    const int kind = (v >> 6) & 3, e = v & 7;
    kc = (v >> 3) & 7; isw = kind == 0; q = ((v >> 7) & 2) | (t & 1);
    const int d = (dest - src - 1) & 7;
    tile = kind == 0 ? ((e + src) & 7) : kind == 3 ? 2 * dest + e : 2 * ((src + 1 + e + (e >= d)) & 7) + (kind - 1);
}

__host__ __device__ __forceinline__ void role_of(int b, int& nt, int& slot, int& g) {
    constexpr int NG = 8 / NTX, X0 = 8 - NG;
    const int x = b & 7, s = b >> 3, np = x < X0 ? 8 : 8 - NTX;
    if (s < np) {
        const int pid = s < 8 - NTX ? s * 8 + x : (8 - NTX) * 8 + (s - (8 - NTX)) * X0 + x;
        slot = NTX == 1 ? pid >> 3 : pid % 7; nt = NTX == 1 ? pid & 7 : pid / 7; g = -1;
    } else {
        const int j = s - np, r = j / NTX + x / NG * NG * 3;
        nt = x % NG * NTX + j % NTX; slot = 0;
        g = r >= 21 ? r : r % 3 == 2 ? 14 + r / 3 : r / 3 * 2 + r % 3;
    }
}

__global__ __launch_bounds__(512, 2) void fused_gemm_rs(const Params P) {
    __shared__ __attribute__((aligned(16))) unsigned char lds_raw[9 * HTB + 256];
    LAS unsigned char* lds = (LAS unsigned char*)lds_raw;
    const int tid = threadIdx.x, wid = __builtin_amdgcn_readfirstlane(tid >> 6), lane = tid & 63;
    const int b = blockIdx.x, self = P.rank;
    unsigned char* const ws = P.ws[self];
    unsigned char* const tmo = ws + OFF_TMO;
    LAS unsigned char* const scr = lds + wid * SCR;
    STAMP(0);
    const u64 tstart = __builtin_amdgcn_s_memrealtime();
    int nt, slot, g;
    role_of(b, nt, slot, g);
    const bool is_pull = g < 0, is_a = !is_pull && g < 14, is_b = g >= 14 && g < 21, is_c = g >= 21;
    const int owner = is_c ? self : (self + 1 + (is_a ? g >> 1 : g - 14)) & 7;
    const int mt = 2 * owner + (is_c ? (g - 21) >> 1 : is_b), hf = is_c ? (g - 21) & 1 : g & 1;
    auto pack_decode = [&](int t, int& isw, int& kc, int& tile) {
        const int blk = t >> 2;
        if (blk < 80) {
            const int m = blk - 16; kc = 0; isw = blk >= 16;
            tile = isw ? ((self + 1 + m) & 7) * 8 + (((m >> 3) + self) & 7) : blk;
        } else if (blk < 248) {
            const int m = blk - 80, r = m % 24; kc = 1 + m / 24; isw = r >= 16;
            tile = isw ? self * 8 + (r & 7) : r;
        } else if (blk < 640) {
            const int m = blk - 248, w = m % 56; isw = 1; kc = 1 + m / 56;
            tile = ((self + 1 + w % 7) & 7) * 8 + ((w / 7 + self) & 7);
        } else {
            const int m = blk - 640; isw = 1; kc = 0;
            tile = ((self + 1 + (m >> 3)) & 7) * 8 + (m & 7);
        }
    };
    const unsigned lw = (lane >> 4) * 8 * (GN * 2) + (lane & 15) * 16, lx = (lane & 15) * (KS * 2) + (lane >> 4) * 16;
    auto load_task = [&](int t, u32x4 (&r)[8]) {
        int isw, kc, tile; pack_decode(t, isw, kc, tile); const int q = t & 3;
        const unsigned char* src = isw ? P.w + ((kc * 64 + (q & 1) * 32) * (GN * 2) + (tile * 256 + (q >> 1) * 128) * 2)
                                       : P.x + ((tile * 256 + q * 64) * (KS * 2) + kc * 128);
        const unsigned lo = isw ? lw : lx, s1 = isw ? GN * 2 : 64, s2 = isw ? GN * 4 : 16 * KS * 2;
#pragma unroll
        for (int i = 0; i < 8; ++i) r[i] = *(const u32x4*)(src + ((i >> 1) * s2 + (i & 1) * s1) + lo);
    };
    auto store_image = [&](unsigned char* blk, int isw, int q, const u32x4 (&c)[8]) {
        unsigned char* d0 = blk + (isw ? (q >> 1) * 8192 + (q & 1) * 1024 : q * 8192) + lane * 16;
        const unsigned s1 = isw ? 2048 : 1024, s4 = isw ? HTB : 4096;
#pragma unroll
        for (int j = 0; j < 8; ++j) st16_sys(d0 + ((j >> 2) * s4 + (j & 3) * s1), c[j]);
    };
    u32x4 cur[8];
    const int task0 = is_pull ? wid * 8 + nt : wid * 144 + (is_a ? g : g - 7) * 8 + nt;
    if (is_a || is_c) {
        if (wid >= HEAD_W) { while (__builtin_amdgcn_s_memrealtime() - tstart < PACK_DELAY) __builtin_amdgcn_s_sleep(2); }
        else __builtin_amdgcn_s_setprio(2);
        load_task(task0, cur);
    }
    const unsigned call = (unsigned)__builtin_amdgcn_readfirstlane(
        __hip_atomic_load((const unsigned*)(ws + OFF_CNT + b * 128), __ATOMIC_RELAXED, __HIP_MEMORY_SCOPE_AGENT)) + 1u;
    const size_t par = call & 1u;
    const unsigned tag4 = (call & 255u) * 0x01010101u;
    const int wr = wid >> 2, wc = wid & 3;

#define READY0 0
    if (is_pull) {
        LAS unsigned* const ctl = (LAS unsigned*)(lds + 9 * HTB + 64);
        if (tid < 4) ctl[tid] = 0u;
        __syncthreads();
        const unsigned share = (WIN + 7 - nt) / 8;
        auto lds_draw = [&](int w) {
            unsigned v = 0;
            if (lane == 0) v = __hip_atomic_fetch_add(&ctl[w], 1u, __ATOMIC_RELAXED, __HIP_MEMORY_SCOPE_WORKGROUP);
            return (unsigned)__builtin_amdgcn_readfirstlane(v);
        };
        auto acquire = [&]() {
            const unsigned turn = lds_draw(0);
            while (__hip_atomic_load(&ctl[1], __ATOMIC_RELAXED, __HIP_MEMORY_SCOPE_WORKGROUP) + share <= turn) __builtin_amdgcn_s_sleep(1);
        };
        auto release = [&]() { if (lane == 0) __hip_atomic_fetch_add(&ctl[1], 1u, __ATOMIC_RELAXED, __HIP_MEMORY_SCOPE_WORKGROUP); };
        const int src = (self + 1 + slot) & 7;
        const unsigned char* const sws = P.ws[src];
        if (wid < PWV) {
            const unsigned ord0 = order_vgpr<0, 64>(0u), ord1 = order_vgpr<64, 118>(0u);
            unsigned* const tick = (unsigned*)(ws + OFF_TICK + src * 128);
            const unsigned tbase = (call - 1u) * NTASK - PW;
            const unsigned char* const hbase = ws + OFF_HIN + (par * 8 + src) * (768 * 16);
            int task = task0, isw, kc, tile, q;
            decode_unit(ord0, ord1, task, src, self, isw, kc, tile, q);
            Packed pk;
            while (__builtin_amdgcn_s_memrealtime() - tstart < PULL_DELAY) __builtin_amdgcn_s_sleep(2);
            wait_header(hbase + ((isw ? kc * 8 + tile : 64 + kc * 16 + tile) * 4 + q) * 16, tmo, call, pk.mask, pk.meta);
            for (;;) {
                const unsigned char* const pl = sws + OFF_RP + (par * 2560 + (isw ? (self * 8 + tile) * 8 + kc : 512 + kc * 16 + tile) * 4 + q) * UNIT;
                acquire();
                fetch(pk, pl, lane);
                FETCH_WAIT(pk, 0);
                release();
                unsigned nn = 1u;
                {
                    u64 keep; const unsigned zero = 0u;
                    asm volatile("s_mov_b64 %1, exec\n\ts_mov_b64 exec, 1\n\t"
                                 "global_atomic_add %0, %2, %0, %3 sc0\n\t"
                                 "s_mov_b64 exec, %1" : "+v"(nn), "=&s"(keep) : "v"(zero), "s"(tick) : "memory");
                }
                int ntask, nisw, nkc, ntile, nq; u32x4 h; const unsigned char* hd;
                unpack(pk, cur, pl, scr, lane, [&]() {
                    asm volatile("s_waitcnt vmcnt(0)" : "+v"(nn) :: "memory");
                    ntask = (int)(__builtin_amdgcn_readfirstlane(nn) - tbase);
                    decode_unit(ord0, ord1, ntask < NTASK ? ntask : 0, src, self, nisw, nkc, ntile, nq);
                    hd = hbase + ((nisw ? nkc * 8 + ntile : 64 + nkc * 16 + ntile) * 4 + nq) * 16;
                    const unsigned zero = 0u;
                    asm volatile("global_load_dwordx4 %0, %1, %2 sc0 sc1" : "=&v"(h) : "v"(zero), "s"(hd) : "memory");
                });
                const int idx = src * 8 + kc;
                store_image(isw ? ws + OFF_WB + (size_t)(idx * 8 + tile) * BLK : ws + OFF_XA + (size_t)(idx * 16 + tile) * BLK, isw, q, cur);
                asm volatile("s_waitcnt vmcnt(0)" : "+v"(h) :: "memory");
                if (lane == 0) st_flag8(ws + (isw ? OFF_FW : OFF_FX) + (tile * 64 + idx) * 4 + q, call);
                if (ntask >= NTASK) break;
                if (!header_ok(h, call, pk.mask, pk.meta)) wait_header(hd, tmo, call, pk.mask, pk.meta);
                task = ntask; isw = nisw; kc = nkc; tile = ntile; q = nq;
            }
            if (lane == 0) __hip_atomic_store(&ctl[3], 1u, __ATOMIC_RELAXED, __HIP_MEMORY_SCOPE_WORKGROUP);
        } else {
            while (__hip_atomic_load(&ctl[3], __ATOMIC_RELAXED, __HIP_MEMORY_SCOPE_WORKGROUP) == 0u) __builtin_amdgcn_s_sleep(8);
        }
        STAMP(1);
        const int gnt = src * 8 + nt, fr = lane & 15, fq = lane >> 4;
        for (;;) {
            const int u = (int)lds_draw(2);
            if (u >= 32) break;
            const int odd = u >> 4, bj = (u >> 3) & 1, uw = u & 7, dt = (slot * 2 + odd) * 8 + nt;
            Packed pk;
            wait_header(ws + OFF_HP + (size_t)(((odd * 64 + gnt) * 8 + uw) * 2 + bj) * 16, tmo, call, pk.mask, pk.meta);
            const unsigned char* pl = sws + OFF_PP + (size_t)((((2 * self + odd) * 8 + nt) * 8 + uw) * 2 + bj) * UNIT;
            acquire();
            const unsigned char* const fp = ws + OFF_FP + ((dt * 8 + uw) * 2 + bj) * 4;
            unsigned pf;
            asm volatile("global_load_dword %0, %1, off sc0 sc1" : "=&v"(pf) : "v"(fp) : "memory");
            fetch(pk, pl, lane);
            FETCH_WAIT(pk, 0);
            asm volatile("" : "+v"(pf));
            release();
            if ((unsigned)__builtin_amdgcn_readfirstlane(pf) < call) wait_word<false>(fp, call, tmo, call);
            u32x4 dg[8]; unsigned tw = 0u;
            load_parked(dg, tw, ws + OFF_PD + (size_t)((dt * 8 + uw) * 2 + bj) * 8192, tmo, lane);
            unpack(pk, cur, pl, scr, lane, []() {});
#pragma unroll
            for (int i = 0; i < 8; ++i) asm volatile("" : "+v"(cur[i]));
            PARKED_WAIT(dg, tw);
            const bool bad = (unsigned)__builtin_amdgcn_readfirstlane(tw) == call;
            unsigned char* dst = P.out + (size_t)(odd * 256 + bj * 128 + (uw & 3) * 32 + fq * 4) * (GN * 2) + (size_t)(gnt * 256 + (uw >> 2) * 128 + fr * 8) * 2;
#pragma unroll
            for (int n = 0; n < 2; ++n)
#pragma unroll
                for (int r = 0; r < 4; ++r) {
                    u32x4 v;
#pragma unroll
                    for (int j = 0; j < 4; ++j) {
                        const unsigned u = cur[n * 4 + r][j], w = dg[n * 4 + r][j];
                        const f32x2 y = (f32x2){__uint_as_float(w << 16), __uint_as_float(w & 0xffff0000u)}
                                      + (f32x2){__uint_as_float(u << 16), __uint_as_float(u & 0xffff0000u)};
                        const f32x2 o = gelu2(y);
                        v[j] = cvt_pk_bf16(o.x, o.y);
                    }
                    if (bad) v = (u32x4){0x7fc07fc0u, 0x7fc07fc0u, 0x7fc07fc0u, 0x7fc07fc0u};
                    st16_sys(dst + (size_t)(n * 16 + r) * (GN * 2), v);
                }
        }
        if (tid == 0) __hip_atomic_store((unsigned*)(ws + OFF_CNT + b * 128), call, __ATOMIC_RELAXED, __HIP_MEMORY_SCOPE_AGENT);
        STAMP(5);
        return;
    }

    if (!is_b) {
        unsigned char* hdst = P.ws[0];
#pragma unroll
        for (int r = 1; r < NR; ++r) hdst = lane == r ? P.ws[r] : hdst;
        hdst += OFF_HIN + (par * 8 + self) * (768 * 16);
        for (int t = task0;;) {
#pragma unroll
            for (int i = 0; i < 8; ++i) asm volatile("" : "+v"(cur[i]));
            const int tn = t < (PK_ALL - 1) * 1152 ? t + 1152 : t >= PK_ALL * 1152 ? t + 704 : PK_ALL * 1152 + wid * 88 + (is_a ? g >> 1 : g - 14) * 8 + nt;
            const bool more = tn < 2784 && (tn < PK_ALL * 1152 || is_c || hf);
            int isw, kc, tile; pack_decode(t, isw, kc, tile); const int q = t & 3, idx = self * 8 + kc;
            if (isw) {
                u32x4 tr[8];
#pragma unroll
                for (int j = 0; j < 8; ++j)
#pragma unroll
                    for (int qd = 0; qd < 4; ++qd)
                        tr[j][qd] = perm(cur[2 * qd + 1][j >> 1], cur[2 * qd][j >> 1], (j & 1) ? 0x07060302u : 0x05040100u);
#pragma unroll
                for (int j = 0; j < 8; ++j) cur[j] = tr[j];
            }
            const bool own = (isw ? tile >> 3 : tile >> 1) == self, diag = t >= 992;
            const bool packed = !own && t < 2560, image = !isw || own || diag;
            u32x4 hd = {0u, 0u, 0u, 0u};
            if (packed) {
                Packed pk; pack(cur, pk, scr, lane);
                send(pk, ws + OFF_RP + (par * 2560 + (isw ? tile * 8 + kc : 512 + kc * 16 + tile) * 4 + q) * UNIT, scr, lane);
                hd = header(pk, call);
            }
            if (image)
                store_image(ws + (diag ? OFF_WD + (size_t)(tile * 8 + kc) * BLK : isw ? OFF_WB + (size_t)(idx * 8 + (tile & 7)) * BLK
                                                                                     : OFF_XA + (size_t)(idx * 16 + tile) * BLK), isw, q, cur);
            load_task(more ? tn : t, cur);
            asm volatile("s_waitcnt vmcnt(8)" ::: "memory");
            if (packed && lane < NR && lane != self && (!isw || lane == (tile >> 3)))
                st16_sys(hdst + ((isw ? kc * 8 + (tile & 7) : 64 + kc * 16 + tile) * 4 + q) * 16, hd);
            if (image && lane == 0)
                st_flag8(ws + (diag ? OFF_FD + (tile * 8 + kc) * 4 : isw ? OFF_FW + ((tile & 7) * 64 + idx) * 4 : OFF_FX + (tile * 64 + idx) * 4) + q, call);
            __builtin_amdgcn_s_setprio(0);
            if (!more) break;
            t = tn;
        }
        STAMP(6);
    }

    const unsigned voff = tid * 16, ldsw = wid * 1024, foff = lane * 4;
    const unsigned pickoff = (unsigned)(size_t)(lds + 9 * HTB);
    const unsigned ldsb = (unsigned)(size_t)lds + ldsw;
#define GLDS(bufoff, gbase) asm volatile("s_add_i32 m0, %0, %1\n\ts_nop 0\n\tglobal_load_lds_dwordx4 %2, %3 sc0 sc1" \
                                         :: "s"(ldsb), "i"(bufoff), "v"(voff), "s"(gbase) : "memory", "scc")
#define STAGE(bufoff, gbase) do { GLDS(bufoff, gbase); GLDS((bufoff) + 8192, (gbase) + 8192); } while (0)
#define LIKELY(c) __builtin_expect(!!(c), 1)
#define RARE(c) __builtin_expect(!!(c), 0)
#define LDA(dst, base, off) do { _Pragma("unroll") for (int m = 0; m < 4; ++m) _Pragma("unroll") for (int k = 0; k < 2; ++k) \
        dst[m][k] = *(const LAS bf16x8*)(lds + (base) + ((off) + m * 2048 + k * 1024)); } while (0)
#define LDB(dst, base, off) do { _Pragma("unroll") for (int n = 0; n < 2; ++n) _Pragma("unroll") for (int k = 0; k < 2; ++k) \
        dst[n][k] = *(const LAS bf16x8*)(lds + (base) + ((off) + n * 2048 + k * 1024)); } while (0)
#define MMA(ai, bj, At, Bt) do { __builtin_amdgcn_s_setprio(1); _Pragma("unroll") for (int m = 0; m < 4; ++m) _Pragma("unroll") for (int n = 0; n < 2; ++n) _Pragma("unroll") for (int k = 0; k < 2; ++k) \
        acc[ai][bj][m][n] = __builtin_amdgcn_mfma_f32_16x16x32_bf16(Bt[n][k], At[m][k], acc[ai][bj][m][n], 0, 0, 0); __builtin_amdgcn_s_setprio(0); } while (0)
#define WAIT_V(n) asm volatile("s_waitcnt vmcnt(" #n ")" ::: "memory")
#define WAIT_L0 do { asm volatile("" ::: "memory"); __builtin_amdgcn_s_waitcnt(0xC07F); asm volatile("" ::: "memory"); } while (0)
#define BAR __builtin_amdgcn_s_barrier()
#define SCHED __builtin_amdgcn_sched_barrier(0)
    typedef unsigned u32x2 __attribute__((ext_vector_type(2)));
#define SNAP() do { if (RARE(fresh > wid)) asm volatile("global_load_dword %0, %2, %3 sc0 sc1\n\tglobal_load_dword %1, %2, %4 sc0 sc1" \
                                                        : "+v"(sna), "+v"(snb) : "v"(foff), "s"(fa), "s"(fb) : "memory"); } while (0)
#define PUBLISH(w, xm) do { if (RARE(fresh > wid)) { asm volatile("" : "+v"(sna), "+v"(snb)); \
            u64 m = __ballot(sna == tag4 && ((snb ^ tag4) & (xm)) == 0u); \
            if (vm == 0 && ((m | avail) & todo) == 0ull) m = poll_ready(todo, fa, fb, tmo, call, tag4, xm); \
            asm volatile("ds_write_b64 %0, %1 offset:%2" :: "v"(pickoff), "v"((u32x2){(unsigned)m, (unsigned)(m >> 32)}), "i"(8 * (w)) : "memory"); } } while (0)
#define LISTEN(w) do { if (RARE(fresh)) asm volatile("ds_read_b64 %0, %1 offset:%2" : "+v"(mv) : "v"(pickoff), "i"(8 * (w)) : "memory"); } while (0)
#define TAKE(i) do { if (RARE(fresh)) { asm volatile("s_waitcnt lgkmcnt(0)" : "+v"(mv) :: "memory"); \
            avail |= todo & (((u64)(unsigned)__builtin_amdgcn_readfirstlane(mv.y) << 32) | (unsigned)__builtin_amdgcn_readfirstlane(mv.x)); } \
        u64 bit_;   \
        asm volatile("s_ff1_i32_b64 %0, %3\n\ts_lshl_b64 %4, 1, %0\n\ts_and_b64 %4, %4, %3\n\ts_andn2_b64 %2, %2, %4\n\ts_andn2_b64 %3, %3, %4\n\t" \
                     "s_min_u32 %0, %0, 64\n\ts_bcnt1_i32_b64 %1, %3\n\ts_cmp_lt_u32 %1, %5\n\ts_cselect_b32 %1, 1, 0\n\t" \
                     "s_cmp_lg_u64 %2, %3\n\ts_cselect_b32 %1, %1, 0" \
                     : "=&s"(i), "=&s"(fresh), "+s"(todo), "+s"(avail), "=&s"(bit_) : "i"(SNAP_AT) : "scc"); } while (0)
#define KSTAMP() do {} while (0)
    f32x4 acc[2][2][4][2];
    auto zero_acc = [&]() {
#pragma unroll
        for (int a = 0; a < 2; ++a)
#pragma unroll
            for (int c = 0; c < 2; ++c)
#pragma unroll
                for (int m = 0; m < 4; ++m)
#pragma unroll
                    for (int n = 0; n < 2; ++n) acc[a][c][m][n] = (f32x4){0.f, 0.f, 0.f, 0.f};
    };
    auto to_bf16 = [&](int c, u32x4 (&v)[8]) {
#pragma unroll
        for (int n = 0; n < 2; ++n)
#pragma unroll
            for (int r = 0; r < 4; ++r) {
                u32x4& o = v[n * 4 + r];
                o.x = cvt_pk_bf16(acc[0][c][0][n][r], acc[0][c][1][n][r]); o.y = cvt_pk_bf16(acc[0][c][2][n][r], acc[0][c][3][n][r]);
                o.z = cvt_pk_bf16(acc[1][c][0][n][r], acc[1][c][1][n][r]); o.w = cvt_pk_bf16(acc[1][c][2][n][r], acc[1][c][3][n][r]);
            }
    };
    auto emit_unit = [&](int c, int bj) {
        const int lz = lane_id();
        const unsigned char* const started = ws + OFF_FW + nt * 256 + owner * 32;
        unsigned og, ot;
        asm volatile("global_load_dword %0, %2, off sc0 sc1\n\tglobal_load_dword %1, %3, off sc1" : "=&v"(og), "=&v"(ot) : "v"(started), "v"(tmo) : "memory");
        u32x4 v[8];
        to_bf16(c, v);
        Packed pk;
        if (wid < 4) __builtin_amdgcn_s_setprio(2);
        pack(v, pk, scr, lz);
        __builtin_amdgcn_s_setprio(0);
        asm volatile("s_waitcnt vmcnt(0)" : "+v"(og), "+v"(ot) :: "memory");
        bool bad = (unsigned)__builtin_amdgcn_readfirstlane(ot) == call;
        if ((unsigned)__builtin_amdgcn_readfirstlane(og) != tag4) bad = wait_word<true>(started, tag4, tmo, call) || bad;
        if (bad) {
#pragma unroll
            for (int i = 0; i < 4; ++i) pk.sm[i] = (u32x4){0x40404040u, 0x40404040u, 0x40404040u, 0x40404040u};
            pk.l1 = pk.st = (u32x4){0u, 0u, 0u, 0u}; pk.mask = 0ull; pk.meta = 0x101u;
        }
        STAMP(4);
        send(pk, ws + OFF_PP + (size_t)(((mt * 8 + nt) * 8 + wid) * 2 + bj) * UNIT, scr, lz);
        asm volatile("s_waitcnt vmcnt(0)" ::: "memory");
        if (lz == 0) st16_sys(P.ws[owner] + OFF_HP + (size_t)((((mt & 1) * 64 + self * 8 + nt) * 8 + wid) * 2 + bj) * 16, header(pk, call));
    };
    auto finish = [&]() {
        if (threadIdx.x == 0) __hip_atomic_store((unsigned*)(ws + OFF_CNT + b * 128), call, __ATOMIC_RELAXED, __HIP_MEMORY_SCOPE_AGENT);
        STAMP(5);
    };
    auto park = [&](int c, int pt, int bj) {
        unsigned char* dp = ws + OFF_PD + (size_t)((pt * 8 + wid) * 2 + bj) * 8192 + lane_id() * 16;
        u32x4 v[8];
        to_bf16(c, v);
#pragma unroll
        for (int i = 0; i < 8; ++i) st16_sys(dp + i * 1024, v[i]);
    };
    STAMP(2);
    if (!is_a) {
      const int nsteps = is_b ? 1 : g < 24 ? 3 : 2;
#pragma unroll 1
      for (int step = 0;; ++step) {
        const bool diag = step < nsteps - 1;
        const int link = (g - 21) * 2 + step, dg = ((self + 1 + link) & 7) * 8 + nt, jm = diag ? 2 * self : mt;
        const int nkt = diag ? 8 : is_b ? 56 : 32;
        u64 todo = diag ? 0xffull << (self * 8) : is_b ? ~(0xffull << (owner * 8)) : 0x5555555555555555ull << hf, avail = todo & READY0;
        const unsigned char* const abase = diag ? ws + OFF_WD + (size_t)(dg * 8 - self * 8) * BLK : ws + OFF_WB + (size_t)nt * BLK;
        const int ashift = diag ? 15 : 18;
        const unsigned char* const fa = diag ? ws + OFF_FD + (dg * 8 - self * 8) * 4 : ws + OFF_FW + nt * 256;
        const int x0 = (is_c && !diag && hf) ? HTB : 0;
        const unsigned char* const bbase = ws + OFF_XA + (size_t)jm * BLK;
        const unsigned char* const fb = ws + OFF_FX + jm * 256;
        const int aoff = wr * 8192 + lane * 16;
        int boff = 4 * HTB + wc * 4096 + lane * 16;
        asm volatile("" : "+v"(boff));
        zero_acc();
        bf16x8 At[4][2], B0[2][2], B1[2][2];
        const unsigned char* a1 = abase;
        unsigned sna = ~tag4, snb = ~tag4; u32x2 mv = {0u, 0u}; int fresh = 1;
        int vm = 0; const bool stampit = !diag; (void)stampit;
        int left = __builtin_amdgcn_readfirstlane(nkt);
        __syncthreads();
        if (wr == 1) BAR;
#define SA(bf, h) (((bf) * 2 + (h)) * HTB)
#define SB(bf, h) ((4 + (bf) * 2 + (h)) * HTB)
#define KTILE(bf) do { \
        LDB(B0, boff, SA(bf, 0)); LDB(B1, boff, SA(bf, 1)); LDA(At, aoff, SA(bf, 0)); SCHED; \
        if (LIKELY(vm & 2)) { STAGE(SA(1 - bf, 1), a1 + HTB); WAIT_V(8); } else WAIT_V(0); \
        PUBLISH(bf, 0xffffffffu); \
        WAIT_L0; BAR; SCHED; LISTEN(bf); SCHED; \
        if (LIKELY(vm & 1)) { MMA(0, 0, At, B0); MMA(0, 1, At, B1); } \
        BAR; SCHED; \
        int i2; TAKE(i2); SCHED; \
        LDA(At, aoff, SA(bf, 1)); SCHED; \
        SNAP(); \
        const unsigned char* a2 = abase + ((size_t)i2 << ashift); const unsigned char* b2 = bbase + ((size_t)i2 << 19); \
        if (LIKELY(i2 < 64)) { STAGE(SB(bf, 0), b2 + x0); STAGE(SB(bf, 1), b2 + (HTB - x0)); STAGE(SA(bf, 0), a2); WAIT_V(8); vm |= 4; } \
        else WAIT_V(2); \
        WAIT_L0; BAR; SCHED; if (LIKELY(vm & 1)) { MMA(1, 0, At, B0); MMA(1, 1, At, B1); } BAR; SCHED; \
        left -= vm & 1; KSTAMP(); vm >>= 1; a1 = a2; } while (0)
        for (;;) {
            KTILE(0);
            if (left == 0) break;
            KTILE(1);
            if (left == 0) break;
        }
        asm volatile("s_waitcnt vmcnt(0) lgkmcnt(0)" : "+v"(sna), "+v"(snb), "+v"(mv) :: "memory");
        if (wr == 0) BAR;
        if (!diag) break;
        park(0, link * 16 + nt, 0); park(1, link * 16 + nt, 1);
        asm volatile("s_waitcnt vmcnt(0)" ::: "memory");
        if (lane_id() < 2) st_flag(ws + OFF_FP + (((link * 16 + nt) * 8 + wid) * 2 + lane_id()) * 4, call);
        STAMP(7);
      }
      {
        if (is_b) {
            STAMP(3);
            emit_unit(0, 0); emit_unit(1, 1);
        } else {
            STAMP(3);
            const int lz = lane_id(), pt = 112 + (mt & 1) * 8 + nt;
            park(1, pt, 1 - hf);
            asm volatile("s_waitcnt vmcnt(0)" ::: "memory");
            if (lz == 0) st_flag(ws + OFF_FP + ((pt * 8 + wid) * 2 + 1 - hf) * 4, call);
            wait_word<false>(ws + OFF_FP + ((pt * 8 + wid) * 2 + hf) * 4, call, tmo, call);
            u32x4 dg[8]; unsigned tw = 0u;
            load_parked(dg, tw, ws + OFF_PD + (size_t)((pt * 8 + wid) * 2 + hf) * 8192, tmo, lz);
            PARKED_WAIT(dg, tw);
            const bool bad = (unsigned)__builtin_amdgcn_readfirstlane(tw) == call;
            unsigned char* dst = P.out + (size_t)((mt & 1) * 256 + hf * 128 + wc * 32 + (lz >> 4) * 4) * (GN * 2) + (size_t)((self * 8 + nt) * 256 + wr * 128 + (lz & 15) * 8) * 2;
#pragma unroll
            for (int n = 0; n < 2; ++n)
#pragma unroll
                for (int r = 0; r < 4; ++r) {
                    u32x4 v;
#pragma unroll
                    for (int j = 0; j < 4; ++j) {
                        const unsigned w = dg[n * 4 + r][j];
                        const f32x2 o = gelu2((f32x2){acc[j >> 1][0][(j & 1) * 2][n][r], acc[j >> 1][0][(j & 1) * 2 + 1][n][r]}
                                              + (f32x2){__uint_as_float(w << 16), __uint_as_float(w & 0xffff0000u)});
                        v[j] = cvt_pk_bf16(o.x, o.y);
                    }
                    if (bad) v = (u32x4){0x7fc07fc0u, 0x7fc07fc0u, 0x7fc07fc0u, 0x7fc07fc0u};
                    st16_sys(dst + (size_t)(n * 16 + r) * (GN * 2), v);
                }
        }
      }
    } else {
#pragma unroll 1
      for (int step = 0; step < 2; ++step) {
        const bool diag = step == 1;
        const int dg = owner * 8 + nt, jm = diag ? 2 * self + 1 : mt;
        const int nkt = diag ? 8 : 56;
        u64 todo = diag ? 0xffull << (self * 8) : ~(0xffull << (owner * 8)), avail = todo & READY0;
        const unsigned char* const abase = diag ? ws + OFF_WD + (size_t)(dg * 8 - self * 8) * BLK : ws + OFF_WB + (size_t)nt * BLK;
        const int ashift = diag ? 15 : 18;
        const unsigned char* const fa = diag ? ws + OFF_FD + (dg * 8 - self * 8) * 4 : ws + OFF_FW + nt * 256;
        const unsigned char* const bbase = ws + OFF_XA + (size_t)jm * BLK + hf * HTB;
        const unsigned char* const fb = ws + OFF_FX + jm * 256;
        const unsigned xm = hf ? 0xffff0000u : 0x0000ffffu;
        int aoff[3], boff[3];
#pragma unroll
        for (int i = 0; i < 3; ++i) {
            aoff[i] = i * 3 * HTB + wr * 8192 + lane * 16; boff[i] = i * 3 * HTB + 2 * HTB + wc * 4096 + lane * 16;
            asm volatile("" : "+v"(aoff[i]), "+v"(boff[i]));
        }
        zero_acc();
        bf16x8 A0[4][2], A1[4][2], B0[2][2];
        unsigned sna = ~tag4, snb = ~tag4; u32x2 mv = {0u, 0u}; int fresh = 1;
        int vm = 0; const bool stampit = !diag; (void)stampit;
        int left = __builtin_amdgcn_readfirstlane(nkt);
        int heard = 0;
        __syncthreads();
        if (wr == 1) BAR;
#define HTILE(s) do { \
        constexpr int nb = ((s) + 2) % 3 * 3 * HTB; \
        const int pub = fresh; \
        fresh = heard; int i2; TAKE(i2); \
        if (i2 < 64) vm |= 4; \
        { const int want = fresh; fresh = pub; PUBLISH(s, xm); fresh = want; } \
        SNAP(); \
        if (LIKELY(vm & 4)) { \
            const unsigned char* a2 = abase + ((size_t)i2 << ashift); const unsigned char* b2 = bbase + ((size_t)i2 << 19); \
            GLDS(nb + 2 * HTB, b2); GLDS(nb + 2 * HTB + 8192, b2 + 8192); GLDS(nb, a2); \
            GLDS(nb + 8192, a2 + 8192); GLDS(nb + 16384, a2 + 16384); GLDS(nb + 24576, a2 + 24576); } \
        SCHED; LDB(B0, boff[s], 0); LDA(A0, aoff[s], 0); LDA(A1, aoff[s], HTB); SCHED; \
        if (wr == 1) { if (LIKELY(vm & 4)) WAIT_V(6); else WAIT_V(0); } \
        WAIT_L0; BAR; SCHED; \
        if (RARE(pub)) asm volatile("ds_read_b64 %0, %1 offset:%2" : "+v"(mv) : "v"(pickoff), "i"(8 * (s)) : "memory"); \
        SCHED; heard = pub; \
        if (LIKELY(vm & 1)) { MMA(0, 0, A0, B0); MMA(1, 0, A1, B0); } \
        if (wr == 0) { if (LIKELY(vm & 4)) WAIT_V(6); else WAIT_V(0); } \
        BAR; SCHED; \
        left -= vm & 1; KSTAMP(); vm >>= 1; } while (0)
        for (;;) {
            HTILE(0);
            if (left == 0) break;
            HTILE(1);
            if (left == 0) break;
            HTILE(2);
            if (left == 0) break;
        }
        asm volatile("s_waitcnt vmcnt(0) lgkmcnt(0)" : "+v"(sna), "+v"(snb), "+v"(mv) :: "memory");
        if (wr == 0) BAR;
        if (diag) {
            const int pt = (g | 1) * 8 + nt;
            park(0, pt, hf);
            asm volatile("s_waitcnt vmcnt(0)" ::: "memory");
            if (lane_id() == 0) st_flag(ws + OFF_FP + ((pt * 8 + wid) * 2 + hf) * 4, call);
            STAMP(7);
        } else {
            STAMP(3);
            emit_unit(0, hf);
            STAMP(8);
        }
      }
    }
    finish();
}

}

extern "C" void kernel_launch(int rank, int nranks, ncclComm_t comm, hipStream_t stream,
                              void* const* d_in, const int* in_sizes, int n_in,
                              void* d_out, int out_size, const int* in_dtypes, int out_dtype,
                              void* const* d_out_all, void* const* d_ws_all, size_t ws_bytes) {
    (void)comm; (void)in_dtypes; (void)out_dtype; (void)d_out_all;
    if (nranks != NR || (unsigned)rank >= (unsigned)NR || n_in < 2 || ws_bytes < WS_NEED
        || in_sizes[0] != GM * KS || in_sizes[1] != KS * GN || out_size != (GM / NR) * GN) {
        fprintf(stderr, "kernel_launch: unexpected problem shape\n");
        return;
    }
    Params p;
    p.x = (const unsigned char*)d_in[0];
    p.w = (const unsigned char*)d_in[1];
    p.out = (unsigned char*)d_out;
    for (int i = 0; i < NR; ++i) p.ws[i] = (unsigned char*)d_ws_all[i];
    p.rank = rank; p.pad = 0;
    hipLaunchKernelGGL(fused_gemm_rs, dim3(256), dim3(512), 0, stream, p);
    HIP_CHECK(hipGetLastError());
}
```

```cpp
#include <hip/hip_runtime.h>
#include <rccl/rccl.h>
#include <cstdio>
#include <cstdint>

#define HIP_CHECK(expr) do { hipError_t _e = (expr); if (_e != hipSuccess) { \
    fprintf(stderr, "HIP error %s at %s:%d\n", hipGetErrorString(_e), __FILE__, __LINE__); \
    return; } } while (0)

namespace {

#define LAS __attribute__((address_space(3)))
#define GAS __attribute__((address_space(1)))
typedef short bf16x8 __attribute__((ext_vector_type(8)));
typedef float f32x4 __attribute__((ext_vector_type(4)));
typedef unsigned u32x4 __attribute__((ext_vector_type(4)));
typedef unsigned long long u64;

constexpr int NR = 8;
constexpr int GM = 4096, GK = 4096, GN = 16384;
constexpr int KS = GK / NR;
constexpr int BLK = 32768;
constexpr int HTB = 16384;
#define PWV 7
#define WIN 16
#define PULL_DELAY 100
#define HEAD_W 2
#define PACK_DELAY 150
#define PK_ALL 2
#define NTX 1
#define SNAP_AT 2
#define ACQ_SLEEP 1
#define GLF 1
#define GLH 0
#define C_ODD 1
#define PLAIN 3
#define HS 128
#define HDR_SLEEP 1
#define MID_Q 2
constexpr int NPULL = 56, PW = 8 * PWV;
constexpr u64 DEADLINE = 50000000ull;
constexpr int NTASK = 704;
constexpr int UNIT = 9344;

constexpr size_t OFF_CNT = 0;
constexpr size_t OFF_TMO = 32768;
constexpr size_t OFF_TICK = 33792;
constexpr size_t OFF_FX = 65536;
constexpr size_t OFF_FW = 81920;
constexpr size_t OFF_FD = 98304;
constexpr size_t OFF_FP = 106496;
constexpr size_t OFF_STAMP = 524288;
constexpr size_t OFF_XA = 1u << 20;
constexpr size_t OFF_WB = OFF_XA + (size_t)64 * 16 * BLK;
constexpr size_t OFF_WD = OFF_WB + (size_t)64 * 8 * BLK;
constexpr size_t OFF_RP = OFF_WD + (size_t)64 * 8 * BLK;
constexpr size_t OFF_PP = OFF_RP + (size_t)5120 * UNIT;
constexpr size_t OFF_PD = OFF_PP + (size_t)2048 * UNIT;
constexpr size_t OFF_HIN = OFF_PD + (size_t)128 * 131072;
constexpr size_t OFF_HP = OFF_HIN + (size_t)16 * 768 * HS;
constexpr size_t WS_NEED = OFF_HP + (size_t)2048 * HS + 4096;
static_assert(WS_NEED <= (size_t)161 * 1048576, "workspace");

struct Params {
    const unsigned char* x;
    const unsigned char* w;
    unsigned char* out;
    unsigned char* own;
    unsigned char* ws[NR];
    int rank, pad;
};

__device__ __forceinline__ void st16_sys(unsigned char* p, u32x4 v) {
    asm volatile("global_store_dwordx4 %0, %1, off sc0 sc1\n\ts_nop 1" :: "v"(p), "v"(v) : "memory");
}
__device__ __forceinline__ void st_flag(unsigned char* p, unsigned v) {
    __hip_atomic_store((unsigned*)p, v, __ATOMIC_RELAXED, __HIP_MEMORY_SCOPE_SYSTEM);
}
__device__ __forceinline__ void st_flag8(unsigned char* p, unsigned v) {
    __hip_atomic_store(p, (unsigned char)v, __ATOMIC_RELAXED, __HIP_MEMORY_SCOPE_SYSTEM);
}
__device__ __forceinline__ unsigned ld_flag(const unsigned char* p) {
    return __hip_atomic_load((const unsigned*)p, __ATOMIC_RELAXED, __HIP_MEMORY_SCOPE_SYSTEM);
}
__device__ __forceinline__ unsigned lane_id() {
    return __builtin_amdgcn_mbcnt_hi(~0u, __builtin_amdgcn_mbcnt_lo(~0u, 0u));
}
__device__ __forceinline__ unsigned cvt_pk_bf16(float lo, float hi) {
    unsigned r; asm volatile("v_cvt_pk_bf16_f32 %0, %1, %2" : "=v"(r) : "v"(lo), "v"(hi)); return r;
}
typedef float f32x2 __attribute__((ext_vector_type(2)));
__device__ __forceinline__ f32x2 gelu2(f32x2 y) {
    const float k1 = -2.0f * 0.7978845608028654f * 1.4426950408889634f;
    const float k3 = k1 * 0.044715f;
    const f32x2 z = y * ((y * y) * k3 + k1);
    const f32x2 d = (f32x2){__builtin_amdgcn_exp2f(z.x), __builtin_amdgcn_exp2f(z.y)} + 1.0f;
    return y * (f32x2){__builtin_amdgcn_rcpf(d.x), __builtin_amdgcn_rcpf(d.y)};
}

template <bool EQ>
__device__ __forceinline__ bool wait_word(const unsigned char* p, unsigned want, unsigned char* tmo, unsigned call) {
    const u64 t0 = __builtin_amdgcn_s_memrealtime();
    for (;;) {
        unsigned g, t;
        asm volatile("global_load_dword %0, %2, off sc0 sc1\n\t"
                     "global_load_dword %1, %3, off sc1\n\t"
                     "s_waitcnt vmcnt(0)" : "=&v"(g), "=&v"(t) : "v"(p), "v"(tmo) : "memory");
        if ((unsigned)__builtin_amdgcn_readfirstlane(t) == call || __builtin_amdgcn_s_memrealtime() - t0 > DEADLINE) {
            __hip_atomic_store((unsigned*)tmo, call, __ATOMIC_RELAXED, __HIP_MEMORY_SCOPE_AGENT);
            return true;
        }
        const unsigned v = __builtin_amdgcn_readfirstlane(g);
        if (EQ ? v == want : v >= want) return false;
        __builtin_amdgcn_s_sleep(2);
    }
}

__device__ __forceinline__ u64 now() {
    u64 t; asm volatile("s_memrealtime %0\n\ts_waitcnt lgkmcnt(0)" : "=s"(t) :: "memory"); return t;
}
__device__ __forceinline__ u64 poll_ready(u64 todo, const unsigned char* fa, const unsigned char* fb,
                                          unsigned char* tmo, unsigned call, unsigned tag4, unsigned xm) {
    const u64 t0 = now();
    for (;;) {
        unsigned a, b, t; const unsigned off = lane_id() * 4u, zero = 0u;
        asm volatile("global_load_dword %0, %3, %5 sc0 sc1\n\t"
                     "global_load_dword %1, %3, %6 sc0 sc1\n\t"
                     "global_load_dword %2, %4, %7 sc1\n\t"
                     "s_waitcnt vmcnt(0)"
                     : "=&v"(a), "=&v"(b), "=&v"(t) : "v"(off), "v"(zero), "s"(fa), "s"(fb), "s"(tmo) : "memory");
        const u64 ready = __ballot(a == tag4 && ((b ^ tag4) & xm) == 0u);
        if ((ready & todo) != 0ull) return ready;
        if ((unsigned)__builtin_amdgcn_readfirstlane(t) == call || now() - t0 > DEADLINE) {
            __hip_atomic_store((unsigned*)tmo, call, __ATOMIC_RELAXED, __HIP_MEMORY_SCOPE_AGENT);
            return ~0ull;
        }
        __builtin_amdgcn_s_sleep(1);
    }
}

__device__ __forceinline__ unsigned bfi(unsigned m, unsigned a, unsigned b) { return (a & m) | (b & ~m); }
__device__ __forceinline__ unsigned perm(unsigned hi, unsigned lo, unsigned sel) { return __builtin_amdgcn_perm(hi, lo, sel); }
#define SDWA2(k) " dst_sel:DWORD dst_unused:UNUSED_PAD src0_sel:BYTE_" #k " src1_sel:BYTE_" #k "\n\t"
#define SDWA1(k) " dst_sel:DWORD dst_unused:UNUSED_PAD src0_sel:DWORD src1_sel:BYTE_" #k "\n\t"
__device__ __forceinline__ unsigned spread_bits(unsigned f, unsigned s) {
    unsigned r, a, b;
    asm("v_lshlrev_b32_sdwa %0, %4, %3" SDWA2(0) "v_lshlrev_b32_sdwa %1, %4, %3" SDWA2(1) "v_lshlrev_b32_sdwa %2, %4, %3" SDWA2(2)
        "v_or3_b32 %0, %0, %1, %2\n\t"
        "v_lshlrev_b32_sdwa %1, %4, %3" SDWA2(3)
        "v_or_b32 %0, %0, %1" : "=&v"(r), "=&v"(a), "=&v"(b) : "v"(f), "v"(s));
    return r;
}
__device__ __forceinline__ unsigned take_codes(unsigned& w, unsigned f) {
    unsigned ln, t, l;
    asm("v_xor_b32_sdwa %2, %0, %4" SDWA1(0) "v_ffbl_b32 %1, %2\n\tv_lshrrev_b32 %0, %1, %0\n\t"
        "v_xor_b32_sdwa %2, %0, %4" SDWA1(1) "v_ffbl_b32 %3, %2\n\tv_lshrrev_b32 %0, %3, %0\n\t"
        "v_xor_b32_sdwa %2, %0, %4" SDWA1(2) "v_lshl_or_b32 %1, %3, 8, %1\n\tv_ffbl_b32 %3, %2\n\tv_lshrrev_b32 %0, %3, %0\n\t"
        "v_xor_b32_sdwa %2, %0, %4" SDWA1(3) "v_lshl_or_b32 %1, %3, 16, %1\n\tv_ffbl_b32 %3, %2\n\tv_lshrrev_b32 %0, %3, %0\n\t"
        "v_lshl_or_b32 %1, %3, 24, %1" : "+v"(w), "=&v"(ln), "=&v"(t), "=&v"(l) : "v"(f));
    return ln;
}
__device__ __forceinline__ unsigned wave_scan(unsigned v) {
    v += (unsigned)__builtin_amdgcn_update_dpp(0, (int)v, 0x111, 0xf, 0xf, false);
    v += (unsigned)__builtin_amdgcn_update_dpp(0, (int)v, 0x112, 0xf, 0xf, false);
    v += (unsigned)__builtin_amdgcn_update_dpp(0, (int)v, 0x114, 0xf, 0xf, false);
    v += (unsigned)__builtin_amdgcn_update_dpp(0, (int)v, 0x118, 0xf, 0xf, false);
    v += (unsigned)__builtin_amdgcn_update_dpp(0, (int)v, 0x142, 0xa, 0xf, false);
    v += (unsigned)__builtin_amdgcn_update_dpp(0, (int)v, 0x143, 0xc, 0xf, false);
    return v;
}
__device__ __forceinline__ unsigned rank_in(u64 mask) {
    return __builtin_amdgcn_mbcnt_hi((unsigned)(mask >> 32), __builtin_amdgcn_mbcnt_lo((unsigned)mask, 0u));
}
constexpr int SCR = 8192;
#define LDS_ORDER() asm volatile("" ::: "memory")

struct Packed { u32x4 sm[4], l1, st; u64 mask; unsigned meta; };

__device__ __forceinline__ void pack(const u32x4 (&d)[8], Packed& o, LAS unsigned char* scr, unsigned lane) {
    *(LAS u32x4*)(scr + lane * 16) = (u32x4){0u, 0u, 0u, 0u};
    unsigned sum = 0; u32x4 er[4];
#pragma unroll
    for (int g = 0; g < 16; ++g) {
        const unsigned d0 = d[g >> 1][(g & 1) * 2], d1 = d[g >> 1][(g & 1) * 2 + 1];
        const unsigned lb = perm(d1, d0, 0x06040200u), hb = perm(d1, d0, 0x07050301u);
        o.sm[g >> 2][g & 3] = bfi(0x7f7f7f7fu, lb, hb);
        const unsigned e = bfi(0x01010101u, lb >> 7, hb << 1);
        er[g >> 2][g & 3] = e;
        sum = __builtin_amdgcn_sad_u8(e, 0u, sum);
    }
    const unsigned r2 = (((unsigned)__builtin_amdgcn_readlane(wave_scan(sum), 63) + 16036u) >> 12) & 255u;
    const unsigned kk = (r2 | 0x80u) * 0x01010101u, cc = (~r2 & 0x80u) * 0x01010101u;
    unsigned bad = 0, top = ~0u, longest = 0, total = 0, piece[8], plen[8], us[16], cs[16];
#pragma unroll
    for (int g = 0; g < 16; ++g) {
        const unsigned e = er[g >> 2][g & 3];
        const unsigned u = (kk - (e & 0x7f7f7f7fu)) ^ (e & 0x80808080u) ^ cc;
        bad |= u;
        us[g] = u; cs[g] = perm(0x07070706u, 0x05040703u, u | ((u << 1) & 0x10101010u));
        top &= cs[g];
    }
    const bool hole = __ballot((top & 0x04040404u) != 0x04040404u) != 0ull;
    const unsigned tx = hole ? 0xfdfdfdfdu : 0xfefefefeu, thi = 0x01000000u ^ tx, tlo = 0x01020609u ^ tx;
#pragma unroll
    for (int q = 0; q < 4; ++q) {
        unsigned l1 = 0, acc = 0, base = 0;
#pragma unroll
        for (int gg = 0; gg < 4; ++gg) {
            const unsigned u = us[q * 4 + gg], c = cs[q * 4 + gg];
            l1 |= (c & 0x03030303u) << (2 * gg);
            const unsigned len4 = (perm(thi, tlo, u | (c & 0x10101010u)) ^ tx) + u;
            const unsigned f = c & (c >> 1) & 0x01010101u;
            unsigned ps = len4 + (len4 << 8); asm("" : "+v"(ps)); ps += ps << 16; ps -= 0x100c0804u;
            const unsigned bits = spread_bits(f, ps);
            if ((gg & 1) == 0) { acc = bits; base = ps >> 24; }
            else {
                acc |= bits << (base & 31u); base += ps >> 24;
                piece[q * 2 + (gg >> 1)] = acc >> 1; plen[q * 2 + (gg >> 1)] = base;
                longest = max(longest, base); total += base;
            }
        }
        o.l1[q] = l1;
    }
    const bool raw = (bad & 0xe0e0e0e0u) != 0u || longest > 31u || total > 112u;
    o.mask = __ballot(raw);
    if (raw) total = 0;
    const unsigned incl = wave_scan(total);
    const unsigned rawoff = (((unsigned)__builtin_amdgcn_readlane(incl, 63) + 512u + 127u) >> 7) << 4;
    o.meta = r2 | ((rawoff + 64u * (unsigned)__builtin_popcountll(o.mask) + 127u) >> 7) << 8 | (hole ? 0x4000u : 0u);
    LDS_ORDER();
    scr[lane] = (unsigned char)total;
    if (!raw) {
        unsigned off = incl - total + 512u;
#pragma unroll
        for (int i = 0; i < 8; ++i) {
            const u64 w = (u64)piece[i] << (off & 31u);
            LAS unsigned* p = (LAS unsigned*)(scr + ((off >> 5) << 2));
            __hip_atomic_fetch_or(p, (unsigned)w, __ATOMIC_RELAXED, __HIP_MEMORY_SCOPE_WORKGROUP);
            __hip_atomic_fetch_or(p + 1, (unsigned)(w >> 32), __ATOMIC_RELAXED, __HIP_MEMORY_SCOPE_WORKGROUP);
            off += plen[i];
        }
    } else {
        LAS u32x4* p = (LAS u32x4*)(scr + rawoff + rank_in(o.mask) * 64u);
#pragma unroll
        for (int i = 0; i < 4; ++i) p[i] = er[i];
    }
    LDS_ORDER();
    o.st = *(const LAS u32x4*)(scr + lane * 16);
    LDS_ORDER();
}
__device__ __forceinline__ void send(const Packed& o, unsigned char* pl, const LAS unsigned char* scr, unsigned lane) {
    unsigned char* p = pl + lane * 16;
    for (unsigned k = 1; k * 8u < ((o.meta >> 8) & 63u); ++k)
        if (k * 64u + lane < ((o.meta >> 8) & 63u) * 8u) st16_sys(p + 5120 + k * 1024, *(const LAS u32x4*)(scr + k * 1024 + lane * 16));
#pragma unroll
    for (int i = 0; i < 4; ++i) st16_sys(p + i * 1024, o.sm[i]);
    st16_sys(p + 4096, o.l1);
    if (lane < ((o.meta >> 8) & 63u) * 8u) st16_sys(p + 5120, o.st);
}
__device__ __forceinline__ u32x4 header(const Packed& o, unsigned call) {
    return (u32x4){(unsigned)o.mask, call, (unsigned)(o.mask >> 32), (call << 15) | o.meta};
}
__device__ __forceinline__ bool header_ok(u32x4 h, unsigned call, u64& mask, unsigned& meta) {
    const unsigned t1 = __builtin_amdgcn_readfirstlane(h.y), t3 = __builtin_amdgcn_readfirstlane(h.w);
    mask = ((u64)(unsigned)__builtin_amdgcn_readfirstlane(h.z) << 32) | (unsigned)__builtin_amdgcn_readfirstlane(h.x);
    meta = t3 & 0x7fffu;
    return t1 == call && (t3 >> 15) == (call & 0x1ffffu);
}
__device__ __forceinline__ void wait_header(const unsigned char* hd, unsigned char* tmo, unsigned call, u64& mask, unsigned& meta) {
    const u64 t0 = __builtin_amdgcn_s_memrealtime();
    for (;;) {
        u32x4 h; unsigned t;
        asm volatile("global_load_dwordx4 %0, %2, off sc0 sc1\n\t"
                     "global_load_dword %1, %3, off sc1\n\t"
                     "s_waitcnt vmcnt(0)" : "=&v"(h), "=&v"(t) : "v"(hd), "v"(tmo) : "memory");
        if (header_ok(h, call, mask, meta)) break;
        if ((unsigned)__builtin_amdgcn_readfirstlane(t) == call || __builtin_amdgcn_s_memrealtime() - t0 > DEADLINE) {
            __hip_atomic_store((unsigned*)tmo, call, __ATOMIC_RELAXED, __HIP_MEMORY_SCOPE_AGENT);
            mask = 0ull; meta = 0u;
            break;
        }
        __builtin_amdgcn_s_sleep(HDR_SLEEP);
    }
}
__device__ __forceinline__ void fetch(Packed& o, const unsigned char* pl, unsigned lane) {
    const unsigned n = min(max((o.meta >> 8) & 63u, 1u), 8u) * 8u;
    const u64 smask = n >= 64u ? ~0ull : (1ull << n) - 1ull;
    u64 keep;
    asm volatile("s_mov_b64 %6, exec\n\t"
                 "global_load_dwordx4 %0, %7, %8 sc0 sc1\n\t"
                 "global_load_dwordx4 %1, %7, %8 offset:1024 sc0 sc1\n\t"
                 "global_load_dwordx4 %2, %7, %8 offset:2048 sc0 sc1\n\t"
                 "global_load_dwordx4 %3, %7, %8 offset:3072 sc0 sc1\n\t"
                 "global_load_dwordx4 %4, %7, %9 sc0 sc1\n\t"
                 "s_mov_b64 exec, %10\n\t"
                 "global_load_dwordx4 %5, %7, %9 offset:1024 sc0 sc1\n\t"
                 "s_mov_b64 exec, %6"
                 : "=&v"(o.sm[0]), "=&v"(o.sm[1]), "=&v"(o.sm[2]), "=&v"(o.sm[3]), "=&v"(o.l1), "=&v"(o.st), "=&s"(keep)
                 : "v"(lane * 16u), "s"(pl), "s"(pl + 4096), "s"(smask) : "memory");
}
#define FETCH_WAIT(o, n) asm volatile("s_waitcnt vmcnt(" #n ")" : "+v"(o.sm[0]), "+v"(o.sm[1]), "+v"(o.sm[2]), "+v"(o.sm[3]), "+v"(o.l1), "+v"(o.st) :: "memory")
template <class Mid>
__device__ __forceinline__ void unpack(const Packed& o, u32x4 (&d)[8], const unsigned char* pl, LAS unsigned char* scr, unsigned lane, Mid mid) {
    *(LAS u32x4*)(scr + lane * 16) = o.st;
    if (((o.meta >> 8) & 63u) > 8u) {
        u32x4 m[4];
        asm volatile("global_load_dwordx4 %0, %4, off sc0 sc1\n\t"
                     "global_load_dwordx4 %1, %4, off offset:1024 sc0 sc1\n\t"
                     "global_load_dwordx4 %2, %4, off offset:2048 sc0 sc1\n\t"
                     "global_load_dwordx4 %3, %4, off offset:3072 sc0 sc1\n\t"
                     "s_waitcnt vmcnt(0)" : "=&v"(m[0]), "=&v"(m[1]), "=&v"(m[2]), "=&v"(m[3]) : "v"(pl + 6144 + lane * 16) : "memory");
#pragma unroll
        for (int k = 0; k < 4; ++k) *(LAS u32x4*)(scr + (k + 1) * 1024 + lane * 16) = m[k];
    }
    LDS_ORDER();
    const unsigned len = scr[lane] & 127u;
    const unsigned incl = wave_scan(len), off = incl - len + 512u;
    const LAS unsigned* p = (const LAS unsigned*)(scr + ((off >> 5) << 2));
    const unsigned w0 = p[0], w1 = p[1], w2 = p[2], w3 = p[3], w4 = p[4];
    unsigned s0 = __builtin_amdgcn_alignbit(w1, w0, off), s1 = __builtin_amdgcn_alignbit(w2, w1, off);
    unsigned s2 = __builtin_amdgcn_alignbit(w3, w2, off), s3 = __builtin_amdgcn_alignbit(w4, w3, off);
    const bool hole = (o.meta & 0x4000u) != 0u;
    const unsigned tx = hole ? 0xf8f8f8f8u : 0xf7f7f7f7u, thi = (hole ? 0x07070008u : 0x08080808u) ^ tx, tlo = 0x0309090au ^ tx;
    const unsigned r7 = (o.meta + 8u) & 255u;
    const unsigned kk = (r7 | 0x80u) * 0x01010101u, cc = (~r7 & 0x80u) * 0x01010101u;
    u32x4 er[4];
#pragma unroll
    for (int q = 0; q < 4; ++q) {
        if (q == MID_Q) { asm volatile("" : "+v"(s0)); mid(); asm volatile("" : "+v"(s0)); }
        const unsigned l1 = o.l1[q], x = l1 & (l1 >> 1);
        unsigned w = 0, used = 0;
#pragma unroll
        for (int gg = 0; gg < 4; ++gg) {
            if ((gg & 1) == 0) { w = (s0 << 1) | 1u; used = 0; }
            const unsigned f = (x >> (2 * gg)) & 0x01010101u;
            const unsigned ln = take_codes(w, f);
            used = __builtin_amdgcn_sad_u8(ln, 0u, used);
            const unsigned y = (perm(thi, tlo, ln | ((ln << 1) & 0x10101010u)) ^ tx) + ln + ((l1 >> (2 * gg)) & 0x03030303u);
            er[q][gg] = (kk - y) ^ cc;
            if (gg & 1) {
                s0 = __builtin_amdgcn_alignbit(s1, s0, used); s1 = __builtin_amdgcn_alignbit(s2, s1, used);
                s2 = __builtin_amdgcn_alignbit(s3, s2, used); s3 >>= used & 31u;
            }
        }
    }
    if (o.mask != 0ull) {
        const unsigned rawoff = (((unsigned)__builtin_amdgcn_readlane(incl, 63) + 512u + 127u) >> 7) << 4;
        if ((o.mask >> lane) & 1ull) {
            const LAS u32x4* r = (const LAS u32x4*)(scr + rawoff + rank_in(o.mask) * 64u);
#pragma unroll
            for (int i = 0; i < 4; ++i) er[i] = r[i];
        }
    }
    LDS_ORDER();
#pragma unroll
    for (int g = 0; g < 16; ++g) {
        const unsigned e = er[g >> 2][g & 3], sm = o.sm[g >> 2][g & 3];
        const unsigned hb = bfi(0x80808080u, sm, e >> 1), lb = bfi(0x7f7f7f7fu, sm, e << 7);
        d[g >> 1][(g & 1) * 2] = perm(hb, lb, 0x05010400u); d[g >> 1][(g & 1) * 2 + 1] = perm(hb, lb, 0x07030602u);
    }
}

__device__ __forceinline__ void load_parked(u32x4 (&dg)[8], unsigned& tw, const unsigned char* dp, const unsigned char* tmo, unsigned lane) {
    asm volatile("global_load_dwordx4 %0, %9, %10 sc0 sc1\n\t"
                 "global_load_dwordx4 %1, %9, %10 offset:1024 sc0 sc1\n\t"
                 "global_load_dwordx4 %2, %9, %10 offset:2048 sc0 sc1\n\t"
                 "global_load_dwordx4 %3, %9, %10 offset:3072 sc0 sc1\n\t"
                 "global_load_dwordx4 %4, %9, %11 sc0 sc1\n\t"
                 "global_load_dwordx4 %5, %9, %11 offset:1024 sc0 sc1\n\t"
                 "global_load_dwordx4 %6, %9, %11 offset:2048 sc0 sc1\n\t"
                 "global_load_dwordx4 %7, %9, %11 offset:3072 sc0 sc1\n\t"
                 "global_load_dword %8, %8, %12 sc1"
                 : "=&v"(dg[0]), "=&v"(dg[1]), "=&v"(dg[2]), "=&v"(dg[3]), "=&v"(dg[4]), "=&v"(dg[5]), "=&v"(dg[6]), "=&v"(dg[7]), "+v"(tw)
                 : "v"(lane * 16u), "s"(dp), "s"(dp + 4096), "s"(tmo) : "memory");
}
#define PARKED_WAIT(dg, tw) asm volatile("s_waitcnt vmcnt(0)" : "+v"(dg[0]), "+v"(dg[1]), "+v"(dg[2]), "+v"(dg[3]), "+v"(dg[4]), "+v"(dg[5]), \
                                         "+v"(dg[6]), "+v"(dg[7]), "+v"(tw) :: "memory")

#define STAMP(i) do {} while (0)

#define ORD_A1 1.15
#define ORD_B0 0.4
#define ORD_BS 1.2
#define ORD_C0 7.0
#define ORD_CS 0.3
#define ORD_W 0.3
struct Order { unsigned short v[354]; };
constexpr Order make_order() {
    double key[352] = {}; unsigned short id[352] = {}; int n = 0;
    for (int kc = 0; kc < 8; ++kc)
        for (int h = 0; h < 2; ++h) {
            for (int e = 0; e < 8; ++e) { key[n] = kc - ORD_W + (e + 0.4) / 8 + h * 1e-3 - (kc == 0 && e < PLAIN ? 9 : 0); id[n++] = (unsigned short)(h << 8 | 0 << 6 | kc << 3 | e); }
            for (int e = 0; e < 6; ++e) { key[n] = (kc + (e + 0.5) / 6) * (h ? ORD_A1 : 1.0); id[n++] = (unsigned short)(h << 8 | 1 << 6 | kc << 3 | e); }
            for (int e = 0; e < 6; ++e) { key[n] = ORD_B0 + (kc + (e + 0.6) / 6) * ORD_BS + h * 1e-3; id[n++] = (unsigned short)(h << 8 | 2 << 6 | kc << 3 | e); }
            for (int e = 0; e < 2; ++e) { key[n] = ORD_C0 + (2 * kc + e + 0.5 * h) * ORD_CS; id[n++] = (unsigned short)(h << 8 | 3 << 6 | kc << 3 | e); }
        }
    Order o = {};
    for (int i = 0; i < 352; ++i) { int r = 0; for (int j = 0; j < 352; ++j) r += (key[j] < key[i]) || (key[j] == key[i] && j < i); o.v[r] = id[i]; }
    return o;
}
constexpr Order ORDER = make_order();
constexpr unsigned order_word(int i) { return ORDER.v[3 * i] | ORDER.v[3 * i + 1] << 10 | (unsigned)ORDER.v[3 * i + 2] << 20; }
template <int I, int N> __device__ __forceinline__ unsigned order_vgpr(unsigned v) {
    if constexpr (I == N) return v;
    else {
        unsigned t;
        asm("s_mov_b32 %1, %2\n\tv_writelane_b32 %0, %1, %3" : "+v"(v), "=&s"(t) : "i"(order_word(I)), "i"(I & 63));
        return order_vgpr<I + 1, N>(v);
    }
}
__device__ __forceinline__ void decode_unit(unsigned ord0, unsigned ord1, int t, int src, int dest, int& isw, int& kc, int& tile, int& q) {
    const int n = t >> 1, dw = n / 3;
    const unsigned w0 = __builtin_amdgcn_readlane(ord0, dw & 63), w1 = __builtin_amdgcn_readlane(ord1, dw & 63);
    const unsigned v = (dw < 64 ? w0 : w1) >> ((n - dw * 3) * 10);
    const int kind = (v >> 6) & 3, e = v & 7;
    kc = (v >> 3) & 7; isw = kind == 0; q = ((v >> 7) & 2) | (t & 1);
    const int d = (dest - src - 1) & 7;
    tile = kind == 0 ? ((e + src) & 7) : kind == 3 ? 2 * dest + e : 2 * ((src + 1 + e + (e >= d)) & 7) + (kind - 1);
}

__host__ __device__ __forceinline__ void role_of(int b, int& nt, int& slot, int& g) {
    constexpr int NG = 8 / NTX, X0 = 8 - NG;
    const int x = b & 7, s = b >> 3, np = x < X0 ? 8 : 8 - NTX;
    if (s < np) {
        const int pid = s < 8 - NTX ? s * 8 + x : (8 - NTX) * 8 + (s - (8 - NTX)) * X0 + x;
        slot = NTX == 1 ? pid >> 3 : pid % 7; nt = NTX == 1 ? pid & 7 : pid / 7; g = -1;
    } else {
        const int j = s - np, r = j / NTX + x / NG * NG * 3;
        nt = x % NG * NTX + j % NTX; slot = 0;
        g = r >= 21 ? r : r % 3 == 2 ? 14 + r / 3 : r / 3 * 2 + r % 3;
    }
}

__global__ __launch_bounds__(512, 2) void fused_gemm_rs(const Params P) {
    __shared__ __attribute__((aligned(16))) unsigned char lds_raw[9 * HTB + 256];
    LAS unsigned char* lds = (LAS unsigned char*)lds_raw;
    const int tid = threadIdx.x, wid = __builtin_amdgcn_readfirstlane(tid >> 6), lane = tid & 63;
    const int b = blockIdx.x, self = P.rank;
    unsigned char* const ws = P.own;
    const GAS unsigned char *px = (const GAS unsigned char*)P.x, *pw = (const GAS unsigned char*)P.w;
    asm volatile("" : "+s"(px), "+s"(pw));
    unsigned count; const unsigned vzero = 0u;
    asm volatile("global_load_dword %0, %1, %2 sc1" : "=&v"(count) : "v"(vzero), "s"(ws + OFF_CNT + b * 128) : "memory");
    unsigned char* const tmo = ws + OFF_TMO;
    LAS unsigned char* const scr = lds + wid * SCR;
    STAMP(0);
    const u64 tstart = __builtin_amdgcn_s_memrealtime();
    int nt, slot, g;
    role_of(b, nt, slot, g);
    const bool is_pull = g < 0, is_a = !is_pull && g < 14, is_b = g >= 14 && g < 21, is_c = g >= 21;
    const int owner = is_c ? self : (self + 1 + (is_a ? g >> 1 : g - 14)) & 7;
    const int mt = 2 * owner + (is_c ? (g - 21) >> 1 : is_b), hf = is_c ? (g - 21) & 1 : g & 1;
    auto pack_decode = [&](int t, int& isw, int& kc, int& tile) {
        const int blk = t >> 2;
        if (blk < 80) {
            const int m = blk - 16; kc = 0; isw = blk >= 16;
            tile = isw ? ((self + 1 + m) & 7) * 8 + (((m >> 3) + self) & 7) : blk;
        } else if (blk < 248) {
            const int m = blk - 80, r = m % 24; kc = 1 + m / 24; isw = r >= 16;
            tile = isw ? self * 8 + (r & 7) : r;
        } else if (blk < 640) {
            const int m = blk - 248, w = m % 56; isw = 1; kc = 1 + m / 56;
            tile = ((self + 1 + w % 7) & 7) * 8 + ((w / 7 + self) & 7);
        } else {
            const int m = blk - 640; isw = 1; kc = 0;
            tile = ((self + 1 + (m >> 3)) & 7) * 8 + (m & 7);
        }
    };
    const unsigned lw = (lane >> 4) * 8 * (GN * 2) + (lane & 15) * 16, lx = (lane & 15) * (KS * 2) + (lane >> 4) * 16;
    auto load_task = [&](int t, u32x4 (&r)[8]) {
        int isw, kc, tile; pack_decode(t, isw, kc, tile); const int q = t & 3;
        const GAS unsigned char* src = isw ? pw + ((kc * 64 + (q & 1) * 32) * (GN * 2) + (tile * 256 + (q >> 1) * 128) * 2)
                                       : px + ((tile * 256 + q * 64) * (KS * 2) + kc * 128);
        const unsigned lo = isw ? lw : lx, s1 = isw ? GN * 2 : 64, s2 = isw ? GN * 4 : 16 * KS * 2;
#pragma unroll
        for (int i = 0; i < 8; ++i) r[i] = *(const GAS u32x4*)(src + ((i >> 1) * s2 + (i & 1) * s1) + lo);
    };
    auto store_image = [&](unsigned char* blk, int isw, int q, const u32x4 (&c)[8]) {
        unsigned char* d0 = blk + (isw ? (q >> 1) * 8192 + (q & 1) * 1024 : q * 8192) + lane * 16;
        const unsigned s1 = isw ? 2048 : 1024, s4 = isw ? HTB : 4096;
#pragma unroll
        for (int j = 0; j < 8; ++j) st16_sys(d0 + ((j >> 2) * s4 + (j & 3) * s1), c[j]);
    };
    u32x4 cur[8];
    const int task0 = is_pull ? wid * 8 + nt : wid * 144 + (is_a ? g : g - 7) * 8 + nt;
    if (is_a || is_c) {
        if (wid >= HEAD_W) {
            while (__builtin_amdgcn_s_memrealtime() - tstart < PACK_DELAY) __builtin_amdgcn_s_sleep(2); }
        else __builtin_amdgcn_s_setprio(2);
        load_task(task0, cur);
    }
    unsigned peer_lo = 0u, peer_hi = 0u;
#pragma unroll
    for (int r = 0; r < NR; ++r)
        asm("v_writelane_b32 %0, %2, %4\n\tv_writelane_b32 %1, %3, %4" : "+v"(peer_lo), "+v"(peer_hi)
            : "s"((unsigned)(u64)P.ws[r]), "s"((unsigned)((u64)P.ws[r] >> 32)), "i"(r));
    asm volatile("s_waitcnt vmcnt(0)" : "+v"(count) :: "memory");
    const unsigned call = (unsigned)__builtin_amdgcn_readfirstlane(count) + 1u;
    const size_t par = call & 1u;
    const unsigned tag4 = (call & 255u) * 0x01010101u;
    const int wr = wid >> 2, wc = wid & 3;

#define READY0 0
    if (is_pull) {
        LAS unsigned* const ctl = (LAS unsigned*)(lds + 9 * HTB + 64);
        if (tid < 4) ctl[tid] = 0u;
        __syncthreads();
        const unsigned share = (WIN + 7 - nt) / 8;
        auto lds_draw = [&](int w) {
            unsigned v = 0;
            if (lane == 0) v = __hip_atomic_fetch_add(&ctl[w], 1u, __ATOMIC_RELAXED, __HIP_MEMORY_SCOPE_WORKGROUP);
            return (unsigned)__builtin_amdgcn_readfirstlane(v);
        };
        auto acquire = [&]() {
            const unsigned turn = lds_draw(0);
            while (__hip_atomic_load(&ctl[1], __ATOMIC_RELAXED, __HIP_MEMORY_SCOPE_WORKGROUP) + share <= turn) __builtin_amdgcn_s_sleep(ACQ_SLEEP);
        };
        auto release = [&]() { if (lane == 0) __hip_atomic_fetch_add(&ctl[1], 1u, __ATOMIC_RELAXED, __HIP_MEMORY_SCOPE_WORKGROUP); };
        const int src = (self + 1 + slot) & 7;
        const unsigned char* const sws = P.ws[src];
        if (wid < PWV) {
            const unsigned ord0 = order_vgpr<0, 64>(0u), ord1 = order_vgpr<64, 118>(0u);
            unsigned* const tick = (unsigned*)(ws + OFF_TICK + src * 128);
            const unsigned tbase = (call - 1u) * NTASK - PW;
            const unsigned char* const hbase = ws + OFF_HIN + (par * 8 + src) * (768 * HS);
            int task = task0, isw, kc, tile, q;
            decode_unit(ord0, ord1, task, src, self, isw, kc, tile, q);
            Packed pk;
            unsigned second = 0u;
            if (task < 4 * PLAIN && lane == 0) second = __hip_atomic_fetch_add(tick, 1u, __ATOMIC_RELAXED, __HIP_MEMORY_SCOPE_AGENT);
            second = __builtin_amdgcn_readfirstlane(second);
            while (__builtin_amdgcn_s_memrealtime() - tstart < PULL_DELAY) __builtin_amdgcn_s_sleep(2);
            wait_header(hbase + ((isw ? kc * 8 + tile : 64 + kc * 16 + tile) * 4 + q) * HS, tmo, call, pk.mask, pk.meta);
            if (task < 4 * PLAIN) {
                unsigned tw = 0u;
                acquire();
                load_parked(cur, tw, sws + OFF_RP + (par * 2560 + (self * 8 + tile) * 32 + q) * UNIT, tmo, lane);
                PARKED_WAIT(cur, tw);
                release();
                store_image(ws + OFF_WB + (size_t)(src * 64 + tile) * BLK, 1, q, cur);
                asm volatile("s_waitcnt vmcnt(0)" ::: "memory");
                if (lane == 0) st_flag8(ws + OFF_FW + (tile * 64 + src * 8) * 4 + q, call);
                task = (int)(second - tbase);
                decode_unit(ord0, ord1, task < NTASK ? task : 0, src, self, isw, kc, tile, q);
                wait_header(hbase + ((isw ? kc * 8 + tile : 64 + kc * 16 + tile) * 4 + q) * HS, tmo, call, pk.mask, pk.meta);
            }
            while (task < NTASK) {
                const unsigned char* const pl = sws + OFF_RP + (par * 2560 + (isw ? (self * 8 + tile) * 8 + kc : 512 + kc * 16 + tile) * 4 + q) * UNIT;
                acquire();
                fetch(pk, pl, lane);
                FETCH_WAIT(pk, 0);
                release();
                unsigned nn = 1u;
                {
                    u64 keep; const unsigned zero = 0u;
                    asm volatile("s_mov_b64 %1, exec\n\ts_mov_b64 exec, 1\n\t"
                                 "global_atomic_add %0, %2, %0, %3 sc0\n\t"
                                 "s_mov_b64 exec, %1" : "+v"(nn), "=&s"(keep) : "v"(zero), "s"(tick) : "memory");
                }
                int ntask, nisw, nkc, ntile, nq; u32x4 h; const unsigned char* hd;
                unpack(pk, cur, pl, scr, lane, [&]() {
                    asm volatile("s_waitcnt vmcnt(0)" : "+v"(nn) :: "memory");
                    ntask = (int)(__builtin_amdgcn_readfirstlane(nn) - tbase);
                    decode_unit(ord0, ord1, ntask < NTASK ? ntask : 0, src, self, nisw, nkc, ntile, nq);
                    hd = hbase + ((nisw ? nkc * 8 + ntile : 64 + nkc * 16 + ntile) * 4 + nq) * HS;
                    const unsigned zero = 0u;
                    asm volatile("global_load_dwordx4 %0, %1, %2 sc0 sc1" : "=&v"(h) : "v"(zero), "s"(hd) : "memory");
                });
                const int idx = src * 8 + kc;
                store_image(isw ? ws + OFF_WB + (size_t)(idx * 8 + tile) * BLK : ws + OFF_XA + (size_t)(idx * 16 + tile) * BLK, isw, q, cur);
                asm volatile("s_waitcnt vmcnt(0)" : "+v"(h) :: "memory");
                if (lane == 0) st_flag8(ws + (isw ? OFF_FW : OFF_FX) + (tile * 64 + idx) * 4 + q, call);
                if (ntask >= NTASK) break;
                if (!header_ok(h, call, pk.mask, pk.meta)) wait_header(hd, tmo, call, pk.mask, pk.meta);
                task = ntask; isw = nisw; kc = nkc; tile = ntile; q = nq;
            }
            if (lane == 0) __hip_atomic_store(&ctl[3], 1u, __ATOMIC_RELAXED, __HIP_MEMORY_SCOPE_WORKGROUP);
        } else {
            while (__hip_atomic_load(&ctl[3], __ATOMIC_RELAXED, __HIP_MEMORY_SCOPE_WORKGROUP) == 0u) __builtin_amdgcn_s_sleep(8);
        }
        STAMP(1);
        const int gnt = src * 8 + nt, fr = lane & 15, fq = lane >> 4;
        auto head_of = [&](int u) { return ws + OFF_HP + (size_t)(((((u >> 4) & 1) * 64 + gnt) * 8 + (u & 7)) * 2 + ((u >> 3) & 1)) * HS; };
        u32x4 h = {0u, 0u, 0u, 0u};
        for (int u = (int)lds_draw(2); u < 32;) {
            const int odd = u >> 4, bj = (u >> 3) & 1, uw = u & 7, dt = (slot * 2 + odd) * 8 + nt;
            Packed pk;
            if (!header_ok(h, call, pk.mask, pk.meta)) wait_header(head_of(u), tmo, call, pk.mask, pk.meta);
            const unsigned char* pl = sws + OFF_PP + (size_t)((((2 * self + odd) * 8 + nt) * 8 + uw) * 2 + bj) * UNIT;
            acquire();
            const unsigned char* const fp = ws + OFF_FP + ((dt * 8 + uw) * 2 + bj) * 4;
            unsigned pf;
            asm volatile("global_load_dword %0, %1, off sc0 sc1" : "=&v"(pf) : "v"(fp) : "memory");
            fetch(pk, pl, lane);
            FETCH_WAIT(pk, 0);
            release();
            asm volatile("" : "+v"(pf));
            if ((unsigned)__builtin_amdgcn_readfirstlane(pf) < call) wait_word<false>(fp, call, tmo, call);
            u32x4 dg[8]; unsigned tw = 0u;
            load_parked(dg, tw, ws + OFF_PD + (size_t)((dt * 8 + uw) * 2 + bj) * 8192, tmo, lane);
            unpack(pk, cur, pl, scr, lane, [&]() {
                u = (int)lds_draw(2);
                const unsigned zero = 0u;
                asm volatile("global_load_dwordx4 %0, %1, %2 sc0 sc1" : "=&v"(h) : "v"(zero), "s"(head_of(u)) : "memory");
            });
#pragma unroll
            for (int i = 0; i < 8; ++i) asm volatile("" : "+v"(cur[i]));
            PARKED_WAIT(dg, tw);
            asm volatile("" : "+v"(h));
            const bool bad = (unsigned)__builtin_amdgcn_readfirstlane(tw) == call;
            unsigned char* dst = P.out + (size_t)(odd * 256 + bj * 128 + (uw & 3) * 32 + fq * 4) * (GN * 2) + (size_t)(gnt * 256 + (uw >> 2) * 128 + fr * 8) * 2;
#pragma unroll
            for (int n = 0; n < 2; ++n)
#pragma unroll
                for (int r = 0; r < 4; ++r) {
                    u32x4 v;
#pragma unroll
                    for (int j = 0; j < 4; ++j) {
                        const unsigned pu = cur[n * 4 + r][j], w = dg[n * 4 + r][j];
                        const f32x2 y = (f32x2){__uint_as_float(w << 16), __uint_as_float(w & 0xffff0000u)}
                                      + (f32x2){__uint_as_float(pu << 16), __uint_as_float(pu & 0xffff0000u)};
                        const f32x2 o = gelu2(y);
                        v[j] = cvt_pk_bf16(o.x, o.y);
                    }
                    if (bad) v = (u32x4){0x7fc07fc0u, 0x7fc07fc0u, 0x7fc07fc0u, 0x7fc07fc0u};
                    st16_sys(dst + (size_t)(n * 16 + r) * (GN * 2), v);
                }
        }
        if (tid == 0) __hip_atomic_store((unsigned*)(ws + OFF_CNT + b * 128), call, __ATOMIC_RELAXED, __HIP_MEMORY_SCOPE_AGENT);
        STAMP(5);
        return;
    }

    if (!is_b) {
        unsigned char* const hdst = (unsigned char*)(((u64)peer_hi << 32) | peer_lo) + OFF_HIN + (par * 8 + self) * (768 * HS);
        for (int t = task0;;) {
#pragma unroll
            for (int i = 0; i < 8; ++i) asm volatile("" : "+v"(cur[i]));
            const int tn = t < (PK_ALL - 1) * 1152 ? t + 1152 : t >= PK_ALL * 1152 ? t + 704 : PK_ALL * 1152 + wid * 88 + (is_a ? g >> 1 : g - 14) * 8 + nt;
            const bool more = tn < 2784 && (tn < PK_ALL * 1152 || is_c || hf);
            int isw, kc, tile; pack_decode(t, isw, kc, tile); const int q = t & 3, idx = self * 8 + kc;
            if (isw) {
                u32x4 tr[8];
#pragma unroll
                for (int j = 0; j < 8; ++j)
#pragma unroll
                    for (int qd = 0; qd < 4; ++qd)
                        tr[j][qd] = perm(cur[2 * qd + 1][j >> 1], cur[2 * qd][j >> 1], (j & 1) ? 0x07060302u : 0x05040100u);
#pragma unroll
                for (int j = 0; j < 8; ++j) cur[j] = tr[j];
            }
            const bool own = (isw ? tile >> 3 : tile >> 1) == self, diag = t >= 992;
            const bool packed = !own && t < 2560, image = !isw || own || diag;
            u32x4 hd = {0u, 0u, 0u, 0u};
            if (packed && t >= 64 && t < 64 + 32 * PLAIN) {
                unsigned char* pl = ws + OFF_RP + (par * 2560 + tile * 32 + q) * UNIT + lane * 16;
#pragma unroll
                for (int i = 0; i < 8; ++i) st16_sys(pl + i * 1024, cur[i]);
                hd = (u32x4){0u, call, 0u, call << 15};
            } else if (packed) {
                Packed pk; pack(cur, pk, scr, lane);
                send(pk, ws + OFF_RP + (par * 2560 + (isw ? tile * 8 + kc : 512 + kc * 16 + tile) * 4 + q) * UNIT, scr, lane);
                hd = header(pk, call);
            }
            if (image)
                store_image(ws + (diag ? OFF_WD + (size_t)(tile * 8 + kc) * BLK : isw ? OFF_WB + (size_t)(idx * 8 + (tile & 7)) * BLK
                                                                                     : OFF_XA + (size_t)(idx * 16 + tile) * BLK), isw, q, cur);
            load_task(more ? tn : t, cur);
            asm volatile("s_waitcnt vmcnt(8)" ::: "memory");
            if (packed && lane < NR && lane != self && (!isw || lane == (tile >> 3)))
                st16_sys(hdst + ((isw ? kc * 8 + (tile & 7) : 64 + kc * 16 + tile) * 4 + q) * HS, hd);
            if (image && lane == 0)
                st_flag8(ws + (diag ? OFF_FD + (tile * 8 + kc) * 4 : isw ? OFF_FW + ((tile & 7) * 64 + idx) * 4 : OFF_FX + (tile * 64 + idx) * 4) + q, call);
            __builtin_amdgcn_s_setprio(0);
            if (!more) break;
            t = tn;
        }
        STAMP(6);
    }

    const unsigned voff = tid * 16, ldsw = wid * 1024, foff = lane * 4;
    const unsigned pickoff = (unsigned)(size_t)(lds + 9 * HTB);
    const unsigned ldsb = (unsigned)(size_t)lds + ldsw;
#define GLDS(bufoff, gbase) asm volatile("s_add_i32 m0, %0, %1\n\ts_nop 0\n\tglobal_load_lds_dwordx4 %2, %3 sc0 sc1" \
                                         :: "s"(ldsb), "i"(bufoff), "v"(voff), "s"(gbase) : "memory", "scc")
#define STAGE(bufoff, gbase) do { GLDS(bufoff, gbase); GLDS((bufoff) + 8192, (gbase) + 8192); } while (0)
    const unsigned lds4 = (unsigned)(size_t)lds + wid * 4096, lds2 = (unsigned)(size_t)lds + wid * 2048;
    const unsigned voff4 = wid * 4096 + lane * 16, voff2 = wid * 2048 + lane * 16;
#define GL4(bufoff, vo, gbase) asm volatile("s_add_i32 m0, %0, %1\n\ts_nop 0\n\tglobal_load_lds_dwordx4 %2, %3 sc0 sc1\n\t" \
        "global_load_lds_dwordx4 %2, %3 offset:1024 sc0 sc1\n\tglobal_load_lds_dwordx4 %2, %3 offset:2048 sc0 sc1\n\t" \
        "global_load_lds_dwordx4 %2, %3 offset:3072 sc0 sc1" :: "s"(lds4), "i"(bufoff), "v"(vo), "s"(gbase) : "memory", "scc")
#define GL2(bufoff, vo, gbase) asm volatile("s_add_i32 m0, %0, %1\n\ts_nop 0\n\tglobal_load_lds_dwordx4 %2, %3 sc0 sc1\n\t" \
        "global_load_lds_dwordx4 %2, %3 offset:1024 sc0 sc1" :: "s"(lds2), "i"(bufoff), "v"(vo), "s"(gbase) : "memory", "scc")
#define LIKELY(c) __builtin_expect(!!(c), 1)
#define RARE(c) __builtin_expect(!!(c), 0)
#define LDA(dst, base, off) do { _Pragma("unroll") for (int m = 0; m < 4; ++m) _Pragma("unroll") for (int k = 0; k < 2; ++k) \
        dst[m][k] = *(const LAS bf16x8*)(lds + (base) + ((off) + m * 2048 + k * 1024)); } while (0)
#define LDB(dst, base, off) do { _Pragma("unroll") for (int n = 0; n < 2; ++n) _Pragma("unroll") for (int k = 0; k < 2; ++k) \
        dst[n][k] = *(const LAS bf16x8*)(lds + (base) + ((off) + n * 2048 + k * 1024)); } while (0)
#define MMA(ai, bj, At, Bt) do { __builtin_amdgcn_s_setprio(1); _Pragma("unroll") for (int m = 0; m < 4; ++m) _Pragma("unroll") for (int n = 0; n < 2; ++n) _Pragma("unroll") for (int k = 0; k < 2; ++k) \
        acc[ai][bj][m][n] = __builtin_amdgcn_mfma_f32_16x16x32_bf16(Bt[n][k], At[m][k], acc[ai][bj][m][n], 0, 0, 0); __builtin_amdgcn_s_setprio(0); } while (0)
#define WAIT_V(n) asm volatile("s_waitcnt vmcnt(" #n ")" ::: "memory")
#define WAIT_L0 do { asm volatile("" ::: "memory"); __builtin_amdgcn_s_waitcnt(0xC07F); asm volatile("" ::: "memory"); } while (0)
#define BAR __builtin_amdgcn_s_barrier()
#define SCHED __builtin_amdgcn_sched_barrier(0)
    typedef unsigned u32x2 __attribute__((ext_vector_type(2)));
#define SNAP() do { if (RARE(fresh > wid)) asm volatile("global_load_dword %0, %2, %3 sc0 sc1\n\tglobal_load_dword %1, %2, %4 sc0 sc1" \
                                                        : "+v"(sna), "+v"(snb) : "v"(foff), "s"(fa), "s"(fb) : "memory"); } while (0)
#define PUBLISH(w, xm) do { if (RARE(fresh > wid)) { asm volatile("" : "+v"(sna), "+v"(snb)); \
            u64 m = __ballot(sna == tag4 && ((snb ^ tag4) & (xm)) == 0u); \
            if (vm == 0 && ((m | avail) & todo) == 0ull) m = poll_ready(todo, fa, fb, tmo, call, tag4, xm); \
            news = (m & todo & ~avail) != 0ull; \
            asm volatile("ds_write_b64 %0, %1 offset:%2" :: "v"(pickoff), "v"((u32x2){(unsigned)m, (unsigned)(m >> 32)}), "i"(8 * (w)) : "memory"); } } while (0)
#define LISTEN(w) do { if (RARE(fresh)) asm volatile("ds_read_b64 %0, %1 offset:%2" : "+v"(mv) : "v"(pickoff), "i"(8 * (w)) : "memory"); } while (0)
#define TAKE(i) do { if (RARE(fresh)) { asm volatile("s_waitcnt lgkmcnt(0)" : "+v"(mv) :: "memory"); \
            avail |= todo & (((u64)(unsigned)__builtin_amdgcn_readfirstlane(mv.y) << 32) | (unsigned)__builtin_amdgcn_readfirstlane(mv.x)); } \
        u64 bit_;   \
        asm volatile("s_ff1_i32_b64 %0, %3\n\ts_lshl_b64 %4, 1, %0\n\ts_and_b64 %4, %4, %3\n\ts_andn2_b64 %2, %2, %4\n\ts_andn2_b64 %3, %3, %4\n\t" \
                     "s_min_u32 %0, %0, 64\n\ts_bcnt1_i32_b64 %1, %3\n\ts_cmp_lt_u32 %1, %5\n\ts_cselect_b32 %1, 1, 0\n\t" \
                     "s_cmp_lg_u64 %2, %3\n\ts_cselect_b32 %1, %1, 0" \
                     : "=&s"(i), "=&s"(fresh), "+s"(todo), "+s"(avail), "=&s"(bit_) : "i"(SNAP_AT) : "scc"); } while (0)
#define KSTAMP() do {} while (0)
    f32x4 acc[2][2][4][2];
    auto zero_acc = [&]() {
#pragma unroll
        for (int a = 0; a < 2; ++a)
#pragma unroll
            for (int c = 0; c < 2; ++c)
#pragma unroll
                for (int m = 0; m < 4; ++m)
#pragma unroll
                    for (int n = 0; n < 2; ++n) acc[a][c][m][n] = (f32x4){0.f, 0.f, 0.f, 0.f};
    };
    auto to_bf16 = [&](int c, u32x4 (&v)[8]) {
#pragma unroll
        for (int n = 0; n < 2; ++n)
#pragma unroll
            for (int r = 0; r < 4; ++r) {
                u32x4& o = v[n * 4 + r];
                o.x = cvt_pk_bf16(acc[0][c][0][n][r], acc[0][c][1][n][r]); o.y = cvt_pk_bf16(acc[0][c][2][n][r], acc[0][c][3][n][r]);
                o.z = cvt_pk_bf16(acc[1][c][0][n][r], acc[1][c][1][n][r]); o.w = cvt_pk_bf16(acc[1][c][2][n][r], acc[1][c][3][n][r]);
            }
    };
    auto emit_unit = [&](int c, int bj) {
        const int lz = lane_id();
        const unsigned char* const started = ws + OFF_FW + nt * 256 + owner * 32;
        unsigned og, ot;
        asm volatile("global_load_dword %0, %2, off sc0 sc1\n\tglobal_load_dword %1, %3, off sc1" : "=&v"(og), "=&v"(ot) : "v"(started), "v"(tmo) : "memory");
        u32x4 v[8];
        to_bf16(c, v);
        Packed pk;
        if (wid < 4) __builtin_amdgcn_s_setprio(2);
        pack(v, pk, scr, lz);
        __builtin_amdgcn_s_setprio(0);
        asm volatile("s_waitcnt vmcnt(0)" : "+v"(og), "+v"(ot) :: "memory");
        bool bad = (unsigned)__builtin_amdgcn_readfirstlane(ot) == call;
        if ((unsigned)__builtin_amdgcn_readfirstlane(og) != tag4) bad = wait_word<true>(started, tag4, tmo, call) || bad;
        if (bad) {
#pragma unroll
            for (int i = 0; i < 4; ++i) pk.sm[i] = (u32x4){0x40404040u, 0x40404040u, 0x40404040u, 0x40404040u};
            pk.l1 = pk.st = (u32x4){0u, 0u, 0u, 0u}; pk.mask = 0ull; pk.meta = 0x101u;
        }
        STAMP(4);
        send(pk, ws + OFF_PP + (size_t)(((mt * 8 + nt) * 8 + wid) * 2 + bj) * UNIT, scr, lz);
        asm volatile("s_waitcnt vmcnt(0)" ::: "memory");
        if (lz == 0) st16_sys(P.ws[owner] + OFF_HP + (size_t)((((mt & 1) * 64 + self * 8 + nt) * 8 + wid) * 2 + bj) * HS, header(pk, call));
    };
    auto finish = [&]() {
        if (threadIdx.x == 0) __hip_atomic_store((unsigned*)(ws + OFF_CNT + b * 128), call, __ATOMIC_RELAXED, __HIP_MEMORY_SCOPE_AGENT);
        STAMP(5);
    };
    auto park = [&](int c, int pt, int bj) {
        unsigned char* dp = ws + OFF_PD + (size_t)((pt * 8 + wid) * 2 + bj) * 8192 + lane_id() * 16;
        u32x4 v[8];
        to_bf16(c, v);
#pragma unroll
        for (int i = 0; i < 8; ++i) st16_sys(dp + i * 1024, v[i]);
    };
    STAMP(2);
    if (!is_a) {
      const int nsteps = is_b ? 1 : 1 + (7 + C_ODD + 24 - g) / 4;
#pragma unroll 1
      for (int step = 0;; ++step) {
        const bool diag = step < nsteps - 1;
        const int dj = step * 4 + g - 21, dodd = dj >= 7, link = dj - 7 * dodd, dg = ((self + 1 + link) & 7) * 8 + nt, jm = diag ? 2 * self + dodd : mt;
        const int nkt = diag ? 8 : is_b ? 56 : 32;
        u64 todo = diag ? 0xffull << (self * 8) : is_b ? ~(0xffull << (owner * 8)) : hf ? 0x55aa55aa55aa55aaull : 0xaa55aa55aa55aa55ull, avail = todo & READY0;
        const unsigned char* const abase = diag ? ws + OFF_WD + (size_t)(dg * 8 - self * 8) * BLK : ws + OFF_WB + (size_t)nt * BLK;
        const int ashift = diag ? 15 : 18;
        const unsigned char* const fa = diag ? ws + OFF_FD + (dg * 8 - self * 8) * 4 : ws + OFF_FW + nt * 256;
        const int x0 = (is_c && !diag && hf) ? HTB : 0;
        const unsigned voffx = voff4 ^ x0, voffh = voff2 + HTB;
        const unsigned char* const bbase = ws + OFF_XA + (size_t)jm * BLK;
        const unsigned char* const fb = ws + OFF_FX + jm * 256;
        const int aoff = wr * 8192 + lane * 16;
        int boff = 4 * HTB + wc * 4096 + lane * 16;
        asm volatile("" : "+v"(boff));
        zero_acc();
        bf16x8 At[4][2], B0[2][2], B1[2][2];
        const unsigned char* a1 = abase;
        unsigned sna = ~tag4, snb = ~tag4; u32x2 mv = {0u, 0u}; int fresh = 1;
        int vm = 0; const bool stampit = !diag; (void)stampit;
        int left = __builtin_amdgcn_readfirstlane(nkt);
        __syncthreads();
        if (wr == 1) BAR;
#define SA(bf, h) (((bf) * 2 + (h)) * HTB)
#define SB(bf, h) ((4 + (bf) * 2 + (h)) * HTB)
#define KTILE(bf) do { \
        LDB(B0, boff, SA(bf, 0)); LDB(B1, boff, SA(bf, 1)); LDA(At, aoff, SA(bf, 0)); SCHED; \
        if (LIKELY(vm & 2)) { if (GLF) GL2(SA(1 - bf, 1), voffh, a1); else STAGE(SA(1 - bf, 1), a1 + HTB); WAIT_V(8); } else WAIT_V(0); \
        bool news; PUBLISH(bf, 0xffffffffu); (void)news; \
        WAIT_L0; BAR; SCHED; LISTEN(bf); SCHED; \
        if (LIKELY(vm & 1)) { MMA(0, 0, At, B0); MMA(0, 1, At, B1); } \
        BAR; SCHED; \
        int i2; TAKE(i2); SCHED; \
        LDA(At, aoff, SA(bf, 1)); SCHED; \
        SNAP(); \
        const unsigned char* a2 = abase + ((size_t)i2 << ashift); const unsigned char* b2 = bbase + ((size_t)i2 << 19); \
        if (LIKELY(i2 < 64)) { if (GLF) { GL4(SB(bf, 0), voffx, b2); GL2(SA(bf, 0), voff2, a2); } else { STAGE(SB(bf, 0), b2 + x0); STAGE(SB(bf, 1), b2 + (HTB - x0)); STAGE(SA(bf, 0), a2); } WAIT_V(8); vm |= 4; } \
        else WAIT_V(2); \
        WAIT_L0; BAR; SCHED; if (LIKELY(vm & 1)) { MMA(1, 0, At, B0); MMA(1, 1, At, B1); } BAR; SCHED; \
        left -= vm & 1; KSTAMP(); vm >>= 1; a1 = a2; } while (0)
        for (;;) {
            KTILE(0);
            if (left == 0) break;
            KTILE(1);
            if (left == 0) break;
        }
        asm volatile("s_waitcnt vmcnt(0) lgkmcnt(0)" : "+v"(sna), "+v"(snb), "+v"(mv) :: "memory");
        if (wr == 0) BAR;
        if (!diag) break;
        park(0, link * 16 + dodd * 8 + nt, 0); park(1, link * 16 + dodd * 8 + nt, 1);
        asm volatile("s_waitcnt vmcnt(0)" ::: "memory");
        if (lane_id() < 2) st_flag(ws + OFF_FP + (((link * 16 + dodd * 8 + nt) * 8 + wid) * 2 + lane_id()) * 4, call);
        STAMP(7);
      }
      {
        if (is_b) {
            STAMP(3);
            emit_unit(0, 0); emit_unit(1, 1);
        } else {
            STAMP(3);
            const int lz = lane_id(), pt = 112 + (mt & 1) * 8 + nt;
            park(1, pt, 1 - hf);
            asm volatile("s_waitcnt vmcnt(0)" ::: "memory");
            if (lz == 0) st_flag(ws + OFF_FP + ((pt * 8 + wid) * 2 + 1 - hf) * 4, call);
            wait_word<false>(ws + OFF_FP + ((pt * 8 + wid) * 2 + hf) * 4, call, tmo, call);
            u32x4 dg[8]; unsigned tw = 0u;
            load_parked(dg, tw, ws + OFF_PD + (size_t)((pt * 8 + wid) * 2 + hf) * 8192, tmo, lz);
            PARKED_WAIT(dg, tw);
            const bool bad = (unsigned)__builtin_amdgcn_readfirstlane(tw) == call;
            unsigned char* dst = P.out + (size_t)((mt & 1) * 256 + hf * 128 + wc * 32 + (lz >> 4) * 4) * (GN * 2) + (size_t)((self * 8 + nt) * 256 + wr * 128 + (lz & 15) * 8) * 2;
#pragma unroll
            for (int n = 0; n < 2; ++n)
#pragma unroll
                for (int r = 0; r < 4; ++r) {
                    u32x4 v;
#pragma unroll
                    for (int j = 0; j < 4; ++j) {
                        const unsigned w = dg[n * 4 + r][j];
                        const f32x2 o = gelu2((f32x2){acc[j >> 1][0][(j & 1) * 2][n][r], acc[j >> 1][0][(j & 1) * 2 + 1][n][r]}
                                              + (f32x2){__uint_as_float(w << 16), __uint_as_float(w & 0xffff0000u)});
                        v[j] = cvt_pk_bf16(o.x, o.y);
                    }
                    if (bad) v = (u32x4){0x7fc07fc0u, 0x7fc07fc0u, 0x7fc07fc0u, 0x7fc07fc0u};
                    st16_sys(dst + (size_t)(n * 16 + r) * (GN * 2), v);
                }
        }
      }
    } else {
#pragma unroll 1
      for (int step = 0; step < ((g >> 1) < C_ODD ? 1 : 2); ++step) {
        const bool diag = step == 1;
        const int dg = owner * 8 + nt, jm = diag ? 2 * self + 1 : mt;
        const int nkt = diag ? 8 : 56;
        u64 todo = diag ? 0xffull << (self * 8) : ~(0xffull << (owner * 8)), avail = todo & READY0;
        const unsigned char* const abase = diag ? ws + OFF_WD + (size_t)(dg * 8 - self * 8) * BLK : ws + OFF_WB + (size_t)nt * BLK;
        const int ashift = diag ? 15 : 18;
        const unsigned char* const fa = diag ? ws + OFF_FD + (dg * 8 - self * 8) * 4 : ws + OFF_FW + nt * 256;
        const unsigned char* const bbase = ws + OFF_XA + (size_t)jm * BLK + hf * HTB;
        const unsigned char* const fb = ws + OFF_FX + jm * 256;
        const unsigned xm = hf ? 0xffff0000u : 0x0000ffffu;
        int aoff[3], boff[3];
#pragma unroll
        for (int i = 0; i < 3; ++i) {
            aoff[i] = i * 3 * HTB + wr * 8192 + lane * 16; boff[i] = i * 3 * HTB + 2 * HTB + wc * 4096 + lane * 16;
            asm volatile("" : "+v"(aoff[i]), "+v"(boff[i]));
        }
        zero_acc();
        bf16x8 A0[4][2], A1[4][2], B0[2][2];
        unsigned sna = ~tag4, snb = ~tag4; u32x2 mv = {0u, 0u}; int fresh = 1;
        int vm = 0; const bool stampit = !diag; (void)stampit;
        int left = __builtin_amdgcn_readfirstlane(nkt);
        int heard = 0;
        __syncthreads();
        if (wr == 1) BAR;
#define HTILE(s) do { \
        constexpr int nb = ((s) + 2) % 3 * 3 * HTB; \
        const int pub = fresh; \
        fresh = heard; int i2; TAKE(i2); \
        if (i2 < 64) vm |= 4; \
        bool news = false;   \
        { const int want = fresh; fresh = pub; PUBLISH(s, xm); fresh = want; } \
        if (!news) SNAP(); \
        if (LIKELY(vm & 4)) { \
            const unsigned char* a2 = abase + ((size_t)i2 << ashift); const unsigned char* b2 = bbase + ((size_t)i2 << 19); \
            if (GLH) { GL2(nb + 2 * HTB, voff2, b2); GL4(nb, voff4, a2); } else { \
            GLDS(nb + 2 * HTB, b2); GLDS(nb + 2 * HTB + 8192, b2 + 8192); GLDS(nb, a2); \
            GLDS(nb + 8192, a2 + 8192); GLDS(nb + 16384, a2 + 16384); GLDS(nb + 24576, a2 + 24576); } } \
        SCHED; LDB(B0, boff[s], 0); LDA(A0, aoff[s], 0); LDA(A1, aoff[s], HTB); SCHED; \
        if (wr == 1) { if (LIKELY(vm & 4)) WAIT_V(6); else WAIT_V(0); } \
        WAIT_L0; BAR; SCHED; \
        if (RARE(pub)) asm volatile("ds_read_b64 %0, %1 offset:%2" : "+v"(mv) : "v"(pickoff), "i"(8 * (s)) : "memory"); \
        SCHED; heard = pub; \
        if (LIKELY(vm & 1)) { MMA(0, 0, A0, B0); MMA(1, 0, A1, B0); } \
        if (wr == 0) { if (LIKELY(vm & 4)) WAIT_V(6); else WAIT_V(0); } \
        BAR; SCHED; \
        left -= vm & 1; KSTAMP(); vm >>= 1; } while (0)
        for (;;) {
            HTILE(0);
            if (left == 0) break;
            HTILE(1);
            if (left == 0) break;
            HTILE(2);
            if (left == 0) break;
        }
        asm volatile("s_waitcnt vmcnt(0) lgkmcnt(0)" : "+v"(sna), "+v"(snb), "+v"(mv) :: "memory");
        if (wr == 0) BAR;
        if (diag) {
            const int pt = (g | 1) * 8 + nt;
            park(0, pt, hf);
            asm volatile("s_waitcnt vmcnt(0)" ::: "memory");
            if (lane_id() == 0) st_flag(ws + OFF_FP + ((pt * 8 + wid) * 2 + hf) * 4, call);
            STAMP(7);
        } else {
            STAMP(3);
            emit_unit(0, hf);
            STAMP(8);
        }
      }
    }
    finish();
}

}

extern "C" void kernel_launch(int rank, int nranks, ncclComm_t comm, hipStream_t stream,
                              void* const* d_in, const int* in_sizes, int n_in,
                              void* d_out, int out_size, const int* in_dtypes, int out_dtype,
                              void* const* d_out_all, void* const* d_ws_all, size_t ws_bytes) {
    (void)comm; (void)in_dtypes; (void)out_dtype; (void)d_out_all;
    if (nranks != NR || (unsigned)rank >= (unsigned)NR || n_in < 2 || ws_bytes < WS_NEED
        || in_sizes[0] != GM * KS || in_sizes[1] != KS * GN || out_size != (GM / NR) * GN) {
        fprintf(stderr, "kernel_launch: unexpected problem shape\n");
        return;
    }
    Params p;
    p.x = (const unsigned char*)d_in[0];
    p.w = (const unsigned char*)d_in[1];
    p.out = (unsigned char*)d_out;
    for (int i = 0; i < NR; ++i) p.ws[i] = (unsigned char*)d_ws_all[i];
    p.own = p.ws[rank]; p.rank = rank; p.pad = 0;
    hipLaunchKernelGGL(fused_gemm_rs, dim3(256), dim3(512), 0, stream, p);
    HIP_CHECK(hipGetLastError());
}
```

```cpp
#include <hip/hip_runtime.h>
#include <rccl/rccl.h>
#include <cstdio>
#include <cstdint>

#define HIP_CHECK(expr) do { hipError_t _e = (expr); if (_e != hipSuccess) { \
    fprintf(stderr, "HIP error %s at %s:%d\n", hipGetErrorString(_e), __FILE__, __LINE__); \
    return; } } while (0)

namespace {

#define LAS __attribute__((address_space(3)))
#define GAS __attribute__((address_space(1)))
typedef short bf16x8 __attribute__((ext_vector_type(8)));
typedef float f32x4 __attribute__((ext_vector_type(4)));
typedef unsigned u32x4 __attribute__((ext_vector_type(4)));
typedef unsigned long long u64;

constexpr int NR = 8;
constexpr int GM = 4096, GK = 4096, GN = 16384;
constexpr int KS = GK / NR;
constexpr int BLK = 32768;
constexpr int HTB = 16384;
#define PWV 7
#define WIN 16
#define PULL_DELAY 100
#define HEAD_W 2
#define PACK_DELAY 150
#define PK_ALL 2
#define NTX 1
#define SNAP_AT 2
#define ACQ_SLEEP 1
#define C_ODD 0
#define PLAIN 3
#define HS 128
#define HDR_SLEEP 1
#define FILL_STOP 6
#define SNAP_H 1
#define RX 8
#define RW 8
#define HELP_KC 0xe0
#define MID_Q 2
constexpr int NPULL = 56, PW = 8 * PWV;
constexpr u64 DEADLINE = 50000000ull;
constexpr int NTASK = 704;
constexpr int UNIT = 9344;

constexpr size_t OFF_CNT = 0;
constexpr size_t OFF_TMO = 32768;
constexpr size_t OFF_TICK = 33792;
constexpr size_t OFF_FX = 131072;
constexpr size_t OFF_FW = OFF_FX + 4096;
constexpr int FR = 8192;
constexpr size_t OFF_FD = 98304;
constexpr size_t OFF_FQ = 104448;
constexpr size_t OFF_FP = 106496;
constexpr size_t OFF_STAMP = 524288;
constexpr size_t OFF_XA = 1u << 20;
constexpr size_t OFF_WB = OFF_XA + (size_t)64 * 16 * BLK;
constexpr size_t OFF_WD = OFF_WB + (size_t)64 * 8 * BLK;
constexpr size_t OFF_RP = OFF_WD + (size_t)64 * 8 * BLK;
constexpr size_t OFF_PP = OFF_RP + (size_t)5120 * UNIT;
constexpr size_t OFF_PD = OFF_PP + (size_t)2048 * UNIT;
constexpr size_t OFF_HIN = OFF_PD + (size_t)144 * 131072;
constexpr size_t OFF_HP = OFF_HIN + (size_t)16 * 768 * HS;
constexpr size_t WS_NEED = OFF_HP + (size_t)2048 * HS + 4096;
static_assert(WS_NEED <= (size_t)161 * 1048576, "workspace");

struct Params {
    const unsigned char* x;
    const unsigned char* w;
    unsigned char* out;
    unsigned char* own;
    unsigned char* ws[NR];
    int rank, pad;
};

#define SBASE "s_nop 4\n\t"
__device__ __forceinline__ void st16_sys(unsigned char* p, u32x4 v) {
    asm volatile("global_store_dwordx4 %0, %1, off sc0 sc1\n\ts_nop 1" :: "v"(p), "v"(v) : "memory");
}
__device__ __forceinline__ void st_flag(unsigned char* p, unsigned v) {
    __hip_atomic_store((unsigned*)p, v, __ATOMIC_RELAXED, __HIP_MEMORY_SCOPE_SYSTEM);
}
__device__ __forceinline__ void st_flag8(unsigned char* p, unsigned v) {
    __hip_atomic_store(p, (unsigned char)v, __ATOMIC_RELAXED, __HIP_MEMORY_SCOPE_SYSTEM);
}
__device__ __forceinline__ unsigned ld_flag(const unsigned char* p) {
    return __hip_atomic_load((const unsigned*)p, __ATOMIC_RELAXED, __HIP_MEMORY_SCOPE_SYSTEM);
}
__device__ __forceinline__ unsigned lane_id() {
    return __builtin_amdgcn_mbcnt_hi(~0u, __builtin_amdgcn_mbcnt_lo(~0u, 0u));
}
__device__ __forceinline__ unsigned cvt_pk_bf16(float lo, float hi) {
    unsigned r; asm volatile("v_cvt_pk_bf16_f32 %0, %1, %2" : "=v"(r) : "v"(lo), "v"(hi)); return r;
}
typedef float f32x2 __attribute__((ext_vector_type(2)));
__device__ __forceinline__ f32x2 gelu2(f32x2 y) {
    const float k1 = -2.0f * 0.7978845608028654f * 1.4426950408889634f;
    const float k3 = k1 * 0.044715f;
    const f32x2 z = y * ((y * y) * k3 + k1);
    const f32x2 d = (f32x2){__builtin_amdgcn_exp2f(z.x), __builtin_amdgcn_exp2f(z.y)} + 1.0f;
    return y * (f32x2){__builtin_amdgcn_rcpf(d.x), __builtin_amdgcn_rcpf(d.y)};
}

template <bool EQ>
__device__ __forceinline__ bool wait_word(const unsigned char* p, unsigned want, unsigned char* tmo, unsigned call) {
    const u64 t0 = __builtin_amdgcn_s_memrealtime();
    for (;;) {
        unsigned g, t;
        asm volatile("global_load_dword %0, %2, off sc0 sc1\n\t"
                     "global_load_dword %1, %3, off sc1\n\t"
                     "s_waitcnt vmcnt(0)" : "=&v"(g), "=&v"(t) : "v"(p), "v"(tmo) : "memory");
        if ((unsigned)__builtin_amdgcn_readfirstlane(t) == call || __builtin_amdgcn_s_memrealtime() - t0 > DEADLINE) {
            __hip_atomic_store((unsigned*)tmo, call, __ATOMIC_RELAXED, __HIP_MEMORY_SCOPE_AGENT);
            return true;
        }
        const unsigned v = __builtin_amdgcn_readfirstlane(g);
        if (EQ ? v == want : v >= want) return false;
        __builtin_amdgcn_s_sleep(2);
    }
}

__device__ __forceinline__ u64 now() {
    u64 t; asm volatile("s_memrealtime %0\n\ts_waitcnt lgkmcnt(0)" : "=s"(t) :: "memory"); return t;
}
__device__ __forceinline__ u64 poll_ready(u64 todo, const unsigned char* ws, unsigned fa, unsigned fb,
                                          unsigned char* tmo, unsigned call, unsigned tag4, unsigned xm) {
    const u64 t0 = now();
    for (;;) {
        unsigned a, b, t; const unsigned zero = 0u;
        asm volatile(SBASE "global_load_dword %0, %3, %6 sc0 sc1\n\t"
                     "global_load_dword %1, %4, %6 sc0 sc1\n\t"
                     "global_load_dword %2, %5, %7 sc1\n\t"
                     "s_waitcnt vmcnt(0)"
                     : "=&v"(a), "=&v"(b), "=&v"(t) : "v"(fa), "v"(fb), "v"(zero), "s"(ws), "s"(tmo) : "memory");
        const u64 ready = __ballot(a == tag4 && ((b ^ tag4) & xm) == 0u);
        if ((ready & todo) != 0ull) return ready;
        if ((unsigned)__builtin_amdgcn_readfirstlane(t) == call || now() - t0 > DEADLINE) {
            __hip_atomic_store((unsigned*)tmo, call, __ATOMIC_RELAXED, __HIP_MEMORY_SCOPE_AGENT);
            return ~0ull;
        }
        __builtin_amdgcn_s_sleep(1);
    }
}

__device__ __forceinline__ unsigned bfi(unsigned m, unsigned a, unsigned b) { return (a & m) | (b & ~m); }
__device__ __forceinline__ unsigned perm(unsigned hi, unsigned lo, unsigned sel) { return __builtin_amdgcn_perm(hi, lo, sel); }
#define SDWA2(k) " dst_sel:DWORD dst_unused:UNUSED_PAD src0_sel:BYTE_" #k " src1_sel:BYTE_" #k "\n\t"
#define SDWA1(k) " dst_sel:DWORD dst_unused:UNUSED_PAD src0_sel:DWORD src1_sel:BYTE_" #k "\n\t"
__device__ __forceinline__ unsigned spread_bits(unsigned f, unsigned s) {
    unsigned r, a, b;
    asm("v_lshlrev_b32_sdwa %0, %4, %3" SDWA2(0) "v_lshlrev_b32_sdwa %1, %4, %3" SDWA2(1) "v_lshlrev_b32_sdwa %2, %4, %3" SDWA2(2)
        "v_or3_b32 %0, %0, %1, %2\n\t"
        "v_lshlrev_b32_sdwa %1, %4, %3" SDWA2(3)
        "v_or_b32 %0, %0, %1" : "=&v"(r), "=&v"(a), "=&v"(b) : "v"(f), "v"(s));
    return r;
}
__device__ __forceinline__ unsigned take_codes(unsigned& w, unsigned f) {
    unsigned ln, t, l;
    asm("v_xor_b32_sdwa %2, %0, %4" SDWA1(0) "v_ffbl_b32 %1, %2\n\tv_lshrrev_b32 %0, %1, %0\n\t"
        "v_xor_b32_sdwa %2, %0, %4" SDWA1(1) "v_ffbl_b32 %3, %2\n\tv_lshrrev_b32 %0, %3, %0\n\t"
        "v_xor_b32_sdwa %2, %0, %4" SDWA1(2) "v_lshl_or_b32 %1, %3, 8, %1\n\tv_ffbl_b32 %3, %2\n\tv_lshrrev_b32 %0, %3, %0\n\t"
        "v_xor_b32_sdwa %2, %0, %4" SDWA1(3) "v_lshl_or_b32 %1, %3, 16, %1\n\tv_ffbl_b32 %3, %2\n\tv_lshrrev_b32 %0, %3, %0\n\t"
        "v_lshl_or_b32 %1, %3, 24, %1" : "+v"(w), "=&v"(ln), "=&v"(t), "=&v"(l) : "v"(f));
    return ln;
}
__device__ __forceinline__ unsigned wave_scan(unsigned v) {
    v += (unsigned)__builtin_amdgcn_update_dpp(0, (int)v, 0x111, 0xf, 0xf, false);
    v += (unsigned)__builtin_amdgcn_update_dpp(0, (int)v, 0x112, 0xf, 0xf, false);
    v += (unsigned)__builtin_amdgcn_update_dpp(0, (int)v, 0x114, 0xf, 0xf, false);
    v += (unsigned)__builtin_amdgcn_update_dpp(0, (int)v, 0x118, 0xf, 0xf, false);
    v += (unsigned)__builtin_amdgcn_update_dpp(0, (int)v, 0x142, 0xa, 0xf, false);
    v += (unsigned)__builtin_amdgcn_update_dpp(0, (int)v, 0x143, 0xc, 0xf, false);
    return v;
}
__device__ __forceinline__ unsigned rank_in(u64 mask) {
    return __builtin_amdgcn_mbcnt_hi((unsigned)(mask >> 32), __builtin_amdgcn_mbcnt_lo((unsigned)mask, 0u));
}
constexpr int SCR = 8192;
#define LDS_ORDER() asm volatile("" ::: "memory")

struct Packed { u32x4 sm[4], l1, st; u64 mask; unsigned meta; };

__device__ __forceinline__ void pack(const u32x4 (&d)[8], Packed& o, LAS unsigned char* scr, unsigned lane) {
    *(LAS u32x4*)(scr + lane * 16) = (u32x4){0u, 0u, 0u, 0u};
    unsigned sum = 0; u32x4 er[4];
#pragma unroll
    for (int g = 0; g < 16; ++g) {
        const unsigned d0 = d[g >> 1][(g & 1) * 2], d1 = d[g >> 1][(g & 1) * 2 + 1];
        const unsigned lb = perm(d1, d0, 0x06040200u), hb = perm(d1, d0, 0x07050301u);
        o.sm[g >> 2][g & 3] = bfi(0x7f7f7f7fu, lb, hb);
        const unsigned e = bfi(0x01010101u, lb >> 7, hb << 1);
        er[g >> 2][g & 3] = e;
        sum = __builtin_amdgcn_sad_u8(e, 0u, sum);
    }
    const unsigned r2 = (((unsigned)__builtin_amdgcn_readlane(wave_scan(sum), 63) + 16036u) >> 12) & 255u;
    const unsigned kk = (r2 | 0x80u) * 0x01010101u, cc = (~r2 & 0x80u) * 0x01010101u;
    unsigned bad = 0, top = ~0u, longest = 0, total = 0, piece[8], plen[8], us[16], cs[16];
#pragma unroll
    for (int g = 0; g < 16; ++g) {
        const unsigned e = er[g >> 2][g & 3];
        const unsigned u = (kk - (e & 0x7f7f7f7fu)) ^ (e & 0x80808080u) ^ cc;
        bad |= u;
        us[g] = u; cs[g] = perm(0x07070706u, 0x05040703u, u | ((u << 1) & 0x10101010u));
        top &= cs[g];
    }
    const bool hole = __ballot((top & 0x04040404u) != 0x04040404u) != 0ull;
    const unsigned tx = hole ? 0xfdfdfdfdu : 0xfefefefeu, thi = 0x01000000u ^ tx, tlo = 0x01020609u ^ tx;
#pragma unroll
    for (int q = 0; q < 4; ++q) {
        unsigned l1 = 0, acc = 0, base = 0;
#pragma unroll
        for (int gg = 0; gg < 4; ++gg) {
            const unsigned u = us[q * 4 + gg], c = cs[q * 4 + gg];
            l1 |= (c & 0x03030303u) << (2 * gg);
            const unsigned len4 = (perm(thi, tlo, u | (c & 0x10101010u)) ^ tx) + u;
            const unsigned f = c & (c >> 1) & 0x01010101u;
            unsigned ps = len4 + (len4 << 8); asm("" : "+v"(ps)); ps += ps << 16; ps -= 0x100c0804u;
            const unsigned bits = spread_bits(f, ps);
            if ((gg & 1) == 0) { acc = bits; base = ps >> 24; }
            else {
                acc |= bits << (base & 31u); base += ps >> 24;
                piece[q * 2 + (gg >> 1)] = acc >> 1; plen[q * 2 + (gg >> 1)] = base;
                longest = max(longest, base); total += base;
            }
        }
        o.l1[q] = l1;
    }
    const bool raw = (bad & 0xe0e0e0e0u) != 0u || longest > 31u || total > 112u;
    o.mask = __ballot(raw);
    if (raw) total = 0;
    const unsigned incl = wave_scan(total);
    const unsigned rawoff = (((unsigned)__builtin_amdgcn_readlane(incl, 63) + 512u + 127u) >> 7) << 4;
    o.meta = r2 | ((rawoff + 64u * (unsigned)__builtin_popcountll(o.mask) + 127u) >> 7) << 8 | (hole ? 0x4000u : 0u);
    LDS_ORDER();
    scr[lane] = (unsigned char)total;
    if (!raw) {
        unsigned off = incl - total + 512u;
#pragma unroll
        for (int i = 0; i < 8; ++i) {
            const u64 w = (u64)piece[i] << (off & 31u);
            LAS unsigned* p = (LAS unsigned*)(scr + ((off >> 5) << 2));
            __hip_atomic_fetch_or(p, (unsigned)w, __ATOMIC_RELAXED, __HIP_MEMORY_SCOPE_WORKGROUP);
            __hip_atomic_fetch_or(p + 1, (unsigned)(w >> 32), __ATOMIC_RELAXED, __HIP_MEMORY_SCOPE_WORKGROUP);
            off += plen[i];
        }
    } else {
        LAS u32x4* p = (LAS u32x4*)(scr + rawoff + rank_in(o.mask) * 64u);
#pragma unroll
        for (int i = 0; i < 4; ++i) p[i] = er[i];
    }
    LDS_ORDER();
    o.st = *(const LAS u32x4*)(scr + lane * 16);
    LDS_ORDER();
}
__device__ __forceinline__ void send(const Packed& o, unsigned char* pl, const LAS unsigned char* scr, unsigned lane) {
    unsigned char* p = pl + lane * 16;
    for (unsigned k = 1; k * 8u < ((o.meta >> 8) & 63u); ++k)
        if (k * 64u + lane < ((o.meta >> 8) & 63u) * 8u) st16_sys(p + 5120 + k * 1024, *(const LAS u32x4*)(scr + k * 1024 + lane * 16));
#pragma unroll
    for (int i = 0; i < 4; ++i) st16_sys(p + i * 1024, o.sm[i]);
    st16_sys(p + 4096, o.l1);
    if (lane < ((o.meta >> 8) & 63u) * 8u) st16_sys(p + 5120, o.st);
}
__device__ __forceinline__ u32x4 header(const Packed& o, unsigned call) {
    return (u32x4){(unsigned)o.mask, call, (unsigned)(o.mask >> 32), (call << 15) | o.meta};
}
__device__ __forceinline__ bool header_ok(u32x4 h, unsigned call, u64& mask, unsigned& meta) {
    const unsigned t1 = __builtin_amdgcn_readfirstlane(h.y), t3 = __builtin_amdgcn_readfirstlane(h.w);
    mask = ((u64)(unsigned)__builtin_amdgcn_readfirstlane(h.z) << 32) | (unsigned)__builtin_amdgcn_readfirstlane(h.x);
    meta = t3 & 0x7fffu;
    return t1 == call && (t3 >> 15) == (call & 0x1ffffu);
}
__device__ __forceinline__ void wait_header(const unsigned char* hd, unsigned char* tmo, unsigned call, u64& mask, unsigned& meta) {
    const u64 t0 = __builtin_amdgcn_s_memrealtime();
    for (;;) {
        u32x4 h; unsigned t;
        asm volatile("global_load_dwordx4 %0, %2, off sc0 sc1\n\t"
                     "global_load_dword %1, %3, off sc1\n\t"
                     "s_waitcnt vmcnt(0)" : "=&v"(h), "=&v"(t) : "v"(hd), "v"(tmo) : "memory");
        if (header_ok(h, call, mask, meta)) break;
        if ((unsigned)__builtin_amdgcn_readfirstlane(t) == call || __builtin_amdgcn_s_memrealtime() - t0 > DEADLINE) {
            __hip_atomic_store((unsigned*)tmo, call, __ATOMIC_RELAXED, __HIP_MEMORY_SCOPE_AGENT);
            mask = 0ull; meta = 0u;
            break;
        }
        __builtin_amdgcn_s_sleep(HDR_SLEEP);
    }
}
__device__ __forceinline__ void fetch(Packed& o, const unsigned char* pl, unsigned lane) {
    const unsigned n = min(max((o.meta >> 8) & 63u, 1u), 8u) * 8u;
    const u64 smask = n >= 64u ? ~0ull : (1ull << n) - 1ull;
    u64 keep;
    asm volatile(SBASE "s_mov_b64 %6, exec\n\t"
                 "global_load_dwordx4 %0, %7, %8 sc0 sc1\n\t"
                 "global_load_dwordx4 %1, %7, %8 offset:1024 sc0 sc1\n\t"
                 "global_load_dwordx4 %2, %7, %8 offset:2048 sc0 sc1\n\t"
                 "global_load_dwordx4 %3, %7, %8 offset:3072 sc0 sc1\n\t"
                 "global_load_dwordx4 %4, %7, %9 sc0 sc1\n\t"
                 "s_mov_b64 exec, %10\n\t"
                 "global_load_dwordx4 %5, %7, %9 offset:1024 sc0 sc1\n\t"
                 "s_mov_b64 exec, %6"
                 : "=&v"(o.sm[0]), "=&v"(o.sm[1]), "=&v"(o.sm[2]), "=&v"(o.sm[3]), "=&v"(o.l1), "=&v"(o.st), "=&s"(keep)
                 : "v"(lane * 16u), "s"(pl), "s"(pl + 4096), "s"(smask) : "memory");
}
#define FETCH_WAIT(o, n) asm volatile("s_waitcnt vmcnt(" #n ")" : "+v"(o.sm[0]), "+v"(o.sm[1]), "+v"(o.sm[2]), "+v"(o.sm[3]), "+v"(o.l1), "+v"(o.st) :: "memory")
template <class Mid>
__device__ __forceinline__ void unpack(const Packed& o, u32x4 (&d)[8], const unsigned char* pl, LAS unsigned char* scr, unsigned lane, Mid mid) {
    *(LAS u32x4*)(scr + lane * 16) = o.st;
    if (((o.meta >> 8) & 63u) > 8u) {
        u32x4 m[4];
        asm volatile("global_load_dwordx4 %0, %4, off sc0 sc1\n\t"
                     "global_load_dwordx4 %1, %4, off offset:1024 sc0 sc1\n\t"
                     "global_load_dwordx4 %2, %4, off offset:2048 sc0 sc1\n\t"
                     "global_load_dwordx4 %3, %4, off offset:3072 sc0 sc1\n\t"
                     "s_waitcnt vmcnt(0)" : "=&v"(m[0]), "=&v"(m[1]), "=&v"(m[2]), "=&v"(m[3]) : "v"(pl + 6144 + lane * 16) : "memory");
#pragma unroll
        for (int k = 0; k < 4; ++k) *(LAS u32x4*)(scr + (k + 1) * 1024 + lane * 16) = m[k];
    }
    LDS_ORDER();
    const unsigned len = scr[lane] & 127u;
    const unsigned incl = wave_scan(len), off = incl - len + 512u;
    const LAS unsigned* p = (const LAS unsigned*)(scr + ((off >> 5) << 2));
    const unsigned w0 = p[0], w1 = p[1], w2 = p[2], w3 = p[3], w4 = p[4];
    unsigned s0 = __builtin_amdgcn_alignbit(w1, w0, off), s1 = __builtin_amdgcn_alignbit(w2, w1, off);
    unsigned s2 = __builtin_amdgcn_alignbit(w3, w2, off), s3 = __builtin_amdgcn_alignbit(w4, w3, off);
    const bool hole = (o.meta & 0x4000u) != 0u;
    const unsigned tx = hole ? 0xf8f8f8f8u : 0xf7f7f7f7u, thi = (hole ? 0x07070008u : 0x08080808u) ^ tx, tlo = 0x0309090au ^ tx;
    const unsigned r7 = (o.meta + 8u) & 255u;
    const unsigned kk = (r7 | 0x80u) * 0x01010101u, cc = (~r7 & 0x80u) * 0x01010101u;
    u32x4 er[4];
#pragma unroll
    for (int q = 0; q < 4; ++q) {
        if (q == MID_Q) { asm volatile("" : "+v"(s0)); mid(); asm volatile("" : "+v"(s0)); }
        const unsigned l1 = o.l1[q], x = l1 & (l1 >> 1);
        unsigned w = 0, used = 0;
#pragma unroll
        for (int gg = 0; gg < 4; ++gg) {
            if ((gg & 1) == 0) { w = (s0 << 1) | 1u; used = 0; }
            const unsigned f = (x >> (2 * gg)) & 0x01010101u;
            const unsigned ln = take_codes(w, f);
            used = __builtin_amdgcn_sad_u8(ln, 0u, used);
            const unsigned y = (perm(thi, tlo, ln | ((ln << 1) & 0x10101010u)) ^ tx) + ln + ((l1 >> (2 * gg)) & 0x03030303u);
            er[q][gg] = (kk - y) ^ cc;
            if (gg & 1) {
                s0 = __builtin_amdgcn_alignbit(s1, s0, used); s1 = __builtin_amdgcn_alignbit(s2, s1, used);
                s2 = __builtin_amdgcn_alignbit(s3, s2, used); s3 >>= used & 31u;
            }
        }
    }
    if (o.mask != 0ull) {
        const unsigned rawoff = (((unsigned)__builtin_amdgcn_readlane(incl, 63) + 512u + 127u) >> 7) << 4;
        if ((o.mask >> lane) & 1ull) {
            const LAS u32x4* r = (const LAS u32x4*)(scr + rawoff + rank_in(o.mask) * 64u);
#pragma unroll
            for (int i = 0; i < 4; ++i) er[i] = r[i];
        }
    }
    LDS_ORDER();
#pragma unroll
    for (int g = 0; g < 16; ++g) {
        const unsigned e = er[g >> 2][g & 3], sm = o.sm[g >> 2][g & 3];
        const unsigned hb = bfi(0x80808080u, sm, e >> 1), lb = bfi(0x7f7f7f7fu, sm, e << 7);
        d[g >> 1][(g & 1) * 2] = perm(hb, lb, 0x05010400u); d[g >> 1][(g & 1) * 2 + 1] = perm(hb, lb, 0x07030602u);
    }
}

__device__ __forceinline__ void load_parked(u32x4 (&dg)[8], unsigned& tw, const unsigned char* dp, const unsigned char* tmo, unsigned lane) {
    asm volatile(SBASE "global_load_dwordx4 %0, %9, %10 sc0 sc1\n\t"
                 "global_load_dwordx4 %1, %9, %10 offset:1024 sc0 sc1\n\t"
                 "global_load_dwordx4 %2, %9, %10 offset:2048 sc0 sc1\n\t"
                 "global_load_dwordx4 %3, %9, %10 offset:3072 sc0 sc1\n\t"
                 "global_load_dwordx4 %4, %9, %11 sc0 sc1\n\t"
                 "global_load_dwordx4 %5, %9, %11 offset:1024 sc0 sc1\n\t"
                 "global_load_dwordx4 %6, %9, %11 offset:2048 sc0 sc1\n\t"
                 "global_load_dwordx4 %7, %9, %11 offset:3072 sc0 sc1\n\t"
                 "global_load_dword %8, %8, %12 sc1"
                 : "=&v"(dg[0]), "=&v"(dg[1]), "=&v"(dg[2]), "=&v"(dg[3]), "=&v"(dg[4]), "=&v"(dg[5]), "=&v"(dg[6]), "=&v"(dg[7]), "+v"(tw)
                 : "v"(lane * 16u), "s"(dp), "s"(dp + 4096), "s"(tmo) : "memory");
}
#define PARKED_WAIT(dg, tw) asm volatile("s_waitcnt vmcnt(0)" : "+v"(dg[0]), "+v"(dg[1]), "+v"(dg[2]), "+v"(dg[3]), "+v"(dg[4]), "+v"(dg[5]), \
                                         "+v"(dg[6]), "+v"(dg[7]), "+v"(tw) :: "memory")

#define STAMP(i) do {} while (0)

#define ORD_A1 1.17
#define ORD_AL 0.0
#define ORD_B0 0.4
#define ORD_BS 1.22
#define ORD_C0 10.5
#define ORD_CS 0.3
#define ORD_W 0.3
struct Order { unsigned short v[354]; };
constexpr Order make_order() {
    double key[352] = {}; unsigned short id[352] = {}; int n = 0;
    for (int kc = 0; kc < 8; ++kc)
        for (int h = 0; h < 2; ++h) {
            for (int e = 0; e < 8; ++e) { const double ka = kc - ORD_W + (e + 0.4) / 8, kb = ORD_B0 + ka * ORD_BS; key[n] = (ka < kb ? ka : kb) + h * 1e-3 - (kc == 0 && e < PLAIN ? 9 : 0); id[n++] = (unsigned short)(h << 8 | 0 << 6 | kc << 3 | e); }
            for (int e = 0; e < 6; ++e) { key[n] = (kc + (e + 0.5) / 6) * (h ? ORD_A1 : 1.0) + (h ? ORD_AL : 0.0); id[n++] = (unsigned short)(h << 8 | 1 << 6 | kc << 3 | e); }
            for (int e = 0; e < 6; ++e) { key[n] = ORD_B0 + (kc + (e + 0.6) / 6) * ORD_BS + h * 1e-3; id[n++] = (unsigned short)(h << 8 | 2 << 6 | kc << 3 | e); }
            for (int e = 0; e < 2; ++e) { key[n] = ORD_C0 + (2 * kc + e + 0.5 * h) * ORD_CS; id[n++] = (unsigned short)(h << 8 | 3 << 6 | kc << 3 | e); }
        }
    Order o = {};
    for (int i = 0; i < 352; ++i) { int r = 0; for (int j = 0; j < 352; ++j) r += (key[j] < key[i]) || (key[j] == key[i] && j < i); o.v[r] = id[i]; }
    return o;
}
constexpr Order ORDER = make_order();
constexpr unsigned order_word(int i) { return ORDER.v[3 * i] | ORDER.v[3 * i + 1] << 10 | (unsigned)ORDER.v[3 * i + 2] << 20; }
template <int I, int N> __device__ __forceinline__ unsigned order_vgpr(unsigned v) {
    if constexpr (I == N) return v;
    else {
        unsigned t;
        asm("s_mov_b32 %1, %2\n\tv_writelane_b32 %0, %1, %3" : "+v"(v), "=&s"(t) : "i"(order_word(I)), "i"(I & 63));
        return order_vgpr<I + 1, N>(v);
    }
}
__device__ __forceinline__ void decode_unit(unsigned ord0, unsigned ord1, int t, int src, int dest, int& isw, int& kc, int& tile, int& q) {
    const int n = t >> 1, dw = n / 3;
    const unsigned w0 = __builtin_amdgcn_readlane(ord0, dw & 63), w1 = __builtin_amdgcn_readlane(ord1, dw & 63);
    const unsigned v = (dw < 64 ? w0 : w1) >> ((n - dw * 3) * 10);
    const int kind = (v >> 6) & 3, e = v & 7;
    kc = (v >> 3) & 7; isw = kind == 0; q = ((v >> 7) & 2) | (t & 1);
    const int d = (dest - src - 1) & 7;
    tile = kind == 0 ? ((e + src) & 7) : kind == 3 ? 2 * dest + e : 2 * ((src + 1 + e + (e >= d)) & 7) + (kind - 1);
}

__host__ __device__ __forceinline__ void role_of(int b, int& nt, int& slot, int& g) {
    constexpr int NG = 8 / NTX, X0 = 8 - NG;
    const int x = b & 7, s = b >> 3, np = x < X0 ? 8 : 8 - NTX;
    if (s < np) {
        const int pid = s < 8 - NTX ? s * 8 + x : (8 - NTX) * 8 + (s - (8 - NTX)) * X0 + x;
        slot = NTX == 1 ? pid >> 3 : pid % 7; nt = NTX == 1 ? pid & 7 : pid / 7; g = -1;
    } else {
        const int j = s - np, r = j / NTX + x / NG * NG * 3;
        nt = x % NG * NTX + j % NTX; slot = 0;
        g = r >= 21 ? r : r % 3 == 2 ? 14 + r / 3 : r / 3 * 2 + r % 3;
    }
}

__global__ __launch_bounds__(512, 2) void fused_gemm_rs(const Params P) {
    __shared__ __attribute__((aligned(16))) unsigned char lds_raw[9 * HTB + 256];
    LAS unsigned char* lds = (LAS unsigned char*)lds_raw;
    const int tid = threadIdx.x, wid = __builtin_amdgcn_readfirstlane(tid >> 6), lane = tid & 63;
    const int b = blockIdx.x, self = P.rank;
    unsigned char* const ws = P.own;
    const GAS unsigned char *px = (const GAS unsigned char*)P.x, *pw = (const GAS unsigned char*)P.w;
    asm volatile("" : "+s"(px), "+s"(pw));
    unsigned count; const unsigned vzero = 0u;
    asm volatile(SBASE "global_load_dword %0, %1, %2 sc1" : "=&v"(count) : "v"(vzero), "s"(ws + OFF_CNT + b * 128) : "memory");
    unsigned char* const tmo = ws + OFF_TMO;
    LAS unsigned char* const scr = lds + wid * SCR;
    STAMP(0);
    const u64 tstart = __builtin_amdgcn_s_memrealtime();
    int nt, slot, g;
    role_of(b, nt, slot, g);
    const bool is_pull = g < 0, is_a = !is_pull && g < 14, is_b = g >= 14 && g < 21, is_c = g >= 21;
    const int owner = is_c ? self : (self + 1 + (is_a ? g >> 1 : g - 14)) & 7;
    const int mt = 2 * owner + (is_c ? (g - 21) >> 1 : is_b), hf = is_c ? (g - 21) & 1 : g & 1;
    auto pack_decode = [&](int t, int& isw, int& kc, int& tile) {
        const int blk = t >> 2;
        if (blk < 80) {
            const int m = blk - 16; kc = 0; isw = blk >= 16;
            tile = isw ? ((self + 1 + m) & 7) * 8 + (((m >> 3) + self) & 7) : blk;
        } else if (blk < 248) {
            const int m = blk - 80, r = m % 24; kc = 1 + m / 24; isw = r >= 16;
            tile = isw ? self * 8 + (r & 7) : r;
        } else if (blk < 640) {
            const int m = blk - 248, w = m % 56; isw = 1; kc = 1 + m / 56;
            tile = ((self + 1 + w % 7) & 7) * 8 + ((w / 7 + self) & 7);
        } else {
            const int m = blk - 640; isw = 1; kc = 0;
            tile = ((self + 1 + (m >> 3)) & 7) * 8 + (m & 7);
        }
    };
    const unsigned lw = (lane >> 4) * 8 * (GN * 2) + (lane & 15) * 16, lx = (lane & 15) * (KS * 2) + (lane >> 4) * 16;
    auto load_task = [&](int t, u32x4 (&r)[8]) {
        int isw, kc, tile; pack_decode(t, isw, kc, tile); const int q = t & 3;
        const GAS unsigned char* src = isw ? pw + ((kc * 64 + (q & 1) * 32) * (GN * 2) + (tile * 256 + (q >> 1) * 128) * 2)
                                       : px + ((tile * 256 + q * 64) * (KS * 2) + kc * 128);
        const unsigned lo = isw ? lw : lx, s1 = isw ? GN * 2 : 64, s2 = isw ? GN * 4 : 16 * KS * 2;
#pragma unroll
        for (int i = 0; i < 8; ++i) r[i] = *(const GAS u32x4*)(src + ((i >> 1) * s2 + (i & 1) * s1) + lo);
    };
    auto store_image = [&](unsigned char* blk, int isw, int q, const u32x4 (&c)[8]) {
        unsigned char* d0 = blk + (isw ? (q >> 1) * 8192 + (q & 1) * 1024 : q * 8192) + lane * 16;
        const unsigned s1 = isw ? 2048 : 1024, s4 = isw ? HTB : 4096;
#pragma unroll
        for (int j = 0; j < 8; ++j) st16_sys(d0 + ((j >> 2) * s4 + (j & 3) * s1), c[j]);
    };
    u32x4 cur[8];
    const int task0 = is_pull ? wid * 8 + nt : wid * 144 + (is_a ? g : g - 7) * 8 + nt;
    if (is_a || is_c) {
        if (wid >= HEAD_W) {
            while (__builtin_amdgcn_s_memrealtime() - tstart < PACK_DELAY) __builtin_amdgcn_s_sleep(2); }
        else __builtin_amdgcn_s_setprio(2);
        load_task(task0, cur);
    }
    unsigned peer_lo = 0u, peer_hi = 0u;
#pragma unroll
    for (int r = 0; r < NR; ++r)
        asm("v_writelane_b32 %0, %2, %4\n\tv_writelane_b32 %1, %3, %4" : "+v"(peer_lo), "+v"(peer_hi)
            : "s"((unsigned)(u64)P.ws[r]), "s"((unsigned)((u64)P.ws[r] >> 32)), "i"(r));
    asm volatile("s_waitcnt vmcnt(0)" : "+v"(count) :: "memory");
    const unsigned call = (unsigned)__builtin_amdgcn_readfirstlane(count) + 1u;
    const size_t par = call & 1u;
    const unsigned tag4 = (call & 255u) * 0x01010101u;
    const int wr = wid >> 2, wc = wid & 3;

#define READY0 0
    if (is_pull) {
        LAS unsigned* const ctl = (LAS unsigned*)(lds + 9 * HTB + 64);
        if (tid < 4) ctl[tid] = 0u;
        __syncthreads();
        const unsigned share = (WIN + 7 - nt) / 8;
        auto lds_draw = [&](int w) {
            unsigned v = 0;
            if (lane == 0) v = __hip_atomic_fetch_add(&ctl[w], 1u, __ATOMIC_RELAXED, __HIP_MEMORY_SCOPE_WORKGROUP);
            return (unsigned)__builtin_amdgcn_readfirstlane(v);
        };
        auto acquire = [&]() {
            const unsigned turn = lds_draw(0);
            while (__hip_atomic_load(&ctl[1], __ATOMIC_RELAXED, __HIP_MEMORY_SCOPE_WORKGROUP) + share <= turn) __builtin_amdgcn_s_sleep(ACQ_SLEEP);
        };
        auto release = [&]() { if (lane == 0) __hip_atomic_fetch_add(&ctl[1], 1u, __ATOMIC_RELAXED, __HIP_MEMORY_SCOPE_WORKGROUP); };
        const int src = (self + 1 + slot) & 7;
        const unsigned char* const sws = P.ws[src];
        if (wid < PWV) {
            const unsigned ord0 = order_vgpr<0, 64>(0u), ord1 = order_vgpr<64, 118>(0u);
            unsigned* const tick = (unsigned*)(ws + OFF_TICK + src * 128);
            const unsigned tbase = (call - 1u) * NTASK - PW;
            const unsigned char* const hbase = ws + OFF_HIN + (par * 8 + src) * (768 * HS);
            int task = task0, isw, kc, tile, q;
            decode_unit(ord0, ord1, task, src, self, isw, kc, tile, q);
            Packed pk;
            unsigned second = 0u;
            if (task < 4 * PLAIN && lane == 0) second = __hip_atomic_fetch_add(tick, 1u, __ATOMIC_RELAXED, __HIP_MEMORY_SCOPE_AGENT);
            second = __builtin_amdgcn_readfirstlane(second);
            while (__builtin_amdgcn_s_memrealtime() - tstart < PULL_DELAY) __builtin_amdgcn_s_sleep(2);
            wait_header(hbase + ((isw ? kc * 8 + tile : 64 + kc * 16 + tile) * 4 + q) * HS, tmo, call, pk.mask, pk.meta);
            if (task < 4 * PLAIN) {
                unsigned tw = 0u;
                acquire();
                load_parked(cur, tw, sws + OFF_RP + (par * 2560 + (self * 8 + tile) * 32 + q) * UNIT, tmo, lane);
                PARKED_WAIT(cur, tw);
                release();
                store_image(ws + OFF_WB + (size_t)(src * 64 + tile) * BLK, 1, q, cur);
                asm volatile("s_waitcnt vmcnt(0)" ::: "memory");
                if (lane < RW) st_flag8(ws + OFF_FW + (tile * 64 + src * 8) * 4 + q + lane * FR, call);
                task = (int)(second - tbase);
                decode_unit(ord0, ord1, task < NTASK ? task : 0, src, self, isw, kc, tile, q);
                wait_header(hbase + ((isw ? kc * 8 + tile : 64 + kc * 16 + tile) * 4 + q) * HS, tmo, call, pk.mask, pk.meta);
            }
            while (task < NTASK) {
                const unsigned char* const pl = sws + OFF_RP + (par * 2560 + (isw ? (self * 8 + tile) * 8 + kc : 512 + kc * 16 + tile) * 4 + q) * UNIT;
                acquire();
                fetch(pk, pl, lane);
                FETCH_WAIT(pk, 0);
                release();
                unsigned nn = 1u;
                {
                    u64 keep; const unsigned zero = 0u;
                    asm volatile(SBASE "s_mov_b64 %1, exec\n\ts_mov_b64 exec, 1\n\t"
                                 "global_atomic_add %0, %2, %0, %3 sc0\n\t"
                                 "s_mov_b64 exec, %1" : "+v"(nn), "=&s"(keep) : "v"(zero), "s"(tick) : "memory");
                }
                int ntask, nisw, nkc, ntile, nq; u32x4 h; const unsigned char* hd;
                unpack(pk, cur, pl, scr, lane, [&]() {
                    asm volatile("s_waitcnt vmcnt(0)" : "+v"(nn) :: "memory");
                    ntask = (int)(__builtin_amdgcn_readfirstlane(nn) - tbase);
                    decode_unit(ord0, ord1, ntask < NTASK ? ntask : 0, src, self, nisw, nkc, ntile, nq);
                    hd = hbase + ((nisw ? nkc * 8 + ntile : 64 + nkc * 16 + ntile) * 4 + nq) * HS;
                    const unsigned zero = 0u;
                    asm volatile(SBASE "global_load_dwordx4 %0, %1, %2 sc0 sc1" : "=&v"(h) : "v"(zero), "s"(hd) : "memory");
                });
                const int idx = src * 8 + kc;
                store_image(isw ? ws + OFF_WB + (size_t)(idx * 8 + tile) * BLK : ws + OFF_XA + (size_t)(idx * 16 + tile) * BLK, isw, q, cur);
                asm volatile("s_waitcnt vmcnt(0)" : "+v"(h) :: "memory");
                if (lane < (isw ? RW : RX)) st_flag8(ws + (isw ? OFF_FW : OFF_FX) + (tile * 64 + idx) * 4 + q + lane * FR, call);
                if (ntask >= NTASK) break;
                if (!header_ok(h, call, pk.mask, pk.meta)) wait_header(hd, tmo, call, pk.mask, pk.meta);
                task = ntask; isw = nisw; kc = nkc; tile = ntile; q = nq;
            }
            if (lane == 0) __hip_atomic_store(&ctl[3], 1u, __ATOMIC_RELAXED, __HIP_MEMORY_SCOPE_WORKGROUP);
        } else {
            while (__hip_atomic_load(&ctl[3], __ATOMIC_RELAXED, __HIP_MEMORY_SCOPE_WORKGROUP) == 0u) __builtin_amdgcn_s_sleep(8);
        }
        STAMP(1);
        const int gnt = src * 8 + nt, fr = lane & 15, fq = lane >> 4;
        auto head_of = [&](int u) { return ws + OFF_HP + (size_t)(((((u >> 4) & 1) * 64 + gnt) * 8 + (u & 7)) * 2 + ((u >> 3) & 1)) * HS; };
        u32x4 h = {0u, 0u, 0u, 0u};
        for (int u = (int)lds_draw(2); u < 32;) {
            const int odd = u >> 4, bj = (u >> 3) & 1, uw = u & 7, dt = (slot * 2 + odd) * 8 + nt;
            Packed pk;
            if (!header_ok(h, call, pk.mask, pk.meta)) wait_header(head_of(u), tmo, call, pk.mask, pk.meta);
            const unsigned char* pl = sws + OFF_PP + (size_t)((((2 * self + odd) * 8 + nt) * 8 + uw) * 2 + bj) * UNIT;
            acquire();
            const unsigned char* const fp = ws + OFF_FP + ((dt * 8 + uw) * 2 + bj) * 4;
            unsigned pf;
            asm volatile("global_load_dword %0, %1, off sc0 sc1" : "=&v"(pf) : "v"(fp) : "memory");
            fetch(pk, pl, lane);
            FETCH_WAIT(pk, 0);
            release();
            asm volatile("" : "+v"(pf));
            if ((unsigned)__builtin_amdgcn_readfirstlane(pf) < call) wait_word<false>(fp, call, tmo, call);
            u32x4 dg[8]; unsigned tw = 0u;
            load_parked(dg, tw, ws + OFF_PD + (size_t)((dt * 8 + uw) * 2 + bj) * 8192, tmo, lane);
            unpack(pk, cur, pl, scr, lane, [&]() {
                u = (int)lds_draw(2);
                const unsigned zero = 0u;
                asm volatile(SBASE "global_load_dwordx4 %0, %1, %2 sc0 sc1" : "=&v"(h) : "v"(zero), "s"(head_of(u)) : "memory");
            });
#pragma unroll
            for (int i = 0; i < 8; ++i) asm volatile("" : "+v"(cur[i]));
            PARKED_WAIT(dg, tw);
            asm volatile("" : "+v"(h));
            const bool bad = (unsigned)__builtin_amdgcn_readfirstlane(tw) == call;
            unsigned char* dst = P.out + (size_t)(odd * 256 + bj * 128 + (uw & 3) * 32 + fq * 4) * (GN * 2) + (size_t)(gnt * 256 + (uw >> 2) * 128 + fr * 8) * 2;
#pragma unroll
            for (int n = 0; n < 2; ++n)
#pragma unroll
                for (int r = 0; r < 4; ++r) {
                    u32x4 v;
#pragma unroll
                    for (int j = 0; j < 4; ++j) {
                        const unsigned pu = cur[n * 4 + r][j], w = dg[n * 4 + r][j];
                        const f32x2 y = (f32x2){__uint_as_float(w << 16), __uint_as_float(w & 0xffff0000u)}
                                      + (f32x2){__uint_as_float(pu << 16), __uint_as_float(pu & 0xffff0000u)};
                        const f32x2 o = gelu2(y);
                        v[j] = cvt_pk_bf16(o.x, o.y);
                    }
                    if (bad) v = (u32x4){0x7fc07fc0u, 0x7fc07fc0u, 0x7fc07fc0u, 0x7fc07fc0u};
                    st16_sys(dst + (size_t)(n * 16 + r) * (GN * 2), v);
                }
        }
        if (tid == 0) __hip_atomic_store((unsigned*)(ws + OFF_CNT + b * 128), call, __ATOMIC_RELAXED, __HIP_MEMORY_SCOPE_AGENT);
        STAMP(5);
        return;
    }

    if (!is_b) {
        unsigned char* const hdst = (unsigned char*)(((u64)peer_hi << 32) | peer_lo) + OFF_HIN + (par * 8 + self) * (768 * HS);
        for (int t = task0;;) {
#pragma unroll
            for (int i = 0; i < 8; ++i) asm volatile("" : "+v"(cur[i]));
            const int tn = t < (PK_ALL - 1) * 1152 ? t + 1152 : t >= PK_ALL * 1152 ? t + 704 : PK_ALL * 1152 + wid * 88 + (is_a ? g >> 1 : g - 14) * 8 + nt;
            const bool more = tn < 2784 && (tn < PK_ALL * 1152 || is_c || hf);
            int isw, kc, tile; pack_decode(t, isw, kc, tile); const int q = t & 3, idx = self * 8 + kc;
            if (isw) {
                u32x4 tr[8];
#pragma unroll
                for (int j = 0; j < 8; ++j)
#pragma unroll
                    for (int qd = 0; qd < 4; ++qd)
                        tr[j][qd] = perm(cur[2 * qd + 1][j >> 1], cur[2 * qd][j >> 1], (j & 1) ? 0x07060302u : 0x05040100u);
#pragma unroll
                for (int j = 0; j < 8; ++j) cur[j] = tr[j];
            }
            const bool own = (isw ? tile >> 3 : tile >> 1) == self, diag = t >= 992;
            const bool packed = !own && t < 2560, image = !isw || own || diag;
            u32x4 hd = {0u, 0u, 0u, 0u};
            if (packed && t >= 64 && t < 64 + 32 * PLAIN) {
                unsigned char* pl = ws + OFF_RP + (par * 2560 + tile * 32 + q) * UNIT + lane * 16;
#pragma unroll
                for (int i = 0; i < 8; ++i) st16_sys(pl + i * 1024, cur[i]);
                hd = (u32x4){0u, call, 0u, call << 15};
            } else if (packed) {
                Packed pk; pack(cur, pk, scr, lane);
                send(pk, ws + OFF_RP + (par * 2560 + (isw ? tile * 8 + kc : 512 + kc * 16 + tile) * 4 + q) * UNIT, scr, lane);
                hd = header(pk, call);
            }
            if (image)
                store_image(ws + (diag ? OFF_WD + (size_t)(tile * 8 + kc) * BLK : isw ? OFF_WB + (size_t)(idx * 8 + (tile & 7)) * BLK
                                                                                     : OFF_XA + (size_t)(idx * 16 + tile) * BLK), isw, q, cur);
            load_task(more ? tn : t, cur);
            asm volatile("s_waitcnt vmcnt(8)" ::: "memory");
            if (packed && lane < NR && lane != self && (!isw || lane == (tile >> 3)))
                st16_sys(hdst + ((isw ? kc * 8 + (tile & 7) : 64 + kc * 16 + tile) * 4 + q) * HS, hd);
            if (image && lane < (diag ? 1 : isw ? RW : RX))
                st_flag8(ws + (diag ? OFF_FD + (tile * 8 + kc) * 4 : isw ? OFF_FW + ((tile & 7) * 64 + idx) * 4 : OFF_FX + (tile * 64 + idx) * 4) + q + lane * FR, call);
            if (!isw && tile == 2 * self + 1 && lane == 0) st_flag8(ws + OFF_FQ + kc * 4 + q, call);
            __builtin_amdgcn_s_setprio(0);
            if (!more) break;
            t = tn;
        }
        STAMP(6);
    }

    const unsigned foff = lane * 4;
    const unsigned pickoff = (unsigned)(size_t)(lds + 9 * HTB);
    const int fxc = (RX > 8 ? nt * 2 + hf : nt % RX) * FR, fwc = (g % RW) * FR;
    const unsigned lds4 = (unsigned)(size_t)lds + wid * 4096, lds2 = (unsigned)(size_t)lds + wid * 2048;
    const unsigned voff4 = wid * 4096 + lane * 16, voff2 = wid * 2048 + lane * 16;
#define GL4(bufoff, vo, gbase) asm volatile("s_add_i32 m0, %0, %1\n\ts_nop 3\n\tglobal_load_lds_dwordx4 %2, %3 sc0 sc1\n\t" \
        "global_load_lds_dwordx4 %2, %3 offset:1024 sc0 sc1\n\tglobal_load_lds_dwordx4 %2, %3 offset:2048 sc0 sc1\n\t" \
        "global_load_lds_dwordx4 %2, %3 offset:3072 sc0 sc1" :: "s"(lds4), "i"(bufoff), "v"(vo), "s"(gbase) : "memory", "scc")
#define GL2(bufoff, vo, gbase) asm volatile("s_add_i32 m0, %0, %1\n\ts_nop 3\n\tglobal_load_lds_dwordx4 %2, %3 sc0 sc1\n\t" \
        "global_load_lds_dwordx4 %2, %3 offset:1024 sc0 sc1" :: "s"(lds2), "i"(bufoff), "v"(vo), "s"(gbase) : "memory", "scc")
#define LIKELY(c) __builtin_expect(!!(c), 1)
#define RARE(c) __builtin_expect(!!(c), 0)
#define LDA(dst, base, off) do { _Pragma("unroll") for (int m = 0; m < 4; ++m) _Pragma("unroll") for (int k = 0; k < 2; ++k) \
        dst[m][k] = *(const LAS bf16x8*)(lds + (base) + ((off) + m * 2048 + k * 1024)); } while (0)
#define LDB(dst, base, off) do { _Pragma("unroll") for (int n = 0; n < 2; ++n) _Pragma("unroll") for (int k = 0; k < 2; ++k) \
        dst[n][k] = *(const LAS bf16x8*)(lds + (base) + ((off) + n * 2048 + k * 1024)); } while (0)
#define MMA(ai, bj, At, Bt) do { __builtin_amdgcn_s_setprio(1); _Pragma("unroll") for (int m = 0; m < 4; ++m) _Pragma("unroll") for (int n = 0; n < 2; ++n) _Pragma("unroll") for (int k = 0; k < 2; ++k) \
        acc[ai][bj][m][n] = __builtin_amdgcn_mfma_f32_16x16x32_bf16(Bt[n][k], At[m][k], acc[ai][bj][m][n], 0, 0, 0); __builtin_amdgcn_s_setprio(0); } while (0)
#define MMQ(ai, bj, At, Bt) do { _Pragma("unroll") for (int k = 0; k < 2; ++k) _Pragma("unroll") for (int m = 0; m < 4; ++m) _Pragma("unroll") for (int n = 0; n < 2; ++n) \
        acc[ai][bj][m][n] = __builtin_amdgcn_mfma_f32_16x16x32_bf16(Bt[n][k], At[m][k], acc[ai][bj][m][n], 0, 0, 0); SCHED; } while (0)
#define WAIT_V(n) asm volatile("s_waitcnt vmcnt(" #n ")" ::: "memory")
#define WAIT_L0 do { asm volatile("" ::: "memory"); __builtin_amdgcn_s_waitcnt(0xC07F); asm volatile("" ::: "memory"); } while (0)
#define BAR __builtin_amdgcn_s_barrier()
#define SCHED __builtin_amdgcn_sched_barrier(0)
    typedef unsigned u32x2 __attribute__((ext_vector_type(2)));
#define SNAP() asm volatile(SBASE "global_load_dword %0, %2, %4 sc0 sc1\n\tglobal_load_dword %1, %3, %4 sc0 sc1" : "+v"(sna), "+v"(snb) : "v"(fa), "v"(fb), "s"(ws) : "memory")
#define PUBLISH(m, w) asm volatile("ds_write_b64 %0, %1 offset:%2" :: "v"(pickoff), "v"((u32x2){(unsigned)(m), (unsigned)((m) >> 32)}), "i"(8 * (w)) : "memory")
#define LISTEN(w) asm volatile("ds_read_b64 %0, %1 offset:%2" : "=v"(mv) : "v"(pickoff), "i"(8 * (w)) : "memory"); SCHED
#define N0 asm volatile("s_waitcnt lgkmcnt(0)" : "+v"(mv) :: "memory"); m_ = ((u64)(unsigned)__builtin_amdgcn_readfirstlane(mv.y) << 32) | (unsigned)__builtin_amdgcn_readfirstlane(mv.x); asm volatile("" : "+s"(m_)); SCHED
#define N5 asm volatile("s_andn2_b64 %0, %0, %2\n\ts_andn2_b64 %1, %1, %2" : "+s"(todo), "+s"(avail) : "s"(m_) : "scc"); SCHED
#define MF(ai, bj, At, Bt, m, k) acc[ai][bj][m][0] = __builtin_amdgcn_mfma_f32_16x16x32_bf16(Bt[0][k], At[m][k], acc[ai][bj][m][0], 0, 0, 0); \
                                 acc[ai][bj][m][1] = __builtin_amdgcn_mfma_f32_16x16x32_bf16(Bt[1][k], At[m][k], acc[ai][bj][m][1], 0, 0, 0); SCHED
    const int w0 = wid == 0;
#define KSTAMP() do {} while (0)
    f32x4 acc[2][2][4][2];
    auto zero_acc = [&]() {
#pragma unroll
        for (int a = 0; a < 2; ++a)
#pragma unroll
            for (int c = 0; c < 2; ++c)
#pragma unroll
                for (int m = 0; m < 4; ++m)
#pragma unroll
                    for (int n = 0; n < 2; ++n) acc[a][c][m][n] = (f32x4){0.f, 0.f, 0.f, 0.f};
    };
    auto to_bf16 = [&](int c, u32x4 (&v)[8]) {
#pragma unroll
        for (int n = 0; n < 2; ++n)
#pragma unroll
            for (int r = 0; r < 4; ++r) {
                u32x4& o = v[n * 4 + r];
                o.x = cvt_pk_bf16(acc[0][c][0][n][r], acc[0][c][1][n][r]); o.y = cvt_pk_bf16(acc[0][c][2][n][r], acc[0][c][3][n][r]);
                o.z = cvt_pk_bf16(acc[1][c][0][n][r], acc[1][c][1][n][r]); o.w = cvt_pk_bf16(acc[1][c][2][n][r], acc[1][c][3][n][r]);
            }
    };
    auto emit_unit = [&](int c, int bj) {
        const int lz = lane_id();
        const unsigned char* const started = ws + OFF_FW + fwc + nt * 256 + owner * 32;
        unsigned og, ot;
        asm volatile("global_load_dword %0, %2, off sc0 sc1\n\tglobal_load_dword %1, %3, off sc1" : "=&v"(og), "=&v"(ot) : "v"(started), "v"(tmo) : "memory");
        u32x4 v[8];
        to_bf16(c, v);
        Packed pk;
        if (wid < 4) __builtin_amdgcn_s_setprio(2);
        pack(v, pk, scr, lz);
        __builtin_amdgcn_s_setprio(0);
        asm volatile("s_waitcnt vmcnt(0)" : "+v"(og), "+v"(ot) :: "memory");
        bool bad = (unsigned)__builtin_amdgcn_readfirstlane(ot) == call;
        if ((unsigned)__builtin_amdgcn_readfirstlane(og) != tag4) bad = wait_word<true>(started, tag4, tmo, call) || bad;
        if (bad) {
#pragma unroll
            for (int i = 0; i < 4; ++i) pk.sm[i] = (u32x4){0x40404040u, 0x40404040u, 0x40404040u, 0x40404040u};
            pk.l1 = pk.st = (u32x4){0u, 0u, 0u, 0u}; pk.mask = 0ull; pk.meta = 0x101u;
        }
        STAMP(4);
        send(pk, ws + OFF_PP + (size_t)(((mt * 8 + nt) * 8 + wid) * 2 + bj) * UNIT, scr, lz);
        asm volatile("s_waitcnt vmcnt(0)" ::: "memory");
        if (lz == 0) st16_sys(P.ws[owner] + OFF_HP + (size_t)((((mt & 1) * 64 + self * 8 + nt) * 8 + wid) * 2 + bj) * HS, header(pk, call));
    };
    auto finish = [&]() {
        if (threadIdx.x == 0) __hip_atomic_store((unsigned*)(ws + OFF_CNT + b * 128), call, __ATOMIC_RELAXED, __HIP_MEMORY_SCOPE_AGENT);
        STAMP(5);
    };
    auto park = [&](int c, int pt, int bj) {
        unsigned char* dp = ws + OFF_PD + (size_t)((pt * 8 + wid) * 2 + bj) * 8192 + lane_id() * 16;
        u32x4 v[8];
        to_bf16(c, v);
#pragma unroll
        for (int i = 0; i < 8; ++i) st16_sys(dp + i * 1024, v[i]);
    };
    const u64 help = (HELP_KC * 0x0101010101010101ull) & ~(0xffull << (self * 8));
    STAMP(2);
    if (!is_a) {
      const int nsteps = is_b ? 1 : 1 + (7 + C_ODD + 24 - g) / 4;
#pragma unroll 1
      for (int step = 0;; ++step) {
        const bool diag = step < nsteps - 1;
        const int dj = step * 4 + g - 21, dodd = dj >= 7, link = dj - 7 * dodd, dg = ((self + 1 + link) & 7) * 8 + nt, jm = diag ? 2 * self + dodd : mt;
                u64 todo = diag ? 0xffull << (self * 8) : is_b ? ~(0xffull << (owner * 8)) : (hf ? 0x55aa55aa55aa55aaull : 0xaa55aa55aa55aa55ull) & ~help, avail = todo & READY0;
        const int nkt = __builtin_popcountll(todo);
        const unsigned char* const abase = diag ? ws + OFF_WD + (size_t)(dg * 8 - self * 8) * BLK : ws + OFF_WB + (size_t)nt * BLK;
        const int ashift = diag ? 15 : 18;
        const unsigned fa = (unsigned)(diag ? OFF_FD + (dg * 8 - self * 8) * 4 : OFF_FW + fwc + nt * 256) + foff;
        const int x0 = (is_c && !diag && hf) ? HTB : 0;
        const unsigned voffx = voff4 ^ x0, voffh = voff2 + HTB;
        const unsigned char* const bbase = ws + OFF_XA + (size_t)jm * BLK;
        const unsigned fb = (unsigned)OFF_FX + fxc + jm * 256 + foff;
        const int aoff = wr * 8192 + lane * 16;
        int boff = 4 * HTB + wc * 4096 + lane * 16;
        asm volatile("" : "+v"(boff));
        zero_acc();
        bf16x8 At[4][2], B0[2][2], B1[2][2];
        const unsigned char *a2 = abase, *b2 = bbase;
        unsigned sna = ~tag4, snb = ~tag4; u32x2 mv; int look = w0;
        int vm = 0; const bool stampit = !diag; (void)stampit;
        int left = __builtin_amdgcn_readfirstlane(nkt);
        if (tid < 6) *(LAS unsigned*)(lds + 9 * HTB + tid * 4) = 0u;
        __syncthreads();
        if (wr == 1) BAR;
#define SA(bf, h) (((bf) * 2 + (h)) * HTB)
#define SB(bf, h) ((4 + (bf) * 2 + (h)) * HTB)
#define F1 asm volatile("s_and_b64 %1, %1, %2\n\ts_or_b64 %0, %0, %1" : "+s"(avail), "+s"(m_) : "s"(todo) : "scc"); SCHED
#define F2 asm volatile("s_ff1_i32_b64 %0, %2\n\ts_lshl_b64 %1, 1, %0" : "=&s"(i_), "=s"(m_) : "s"(avail) : "scc"); SCHED
#define F3 asm volatile("s_and_b64 %0, %0, %2\n\ts_cselect_b32 %1, 4, 0" : "+s"(m_), "=s"(vb_) : "s"(avail) : "scc"); SCHED
#define F6 a2 = abase + ((unsigned)i_ << ashift); asm volatile("" : "+s"(a2)); SCHED
#define F7 b2 = bbase + ((unsigned)i_ << 19); asm volatile("" : "+s"(b2)); SCHED
#define F8 asm volatile("s_bcnt1_i32_b64 %0, %1\n\ts_cmp_lt_u32 %0, %2\n\ts_cselect_b32 %0, %3, 0" : "=&s"(look) : "s"(avail), "i"(SNAP_AT), "s"(w0) : "scc"); \
           vm = (vm | vb_) >> 1; asm volatile("" : "+s"(vm)); SCHED
#define KTILE(bf) do { \
        LDB(B0, boff, SA(bf, 0)); LDB(B1, boff, SA(bf, 1)); LDA(At, aoff, SA(bf, 0)); SCHED; \
        if (LIKELY(vm & 2)) { GL2(SA(1 - bf, 1), voffh, a2); WAIT_V(8); } else WAIT_V(0); \
        if (RARE(look)) { asm volatile("" : "+v"(sna), "+v"(snb)); \
            u64 m = __ballot(sna == tag4 && snb == tag4); \
            if (vm == 0 && ((m | avail) & todo) == 0ull) m = poll_ready(todo, ws, fa, fb, tmo, call, tag4, 0xffffffffu); \
            PUBLISH(m, bf); } \
        int cur_ = vm & 1; asm volatile("" : "+s"(cur_)); __builtin_amdgcn_s_setprio(1); \
        WAIT_L0; BAR; SCHED; LISTEN(bf); \
        u64 m_; int i_, vb_; \
        if (LIKELY(cur_)) { \
            MF(0, 0, At, B0, 0, 0); MF(0, 0, At, B0, 1, 0); MF(0, 0, At, B0, 2, 0); MF(0, 0, At, B0, 3, 0); N0; \
            MF(0, 0, At, B0, 0, 1); F1; MF(0, 0, At, B0, 1, 1); F2; MF(0, 0, At, B0, 2, 1); F3; MF(0, 0, At, B0, 3, 1); N5; \
            MF(0, 1, At, B1, 0, 0); F6; MF(0, 1, At, B1, 1, 0); F7; MF(0, 1, At, B1, 2, 0); F8; MF(0, 1, At, B1, 3, 0); \
            MF(0, 1, At, B1, 0, 1); MF(0, 1, At, B1, 1, 1); MF(0, 1, At, B1, 2, 1); MF(0, 1, At, B1, 3, 1); } \
        asm volatile("" : "+s"(cur_)); \
        if (RARE(!cur_)) { N0; F1; F2; F3; N5; F6; F7; F8; } \
        BAR; __builtin_amdgcn_s_setprio(0); SCHED; \
        LDA(At, aoff, SA(bf, 1)); SCHED; \
        if (RARE(look)) SNAP(); \
        if (LIKELY(vm & 2)) { GL4(SB(bf, 0), voffx, b2); GL2(SA(bf, 0), voff2, a2); WAIT_V(8); } else WAIT_V(2); \
        left -= cur_; KSTAMP(); \
        asm volatile("" : "+s"(cur_), "+s"(left)); __builtin_amdgcn_s_setprio(1); \
        WAIT_L0; BAR; SCHED; if (LIKELY(cur_)) { MMQ(1, 0, At, B0); MMQ(1, 1, At, B1); } BAR; __builtin_amdgcn_s_setprio(0); SCHED; } while (0)
        for (;;) {
            KTILE(0);
            if (left == 0) break;
            KTILE(1);
            if (left == 0) break;
        }
        asm volatile("s_waitcnt vmcnt(0)\n\ts_nop 15\n\ts_nop 7" : "+v"(sna), "+v"(snb) :: "memory");
        if (wr == 0) BAR;
        if (!diag) break;
        park(0, link * 16 + dodd * 8 + nt, 0); park(1, link * 16 + dodd * 8 + nt, 1);
        asm volatile("s_waitcnt vmcnt(0)" ::: "memory");
        if (lane_id() < 2) st_flag(ws + OFF_FP + (((link * 16 + dodd * 8 + nt) * 8 + wid) * 2 + lane_id()) * 4, call);
        STAMP(7);
      }
      {
        if (is_b) {
            STAMP(3);
            emit_unit(0, 0); emit_unit(1, 1);
        } else {
            STAMP(3);
            const int lz = lane_id(), pt = 112 + (mt & 1) * 8 + nt;
            park(1, pt, 1 - hf);
            asm volatile("s_waitcnt vmcnt(0)" ::: "memory");
            if (lz == 0) st_flag(ws + OFF_FP + ((pt * 8 + wid) * 2 + 1 - hf) * 4, call);
            wait_word<false>(ws + OFF_FP + ((pt * 8 + wid) * 2 + hf) * 4, call, tmo, call);
            if (HELP_KC) wait_word<false>(ws + OFF_FP + (((pt + 16) * 8 + wid) * 2 + hf) * 4, call, tmo, call);
            u32x4 dg[8], dh[8]; unsigned tw = 0u, tv = 0u;
            load_parked(dg, tw, ws + OFF_PD + (size_t)((pt * 8 + wid) * 2 + hf) * 8192, tmo, lz);
            load_parked(dh, tv, ws + OFF_PD + (size_t)(((pt + 16) * 8 + wid) * 2 + hf) * 8192, tmo, lz);
            PARKED_WAIT(dg, tw); PARKED_WAIT(dh, tv);
            const bool bad = (unsigned)__builtin_amdgcn_readfirstlane(tw) == call;
            unsigned char* dst = P.out + (size_t)((mt & 1) * 256 + hf * 128 + wc * 32 + (lz >> 4) * 4) * (GN * 2) + (size_t)((self * 8 + nt) * 256 + wr * 128 + (lz & 15) * 8) * 2;
#pragma unroll
            for (int n = 0; n < 2; ++n)
#pragma unroll
                for (int r = 0; r < 4; ++r) {
                    u32x4 v;
#pragma unroll
                    for (int j = 0; j < 4; ++j) {
                        const unsigned w = dg[n * 4 + r][j], x = HELP_KC ? dh[n * 4 + r][j] : 0u;
                        const f32x2 o = gelu2((f32x2){acc[j >> 1][0][(j & 1) * 2][n][r], acc[j >> 1][0][(j & 1) * 2 + 1][n][r]}
                                              + (f32x2){__uint_as_float(w << 16), __uint_as_float(w & 0xffff0000u)}
                                              + (f32x2){__uint_as_float(x << 16), __uint_as_float(x & 0xffff0000u)});
                        v[j] = cvt_pk_bf16(o.x, o.y);
                    }
                    if (bad) v = (u32x4){0x7fc07fc0u, 0x7fc07fc0u, 0x7fc07fc0u, 0x7fc07fc0u};
                    st16_sys(dst + (size_t)(n * 16 + r) * (GN * 2), v);
                }
        }
      }
    } else {
      const int dg = owner * 8 + nt;
      const u64 dmask = 0xffull << (owner * 8);
      const bool mine = (g >> 1) >= C_ODD;
      u64 todo = mine ? ~0ull : ~dmask, avail = todo & READY0;
      const unsigned char* const abase = ws + OFF_WB + (size_t)nt * BLK;
      const unsigned char* const bbase = ws + OFF_XA + (size_t)mt * BLK + hf * HTB;
      const unsigned char* const adiag = ws + OFF_WD + (size_t)(dg * 8 - owner * 8) * BLK;
      const unsigned char* const bdiag = ws + OFF_XA + (size_t)(2 * self + 1) * BLK + hf * HTB + (size_t)(self - owner) * (8 * 16 * (size_t)BLK);
      const bool dl = (lane >> 3) == owner;
      unsigned fa = (unsigned)(dl ? OFF_FD + (dg * 8 - owner * 8) * 4 : OFF_FW + fwc + nt * 256) + foff;
      unsigned fb = (unsigned)(dl ? OFF_FQ - owner * 32 : OFF_FX + fxc + mt * 256) + foff;
      unsigned xm = hf ? 0xffff0000u : 0x0000ffffu;
      const int cm = 2 * self + (g >> 2), ch = (g >> 1) & 1;
      zero_acc();
#pragma unroll 1
      for (int step = 0;; ++step) {
        int aoff[3], boff[3];
#pragma unroll
        for (int i = 0; i < 3; ++i) {
            aoff[i] = i * 3 * HTB + wr * 8192 + lane * 16; boff[i] = i * 3 * HTB + 2 * HTB + wc * 4096 + lane * 16;
            asm volatile("" : "+v"(aoff[i]), "+v"(boff[i]));
        }
        bf16x8 A0[4][2], A1[4][2], B0[2][2];
        unsigned sna = ~tag4, snb = ~tag4; u32x2 mv;
        int vm = 0;
        int look = w0, more = 1;
        const unsigned char *a2 = ws, *b2 = ws;
        const u64 low = step ? 0ull : dmask;
        const unsigned char* const am = step == 1 ? adiag : abase;
        const unsigned char* const bm = step == 1 ? bdiag : step ? ws + OFF_XA + (size_t)cm * BLK + ch * HTB : bbase;
        const int shm = step == 1 ? 15 : 18, vbm = step == 1 ? 0x44 : 4;
        int mleft = __builtin_popcountll(todo & ~low);
        u64 fill = ~0ull;
        if (tid < 6) *(LAS unsigned*)(lds + 9 * HTB + tid * 4) = 0u;
        __syncthreads();
        if (wr == 1) BAR;
#define HSTAMP() do {} while (0)
#define SEGT(i) do {} while (0)
#define N1 asm volatile("s_and_b64 %1, %1, %2\n\ts_or_b64 %0, %0, %1\n\ts_and_b64 %1, %0, %3" : "+s"(avail), "+s"(m_) : "s"(todo), "s"(fill) : "scc"); SCHED
#define N2 asm volatile("s_andn2_b64 %0, %7, %8\n\ts_cselect_b64 %0, %0, %6\n\ts_cselect_b64 %1, %9, %10\n\ts_cselect_b64 %2, %11, %12\n\t" \
                        "s_cselect_b32 %3, %13, 15\n\ts_cselect_b32 %4, %14, 0x44\n\ts_subb_u32 %5, %5, 0" \
                        : "=&s"(c_), "=&s"(a2), "=&s"(b2), "=&s"(sh_), "=&s"(vb_), "+s"(mleft) \
                        : "s"(m_), "s"(avail), "s"(low), "s"(am), "s"(adiag), "s"(bm), "s"(bdiag), "s"(shm), "s"(vbm) : "scc"); SCHED
#define N3 asm volatile("s_ff1_i32_b64 %0, %2\n\ts_lshl_b64 %1, 1, %0" : "=&s"(i_), "=s"(m_) : "s"(c_) : "scc"); SCHED
#define N4 asm volatile("s_and_b64 %0, %0, %2\n\ts_cselect_b32 %1, %1, 0" : "+s"(m_), "+s"(vb_) : "s"(c_) : "scc"); SCHED
#define N6 vm = ((vm >> 1) & 0x33) | vb_; more = vm | mleft; asm volatile("" : "+s"(vm), "+s"(more)); SCHED
#define N7 a2 += (unsigned)i_ << sh_; asm volatile("" : "+s"(a2)); SCHED
#define N8 b2 += (unsigned)i_ << 19; asm volatile("" : "+s"(b2)); SCHED
#define N9 asm volatile("s_cmp_gt_i32 %3, %4\n\ts_cselect_b64 %0, -1, 0\n\ts_andn2_b64 %2, %5, %6\n\ts_bcnt1_i32_b64 %1, %2\n\ts_cmp_lt_u32 %1, %7\n\ts_cselect_b32 %1, %8, 0" \
                        : "=&s"(fill), "=&s"(look), "=&s"(c_) : "s"(mleft), "i"(FILL_STOP), "s"(avail), "s"(low), "i"(SNAP_H), "s"(w0) : "scc"); SCHED
#define MF2(ai, At, m, k) MF(ai, 0, At, B0, m, k)
#define HTILE(s) do { \
        constexpr int nb = ((s) + 2) % 3 * 3 * HTB; \
        if (LIKELY(vm & 4)) { GL2(nb + 2 * HTB, voff2, b2); GL4(nb, voff4, a2); } \
        SEGT(0); SCHED; LDB(B0, boff[s], 0); LDA(A0, aoff[s], 0); LDA(A1, aoff[s], HTB); SCHED; \
        asm volatile("s_bitcmp1_b32 %2, 2\n\ts_cbranch_scc1 1f\n\ts_waitcnt vmcnt(0)\n1:\n\ts_waitcnt vmcnt(6)" : "+v"(sna), "+v"(snb) : "s"(vm) : "memory", "scc"); \
        if (RARE(look)) {   \
            u64 m = __ballot(sna == tag4 && ((snb ^ tag4) & xm) == 0u); \
            const u64 want = todo & (fill | ~low); \
            if (vm == 0 && ((m | avail) & want) == 0ull) m = poll_ready(want, ws, fa, fb, tmo, call, tag4, xm); \
            PUBLISH(m, s); \
            if ((m & todo & ~avail & ~low) == 0ull) SNAP(); } \
        WAIT_L0; SEGT(1); BAR; SEGT(2); SCHED; \
        LISTEN(s); \
        int kind = vm & 17; asm volatile("" : "+s"(kind)); \
        u64 m_, c_; int i_, sh_, vb_; \
        if (LIKELY(kind == 1)) { \
            __builtin_amdgcn_s_setprio(1); \
            MF2(0, A0, 0, 0); MF2(0, A0, 1, 0); MF2(0, A0, 2, 0); MF2(0, A0, 3, 0); N0; \
            MF2(0, A0, 0, 1); N1; MF2(0, A0, 1, 1); N2; MF2(0, A0, 2, 1); MF2(0, A0, 3, 1); N3; \
            MF2(1, A1, 0, 0); N4; MF2(1, A1, 1, 0); N5; MF2(1, A1, 2, 0); N6; MF2(1, A1, 3, 0); N7; \
            MF2(1, A1, 0, 1); N8; MF2(1, A1, 1, 1); N9; MF2(1, A1, 2, 1); MF2(1, A1, 3, 1); \
            __builtin_amdgcn_s_setprio(0); } \
        asm volatile("" : "+s"(kind)); \
        if (RARE(kind == 17)) { MMA(0, 1, A0, B0); MMA(1, 1, A1, B0); } \
        asm volatile("" : "+s"(kind)); \
        if (RARE(kind != 1)) { N0; N1; N2; N3; N4; N5; N6; N7; N8; N9; } \
        SEGT(3); HSTAMP(); SEGT(4); BAR; SEGT(5); SCHED; } while (0)
        for (;;) {
            HTILE(0);
            if (more == 0) break;
            HTILE(1);
            if (more == 0) break;
            HTILE(2);
            if (more == 0) break;
        }
        asm volatile("s_waitcnt vmcnt(0)\n\ts_nop 15\n\ts_nop 7" : "+v"(sna), "+v"(snb) :: "memory");
        if (wr == 0) BAR;
        if (step == 2) break;
        if (step == 0) {
            STAMP(3);
            emit_unit(0, hf);
            STAMP(8);
        }
#pragma unroll
        for (int i = 0; i < 16; ++i) acc[i >> 3][0][(i >> 1) & 3][i & 1] = (f32x4){0.f, 0.f, 0.f, 0.f};
        if (todo == 0ull) {
            const int pt = (g | 1) * 8 + nt;
            if (mine) {
                park(1, pt, hf);
                asm volatile("s_waitcnt vmcnt(0)" ::: "memory");
                if (lane_id() == 0) st_flag(ws + OFF_FP + ((pt * 8 + wid) * 2 + hf) * 4, call);
            }
            STAMP(7);
            if (!HELP_KC || hf || g >= 8) { finish(); return; }
            step = 1; todo = help; avail = 0ull;
            fa = (unsigned)OFF_FW + fwc + nt * 256 + foff; fb = (unsigned)OFF_FX + fxc + cm * 256 + foff;
            xm = ch ? 0xffff0000u : 0x0000ffffu;
        }
      }
      const int pt = 128 + (cm & 1) * 8 + nt;
      park(0, pt, ch);
      asm volatile("s_waitcnt vmcnt(0)" ::: "memory");
      if (lane_id() == 0) st_flag(ws + OFF_FP + ((pt * 8 + wid) * 2 + ch) * 4, call);
    }
    finish();
}

}

extern "C" void kernel_launch(int rank, int nranks, ncclComm_t comm, hipStream_t stream,
                              void* const* d_in, const int* in_sizes, int n_in,
                              void* d_out, int out_size, const int* in_dtypes, int out_dtype,
                              void* const* d_out_all, void* const* d_ws_all, size_t ws_bytes) {
    (void)comm; (void)in_dtypes; (void)out_dtype; (void)d_out_all;
    if (nranks != NR || (unsigned)rank >= (unsigned)NR || n_in < 2 || ws_bytes < WS_NEED
        || in_sizes[0] != GM * KS || in_sizes[1] != KS * GN || out_size != (GM / NR) * GN) {
        fprintf(stderr, "kernel_launch: unexpected problem shape\n");
        return;
    }
    Params p;
    p.x = (const unsigned char*)d_in[0];
    p.w = (const unsigned char*)d_in[1];
    p.out = (unsigned char*)d_out;
    for (int i = 0; i < NR; ++i) p.ws[i] = (unsigned char*)d_ws_all[i];
    p.own = p.ws[rank]; p.rank = rank; p.pad = 0;
    hipLaunchKernelGGL(fused_gemm_rs, dim3(256), dim3(512), 0, stream, p);
    HIP_CHECK(hipGetLastError());
}
```

```cpp
#include <hip/hip_runtime.h>
#include <rccl/rccl.h>
#include <cstdio>
#include <cstdint>

#define HIP_CHECK(expr) do { hipError_t _e = (expr); if (_e != hipSuccess) { \
    fprintf(stderr, "HIP error %s at %s:%d\n", hipGetErrorString(_e), __FILE__, __LINE__); \
    return; } } while (0)

namespace {

#define LAS __attribute__((address_space(3)))
#define GAS __attribute__((address_space(1)))
typedef short bf16x8 __attribute__((ext_vector_type(8)));
typedef float f32x4 __attribute__((ext_vector_type(4)));
typedef unsigned u32x4 __attribute__((ext_vector_type(4)));
typedef unsigned long long u64;

constexpr int NR = 8;
constexpr int GM = 4096, GK = 4096, GN = 16384;
constexpr int KS = GK / NR;
constexpr int BLK = 32768;
constexpr int HTB = 16384;
#define PWV 8
#define WIN 16
#define PULL_DELAY 100
#define HEAD_W 2
#define PACK_DELAY 150
#define PK_ALL 2
#define NTX 1
#define SNAP_AT 2
#define ACQ_SLEEP 1
#define C_ODD 1
#define PLAIN 2
#define HS 128
#define HDR_SLEEP 1
#define FILL_STOP 6
#define SNAP_H 1
#define RX 8
#define RW 8
#define HELP_KC 0xe0
#define TOUCH 7
#define PLAIN_U 0xc0000000u
#define MID_Q 2
constexpr int NPULL = 56, PW = 8 * PWV;
constexpr u64 DEADLINE = 50000000ull;
constexpr int NTASK = 704;
constexpr int UNIT = 9344;

constexpr size_t OFF_CNT = 0;
constexpr size_t OFF_TMO = 32768;
constexpr size_t OFF_TICK = 33792;
constexpr size_t OFF_FX = 131072;
constexpr size_t OFF_FW = OFF_FX + 4096;
constexpr int FR = 8192;
constexpr size_t OFF_FD = 98304;
constexpr size_t OFF_FQ = 104448;
constexpr size_t OFF_FP = 106496;
constexpr size_t OFF_STAMP = 524288;
constexpr size_t OFF_XA = 1u << 20;
constexpr size_t OFF_WB = OFF_XA + (size_t)64 * 16 * BLK;
constexpr size_t OFF_WD = OFF_WB + (size_t)64 * 8 * BLK;
constexpr size_t OFF_RP = OFF_WD + (size_t)64 * 8 * BLK;
constexpr size_t OFF_PP = OFF_RP + (size_t)5120 * UNIT;
constexpr size_t OFF_PD = OFF_PP + (size_t)2048 * UNIT;
constexpr size_t OFF_HIN = OFF_PD + (size_t)144 * 131072;
constexpr size_t OFF_HP = OFF_HIN + (size_t)16 * 768 * HS;
constexpr size_t WS_NEED = OFF_HP + (size_t)2048 * HS + 4096;
static_assert(WS_NEED <= (size_t)161 * 1048576, "workspace");

struct Params {
    const unsigned char* x;
    const unsigned char* w;
    unsigned char* out;
    unsigned char* own;
    unsigned char* ws[NR];
    int rank, pad;
};

#define SBASE "s_nop 4\n\t"
__device__ __forceinline__ void st16_sys(unsigned char* p, u32x4 v) {
    asm volatile("global_store_dwordx4 %0, %1, off sc0 sc1\n\ts_nop 1" :: "v"(p), "v"(v) : "memory");
}
__device__ __forceinline__ void st_flag(unsigned char* p, unsigned v) {
    __hip_atomic_store((unsigned*)p, v, __ATOMIC_RELAXED, __HIP_MEMORY_SCOPE_SYSTEM);
}
__device__ __forceinline__ void st_flag8(unsigned char* p, unsigned v) {
    __hip_atomic_store(p, (unsigned char)v, __ATOMIC_RELAXED, __HIP_MEMORY_SCOPE_SYSTEM);
}
__device__ __forceinline__ unsigned ld_flag(const unsigned char* p) {
    return __hip_atomic_load((const unsigned*)p, __ATOMIC_RELAXED, __HIP_MEMORY_SCOPE_SYSTEM);
}
__device__ __forceinline__ unsigned lane_id() {
    return __builtin_amdgcn_mbcnt_hi(~0u, __builtin_amdgcn_mbcnt_lo(~0u, 0u));
}
__device__ __forceinline__ unsigned cvt_pk_bf16(float lo, float hi) {
    unsigned r; asm volatile("v_cvt_pk_bf16_f32 %0, %1, %2" : "=v"(r) : "v"(lo), "v"(hi)); return r;
}
typedef float f32x2 __attribute__((ext_vector_type(2)));
__device__ __forceinline__ f32x2 gelu2(f32x2 y) {
    const float k1 = -2.0f * 0.7978845608028654f * 1.4426950408889634f;
    const float k3 = k1 * 0.044715f;
    const f32x2 z = y * ((y * y) * k3 + k1);
    const f32x2 d = (f32x2){__builtin_amdgcn_exp2f(z.x), __builtin_amdgcn_exp2f(z.y)} + 1.0f;
    return y * (f32x2){__builtin_amdgcn_rcpf(d.x), __builtin_amdgcn_rcpf(d.y)};
}

template <bool EQ>
__device__ __forceinline__ bool wait_word(const unsigned char* p, unsigned want, unsigned char* tmo, unsigned call) {
    const u64 t0 = __builtin_amdgcn_s_memrealtime();
    for (;;) {
        unsigned g, t;
        asm volatile("global_load_dword %0, %2, off sc0 sc1\n\t"
                     "global_load_dword %1, %3, off sc1\n\t"
                     "s_waitcnt vmcnt(0)" : "=&v"(g), "=&v"(t) : "v"(p), "v"(tmo) : "memory");
        if ((unsigned)__builtin_amdgcn_readfirstlane(t) == call || __builtin_amdgcn_s_memrealtime() - t0 > DEADLINE) {
            __hip_atomic_store((unsigned*)tmo, call, __ATOMIC_RELAXED, __HIP_MEMORY_SCOPE_AGENT);
            return true;
        }
        const unsigned v = __builtin_amdgcn_readfirstlane(g);
        if (EQ ? v == want : v >= want) return false;
        __builtin_amdgcn_s_sleep(2);
    }
}

__device__ __forceinline__ u64 now() {
    u64 t; asm volatile("s_memrealtime %0\n\ts_waitcnt lgkmcnt(0)" : "=s"(t) :: "memory"); return t;
}
__device__ __forceinline__ u64 poll_ready(u64 todo, const unsigned char* ws, unsigned fa, unsigned fb,
                                          unsigned char* tmo, unsigned call, unsigned tag4, unsigned xm) {
    const u64 t0 = now();
    for (;;) {
        unsigned a, b, t; const unsigned zero = 0u;
        asm volatile(SBASE "global_load_dword %0, %3, %6 sc0 sc1\n\t"
                     "global_load_dword %1, %4, %6 sc0 sc1\n\t"
                     "global_load_dword %2, %5, %7 sc1\n\t"
                     "s_waitcnt vmcnt(0)"
                     : "=&v"(a), "=&v"(b), "=&v"(t) : "v"(fa), "v"(fb), "v"(zero), "s"(ws), "s"(tmo) : "memory");
        const u64 ready = __ballot(a == tag4 && ((b ^ tag4) & xm) == 0u);
        if ((ready & todo) != 0ull) return ready;
        if ((unsigned)__builtin_amdgcn_readfirstlane(t) == call || now() - t0 > DEADLINE) {
            __hip_atomic_store((unsigned*)tmo, call, __ATOMIC_RELAXED, __HIP_MEMORY_SCOPE_AGENT);
            return ~0ull;
        }
        __builtin_amdgcn_s_sleep(1);
    }
}

__device__ __forceinline__ unsigned bfi(unsigned m, unsigned a, unsigned b) { return (a & m) | (b & ~m); }
__device__ __forceinline__ unsigned perm(unsigned hi, unsigned lo, unsigned sel) { return __builtin_amdgcn_perm(hi, lo, sel); }
#define SDWA2(k) " dst_sel:DWORD dst_unused:UNUSED_PAD src0_sel:BYTE_" #k " src1_sel:BYTE_" #k "\n\t"
#define SDWA1(k) " dst_sel:DWORD dst_unused:UNUSED_PAD src0_sel:DWORD src1_sel:BYTE_" #k "\n\t"
__device__ __forceinline__ unsigned spread_bits(unsigned f, unsigned s) {
    unsigned r, a, b;
    asm("v_lshlrev_b32_sdwa %0, %4, %3" SDWA2(0) "v_lshlrev_b32_sdwa %1, %4, %3" SDWA2(1) "v_lshlrev_b32_sdwa %2, %4, %3" SDWA2(2)
        "v_or3_b32 %0, %0, %1, %2\n\t"
        "v_lshlrev_b32_sdwa %1, %4, %3" SDWA2(3)
        "v_or_b32 %0, %0, %1" : "=&v"(r), "=&v"(a), "=&v"(b) : "v"(f), "v"(s));
    return r;
}
__device__ __forceinline__ unsigned take_codes(unsigned& w, unsigned f) {
    unsigned ln, t, l;
    asm("v_xor_b32_sdwa %2, %0, %4" SDWA1(0) "v_ffbl_b32 %1, %2\n\tv_lshrrev_b32 %0, %1, %0\n\t"
        "v_xor_b32_sdwa %2, %0, %4" SDWA1(1) "v_ffbl_b32 %3, %2\n\tv_lshrrev_b32 %0, %3, %0\n\t"
        "v_xor_b32_sdwa %2, %0, %4" SDWA1(2) "v_lshl_or_b32 %1, %3, 8, %1\n\tv_ffbl_b32 %3, %2\n\tv_lshrrev_b32 %0, %3, %0\n\t"
        "v_xor_b32_sdwa %2, %0, %4" SDWA1(3) "v_lshl_or_b32 %1, %3, 16, %1\n\tv_ffbl_b32 %3, %2\n\tv_lshrrev_b32 %0, %3, %0\n\t"
        "v_lshl_or_b32 %1, %3, 24, %1" : "+v"(w), "=&v"(ln), "=&v"(t), "=&v"(l) : "v"(f));
    return ln;
}
__device__ __forceinline__ unsigned wave_scan(unsigned v) {
    v += (unsigned)__builtin_amdgcn_update_dpp(0, (int)v, 0x111, 0xf, 0xf, false);
    v += (unsigned)__builtin_amdgcn_update_dpp(0, (int)v, 0x112, 0xf, 0xf, false);
    v += (unsigned)__builtin_amdgcn_update_dpp(0, (int)v, 0x114, 0xf, 0xf, false);
    v += (unsigned)__builtin_amdgcn_update_dpp(0, (int)v, 0x118, 0xf, 0xf, false);
    v += (unsigned)__builtin_amdgcn_update_dpp(0, (int)v, 0x142, 0xa, 0xf, false);
    v += (unsigned)__builtin_amdgcn_update_dpp(0, (int)v, 0x143, 0xc, 0xf, false);
    return v;
}
__device__ __forceinline__ unsigned rank_in(u64 mask) {
    return __builtin_amdgcn_mbcnt_hi((unsigned)(mask >> 32), __builtin_amdgcn_mbcnt_lo((unsigned)mask, 0u));
}
constexpr int SCR = 8192;
#define LDS_ORDER() asm volatile("" ::: "memory")

struct Packed { u32x4 sm[4], l1, st; u64 mask; unsigned meta; };

__device__ __forceinline__ void pack(const u32x4 (&d)[8], Packed& o, LAS unsigned char* scr, unsigned lane) {
    *(LAS u32x4*)(scr + lane * 16) = (u32x4){0u, 0u, 0u, 0u};
    unsigned sum = 0; u32x4 er[4];
#pragma unroll
    for (int g = 0; g < 16; ++g) {
        const unsigned d0 = d[g >> 1][(g & 1) * 2], d1 = d[g >> 1][(g & 1) * 2 + 1];
        const unsigned lb = perm(d1, d0, 0x06040200u), hb = perm(d1, d0, 0x07050301u);
        o.sm[g >> 2][g & 3] = bfi(0x7f7f7f7fu, lb, hb);
        const unsigned e = bfi(0x01010101u, lb >> 7, hb << 1);
        er[g >> 2][g & 3] = e;
        if (g % 5 == 0) sum = __builtin_amdgcn_sad_u8(e, 0u, sum);
    }
    const unsigned r2 = min(max(((unsigned)__builtin_amdgcn_readlane(wave_scan(sum), 63) + 4009u) >> 10, 31u), 255u);
    const unsigned kk = r2 * 0x01010101u;
    unsigned bad = 0, top = ~0u, longest = 0, total = 0, piece[8], plen[8], us[16], cs[16];
#pragma unroll
    for (int g = 0; g < 16; ++g) {
        const unsigned u = kk - er[g >> 2][g & 3];
        bad |= u;
        us[g] = u; cs[g] = perm(0x0f0f0f0cu, 0x0a080f07u, u | ((u << 1) & 0x10101010u));
        top &= cs[g];
    }
    const bool hole = __ballot((top & 0x08080808u) != 0x08080808u) != 0ull;
    const unsigned tx = hole ? 0xfdfdfdfdu : 0xfefefefeu, thi = 0x01000000u ^ tx, tlo = 0x01020609u ^ tx;
#pragma unroll
    for (int q = 0; q < 4; ++q) {
        unsigned l1 = 0, acc = 0, base = 0;
#pragma unroll
        for (int gg = 0; gg < 4; ++gg) {
            const unsigned u = us[q * 4 + gg], c = cs[q * 4 + gg];
            l1 |= gg ? (c & 0x06060606u) << (2 * gg - 1) : (c & 0x06060606u) >> 1;
            const unsigned len4 = (perm(thi, tlo, u | (c & 0x10101010u)) ^ tx) + u;
            const unsigned f = c & 0x01010101u;
            unsigned ps = len4 + (len4 << 8); asm("" : "+v"(ps)); ps += ps << 16; ps -= 0x100c0804u;
            const unsigned bits = spread_bits(f, ps);
            if ((gg & 1) == 0) { acc = bits; base = ps >> 24; }
            else {
                acc |= bits << (base & 31u); base += ps >> 24;
                piece[q * 2 + (gg >> 1)] = acc >> 1; plen[q * 2 + (gg >> 1)] = base;
                longest = max(longest, base); total += base;
            }
        }
        o.l1[q] = l1;
    }
    const bool raw = (bad & 0xe0e0e0e0u) != 0u || longest > 31u || total > 112u;
    o.mask = __ballot(raw);
    if (raw) total = 0;
    const unsigned incl = wave_scan(total);
    const unsigned rawoff = (((unsigned)__builtin_amdgcn_readlane(incl, 63) + 512u + 127u) >> 7) << 4;
    o.meta = r2 | ((rawoff + 64u * (unsigned)__builtin_popcountll(o.mask) + 127u) >> 7) << 8 | (hole ? 0x4000u : 0u);
    LDS_ORDER();
    scr[lane] = (unsigned char)total;
    if (!raw) {
        unsigned off = incl - total + 512u;
#pragma unroll
        for (int i = 0; i < 8; ++i) {
            const u64 w = (u64)piece[i] << (off & 31u);
            LAS unsigned* p = (LAS unsigned*)(scr + ((off >> 5) << 2));
            __hip_atomic_fetch_or(p, (unsigned)w, __ATOMIC_RELAXED, __HIP_MEMORY_SCOPE_WORKGROUP);
            __hip_atomic_fetch_or(p + 1, (unsigned)(w >> 32), __ATOMIC_RELAXED, __HIP_MEMORY_SCOPE_WORKGROUP);
            off += plen[i];
        }
    } else {
        LAS u32x4* p = (LAS u32x4*)(scr + rawoff + rank_in(o.mask) * 64u);
#pragma unroll
        for (int i = 0; i < 4; ++i) p[i] = er[i];
    }
    LDS_ORDER();
    o.st = *(const LAS u32x4*)(scr + lane * 16);
    LDS_ORDER();
}
__device__ __forceinline__ void send(const Packed& o, unsigned char* pl, const LAS unsigned char* scr, unsigned lane) {
    unsigned char* p = pl + lane * 16;
    for (unsigned k = 1; k * 8u < ((o.meta >> 8) & 63u); ++k)
        if (k * 64u + lane < ((o.meta >> 8) & 63u) * 8u) st16_sys(p + 5120 + k * 1024, *(const LAS u32x4*)(scr + k * 1024 + lane * 16));
#pragma unroll
    for (int i = 0; i < 4; ++i) st16_sys(p + i * 1024, o.sm[i]);
    st16_sys(p + 4096, o.l1);
    if (lane < ((o.meta >> 8) & 63u) * 8u) st16_sys(p + 5120, o.st);
}
__device__ __forceinline__ u32x4 header(const Packed& o, unsigned call) {
    return (u32x4){(unsigned)o.mask, call, (unsigned)(o.mask >> 32), (call << 15) | o.meta};
}
__device__ __forceinline__ bool header_ok(u32x4 h, unsigned call, u64& mask, unsigned& meta) {
    const unsigned t1 = __builtin_amdgcn_readfirstlane(h.y), t3 = __builtin_amdgcn_readfirstlane(h.w);
    mask = ((u64)(unsigned)__builtin_amdgcn_readfirstlane(h.z) << 32) | (unsigned)__builtin_amdgcn_readfirstlane(h.x);
    meta = t3 & 0x7fffu;
    return t1 == call && (t3 >> 15) == (call & 0x1ffffu);
}
__device__ __forceinline__ void wait_header(const unsigned char* hd, unsigned char* tmo, unsigned call, u64& mask, unsigned& meta) {
    const u64 t0 = __builtin_amdgcn_s_memrealtime();
    for (;;) {
        u32x4 h; unsigned t;
        asm volatile("global_load_dwordx4 %0, %2, off sc0 sc1\n\t"
                     "global_load_dword %1, %3, off sc1\n\t"
                     "s_waitcnt vmcnt(0)" : "=&v"(h), "=&v"(t) : "v"(hd), "v"(tmo) : "memory");
        if (header_ok(h, call, mask, meta)) break;
        if ((unsigned)__builtin_amdgcn_readfirstlane(t) == call || __builtin_amdgcn_s_memrealtime() - t0 > DEADLINE) {
            __hip_atomic_store((unsigned*)tmo, call, __ATOMIC_RELAXED, __HIP_MEMORY_SCOPE_AGENT);
            mask = 0ull; meta = 0u;
            break;
        }
        __builtin_amdgcn_s_sleep(HDR_SLEEP);
    }
}
__device__ __forceinline__ void fetch(Packed& o, const unsigned char* pl, unsigned lane) {
    const unsigned n = min(max((o.meta >> 8) & 63u, 1u), 8u) * 8u;
    const u64 smask = n >= 64u ? ~0ull : (1ull << n) - 1ull;
    u64 keep;
    asm volatile(SBASE "s_mov_b64 %6, exec\n\t"
                 "global_load_dwordx4 %0, %7, %8 sc0 sc1\n\t"
                 "global_load_dwordx4 %1, %7, %8 offset:1024 sc0 sc1\n\t"
                 "global_load_dwordx4 %2, %7, %8 offset:2048 sc0 sc1\n\t"
                 "global_load_dwordx4 %3, %7, %8 offset:3072 sc0 sc1\n\t"
                 "global_load_dwordx4 %4, %7, %9 sc0 sc1\n\t"
                 "s_mov_b64 exec, %10\n\t"
                 "global_load_dwordx4 %5, %7, %9 offset:1024 sc0 sc1\n\t"
                 "s_mov_b64 exec, %6"
                 : "=&v"(o.sm[0]), "=&v"(o.sm[1]), "=&v"(o.sm[2]), "=&v"(o.sm[3]), "=&v"(o.l1), "=&v"(o.st), "=&s"(keep)
                 : "v"(lane * 16u), "s"(pl), "s"(pl + 4096), "s"(smask) : "memory");
}
#define FETCH_WAIT(o, n) asm volatile("s_waitcnt vmcnt(" #n ")" : "+v"(o.sm[0]), "+v"(o.sm[1]), "+v"(o.sm[2]), "+v"(o.sm[3]), "+v"(o.l1), "+v"(o.st) :: "memory")
template <class Mid>
__device__ __forceinline__ void unpack(const Packed& o, u32x4 (&d)[8], const unsigned char* pl, LAS unsigned char* scr, unsigned lane, Mid mid) {
    *(LAS u32x4*)(scr + lane * 16) = o.st;
    if (((o.meta >> 8) & 63u) > 8u) {
        u32x4 m[4];
        asm volatile("global_load_dwordx4 %0, %4, off sc0 sc1\n\t"
                     "global_load_dwordx4 %1, %4, off offset:1024 sc0 sc1\n\t"
                     "global_load_dwordx4 %2, %4, off offset:2048 sc0 sc1\n\t"
                     "global_load_dwordx4 %3, %4, off offset:3072 sc0 sc1\n\t"
                     "s_waitcnt vmcnt(0)" : "=&v"(m[0]), "=&v"(m[1]), "=&v"(m[2]), "=&v"(m[3]) : "v"(pl + 6144 + lane * 16) : "memory");
#pragma unroll
        for (int k = 0; k < 4; ++k) *(LAS u32x4*)(scr + (k + 1) * 1024 + lane * 16) = m[k];
    }
    LDS_ORDER();
    const unsigned len = scr[lane] & 127u;
    const unsigned incl = wave_scan(len), off = incl - len + 512u;
    const LAS unsigned* p = (const LAS unsigned*)(scr + ((off >> 5) << 2));
    const unsigned w0 = p[0], w1 = p[1], w2 = p[2], w3 = p[3], w4 = p[4];
    unsigned s0 = __builtin_amdgcn_alignbit(w1, w0, off), s1 = __builtin_amdgcn_alignbit(w2, w1, off);
    unsigned s2 = __builtin_amdgcn_alignbit(w3, w2, off), s3 = __builtin_amdgcn_alignbit(w4, w3, off);
    const bool hole = (o.meta & 0x4000u) != 0u;
    const unsigned tx = hole ? 0xf8f8f8f8u : 0xf7f7f7f7u, thi = (hole ? 0x07070008u : 0x08080808u) ^ tx, tlo = 0x0309090au ^ tx;
    const unsigned kk = ((o.meta & 255u) + 8u) * 0x01010101u;
    u32x4 er[4];
#pragma unroll
    for (int q = 0; q < 4; ++q) {
        if (q == MID_Q) { asm volatile("" : "+v"(s0)); mid(); asm volatile("" : "+v"(s0)); }
        const unsigned l1 = o.l1[q], x = l1 & (l1 >> 1);
        unsigned w = 0, used = 0;
#pragma unroll
        for (int gg = 0; gg < 4; ++gg) {
            if ((gg & 1) == 0) { w = (s0 << 1) | 1u; used = 0; }
            const unsigned f = (x >> (2 * gg)) & 0x01010101u;
            const unsigned ln = take_codes(w, f);
            used = __builtin_amdgcn_sad_u8(ln, 0u, used);
            const unsigned y = (perm(thi, tlo, ln | ((ln << 1) & 0x10101010u)) ^ tx) + ln + ((l1 >> (2 * gg)) & 0x03030303u);
            er[q][gg] = kk - y;
            if (gg & 1) {
                s0 = __builtin_amdgcn_alignbit(s1, s0, used); s1 = __builtin_amdgcn_alignbit(s2, s1, used);
                s2 = __builtin_amdgcn_alignbit(s3, s2, used); s3 >>= used & 31u;
            }
        }
    }
    if (o.mask != 0ull) {
        const unsigned rawoff = (((unsigned)__builtin_amdgcn_readlane(incl, 63) + 512u + 127u) >> 7) << 4;
        if ((o.mask >> lane) & 1ull) {
            const LAS u32x4* r = (const LAS u32x4*)(scr + rawoff + rank_in(o.mask) * 64u);
#pragma unroll
            for (int i = 0; i < 4; ++i) er[i] = r[i];
        }
    }
    LDS_ORDER();
#pragma unroll
    for (int g = 0; g < 16; ++g) {
        const unsigned e = er[g >> 2][g & 3], sm = o.sm[g >> 2][g & 3];
        const unsigned hb = bfi(0x80808080u, sm, e >> 1), lb = bfi(0x7f7f7f7fu, sm, e << 7);
        d[g >> 1][(g & 1) * 2] = perm(hb, lb, 0x05010400u); d[g >> 1][(g & 1) * 2 + 1] = perm(hb, lb, 0x07030602u);
    }
}

__device__ __forceinline__ void load_parked(u32x4 (&dg)[8], unsigned& tw, const unsigned char* dp, const unsigned char* tmo, unsigned lane) {
    asm volatile(SBASE "global_load_dwordx4 %0, %9, %10 sc0 sc1\n\t"
                 "global_load_dwordx4 %1, %9, %10 offset:1024 sc0 sc1\n\t"
                 "global_load_dwordx4 %2, %9, %10 offset:2048 sc0 sc1\n\t"
                 "global_load_dwordx4 %3, %9, %10 offset:3072 sc0 sc1\n\t"
                 "global_load_dwordx4 %4, %9, %11 sc0 sc1\n\t"
                 "global_load_dwordx4 %5, %9, %11 offset:1024 sc0 sc1\n\t"
                 "global_load_dwordx4 %6, %9, %11 offset:2048 sc0 sc1\n\t"
                 "global_load_dwordx4 %7, %9, %11 offset:3072 sc0 sc1\n\t"
                 "global_load_dword %8, %8, %12 sc1"
                 : "=&v"(dg[0]), "=&v"(dg[1]), "=&v"(dg[2]), "=&v"(dg[3]), "=&v"(dg[4]), "=&v"(dg[5]), "=&v"(dg[6]), "=&v"(dg[7]), "+v"(tw)
                 : "v"(lane * 16u), "s"(dp), "s"(dp + 4096), "s"(tmo) : "memory");
}
#define PARKED_WAIT(dg, tw) asm volatile("s_waitcnt vmcnt(0)" : "+v"(dg[0]), "+v"(dg[1]), "+v"(dg[2]), "+v"(dg[3]), "+v"(dg[4]), "+v"(dg[5]), \
                                         "+v"(dg[6]), "+v"(dg[7]), "+v"(tw) :: "memory")

#define STAMP(i) do {} while (0)

#define ORD_A1 1.17
#define ORD_AL 0.0
#define ORD_B0 0.4
#define ORD_BS 1.22
#define ORD_C0 10.5
#define ORD_CS 0.3
#define ORD_W 0.3
struct Order { unsigned short v[354]; };
constexpr Order make_order() {
    double key[352] = {}; unsigned short id[352] = {}; int n = 0;
    for (int kc = 0; kc < 8; ++kc)
        for (int h = 0; h < 2; ++h) {
            for (int e = 0; e < 8; ++e) { const double ka = kc - ORD_W + (e + 0.4) / 8, kb = ORD_B0 + ka * ORD_BS; key[n] = (ka < kb ? ka : kb) + h * 1e-3 - (kc == 0 && e < PLAIN ? 9 : 0); id[n++] = (unsigned short)(h << 8 | 0 << 6 | kc << 3 | e); }
            for (int e = 0; e < 6; ++e) { key[n] = (kc + (e + 0.5) / 6) * (h ? ORD_A1 : 1.0) + (h ? ORD_AL : 0.0); id[n++] = (unsigned short)(h << 8 | 1 << 6 | kc << 3 | e); }
            for (int e = 0; e < 6; ++e) { key[n] = ORD_B0 + (kc + (e + 0.6) / 6) * ORD_BS + h * 1e-3; id[n++] = (unsigned short)(h << 8 | 2 << 6 | kc << 3 | e); }
            for (int e = 0; e < 2; ++e) { key[n] = ORD_C0 + (2 * kc + e + 0.5 * h) * ORD_CS; id[n++] = (unsigned short)(h << 8 | 3 << 6 | kc << 3 | e); }
        }
    Order o = {};
    for (int i = 0; i < 352; ++i) { int r = 0; for (int j = 0; j < 352; ++j) r += (key[j] < key[i]) || (key[j] == key[i] && j < i); o.v[r] = id[i]; }
    return o;
}
constexpr Order ORDER = make_order();
constexpr unsigned order_word(int i) { return ORDER.v[3 * i] | ORDER.v[3 * i + 1] << 10 | (unsigned)ORDER.v[3 * i + 2] << 20; }
template <int I, int N> __device__ __forceinline__ unsigned order_vgpr(unsigned v) {
    if constexpr (I == N) return v;
    else {
        unsigned t;
        asm("s_mov_b32 %1, %2\n\tv_writelane_b32 %0, %1, %3" : "+v"(v), "=&s"(t) : "i"(order_word(I)), "i"(I & 63));
        return order_vgpr<I + 1, N>(v);
    }
}
__device__ __forceinline__ void decode_unit(unsigned ord0, unsigned ord1, int t, int src, int dest, int& isw, int& kc, int& tile, int& q) {
    const int n = t >> 1, dw = n / 3;
    const unsigned w0 = __builtin_amdgcn_readlane(ord0, dw & 63), w1 = __builtin_amdgcn_readlane(ord1, dw & 63);
    const unsigned v = (dw < 64 ? w0 : w1) >> ((n - dw * 3) * 10);
    const int kind = (v >> 6) & 3, e = v & 7;
    kc = (v >> 3) & 7; isw = kind == 0; q = ((v >> 7) & 2) | (t & 1);
    const int d = (dest - src - 1) & 7;
    tile = kind == 0 ? ((e + src) & 7) : kind == 3 ? 2 * dest + e : 2 * ((src + 1 + e + (e >= d)) & 7) + (kind - 1);
}

__host__ __device__ __forceinline__ void role_of(int b, int& nt, int& slot, int& g) {
    constexpr int NG = 8 / NTX, X0 = 8 - NG;
    const int x = b & 7, s = b >> 3, np = x < X0 ? 8 : 8 - NTX;
    if (s < np) {
        const int pid = s < 8 - NTX ? s * 8 + x : (8 - NTX) * 8 + (s - (8 - NTX)) * X0 + x;
        slot = NTX == 1 ? pid >> 3 : pid % 7; nt = NTX == 1 ? pid & 7 : pid / 7; g = -1;
    } else {
        const int j = s - np, r = j / NTX + x / NG * NG * 3;
        nt = x % NG * NTX + j % NTX; slot = 0;
        g = r >= 21 ? r : r % 3 == 2 ? 14 + r / 3 : r / 3 * 2 + r % 3;
    }
}

__global__ __launch_bounds__(512, 2) void fused_gemm_rs(const Params P) {
    __shared__ __attribute__((aligned(16))) unsigned char lds_raw[9 * HTB + 256];
    LAS unsigned char* lds = (LAS unsigned char*)lds_raw;
    const int tid = threadIdx.x, wid = __builtin_amdgcn_readfirstlane(tid >> 6), lane = tid & 63;
    const int b = blockIdx.x, self = P.rank;
    unsigned char* const ws = P.own;
    const GAS unsigned char *px = (const GAS unsigned char*)P.x, *pw = (const GAS unsigned char*)P.w;
    asm volatile("" : "+s"(px), "+s"(pw));
    unsigned count; const unsigned vzero = 0u;
    asm volatile(SBASE "global_load_dword %0, %1, %2 sc1" : "=&v"(count) : "v"(vzero), "s"(ws + OFF_CNT + b * 128) : "memory");
    unsigned char* const tmo = ws + OFF_TMO;
    LAS unsigned char* const scr = lds + wid * SCR;
    STAMP(0);
    const u64 tstart = __builtin_amdgcn_s_memrealtime();
    int nt, slot, g;
    role_of(b, nt, slot, g);
    const bool is_pull = g < 0, is_a = !is_pull && g < 14, is_b = g >= 14 && g < 21, is_c = g >= 21;
    const int owner = is_c ? self : (self + 1 + (is_a ? g >> 1 : g - 14)) & 7;
    const int mt = 2 * owner + (is_c ? (g - 21) >> 1 : is_b), hf = is_c ? (g - 21) & 1 : g & 1;
    auto pack_decode = [&](int t, int& isw, int& kc, int& tile) {
        const int blk = t >> 2;
        if (blk < 80) {
            const int m = blk - 16; kc = 0; isw = blk >= 16;
            tile = isw ? ((self + 1 + m) & 7) * 8 + (((m >> 3) + self) & 7) : blk;
        } else if (blk < 248) {
            const int m = blk - 80, r = m % 24; kc = 1 + m / 24; isw = r >= 16;
            tile = isw ? self * 8 + (r & 7) : r;
        } else if (blk < 640) {
            const int m = blk - 248, w = m % 56; isw = 1; kc = 1 + m / 56;
            tile = ((self + 1 + w % 7) & 7) * 8 + ((w / 7 + self) & 7);
        } else {
            const int m = blk - 640; isw = 1; kc = 0;
            tile = ((self + 1 + (m >> 3)) & 7) * 8 + (m & 7);
        }
    };
    const unsigned lw = (lane >> 4) * 8 * (GN * 2) + (lane & 15) * 16, lx = (lane & 15) * (KS * 2) + (lane >> 4) * 16;
    auto load_task = [&](int t, u32x4 (&r)[8]) {
        int isw, kc, tile; pack_decode(t, isw, kc, tile); const int q = t & 3;
        const GAS unsigned char* src = isw ? pw + ((kc * 64 + (q & 1) * 32) * (GN * 2) + (tile * 256 + (q >> 1) * 128) * 2)
                                       : px + ((tile * 256 + q * 64) * (KS * 2) + kc * 128);
        const unsigned lo = isw ? lw : lx, s1 = isw ? GN * 2 : 64, s2 = isw ? GN * 4 : 16 * KS * 2;
#pragma unroll
        for (int i = 0; i < 8; ++i) r[i] = *(const GAS u32x4*)(src + ((i >> 1) * s2 + (i & 1) * s1) + lo);
    };
    auto store_image = [&](unsigned char* blk, int isw, int q, const u32x4 (&c)[8]) {
        unsigned char* d0 = blk + (isw ? (q >> 1) * 8192 + (q & 1) * 1024 : q * 8192) + lane * 16;
        const unsigned s1 = isw ? 2048 : 1024, s4 = isw ? HTB : 4096;
#pragma unroll
        for (int j = 0; j < 8; ++j) st16_sys(d0 + ((j >> 2) * s4 + (j & 3) * s1), c[j]);
    };
    u32x4 cur[8];
    const int task0 = is_pull ? wid * 8 + nt : wid * 144 + (is_a ? g : g - 7) * 8 + nt;
    unsigned peer_lo = 0u, peer_hi = 0u;
    auto peers = [&]() {
#pragma unroll
        for (int r = 0; r < NR; ++r)
            asm("v_writelane_b32 %0, %2, %4\n\tv_writelane_b32 %1, %3, %4" : "+v"(peer_lo), "+v"(peer_hi)
                : "s"((unsigned)(u64)P.ws[r]), "s"((unsigned)((u64)P.ws[r] >> 32)), "i"(r));
    };
    if (is_a || is_c) {
        if (wid >= HEAD_W) {
            if (wid < 2 * HEAD_W) {
                peers();
                int isw, kc, tile; const int t = task0 - HEAD_W * 144; pack_decode(t, isw, kc, tile);
                const size_t unit = (isw ? tile * 8 : 512 + tile) * 4 + (t & 3), head = (isw ? tile & 7 : 64 + tile) * 4 + (t & 3);
                const unsigned char* a = lane < NR ? (unsigned char*)(((u64)peer_hi << 32) | peer_lo) + OFF_HIN + (self * 768 + head) * HS + 64
                                       : lane < 10 ? ws + OFF_RP + ((lane & 1) * 2560 + unit) * UNIT
                                       : ws + (isw ? OFF_WB + (size_t)(self * 64 + (tile & 7)) * BLK : OFF_XA + (size_t)(self * 128 + tile) * BLK) + lane * 128;
                if (lane < 11 && (lane >= NR || !isw || lane == (tile >> 3))) { unsigned z; asm volatile("global_load_dword %0, %1, off sc0 sc1\n\ts_waitcnt vmcnt(0)" : "=&v"(z) : "v"(a) : "memory"); }
            }
            while (__builtin_amdgcn_s_memrealtime() - tstart < PACK_DELAY) __builtin_amdgcn_s_sleep(2); }
        else __builtin_amdgcn_s_setprio(2);
        load_task(task0, cur);
    }
    if (is_b && wid == 7 && lane < 16) {
        const unsigned char* ows = P.ws[owner];
        const unsigned char* a = lane < 2 ? ws + OFF_PP + (size_t)(((2 * owner + lane) * 8 + nt) * 16) * UNIT
                               : lane < 4 ? ows + OFF_PP + (size_t)(((2 * self + (lane & 1)) * 8 + nt) * 16) * UNIT
                               : lane < 5 ? ows + OFF_HP + (size_t)((self * 8 + nt) * 16) * HS + 64
                               : lane < 6 ? ws + OFF_HP + (size_t)((owner * 8 + nt) * 16) * HS + 64
                               : lane < 8 ? ws + OFF_PD + (size_t)(((g - 14) * 2 + (lane & 1)) * 8 + nt) * 131072
                               : P.out + (size_t)(lane - 8) * 64 * (GN * 2) + (owner * 8 + nt) * 512;
        unsigned z; asm volatile("global_load_dword %0, %1, off sc0 sc1\n\ts_waitcnt vmcnt(0)" : "=&v"(z) : "v"(a) : "memory");
    }
    peers();
    asm volatile("s_waitcnt vmcnt(0)" : "+v"(count) :: "memory");
    const unsigned call = (unsigned)__builtin_amdgcn_readfirstlane(count) + 1u;
    const size_t par = call & 1u;
    const unsigned tag4 = (call & 255u) * 0x01010101u;
    const int wr = wid >> 2, wc = wid & 3;

#define READY0 0
    if (is_pull) {
        LAS unsigned* const ctl = (LAS unsigned*)(lds + 9 * HTB + 64);
        if (tid < 4) ctl[tid] = 0u;
        __syncthreads();
        const unsigned share = (WIN + 7 - nt) / 8;
        auto lds_draw = [&](int w) {
            unsigned v = 0;
            if (lane == 0) v = __hip_atomic_fetch_add(&ctl[w], 1u, __ATOMIC_RELAXED, __HIP_MEMORY_SCOPE_WORKGROUP);
            return (unsigned)__builtin_amdgcn_readfirstlane(v);
        };
        auto acquire = [&]() {
            const unsigned turn = lds_draw(0);
            while (__hip_atomic_load(&ctl[1], __ATOMIC_RELAXED, __HIP_MEMORY_SCOPE_WORKGROUP) + share <= turn) __builtin_amdgcn_s_sleep(ACQ_SLEEP);
        };
        auto release = [&]() { if (lane == 0) __hip_atomic_fetch_add(&ctl[1], 1u, __ATOMIC_RELAXED, __HIP_MEMORY_SCOPE_WORKGROUP); };
        const int src = (self + 1 + slot) & 7;
        const unsigned char* const sws = P.ws[src];
        if (wid < PWV) {
            const unsigned ord0 = order_vgpr<0, 64>(0u), ord1 = order_vgpr<64, 118>(0u);
            unsigned* const tick = (unsigned*)(ws + OFF_TICK + src * 128);
            const unsigned tbase = (call - 1u) * NTASK - PW;
            const unsigned char* const hbase = ws + OFF_HIN + (par * 8 + src) * (768 * HS);
            int task = task0, isw, kc, tile, q;
            decode_unit(ord0, ord1, task, src, self, isw, kc, tile, q);
            Packed pk;
            {
                const unsigned char* a = lane ? ws + (isw ? OFF_WB + (size_t)((src * 8 + kc) * 8 + tile) * BLK : OFF_XA + (size_t)((src * 8 + kc) * 16 + tile) * BLK) + q * 8192
                    : sws + OFF_RP + (par * 2560 + (isw ? (self * 8 + tile) * 8 + kc : 512 + kc * 16 + tile) * 4 + q) * UNIT + 8320 + self * 128;
                if (lane < 2) { unsigned z; asm volatile("global_load_dword %0, %1, off sc0 sc1\n\ts_waitcnt vmcnt(0)" : "=&v"(z) : "v"(a) : "memory"); }
            }
            unsigned second = 0u;
            if (task < 4 * PLAIN && lane == 0) second = __hip_atomic_fetch_add(tick, 1u, __ATOMIC_RELAXED, __HIP_MEMORY_SCOPE_AGENT);
            second = __builtin_amdgcn_readfirstlane(second);
            while (__builtin_amdgcn_s_memrealtime() - tstart < PULL_DELAY) __builtin_amdgcn_s_sleep(2);
            wait_header(hbase + ((isw ? kc * 8 + tile : 64 + kc * 16 + tile) * 4 + q) * HS, tmo, call, pk.mask, pk.meta);
            if (task < 4 * PLAIN) {
                unsigned tw = 0u;
                acquire();
                load_parked(cur, tw, sws + OFF_RP + (par * 2560 + (self * 8 + tile) * 32 + q) * UNIT, tmo, lane);
                PARKED_WAIT(cur, tw);
                release();
                store_image(ws + OFF_WB + (size_t)(src * 64 + tile) * BLK, 1, q, cur);
                asm volatile("s_waitcnt vmcnt(0)" ::: "memory");
                if (lane < RW) st_flag8(ws + OFF_FW + (tile * 64 + src * 8) * 4 + q + lane * FR, call);
                task = (int)(second - tbase);
                decode_unit(ord0, ord1, task < NTASK ? task : 0, src, self, isw, kc, tile, q);
                wait_header(hbase + ((isw ? kc * 8 + tile : 64 + kc * 16 + tile) * 4 + q) * HS, tmo, call, pk.mask, pk.meta);
            }
            while (task < NTASK) {
                const unsigned char* const pl = sws + OFF_RP + (par * 2560 + (isw ? (self * 8 + tile) * 8 + kc : 512 + kc * 16 + tile) * 4 + q) * UNIT;
                acquire();
                fetch(pk, pl, lane);
                FETCH_WAIT(pk, 0);
                release();
                unsigned nn = 1u;
                {
                    u64 keep; const unsigned zero = 0u;
                    asm volatile(SBASE "s_mov_b64 %1, exec\n\ts_mov_b64 exec, 1\n\t"
                                 "global_atomic_add %0, %2, %0, %3 sc0\n\t"
                                 "s_mov_b64 exec, %1" : "+v"(nn), "=&s"(keep) : "v"(zero), "s"(tick) : "memory");
                }
                int ntask, nisw, nkc, ntile, nq; u32x4 h; const unsigned char* hd;
                unpack(pk, cur, pl, scr, lane, [&]() {
                    asm volatile("s_waitcnt vmcnt(0)" : "+v"(nn) :: "memory");
                    ntask = (int)(__builtin_amdgcn_readfirstlane(nn) - tbase);
                    decode_unit(ord0, ord1, ntask < NTASK ? ntask : 0, src, self, nisw, nkc, ntile, nq);
                    hd = hbase + ((nisw ? nkc * 8 + ntile : 64 + nkc * 16 + ntile) * 4 + nq) * HS;
                    const unsigned zero = 0u;
                    asm volatile(SBASE "global_load_dwordx4 %0, %1, %2 sc0 sc1" : "=&v"(h) : "v"(zero), "s"(hd) : "memory");
                });
                const int idx = src * 8 + kc;
                store_image(isw ? ws + OFF_WB + (size_t)(idx * 8 + tile) * BLK : ws + OFF_XA + (size_t)(idx * 16 + tile) * BLK, isw, q, cur);
                asm volatile("s_waitcnt vmcnt(0)" : "+v"(h) :: "memory");
                if (lane < (isw ? RW : RX)) st_flag8(ws + (isw ? OFF_FW : OFF_FX) + (tile * 64 + idx) * 4 + q + lane * FR, call);
                if (ntask >= NTASK) break;
                if (!header_ok(h, call, pk.mask, pk.meta)) wait_header(hd, tmo, call, pk.mask, pk.meta);
                task = ntask; isw = nisw; kc = nkc; tile = ntile; q = nq;
            }
            if (lane == 0) __hip_atomic_store(&ctl[3], 1u, __ATOMIC_RELAXED, __HIP_MEMORY_SCOPE_WORKGROUP);
        } else {
            while (__hip_atomic_load(&ctl[3], __ATOMIC_RELAXED, __HIP_MEMORY_SCOPE_WORKGROUP) == 0u) __builtin_amdgcn_s_sleep(8);
        }
        STAMP(1);
        const int gnt = src * 8 + nt, fr = lane & 15, fq = lane >> 4;
        auto head_of = [&](int u) { return ws + OFF_HP + (size_t)(((((u >> 4) & 1) * 64 + gnt) * 8 + (u & 7)) * 2 + ((u >> 3) & 1)) * HS; };
        u32x4 h = {0u, 0u, 0u, 0u};
#define USTAMP() do {} while (0)
        for (int u = (int)lds_draw(2); u < 32;) {
            const int odd = u >> 4, bj = (u >> 3) & 1, uw = u & 7, dt = (slot * 2 + odd) * 8 + nt;
            Packed pk;
            const unsigned char* pl = sws + OFF_PP + (size_t)((((2 * self + odd) * 8 + nt) * 8 + uw) * 2 + bj) * UNIT;
            const unsigned char* const fp = ws + OFF_FP + ((dt * 8 + uw) * 2 + bj) * 4;
            const unsigned char* const dp = ws + OFF_PD + (size_t)((dt * 8 + uw) * 2 + bj) * 8192;
            u32x4 dg[8]; unsigned tw = 0u;
            if ((PLAIN_U >> u) & 1u) {
                wait_word<false>(fp, call, tmo, call);
                load_parked(dg, tw, dp, tmo, lane);
                PARKED_WAIT(dg, tw);
                if (!header_ok(h, call, pk.mask, pk.meta)) wait_header(head_of(u), tmo, call, pk.mask, pk.meta);
                acquire();
                tw = 0u;
                load_parked(cur, tw, pl, tmo, lane);
                PARKED_WAIT(cur, tw);
                release();
                USTAMP();
                u = (int)lds_draw(2); h = (u32x4){0u, 0u, 0u, 0u};
            } else {
                if (!header_ok(h, call, pk.mask, pk.meta)) wait_header(head_of(u), tmo, call, pk.mask, pk.meta);
                acquire();
                unsigned pf;
                asm volatile("global_load_dword %0, %1, off sc0 sc1" : "=&v"(pf) : "v"(fp) : "memory");
                fetch(pk, pl, lane);
                FETCH_WAIT(pk, 0);
                release();
                asm volatile("" : "+v"(pf));
                USTAMP();
                if ((unsigned)__builtin_amdgcn_readfirstlane(pf) < call) wait_word<false>(fp, call, tmo, call);
                load_parked(dg, tw, dp, tmo, lane);
                unpack(pk, cur, pl, scr, lane, [&]() {
                    u = (int)lds_draw(2);
                    const unsigned zero = 0u;
                    asm volatile(SBASE "global_load_dwordx4 %0, %1, %2 sc0 sc1" : "=&v"(h) : "v"(zero), "s"(head_of(u)) : "memory");
                });
#pragma unroll
                for (int i = 0; i < 8; ++i) asm volatile("" : "+v"(cur[i]));
                PARKED_WAIT(dg, tw);
                asm volatile("" : "+v"(h));
            }
            const bool bad = (unsigned)__builtin_amdgcn_readfirstlane(tw) == call || ((pk.meta >> 8) & 63u) == 63u;
            unsigned char* dst = P.out + (size_t)(odd * 256 + bj * 128 + (uw & 3) * 32 + fq * 4) * (GN * 2) + (size_t)(gnt * 256 + (uw >> 2) * 128 + fr * 8) * 2;
#pragma unroll
            for (int n = 0; n < 2; ++n)
#pragma unroll
                for (int r = 0; r < 4; ++r) {
                    u32x4 v;
#pragma unroll
                    for (int j = 0; j < 4; ++j) {
                        const unsigned pu = cur[n * 4 + r][j], w = dg[n * 4 + r][j];
                        const f32x2 y = (f32x2){__uint_as_float(w << 16), __uint_as_float(w & 0xffff0000u)}
                                      + (f32x2){__uint_as_float(pu << 16), __uint_as_float(pu & 0xffff0000u)};
                        const f32x2 o = gelu2(y);
                        v[j] = cvt_pk_bf16(o.x, o.y);
                    }
                    if (bad) v = (u32x4){0x7fc07fc0u, 0x7fc07fc0u, 0x7fc07fc0u, 0x7fc07fc0u};
                    st16_sys(dst + (size_t)(n * 16 + r) * (GN * 2), v);
                }
        }
        if (tid == 0) __hip_atomic_store((unsigned*)(ws + OFF_CNT + b * 128), call, __ATOMIC_RELAXED, __HIP_MEMORY_SCOPE_AGENT);
        STAMP(5);
        return;
    }

    if (!is_b) {
        unsigned char* const hdst = (unsigned char*)(((u64)peer_hi << 32) | peer_lo) + OFF_HIN + (par * 8 + self) * (768 * HS);
        for (int t = task0;;) {
#pragma unroll
            for (int i = 0; i < 8; ++i) asm volatile("" : "+v"(cur[i]));
            const int tn = t < (PK_ALL - 1) * 1152 ? t + 1152 : t >= PK_ALL * 1152 ? t + 704 : PK_ALL * 1152 + wid * 88 + (is_a ? g >> 1 : g - 14) * 8 + nt;
            const bool more = tn < 2784 && (tn < PK_ALL * 1152 || is_c || hf);
            int isw, kc, tile; pack_decode(t, isw, kc, tile); const int q = t & 3, idx = self * 8 + kc;
            if (isw) {
                u32x4 tr[8];
#pragma unroll
                for (int j = 0; j < 8; ++j)
#pragma unroll
                    for (int qd = 0; qd < 4; ++qd)
                        tr[j][qd] = perm(cur[2 * qd + 1][j >> 1], cur[2 * qd][j >> 1], (j & 1) ? 0x07060302u : 0x05040100u);
#pragma unroll
                for (int j = 0; j < 8; ++j) cur[j] = tr[j];
            }
            const bool own = (isw ? tile >> 3 : tile >> 1) == self, diag = t >= 992;
            const bool packed = !own && t < 2560, image = !isw || own || diag;
            u32x4 hd = {0u, 0u, 0u, 0u};
            if (packed && t >= 64 && t < 64 + 32 * PLAIN) {
                unsigned char* pl = ws + OFF_RP + (par * 2560 + tile * 32 + q) * UNIT + lane * 16;
#pragma unroll
                for (int i = 0; i < 8; ++i) st16_sys(pl + i * 1024, cur[i]);
                hd = (u32x4){0u, call, 0u, call << 15};
            } else if (packed) {
                Packed pk; pack(cur, pk, scr, lane);
                send(pk, ws + OFF_RP + (par * 2560 + (isw ? tile * 8 + kc : 512 + kc * 16 + tile) * 4 + q) * UNIT, scr, lane);
                hd = header(pk, call);
            }
            if (image)
                store_image(ws + (diag ? OFF_WD + (size_t)(tile * 8 + kc) * BLK : isw ? OFF_WB + (size_t)(idx * 8 + (tile & 7)) * BLK
                                                                                     : OFF_XA + (size_t)(idx * 16 + tile) * BLK), isw, q, cur);
            load_task(more ? tn : t, cur);
            asm volatile("s_waitcnt vmcnt(8)" ::: "memory");
            if (packed && lane < NR && lane != self && (!isw || lane == (tile >> 3)))
                st16_sys(hdst + ((isw ? kc * 8 + (tile & 7) : 64 + kc * 16 + tile) * 4 + q) * HS, hd);
            if (image && lane < (diag ? 1 : isw ? RW : RX))
                st_flag8(ws + (diag ? OFF_FD + (tile * 8 + kc) * 4 : isw ? OFF_FW + ((tile & 7) * 64 + idx) * 4 : OFF_FX + (tile * 64 + idx) * 4) + q + lane * FR, call);
            if (!isw && tile == 2 * self + 1 && lane == 0) st_flag8(ws + OFF_FQ + kc * 4 + q, call);
            __builtin_amdgcn_s_setprio(0);
            if (!more) break;
            t = tn;
        }
        STAMP(6);
    }

    const unsigned foff = lane * 4;
    const unsigned pickoff = (unsigned)(size_t)(lds + 9 * HTB);
    const int fxc = (RX > 8 ? nt * 2 + hf : nt % RX) * FR, fwc = (g % RW) * FR;
    const unsigned lds4 = (unsigned)(size_t)lds + wid * 4096, lds2 = (unsigned)(size_t)lds + wid * 2048;
    const unsigned voff4 = wid * 4096 + lane * 16, voff2 = wid * 2048 + lane * 16;
#define GL4(bufoff, vo, gbase) asm volatile("s_add_i32 m0, %0, %1\n\ts_nop 3\n\tglobal_load_lds_dwordx4 %2, %3 sc0 sc1\n\t" \
        "global_load_lds_dwordx4 %2, %3 offset:1024 sc0 sc1\n\tglobal_load_lds_dwordx4 %2, %3 offset:2048 sc0 sc1\n\t" \
        "global_load_lds_dwordx4 %2, %3 offset:3072 sc0 sc1" :: "s"(lds4), "i"(bufoff), "v"(vo), "s"(gbase) : "memory", "scc")
#define GL2(bufoff, vo, gbase) asm volatile("s_add_i32 m0, %0, %1\n\ts_nop 3\n\tglobal_load_lds_dwordx4 %2, %3 sc0 sc1\n\t" \
        "global_load_lds_dwordx4 %2, %3 offset:1024 sc0 sc1" :: "s"(lds2), "i"(bufoff), "v"(vo), "s"(gbase) : "memory", "scc")
#define LIKELY(c) __builtin_expect(!!(c), 1)
#define RARE(c) __builtin_expect(!!(c), 0)
#define LDA(dst, base, off) do { _Pragma("unroll") for (int m = 0; m < 4; ++m) _Pragma("unroll") for (int k = 0; k < 2; ++k) \
        dst[m][k] = *(const LAS bf16x8*)(lds + (base) + ((off) + m * 2048 + k * 1024)); } while (0)
#define LDB(dst, base, off) do { _Pragma("unroll") for (int n = 0; n < 2; ++n) _Pragma("unroll") for (int k = 0; k < 2; ++k) \
        dst[n][k] = *(const LAS bf16x8*)(lds + (base) + ((off) + n * 2048 + k * 1024)); } while (0)
#define MMA(ai, bj, At, Bt) do { __builtin_amdgcn_s_setprio(1); _Pragma("unroll") for (int m = 0; m < 4; ++m) _Pragma("unroll") for (int n = 0; n < 2; ++n) _Pragma("unroll") for (int k = 0; k < 2; ++k) \
        acc[ai][bj][m][n] = __builtin_amdgcn_mfma_f32_16x16x32_bf16(Bt[n][k], At[m][k], acc[ai][bj][m][n], 0, 0, 0); __builtin_amdgcn_s_setprio(0); } while (0)
#define MMQ(ai, bj, At, Bt) do { _Pragma("unroll") for (int k = 0; k < 2; ++k) _Pragma("unroll") for (int m = 0; m < 4; ++m) _Pragma("unroll") for (int n = 0; n < 2; ++n) \
        acc[ai][bj][m][n] = __builtin_amdgcn_mfma_f32_16x16x32_bf16(Bt[n][k], At[m][k], acc[ai][bj][m][n], 0, 0, 0); SCHED; } while (0)
#define WAIT_V(n) asm volatile("s_waitcnt vmcnt(" #n ")" ::: "memory")
#define WAIT_L0 do { asm volatile("" ::: "memory"); __builtin_amdgcn_s_waitcnt(0xC07F); asm volatile("" ::: "memory"); } while (0)
#define BAR __builtin_amdgcn_s_barrier()
#define SCHED __builtin_amdgcn_sched_barrier(0)
    typedef unsigned u32x2 __attribute__((ext_vector_type(2)));
#define SNAP() asm volatile(SBASE "global_load_dword %0, %2, %4 sc0 sc1\n\tglobal_load_dword %1, %3, %4 sc0 sc1" : "+v"(sna), "+v"(snb) : "v"(fa), "v"(fb), "s"(ws) : "memory")
#define PUBLISH(m, w) asm volatile("ds_write_b64 %0, %1 offset:%2" :: "v"(pickoff), "v"((u32x2){(unsigned)(m), (unsigned)((m) >> 32)}), "i"(8 * (w)) : "memory")
#define LISTEN(w) asm volatile("ds_read_b64 %0, %1 offset:%2" : "=v"(mv) : "v"(pickoff), "i"(8 * (w)) : "memory"); SCHED
#define N0 asm volatile("s_waitcnt lgkmcnt(0)" : "+v"(mv) :: "memory"); m_ = ((u64)(unsigned)__builtin_amdgcn_readfirstlane(mv.y) << 32) | (unsigned)__builtin_amdgcn_readfirstlane(mv.x); asm volatile("" : "+s"(m_)); SCHED
#define N5 asm volatile("s_andn2_b64 %0, %0, %2\n\ts_andn2_b64 %1, %1, %2" : "+s"(todo), "+s"(avail) : "s"(m_) : "scc"); SCHED
#define MF(ai, bj, At, Bt, m, k) acc[ai][bj][m][0] = __builtin_amdgcn_mfma_f32_16x16x32_bf16(Bt[0][k], At[m][k], acc[ai][bj][m][0], 0, 0, 0); \
                                 acc[ai][bj][m][1] = __builtin_amdgcn_mfma_f32_16x16x32_bf16(Bt[1][k], At[m][k], acc[ai][bj][m][1], 0, 0, 0); SCHED
    const int w0 = wid == 0;
#define KSTAMP() do {} while (0)
    f32x4 acc[2][2][4][2];
    auto zero_acc = [&]() {
#pragma unroll
        for (int a = 0; a < 2; ++a)
#pragma unroll
            for (int c = 0; c < 2; ++c)
#pragma unroll
                for (int m = 0; m < 4; ++m)
#pragma unroll
                    for (int n = 0; n < 2; ++n) acc[a][c][m][n] = (f32x4){0.f, 0.f, 0.f, 0.f};
    };
    auto to_bf16 = [&](int c, u32x4 (&v)[8]) {
#pragma unroll
        for (int n = 0; n < 2; ++n)
#pragma unroll
            for (int r = 0; r < 4; ++r) {
                u32x4& o = v[n * 4 + r];
                o.x = cvt_pk_bf16(acc[0][c][0][n][r], acc[0][c][1][n][r]); o.y = cvt_pk_bf16(acc[0][c][2][n][r], acc[0][c][3][n][r]);
                o.z = cvt_pk_bf16(acc[1][c][0][n][r], acc[1][c][1][n][r]); o.w = cvt_pk_bf16(acc[1][c][2][n][r], acc[1][c][3][n][r]);
            }
    };
    auto emit_unit = [&](int c, int bj) {
        const int lz = lane_id();
        const unsigned char* const started = ws + OFF_FW + fwc + nt * 256 + owner * 32;
        unsigned og, ot;
        asm volatile("global_load_dword %0, %2, off sc0 sc1\n\tglobal_load_dword %1, %3, off sc1" : "=&v"(og), "=&v"(ot) : "v"(started), "v"(tmo) : "memory");
        u32x4 v[8];
        to_bf16(c, v);
        Packed pk;
        if (wid < 4) __builtin_amdgcn_s_setprio(2);
        const bool plain = (PLAIN_U >> ((mt & 1) * 16 + bj * 8 + wid)) & 1u;
        if (!plain) pack(v, pk, scr, lz);
        __builtin_amdgcn_s_setprio(0);
        asm volatile("s_waitcnt vmcnt(0)" : "+v"(og), "+v"(ot) :: "memory");
        bool bad = (unsigned)__builtin_amdgcn_readfirstlane(ot) == call;
        if ((unsigned)__builtin_amdgcn_readfirstlane(og) != tag4) bad = wait_word<true>(started, tag4, tmo, call) || bad;
        STAMP(4);
        unsigned char* const pl = ws + OFF_PP + (size_t)(((mt * 8 + nt) * 8 + wid) * 2 + bj) * UNIT;
        if (plain) {
#pragma unroll
            for (int i = 0; i < 8; ++i) st16_sys(pl + i * 1024 + lz * 16, v[i]);
            pk.mask = 0ull; pk.meta = 0u;
        } else send(pk, pl, scr, lz);
        asm volatile("s_waitcnt vmcnt(0)" ::: "memory");
        if (bad) pk.meta |= 63u << 8;
        if (lz == 0) st16_sys(P.ws[owner] + OFF_HP + (size_t)((((mt & 1) * 64 + self * 8 + nt) * 8 + wid) * 2 + bj) * HS, header(pk, call));
    };
    auto finish = [&]() {
        if (threadIdx.x == 0) __hip_atomic_store((unsigned*)(ws + OFF_CNT + b * 128), call, __ATOMIC_RELAXED, __HIP_MEMORY_SCOPE_AGENT);
        STAMP(5);
    };
    auto park = [&](int c, int pt, int bj) {
        unsigned char* dp = ws + OFF_PD + (size_t)((pt * 8 + wid) * 2 + bj) * 8192 + lane_id() * 16;
        u32x4 v[8];
        to_bf16(c, v);
#pragma unroll
        for (int i = 0; i < 8; ++i) st16_sys(dp + i * 1024, v[i]);
    };
    const u64 help = (HELP_KC * 0x0101010101010101ull) & ~(0xffull << (self * 8));
    STAMP(2);
    if (!is_a) {
      const int nsteps = is_b ? 1 : 1 + (7 + C_ODD + 24 - g) / 4;
#pragma unroll 1
      for (int step = 0;; ++step) {
        const bool diag = step < nsteps - 1;
        const int dj = step * 4 + g - 21, dodd = dj >= 7, link = dj - 7 * dodd, dg = ((self + 1 + link) & 7) * 8 + nt, jm = diag ? 2 * self + dodd : mt;
                u64 todo = diag ? 0xffull << (self * 8) : is_b ? ~(0xffull << (owner * 8)) : (hf ? 0x55aa55aa55aa55aaull : 0xaa55aa55aa55aa55ull) & ~help, avail = todo & READY0;
        const int nkt = __builtin_popcountll(todo);
        const unsigned char* const abase = diag ? ws + OFF_WD + (size_t)(dg * 8 - self * 8) * BLK : ws + OFF_WB + (size_t)nt * BLK;
        const int ashift = diag ? 15 : 18;
        const unsigned fa = (unsigned)(diag ? OFF_FD + (dg * 8 - self * 8) * 4 : OFF_FW + fwc + nt * 256) + foff;
        const int x0 = (is_c && !diag && hf) ? HTB : 0;
        const unsigned voffx = voff4 ^ x0, voffh = voff2 + HTB;
        const unsigned char* const bbase = ws + OFF_XA + (size_t)jm * BLK;
        const unsigned fb = (unsigned)OFF_FX + fxc + jm * 256 + foff;
        const int aoff = wr * 8192 + lane * 16;
        int boff = 4 * HTB + wc * 4096 + lane * 16;
        asm volatile("" : "+v"(boff));
        zero_acc();
        bf16x8 At[4][2], B0[2][2], B1[2][2];
        const unsigned char *a2 = abase, *b2 = bbase;
        unsigned sna = ~tag4, snb = ~tag4; u32x2 mv; int look = w0;
        int vm = 0; const bool stampit = !diag; (void)stampit;
        int left = __builtin_amdgcn_readfirstlane(nkt);
        if (tid < 6) *(LAS unsigned*)(lds + 9 * HTB + tid * 4) = 0u;
        __syncthreads();
        if (wr == 1) BAR;
#define SA(bf, h) (((bf) * 2 + (h)) * HTB)
#define SB(bf, h) ((4 + (bf) * 2 + (h)) * HTB)
#define F1 asm volatile("s_and_b64 %1, %1, %2\n\ts_or_b64 %0, %0, %1" : "+s"(avail), "+s"(m_) : "s"(todo) : "scc"); SCHED
#define F2 asm volatile("s_ff1_i32_b64 %0, %2\n\ts_lshl_b64 %1, 1, %0" : "=&s"(i_), "=s"(m_) : "s"(avail) : "scc"); SCHED
#define F3 asm volatile("s_and_b64 %0, %0, %2\n\ts_cselect_b32 %1, 4, 0" : "+s"(m_), "=s"(vb_) : "s"(avail) : "scc"); SCHED
#define F6 a2 = abase + ((unsigned)i_ << ashift); asm volatile("" : "+s"(a2)); SCHED
#define F7 b2 = bbase + ((unsigned)i_ << 19); asm volatile("" : "+s"(b2)); SCHED
#define F8 asm volatile("s_bcnt1_i32_b64 %0, %1\n\ts_cmp_lt_u32 %0, %2\n\ts_cselect_b32 %0, %3, 0" : "=&s"(look) : "s"(avail), "i"(SNAP_AT), "s"(w0) : "scc"); \
           vm = (vm | vb_) >> 1; asm volatile("" : "+s"(vm)); SCHED
#define KTILE(bf) do { \
        LDB(B0, boff, SA(bf, 0)); LDB(B1, boff, SA(bf, 1)); LDA(At, aoff, SA(bf, 0)); SCHED; \
        if (LIKELY(vm & 2)) { GL2(SA(1 - bf, 1), voffh, a2); WAIT_V(8); } else WAIT_V(0); \
        if (RARE(look)) { asm volatile("" : "+v"(sna), "+v"(snb)); \
            u64 m = __ballot(sna == tag4 && snb == tag4); \
            if (vm == 0 && ((m | avail) & todo) == 0ull) m = poll_ready(todo, ws, fa, fb, tmo, call, tag4, 0xffffffffu); \
            PUBLISH(m, bf); } \
        int cur_ = vm & 1; asm volatile("" : "+s"(cur_)); __builtin_amdgcn_s_setprio(1); \
        WAIT_L0; BAR; SCHED; LISTEN(bf); \
        u64 m_; int i_, vb_; \
        if (LIKELY(cur_)) { \
            MF(0, 0, At, B0, 0, 0); MF(0, 0, At, B0, 1, 0); MF(0, 0, At, B0, 2, 0); MF(0, 0, At, B0, 3, 0); N0; \
            MF(0, 0, At, B0, 0, 1); F1; MF(0, 0, At, B0, 1, 1); F2; MF(0, 0, At, B0, 2, 1); F3; MF(0, 0, At, B0, 3, 1); N5; \
            MF(0, 1, At, B1, 0, 0); F6; MF(0, 1, At, B1, 1, 0); F7; MF(0, 1, At, B1, 2, 0); F8; MF(0, 1, At, B1, 3, 0); \
            MF(0, 1, At, B1, 0, 1); MF(0, 1, At, B1, 1, 1); MF(0, 1, At, B1, 2, 1); MF(0, 1, At, B1, 3, 1); } \
        asm volatile("" : "+s"(cur_)); \
        if (RARE(!cur_)) { N0; F1; F2; F3; N5; F6; F7; F8; } \
        BAR; __builtin_amdgcn_s_setprio(0); SCHED; \
        LDA(At, aoff, SA(bf, 1)); SCHED; \
        if (RARE(look)) SNAP(); \
        if (LIKELY(vm & 2)) { GL4(SB(bf, 0), voffx, b2); GL2(SA(bf, 0), voff2, a2); WAIT_V(8); } else WAIT_V(2); \
        left -= cur_; KSTAMP(); \
        asm volatile("" : "+s"(cur_), "+s"(left)); __builtin_amdgcn_s_setprio(1); \
        WAIT_L0; BAR; SCHED; if (LIKELY(cur_)) { MMQ(1, 0, At, B0); MMQ(1, 1, At, B1); } BAR; __builtin_amdgcn_s_setprio(0); SCHED; } while (0)
        for (;;) {
            KTILE(0);
            if (left == 0) break;
            KTILE(1);
            if (left == 0) break;
        }
        asm volatile("s_waitcnt vmcnt(0)\n\ts_nop 15\n\ts_nop 7" : "+v"(sna), "+v"(snb) :: "memory");
        if (wr == 0) BAR;
        if (!diag) break;
        park(0, link * 16 + dodd * 8 + nt, 0); park(1, link * 16 + dodd * 8 + nt, 1);
        asm volatile("s_waitcnt vmcnt(0)" ::: "memory");
        if (lane_id() < 2) st_flag(ws + OFF_FP + (((link * 16 + dodd * 8 + nt) * 8 + wid) * 2 + lane_id()) * 4, call);
        STAMP(7);
      }
      {
        if (is_b) {
            STAMP(3);
            emit_unit(0, 0); emit_unit(1, 1);
        } else {
            STAMP(3);
            const int lz = lane_id(), pt = 112 + (mt & 1) * 8 + nt;
            park(1, pt, 1 - hf);
            asm volatile("s_waitcnt vmcnt(0)" ::: "memory");
            if (lz == 0) st_flag(ws + OFF_FP + ((pt * 8 + wid) * 2 + 1 - hf) * 4, call);
            wait_word<false>(ws + OFF_FP + ((pt * 8 + wid) * 2 + hf) * 4, call, tmo, call);
            if (HELP_KC) wait_word<false>(ws + OFF_FP + (((pt + 16) * 8 + wid) * 2 + hf) * 4, call, tmo, call);
            u32x4 dg[8], dh[8]; unsigned tw = 0u, tv = 0u;
            load_parked(dg, tw, ws + OFF_PD + (size_t)((pt * 8 + wid) * 2 + hf) * 8192, tmo, lz);
            load_parked(dh, tv, ws + OFF_PD + (size_t)(((pt + 16) * 8 + wid) * 2 + hf) * 8192, tmo, lz);
            PARKED_WAIT(dg, tw); PARKED_WAIT(dh, tv);
            const bool bad = (unsigned)__builtin_amdgcn_readfirstlane(tw) == call;
            unsigned char* dst = P.out + (size_t)((mt & 1) * 256 + hf * 128 + wc * 32 + (lz >> 4) * 4) * (GN * 2) + (size_t)((self * 8 + nt) * 256 + wr * 128 + (lz & 15) * 8) * 2;
#pragma unroll
            for (int n = 0; n < 2; ++n)
#pragma unroll
                for (int r = 0; r < 4; ++r) {
                    u32x4 v;
#pragma unroll
                    for (int j = 0; j < 4; ++j) {
                        const unsigned w = dg[n * 4 + r][j], x = HELP_KC ? dh[n * 4 + r][j] : 0u;
                        const f32x2 o = gelu2((f32x2){acc[j >> 1][0][(j & 1) * 2][n][r], acc[j >> 1][0][(j & 1) * 2 + 1][n][r]}
                                              + (f32x2){__uint_as_float(w << 16), __uint_as_float(w & 0xffff0000u)}
                                              + (f32x2){__uint_as_float(x << 16), __uint_as_float(x & 0xffff0000u)});
                        v[j] = cvt_pk_bf16(o.x, o.y);
                    }
                    if (bad) v = (u32x4){0x7fc07fc0u, 0x7fc07fc0u, 0x7fc07fc0u, 0x7fc07fc0u};
                    st16_sys(dst + (size_t)(n * 16 + r) * (GN * 2), v);
                }
        }
      }
    } else {
      const int dg = owner * 8 + nt;
      const u64 dmask = 0xffull << (owner * 8);
      const bool mine = (g >> 1) >= C_ODD;
      u64 todo = mine ? ~0ull : ~dmask, avail = todo & READY0;
      const unsigned char* const abase = ws + OFF_WB + (size_t)nt * BLK;
      const unsigned char* const bbase = ws + OFF_XA + (size_t)mt * BLK + hf * HTB;
      const unsigned char* const adiag = ws + OFF_WD + (size_t)(dg * 8 - owner * 8) * BLK;
      const unsigned char* const bdiag = ws + OFF_XA + (size_t)(2 * self + 1) * BLK + hf * HTB + (size_t)(self - owner) * (8 * 16 * (size_t)BLK);
      const bool dl = (lane >> 3) == owner;
      unsigned fa = (unsigned)(dl ? OFF_FD + (dg * 8 - owner * 8) * 4 : OFF_FW + fwc + nt * 256) + foff;
      unsigned fb = (unsigned)(dl ? OFF_FQ - owner * 32 : OFF_FX + fxc + mt * 256) + foff;
      unsigned xm = hf ? 0xffff0000u : 0x0000ffffu;
      const int cm = 2 * self + (g >> 2), ch = (g >> 1) & 1;
      zero_acc();
#pragma unroll 1
      for (int step = 0;; ++step) {
        int aoff[3], boff[3];
#pragma unroll
        for (int i = 0; i < 3; ++i) {
            aoff[i] = i * 3 * HTB + wr * 8192 + lane * 16; boff[i] = i * 3 * HTB + 2 * HTB + wc * 4096 + lane * 16;
            asm volatile("" : "+v"(aoff[i]), "+v"(boff[i]));
        }
        bf16x8 A0[4][2], A1[4][2], B0[2][2];
        unsigned sna = ~tag4, snb = ~tag4; u32x2 mv;
        int vm = 0;
        int look = w0, more = 1;
        const unsigned char *a2 = ws, *b2 = ws;
        const u64 low = step ? 0ull : dmask;
        const unsigned char* const am = step == 1 ? adiag : abase;
        const unsigned char* const bm = step == 1 ? bdiag : step ? ws + OFF_XA + (size_t)cm * BLK + ch * HTB : bbase;
        const int shm = step == 1 ? 15 : 18, vbm = step == 1 ? 0x44 : 4;
        int mleft = __builtin_popcountll(todo & ~low);
        u64 fill = ~0ull;
        if (tid < 6) *(LAS unsigned*)(lds + 9 * HTB + tid * 4) = 0u;
        __syncthreads();
        if (wr == 1) BAR;
#define HSTAMP() do {} while (0)
#define SEGT(i) do {} while (0)
#define N1 asm volatile("s_and_b64 %1, %1, %2\n\ts_or_b64 %0, %0, %1\n\ts_and_b64 %1, %0, %3" : "+s"(avail), "+s"(m_) : "s"(todo), "s"(fill) : "scc"); SCHED
#define N2 asm volatile("s_andn2_b64 %0, %7, %8\n\ts_cselect_b64 %0, %0, %6\n\ts_cselect_b64 %1, %9, %10\n\ts_cselect_b64 %2, %11, %12\n\t" \
                        "s_cselect_b32 %3, %13, 15\n\ts_cselect_b32 %4, %14, 0x44\n\ts_subb_u32 %5, %5, 0" \
                        : "=&s"(c_), "=&s"(a2), "=&s"(b2), "=&s"(sh_), "=&s"(vb_), "+s"(mleft) \
                        : "s"(m_), "s"(avail), "s"(low), "s"(am), "s"(adiag), "s"(bm), "s"(bdiag), "s"(shm), "s"(vbm) : "scc"); SCHED
#define N3 asm volatile("s_ff1_i32_b64 %0, %2\n\ts_lshl_b64 %1, 1, %0" : "=&s"(i_), "=s"(m_) : "s"(c_) : "scc"); SCHED
#define N4 asm volatile("s_and_b64 %0, %0, %2\n\ts_cselect_b32 %1, %1, 0" : "+s"(m_), "+s"(vb_) : "s"(c_) : "scc"); SCHED
#define N6 vm = ((vm >> 1) & 0x33) | vb_; more = vm | mleft; asm volatile("" : "+s"(vm), "+s"(more)); SCHED
#define N7 a2 += (unsigned)i_ << sh_; asm volatile("" : "+s"(a2)); SCHED
#define N8 b2 += (unsigned)i_ << 19; asm volatile("" : "+s"(b2)); SCHED
#define N9 asm volatile("s_cmp_gt_i32 %3, %4\n\ts_cselect_b64 %0, -1, 0\n\ts_andn2_b64 %2, %5, %6\n\ts_bcnt1_i32_b64 %1, %2\n\ts_cmp_lt_u32 %1, %7\n\ts_cselect_b32 %1, %8, 0" \
                        : "=&s"(fill), "=&s"(look), "=&s"(c_) : "s"(mleft), "i"(FILL_STOP), "s"(avail), "s"(low), "i"(SNAP_H), "s"(w0) : "scc"); SCHED
#define MF2(ai, At, m, k) MF(ai, 0, At, B0, m, k)
#define HTILE(s) do { \
        constexpr int nb = ((s) + 2) % 3 * 3 * HTB; \
        if (LIKELY(vm & 4)) { GL2(nb + 2 * HTB, voff2, b2); GL4(nb, voff4, a2); } \
        SEGT(0); SCHED; LDB(B0, boff[s], 0); LDA(A0, aoff[s], 0); LDA(A1, aoff[s], HTB); SCHED; \
        asm volatile("s_bitcmp1_b32 %2, 2\n\ts_cbranch_scc1 1f\n\ts_waitcnt vmcnt(0)\n1:\n\ts_waitcnt vmcnt(6)" : "+v"(sna), "+v"(snb) : "s"(vm) : "memory", "scc"); \
        if (RARE(look)) {   \
            u64 m = __ballot(sna == tag4 && ((snb ^ tag4) & xm) == 0u); \
            const u64 want = todo & (fill | ~low); \
            if (vm == 0 && ((m | avail) & want) == 0ull) m = poll_ready(want, ws, fa, fb, tmo, call, tag4, xm); \
            PUBLISH(m, s); \
            if ((m & todo & ~avail & ~low) == 0ull) SNAP(); } \
        WAIT_L0; SEGT(1); BAR; SEGT(2); SCHED; \
        LISTEN(s); \
        int kind = vm & 17; asm volatile("" : "+s"(kind)); \
        u64 m_, c_; int i_, sh_, vb_; \
        if (LIKELY(kind == 1)) { \
            __builtin_amdgcn_s_setprio(1); \
            MF2(0, A0, 0, 0); MF2(0, A0, 1, 0); MF2(0, A0, 2, 0); MF2(0, A0, 3, 0); N0; \
            MF2(0, A0, 0, 1); N1; MF2(0, A0, 1, 1); N2; MF2(0, A0, 2, 1); MF2(0, A0, 3, 1); N3; \
            MF2(1, A1, 0, 0); N4; MF2(1, A1, 1, 0); N5; MF2(1, A1, 2, 0); N6; MF2(1, A1, 3, 0); N7; \
            MF2(1, A1, 0, 1); N8; MF2(1, A1, 1, 1); N9; MF2(1, A1, 2, 1); MF2(1, A1, 3, 1); \
            __builtin_amdgcn_s_setprio(0); } \
        asm volatile("" : "+s"(kind)); \
        if (RARE(kind == 17)) { MMA(0, 1, A0, B0); MMA(1, 1, A1, B0); } \
        asm volatile("" : "+s"(kind)); \
        if (RARE(kind != 1)) { N0; N1; N2; N3; N4; N5; N6; N7; N8; N9; } \
        SEGT(3); HSTAMP(); SEGT(4); BAR; SEGT(5); SCHED; } while (0)
        for (;;) {
            HTILE(0);
            if (more == 0) break;
            HTILE(1);
            if (more == 0) break;
            HTILE(2);
            if (more == 0) break;
        }
        asm volatile("s_waitcnt vmcnt(0)\n\ts_nop 15\n\ts_nop 7" : "+v"(sna), "+v"(snb) :: "memory");
        if (wr == 0) BAR;
        if (step == 2) break;
        if (step == 0) {
            STAMP(3);
            emit_unit(0, hf);
            STAMP(8);
        }
#pragma unroll
        for (int i = 0; i < 16; ++i) acc[i >> 3][0][(i >> 1) & 3][i & 1] = (f32x4){0.f, 0.f, 0.f, 0.f};
        if (todo == 0ull) {
            const int pt = (g | 1) * 8 + nt;
            if (mine) {
                park(1, pt, hf);
                asm volatile("s_waitcnt vmcnt(0)" ::: "memory");
                if (lane_id() == 0) st_flag(ws + OFF_FP + ((pt * 8 + wid) * 2 + hf) * 4, call);
            }
            STAMP(7);
            if (!HELP_KC || hf || g >= 8) { finish(); return; }
            step = 1; todo = help; avail = 0ull;
            fa = (unsigned)OFF_FW + fwc + nt * 256 + foff; fb = (unsigned)OFF_FX + fxc + cm * 256 + foff;
            xm = ch ? 0xffff0000u : 0x0000ffffu;
        }
      }
      const int pt = 128 + (cm & 1) * 8 + nt;
      park(0, pt, ch);
      asm volatile("s_waitcnt vmcnt(0)" ::: "memory");
      if (lane_id() == 0) st_flag(ws + OFF_FP + ((pt * 8 + wid) * 2 + ch) * 4, call);
    }
    finish();
}

}

extern "C" void kernel_launch(int rank, int nranks, ncclComm_t comm, hipStream_t stream,
                              void* const* d_in, const int* in_sizes, int n_in,
                              void* d_out, int out_size, const int* in_dtypes, int out_dtype,
                              void* const* d_out_all, void* const* d_ws_all, size_t ws_bytes) {
    (void)comm; (void)in_dtypes; (void)out_dtype; (void)d_out_all;
    if (nranks != NR || (unsigned)rank >= (unsigned)NR || n_in < 2 || ws_bytes < WS_NEED
        || in_sizes[0] != GM * KS || in_sizes[1] != KS * GN || out_size != (GM / NR) * GN) {
        fprintf(stderr, "kernel_launch: unexpected problem shape\n");
        return;
    }
    Params p;
    p.x = (const unsigned char*)d_in[0];
    p.w = (const unsigned char*)d_in[1];
    p.out = (unsigned char*)d_out;
    for (int i = 0; i < NR; ++i) p.ws[i] = (unsigned char*)d_ws_all[i];
    p.own = p.ws[rank]; p.rank = rank; p.pad = 0;
    hipLaunchKernelGGL(fused_gemm_rs, dim3(256), dim3(512), 0, stream, p);
    HIP_CHECK(hipGetLastError());
}
```

```cpp
#include <hip/hip_runtime.h>
#include <rccl/rccl.h>
#include <cstdio>
#include <cstdint>

#define HIP_CHECK(expr) do { hipError_t _e = (expr); if (_e != hipSuccess) { \
    fprintf(stderr, "HIP error %s at %s:%d\n", hipGetErrorString(_e), __FILE__, __LINE__); \
    return; } } while (0)

namespace {

#define LAS __attribute__((address_space(3)))
#define GAS __attribute__((address_space(1)))
typedef short bf16x8 __attribute__((ext_vector_type(8)));
typedef float f32x4 __attribute__((ext_vector_type(4)));
typedef unsigned u32x4 __attribute__((ext_vector_type(4)));
typedef unsigned long long u64;

constexpr int NR = 8;
constexpr int GM = 4096, GK = 4096, GN = 16384;
constexpr int KS = GK / NR;
constexpr int BLK = 32768;
constexpr int HTB = 16384;
constexpr int WIN = 16;
constexpr int PW = 64;
constexpr int PULL_DELAY = 100, PACK_DELAY = 150;
constexpr int HEAD_W = 2;
constexpr int PK_ALL = 2;
constexpr int SNAP_AT = 2, SNAP_H = 1;
constexpr int FILL_STOP = 6;
constexpr int C_ODD = 1;
constexpr int PLAIN = 2;
constexpr unsigned PLAIN_U = 0xc0000000u;
constexpr u64 HELP_KC = 0xe0;
constexpr int HS = 128;
constexpr int HFULL = 2;
constexpr int RX = 8, RW = 8;
constexpr u64 DEADLINE = 50000000ull;
constexpr int NTASK = 704;
constexpr int UNIT = 9344;

constexpr size_t OFF_CNT = 0;
constexpr size_t OFF_TMO = 32768;
constexpr size_t OFF_TICK = 33792;
constexpr size_t OFF_FX = 131072;
constexpr size_t OFF_FW = OFF_FX + 4096;
constexpr int FR = 8192;
constexpr size_t OFF_FD = 98304;
constexpr size_t OFF_FQ = 104448;
constexpr size_t OFF_FP = 106496;
constexpr size_t OFF_XA = 1u << 20;
constexpr size_t OFF_WB = OFF_XA + (size_t)64 * 16 * BLK;
constexpr size_t OFF_WD = OFF_WB + (size_t)64 * 8 * BLK;
constexpr size_t OFF_RP = OFF_WD + (size_t)64 * 8 * BLK;
constexpr size_t OFF_PP = OFF_RP + (size_t)5120 * UNIT;
constexpr size_t OFF_PD = OFF_PP + (size_t)2048 * UNIT;
constexpr size_t OFF_HIN = OFF_PD + (size_t)144 * 131072;
constexpr size_t OFF_HP = OFF_HIN + (size_t)16 * 768 * HS;
constexpr size_t WS_NEED = OFF_HP + (size_t)2048 * HS + 4096;
static_assert(WS_NEED <= (size_t)161 * 1048576, "workspace");

struct Params {
    const unsigned char* x;
    const unsigned char* w;
    unsigned char* out;
    unsigned char* own;
    unsigned char* ws[NR];
    int rank, pad;
};

#define SBASE "s_nop 4\n\t"
__device__ __forceinline__ void st16_sys(unsigned char* p, u32x4 v) {
    asm volatile("global_store_dwordx4 %0, %1, off sc0 sc1\n\ts_nop 1" :: "v"(p), "v"(v) : "memory");
}
__device__ __forceinline__ void st_flag(unsigned char* p, unsigned v) {
    __hip_atomic_store((unsigned*)p, v, __ATOMIC_RELAXED, __HIP_MEMORY_SCOPE_SYSTEM);
}
__device__ __forceinline__ void st_flag8(unsigned char* p, unsigned v) {
    __hip_atomic_store(p, (unsigned char)v, __ATOMIC_RELAXED, __HIP_MEMORY_SCOPE_SYSTEM);
}
__device__ __forceinline__ unsigned lane_id() {
    return __builtin_amdgcn_mbcnt_hi(~0u, __builtin_amdgcn_mbcnt_lo(~0u, 0u));
}
__device__ __forceinline__ unsigned cvt_pk_bf16(float lo, float hi) {
    unsigned r; asm volatile("v_cvt_pk_bf16_f32 %0, %1, %2" : "=v"(r) : "v"(lo), "v"(hi)); return r;
}
typedef float f32x2 __attribute__((ext_vector_type(2)));
__device__ __forceinline__ f32x2 gelu2(f32x2 y) {
    const float k1 = -2.0f * 0.7978845608028654f * 1.4426950408889634f;
    const float k3 = k1 * 0.044715f;
    const f32x2 z = y * ((y * y) * k3 + k1);
    const f32x2 d = (f32x2){__builtin_amdgcn_exp2f(z.x), __builtin_amdgcn_exp2f(z.y)} + 1.0f;
    return y * (f32x2){__builtin_amdgcn_rcpf(d.x), __builtin_amdgcn_rcpf(d.y)};
}

template <bool EQ>
__device__ __forceinline__ bool wait_word(const unsigned char* p, unsigned want, unsigned char* tmo, unsigned call) {
    const u64 t0 = __builtin_amdgcn_s_memrealtime();
    for (;;) {
        unsigned g, t;
        asm volatile("global_load_dword %0, %2, off sc0 sc1\n\t"
                     "global_load_dword %1, %3, off sc1\n\t"
                     "s_waitcnt vmcnt(0)" : "=&v"(g), "=&v"(t) : "v"(p), "v"(tmo) : "memory");
        if ((unsigned)__builtin_amdgcn_readfirstlane(t) == call || __builtin_amdgcn_s_memrealtime() - t0 > DEADLINE) {
            __hip_atomic_store((unsigned*)tmo, call, __ATOMIC_RELAXED, __HIP_MEMORY_SCOPE_AGENT);
            return true;
        }
        const unsigned v = __builtin_amdgcn_readfirstlane(g);
        if (EQ ? v == want : v >= want) return false;
        __builtin_amdgcn_s_sleep(2);
    }
}

__device__ __forceinline__ u64 now() {
    u64 t; asm volatile("s_memrealtime %0\n\ts_waitcnt lgkmcnt(0)" : "=s"(t) :: "memory"); return t;
}
__device__ __forceinline__ u64 poll_ready(u64 todo, const unsigned char* ws, unsigned fa, unsigned fb,
                                          unsigned char* tmo, unsigned call, unsigned tag4, unsigned xm) {
    const u64 t0 = now();
    for (;;) {
        unsigned a, b, t; const unsigned zero = 0u;
        asm volatile(SBASE "global_load_dword %0, %3, %6 sc0 sc1\n\t"
                     "global_load_dword %1, %4, %6 sc0 sc1\n\t"
                     "global_load_dword %2, %5, %7 sc1\n\t"
                     "s_waitcnt vmcnt(0)"
                     : "=&v"(a), "=&v"(b), "=&v"(t) : "v"(fa), "v"(fb), "v"(zero), "s"(ws), "s"(tmo) : "memory");
        const u64 ready = __ballot(a == tag4 && ((b ^ tag4) & xm) == 0u);
        if ((ready & todo) != 0ull) return ready;
        if ((unsigned)__builtin_amdgcn_readfirstlane(t) == call || now() - t0 > DEADLINE) {
            __hip_atomic_store((unsigned*)tmo, call, __ATOMIC_RELAXED, __HIP_MEMORY_SCOPE_AGENT);
            return ~0ull;
        }
        __builtin_amdgcn_s_sleep(1);
    }
}

__device__ __forceinline__ unsigned bfi(unsigned m, unsigned a, unsigned b) { return (a & m) | (b & ~m); }
__device__ __forceinline__ unsigned perm(unsigned hi, unsigned lo, unsigned sel) { return __builtin_amdgcn_perm(hi, lo, sel); }
#define SDWA2(k) " dst_sel:DWORD dst_unused:UNUSED_PAD src0_sel:BYTE_" #k " src1_sel:BYTE_" #k "\n\t"
#define SDWA1(k) " dst_sel:DWORD dst_unused:UNUSED_PAD src0_sel:DWORD src1_sel:BYTE_" #k "\n\t"
__device__ __forceinline__ unsigned spread_bits(unsigned f, unsigned s) {
    unsigned r, a, b;
    asm("v_lshlrev_b32_sdwa %0, %4, %3" SDWA2(0) "v_lshlrev_b32_sdwa %1, %4, %3" SDWA2(1) "v_lshlrev_b32_sdwa %2, %4, %3" SDWA2(2)
        "v_or3_b32 %0, %0, %1, %2\n\t"
        "v_lshlrev_b32_sdwa %1, %4, %3" SDWA2(3)
        "v_or_b32 %0, %0, %1" : "=&v"(r), "=&v"(a), "=&v"(b) : "v"(f), "v"(s));
    return r;
}
__device__ __forceinline__ unsigned take_codes(unsigned& w, unsigned f) {
    unsigned ln, t, l;
    asm("v_xor_b32_sdwa %2, %0, %4" SDWA1(0) "v_ffbl_b32 %1, %2\n\tv_lshrrev_b32 %0, %1, %0\n\t"
        "v_xor_b32_sdwa %2, %0, %4" SDWA1(1) "v_ffbl_b32 %3, %2\n\tv_lshrrev_b32 %0, %3, %0\n\t"
        "v_xor_b32_sdwa %2, %0, %4" SDWA1(2) "v_lshl_or_b32 %1, %3, 8, %1\n\tv_ffbl_b32 %3, %2\n\tv_lshrrev_b32 %0, %3, %0\n\t"
        "v_xor_b32_sdwa %2, %0, %4" SDWA1(3) "v_lshl_or_b32 %1, %3, 16, %1\n\tv_ffbl_b32 %3, %2\n\tv_lshrrev_b32 %0, %3, %0\n\t"
        "v_lshl_or_b32 %1, %3, 24, %1" : "+v"(w), "=&v"(ln), "=&v"(t), "=&v"(l) : "v"(f));
    return ln;
}
__device__ __forceinline__ unsigned wave_scan(unsigned v) {
    v += (unsigned)__builtin_amdgcn_update_dpp(0, (int)v, 0x111, 0xf, 0xf, false);
    v += (unsigned)__builtin_amdgcn_update_dpp(0, (int)v, 0x112, 0xf, 0xf, false);
    v += (unsigned)__builtin_amdgcn_update_dpp(0, (int)v, 0x114, 0xf, 0xf, false);
    v += (unsigned)__builtin_amdgcn_update_dpp(0, (int)v, 0x118, 0xf, 0xf, false);
    v += (unsigned)__builtin_amdgcn_update_dpp(0, (int)v, 0x142, 0xa, 0xf, false);
    v += (unsigned)__builtin_amdgcn_update_dpp(0, (int)v, 0x143, 0xc, 0xf, false);
    return v;
}
__device__ __forceinline__ unsigned rank_in(u64 mask) {
    return __builtin_amdgcn_mbcnt_hi((unsigned)(mask >> 32), __builtin_amdgcn_mbcnt_lo((unsigned)mask, 0u));
}
constexpr int SCR = 8192;
#define LDS_ORDER() asm volatile("" ::: "memory")

struct Packed { u32x4 sm[4], l1, st; u64 mask; unsigned meta; };

__device__ __forceinline__ void pack(const u32x4 (&d)[8], Packed& o, LAS unsigned char* scr, unsigned lane) {
    *(LAS u32x4*)(scr + lane * 16) = (u32x4){0u, 0u, 0u, 0u};
    unsigned sum = 0; u32x4 er[4];
#pragma unroll
    for (int g = 0; g < 16; ++g) {
        const unsigned d0 = d[g >> 1][(g & 1) * 2], d1 = d[g >> 1][(g & 1) * 2 + 1];
        const unsigned lb = perm(d1, d0, 0x06040200u), hb = perm(d1, d0, 0x07050301u);
        o.sm[g >> 2][g & 3] = bfi(0x7f7f7f7fu, lb, hb);
        const unsigned e = bfi(0x01010101u, lb >> 7, hb << 1);
        er[g >> 2][g & 3] = e;
        if (g % 5 == 0) sum = __builtin_amdgcn_sad_u8(e, 0u, sum);
    }
    const unsigned r2 = min(max(((unsigned)__builtin_amdgcn_readlane(wave_scan(sum), 63) + 4009u) >> 10, 31u), 255u);
    const unsigned kk = r2 * 0x01010101u;
    unsigned bad = 0, top = ~0u, longest = 0, total = 0, piece[8], plen[8], us[16], cs[16];
#pragma unroll
    for (int g = 0; g < 16; ++g) {
        const unsigned u = kk - er[g >> 2][g & 3];
        bad |= u;
        us[g] = u; cs[g] = perm(0x0f0f0f0cu, 0x0a080f07u, u | ((u << 1) & 0x10101010u));
        top &= cs[g];
    }
    const bool hole = __ballot((top & 0x08080808u) != 0x08080808u) != 0ull;
    const unsigned tx = hole ? 0xfdfdfdfdu : 0xfefefefeu, thi = 0x01000000u ^ tx, tlo = 0x01020609u ^ tx;
#pragma unroll
    for (int q = 0; q < 4; ++q) {
        unsigned l1 = 0, acc = 0, base = 0;
#pragma unroll
        for (int gg = 0; gg < 4; ++gg) {
            const unsigned u = us[q * 4 + gg], c = cs[q * 4 + gg];
            l1 |= gg ? (c & 0x06060606u) << (2 * gg - 1) : (c & 0x06060606u) >> 1;
            const unsigned len4 = (perm(thi, tlo, u | (c & 0x10101010u)) ^ tx) + u;
            const unsigned f = c & 0x01010101u;
            unsigned ps = len4 + (len4 << 8); asm("" : "+v"(ps)); ps += ps << 16; ps -= 0x100c0804u;
            const unsigned bits = spread_bits(f, ps);
            if ((gg & 1) == 0) { acc = bits; base = ps >> 24; }
            else {
                acc |= bits << (base & 31u); base += ps >> 24;
                piece[q * 2 + (gg >> 1)] = acc >> 1; plen[q * 2 + (gg >> 1)] = base;
                longest = max(longest, base); total += base;
            }
        }
        o.l1[q] = l1;
    }
    const bool raw = (bad & 0xe0e0e0e0u) != 0u || longest > 31u || total > 112u;
    o.mask = __ballot(raw);
    if (raw) total = 0;
    const unsigned incl = wave_scan(total);
    const unsigned rawoff = (((unsigned)__builtin_amdgcn_readlane(incl, 63) + 512u + 127u) >> 7) << 4;
    o.meta = r2 | ((rawoff + 64u * (unsigned)__builtin_popcountll(o.mask) + 127u) >> 7) << 8 | (hole ? 0x4000u : 0u);
    LDS_ORDER();
    scr[lane] = (unsigned char)total;
    if (!raw) {
        unsigned off = incl - total + 512u;
#pragma unroll
        for (int i = 0; i < 8; ++i) {
            const u64 w = (u64)piece[i] << (off & 31u);
            LAS unsigned* p = (LAS unsigned*)(scr + ((off >> 5) << 2));
            __hip_atomic_fetch_or(p, (unsigned)w, __ATOMIC_RELAXED, __HIP_MEMORY_SCOPE_WORKGROUP);
            __hip_atomic_fetch_or(p + 1, (unsigned)(w >> 32), __ATOMIC_RELAXED, __HIP_MEMORY_SCOPE_WORKGROUP);
            off += plen[i];
        }
    } else {
        LAS u32x4* p = (LAS u32x4*)(scr + rawoff + rank_in(o.mask) * 64u);
#pragma unroll
        for (int i = 0; i < 4; ++i) p[i] = er[i];
    }
    LDS_ORDER();
    o.st = *(const LAS u32x4*)(scr + lane * 16);
    LDS_ORDER();
}
__device__ __forceinline__ void send(const Packed& o, unsigned char* pl, const LAS unsigned char* scr, unsigned lane) {
    unsigned char* p = pl + lane * 16;
    for (unsigned k = 1; k * 8u < ((o.meta >> 8) & 63u); ++k)
        if (k * 64u + lane < ((o.meta >> 8) & 63u) * 8u) st16_sys(p + 5120 + k * 1024, *(const LAS u32x4*)(scr + k * 1024 + lane * 16));
#pragma unroll
    for (int i = 0; i < 4; ++i) st16_sys(p + i * 1024, o.sm[i]);
    st16_sys(p + 4096, o.l1);
    if (lane < ((o.meta >> 8) & 63u) * 8u) st16_sys(p + 5120, o.st);
}
__device__ __forceinline__ u32x4 header(const Packed& o, unsigned call) {
    return (u32x4){(unsigned)o.mask, call, (unsigned)(o.mask >> 32), (call << 15) | o.meta};
}
__device__ __forceinline__ bool header_ok(u32x4 h, unsigned call, u64& mask, unsigned& meta) {
    const unsigned t1 = __builtin_amdgcn_readfirstlane(h.y), t3 = __builtin_amdgcn_readfirstlane(h.w);
    mask = ((u64)(unsigned)__builtin_amdgcn_readfirstlane(h.z) << 32) | (unsigned)__builtin_amdgcn_readfirstlane(h.x);
    meta = t3 & 0x7fffu;
    return t1 == call && (t3 >> 15) == (call & 0x1ffffu);
}
__device__ __forceinline__ void wait_header(const unsigned char* hd, unsigned char* tmo, unsigned call, u64& mask, unsigned& meta) {
    const u64 t0 = __builtin_amdgcn_s_memrealtime();
    for (;;) {
        u32x4 h; unsigned t;
        asm volatile("global_load_dwordx4 %0, %2, off sc0 sc1\n\t"
                     "global_load_dword %1, %3, off sc1\n\t"
                     "s_waitcnt vmcnt(0)" : "=&v"(h), "=&v"(t) : "v"(hd), "v"(tmo) : "memory");
        if (header_ok(h, call, mask, meta)) break;
        if ((unsigned)__builtin_amdgcn_readfirstlane(t) == call || __builtin_amdgcn_s_memrealtime() - t0 > DEADLINE) {
            __hip_atomic_store((unsigned*)tmo, call, __ATOMIC_RELAXED, __HIP_MEMORY_SCOPE_AGENT);
            mask = 0ull; meta = 0u;
            break;
        }
        __builtin_amdgcn_s_sleep(1);
    }
}
__device__ __forceinline__ void fetch(Packed& o, const unsigned char* pl, unsigned lane) {
    const unsigned n = min(max((o.meta >> 8) & 63u, 1u), 8u) * 8u;
    const u64 smask = n >= 64u ? ~0ull : (1ull << n) - 1ull;
    u64 keep;
    asm volatile(SBASE "s_mov_b64 %6, exec\n\t"
                 "global_load_dwordx4 %0, %7, %8 sc0 sc1\n\t"
                 "global_load_dwordx4 %1, %7, %8 offset:1024 sc0 sc1\n\t"
                 "global_load_dwordx4 %2, %7, %8 offset:2048 sc0 sc1\n\t"
                 "global_load_dwordx4 %3, %7, %8 offset:3072 sc0 sc1\n\t"
                 "global_load_dwordx4 %4, %7, %9 sc0 sc1\n\t"
                 "s_mov_b64 exec, %10\n\t"
                 "global_load_dwordx4 %5, %7, %9 offset:1024 sc0 sc1\n\t"
                 "s_mov_b64 exec, %6"
                 : "=&v"(o.sm[0]), "=&v"(o.sm[1]), "=&v"(o.sm[2]), "=&v"(o.sm[3]), "=&v"(o.l1), "=&v"(o.st), "=&s"(keep)
                 : "v"(lane * 16u), "s"(pl), "s"(pl + 4096), "s"(smask) : "memory");
}
#define FETCH_WAIT(o, n) asm volatile("s_waitcnt vmcnt(" #n ")" : "+v"(o.sm[0]), "+v"(o.sm[1]), "+v"(o.sm[2]), "+v"(o.sm[3]), "+v"(o.l1), "+v"(o.st) :: "memory")
template <class Mid>
__device__ __forceinline__ void unpack(const Packed& o, u32x4 (&d)[8], const unsigned char* pl, LAS unsigned char* scr, unsigned lane, Mid mid) {
    *(LAS u32x4*)(scr + lane * 16) = o.st;
    if (((o.meta >> 8) & 63u) > 8u) {
        u32x4 m[4];
        asm volatile("global_load_dwordx4 %0, %4, off sc0 sc1\n\t"
                     "global_load_dwordx4 %1, %4, off offset:1024 sc0 sc1\n\t"
                     "global_load_dwordx4 %2, %4, off offset:2048 sc0 sc1\n\t"
                     "global_load_dwordx4 %3, %4, off offset:3072 sc0 sc1\n\t"
                     "s_waitcnt vmcnt(0)" : "=&v"(m[0]), "=&v"(m[1]), "=&v"(m[2]), "=&v"(m[3]) : "v"(pl + 6144 + lane * 16) : "memory");
#pragma unroll
        for (int k = 0; k < 4; ++k) *(LAS u32x4*)(scr + (k + 1) * 1024 + lane * 16) = m[k];
    }
    LDS_ORDER();
    const unsigned len = scr[lane] & 127u;
    const unsigned incl = wave_scan(len), off = incl - len + 512u;
    const LAS unsigned* p = (const LAS unsigned*)(scr + ((off >> 5) << 2));
    const unsigned w0 = p[0], w1 = p[1], w2 = p[2], w3 = p[3], w4 = p[4];
    unsigned s0 = __builtin_amdgcn_alignbit(w1, w0, off), s1 = __builtin_amdgcn_alignbit(w2, w1, off);
    unsigned s2 = __builtin_amdgcn_alignbit(w3, w2, off), s3 = __builtin_amdgcn_alignbit(w4, w3, off);
    const bool hole = (o.meta & 0x4000u) != 0u;
    const unsigned tx = hole ? 0xf8f8f8f8u : 0xf7f7f7f7u, thi = (hole ? 0x07070008u : 0x08080808u) ^ tx, tlo = 0x0309090au ^ tx;
    const unsigned kk = ((o.meta & 255u) + 8u) * 0x01010101u;
    u32x4 er[4];
#pragma unroll
    for (int q = 0; q < 4; ++q) {
        if (q == 2) { asm volatile("" : "+v"(s0)); mid(); asm volatile("" : "+v"(s0)); }
        const unsigned l1 = o.l1[q], x = l1 & (l1 >> 1);
        unsigned w = 0, used = 0;
#pragma unroll
        for (int gg = 0; gg < 4; ++gg) {
            if ((gg & 1) == 0) { w = (s0 << 1) | 1u; used = 0; }
            const unsigned f = (x >> (2 * gg)) & 0x01010101u;
            const unsigned ln = take_codes(w, f);
            used = __builtin_amdgcn_sad_u8(ln, 0u, used);
            const unsigned y = (perm(thi, tlo, ln | ((ln << 1) & 0x10101010u)) ^ tx) + ln + ((l1 >> (2 * gg)) & 0x03030303u);
            er[q][gg] = kk - y;
            if (gg & 1) {
                s0 = __builtin_amdgcn_alignbit(s1, s0, used); s1 = __builtin_amdgcn_alignbit(s2, s1, used);
                s2 = __builtin_amdgcn_alignbit(s3, s2, used); s3 >>= used & 31u;
            }
        }
    }
    if (o.mask != 0ull) {
        const unsigned rawoff = (((unsigned)__builtin_amdgcn_readlane(incl, 63) + 512u + 127u) >> 7) << 4;
        if ((o.mask >> lane) & 1ull) {
            const LAS u32x4* r = (const LAS u32x4*)(scr + rawoff + rank_in(o.mask) * 64u);
#pragma unroll
            for (int i = 0; i < 4; ++i) er[i] = r[i];
        }
    }
    LDS_ORDER();
#pragma unroll
    for (int g = 0; g < 16; ++g) {
        const unsigned e = er[g >> 2][g & 3], sm = o.sm[g >> 2][g & 3];
        const unsigned hb = bfi(0x80808080u, sm, e >> 1), lb = bfi(0x7f7f7f7fu, sm, e << 7);
        d[g >> 1][(g & 1) * 2] = perm(hb, lb, 0x05010400u); d[g >> 1][(g & 1) * 2 + 1] = perm(hb, lb, 0x07030602u);
    }
}

__device__ __forceinline__ void load_parked(u32x4 (&dg)[8], unsigned& tw, const unsigned char* dp, const unsigned char* tmo, unsigned lane) {
    asm volatile(SBASE "global_load_dwordx4 %0, %9, %10 sc0 sc1\n\t"
                 "global_load_dwordx4 %1, %9, %10 offset:1024 sc0 sc1\n\t"
                 "global_load_dwordx4 %2, %9, %10 offset:2048 sc0 sc1\n\t"
                 "global_load_dwordx4 %3, %9, %10 offset:3072 sc0 sc1\n\t"
                 "global_load_dwordx4 %4, %9, %11 sc0 sc1\n\t"
                 "global_load_dwordx4 %5, %9, %11 offset:1024 sc0 sc1\n\t"
                 "global_load_dwordx4 %6, %9, %11 offset:2048 sc0 sc1\n\t"
                 "global_load_dwordx4 %7, %9, %11 offset:3072 sc0 sc1\n\t"
                 "global_load_dword %8, %8, %12 sc1"
                 : "=&v"(dg[0]), "=&v"(dg[1]), "=&v"(dg[2]), "=&v"(dg[3]), "=&v"(dg[4]), "=&v"(dg[5]), "=&v"(dg[6]), "=&v"(dg[7]), "+v"(tw)
                 : "v"(lane * 16u), "s"(dp), "s"(dp + 4096), "s"(tmo) : "memory");
}
#define PARKED_WAIT(dg, tw) asm volatile("s_waitcnt vmcnt(0)" : "+v"(dg[0]), "+v"(dg[1]), "+v"(dg[2]), "+v"(dg[3]), "+v"(dg[4]), "+v"(dg[5]), \
                                         "+v"(dg[6]), "+v"(dg[7]), "+v"(tw) :: "memory")


constexpr double ORD_W = 0.3, ORD_A1 = 1.17, ORD_B0 = 0.4, ORD_BS = 1.22, ORD_C0 = 10.5, ORD_CS = 0.3;
struct Order { unsigned short v[354]; };
constexpr Order make_order() {
    double key[352] = {}; unsigned short id[352] = {}; int n = 0;
    for (int kc = 0; kc < 8; ++kc)
        for (int h = 0; h < 2; ++h) {
            for (int e = 0; e < 8; ++e) { key[n] = kc - ORD_W + (e + 0.4) / 8 + h * 1e-3 - (kc == 0 && e < PLAIN ? 9 : 0); id[n++] = (unsigned short)(h << 8 | 0 << 6 | kc << 3 | e); }
            for (int e = 0; e < 6; ++e) { key[n] = (kc + (e + 0.5) / 6) * (h ? ORD_A1 : 1.0); id[n++] = (unsigned short)(h << 8 | 1 << 6 | kc << 3 | e); }
            for (int e = 0; e < 6; ++e) { key[n] = ORD_B0 + (kc + (e + 0.6) / 6) * ORD_BS + h * 1e-3; id[n++] = (unsigned short)(h << 8 | 2 << 6 | kc << 3 | e); }
            for (int e = 0; e < 2; ++e) { key[n] = ORD_C0 + (2 * kc + e + 0.5 * h) * ORD_CS; id[n++] = (unsigned short)(h << 8 | 3 << 6 | kc << 3 | e); }
        }
    Order o = {};
    for (int i = 0; i < 352; ++i) { int r = 0; for (int j = 0; j < 352; ++j) r += (key[j] < key[i]) || (key[j] == key[i] && j < i); o.v[r] = id[i]; }
    return o;
}
constexpr Order ORDER = make_order();
constexpr unsigned order_word(int i) { return ORDER.v[3 * i] | ORDER.v[3 * i + 1] << 10 | (unsigned)ORDER.v[3 * i + 2] << 20; }
template <int I, int N> __device__ __forceinline__ unsigned order_vgpr(unsigned v) {
    if constexpr (I == N) return v;
    else {
        unsigned t;
        asm("s_mov_b32 %1, %2\n\tv_writelane_b32 %0, %1, %3" : "+v"(v), "=&s"(t) : "i"(order_word(I)), "i"(I & 63));
        return order_vgpr<I + 1, N>(v);
    }
}
__device__ __forceinline__ void decode_unit(unsigned ord0, unsigned ord1, int t, int src, int dest, int& isw, int& kc, int& tile, int& q) {
    const int n = t >> 1, dw = n / 3;
    const unsigned w0 = __builtin_amdgcn_readlane(ord0, dw & 63), w1 = __builtin_amdgcn_readlane(ord1, dw & 63);
    const unsigned v = (dw < 64 ? w0 : w1) >> ((n - dw * 3) * 10);
    const int kind = (v >> 6) & 3, e = v & 7;
    kc = (v >> 3) & 7; isw = kind == 0; q = ((v >> 7) & 2) | (t & 1);
    const int d = (dest - src - 1) & 7;
    tile = kind == 0 ? ((e + src) & 7) : kind == 3 ? 2 * dest + e : 2 * ((src + 1 + e + (e >= d)) & 7) + (kind - 1);
}

__global__ __launch_bounds__(512, 2) void fused_gemm_rs(const Params P) {
    __shared__ __attribute__((aligned(16))) unsigned char lds_raw[9 * HTB + 256];
    LAS unsigned char* lds = (LAS unsigned char*)lds_raw;
    const int tid = threadIdx.x, wid = __builtin_amdgcn_readfirstlane(tid >> 6), lane = tid & 63;
    const int b = blockIdx.x, self = P.rank;
    unsigned char* const ws = P.own;
    const GAS unsigned char *px = (const GAS unsigned char*)P.x, *pw = (const GAS unsigned char*)P.w;
    asm volatile("" : "+s"(px), "+s"(pw));
    unsigned count; const unsigned vzero = 0u;
    asm volatile(SBASE "global_load_dword %0, %1, %2 sc1" : "=&v"(count) : "v"(vzero), "s"(ws + OFF_CNT + b * 128) : "memory");
    unsigned char* const tmo = ws + OFF_TMO;
    LAS unsigned char* const scr = lds + wid * SCR;
    const u64 tstart = __builtin_amdgcn_s_memrealtime();
    const int nt = b & 7, slot = b >> 3, r3 = slot - 7;
    const int g = r3 < 0 ? -1 : r3 >= 21 ? r3 : r3 % 3 == 2 ? 14 + r3 / 3 : r3 / 3 * 2 + r3 % 3;
    const bool is_pull = g < 0, is_a = !is_pull && g < 14, is_b = g >= 14 && g < 21, is_c = g >= 21;
    const int owner = is_c ? self : (self + 1 + (is_a ? g >> 1 : g - 14)) & 7;
    const int mt = 2 * owner + (is_c ? (g - 21) >> 1 : is_b), hf = is_c ? (g - 21) & 1 : g & 1;
    auto pack_decode = [&](int t, int& isw, int& kc, int& tile) {
        const int blk = t >> 2;
        if (blk < 80) {
            const int m = blk - 16; kc = 0; isw = blk >= 16;
            tile = isw ? ((self + 1 + m) & 7) * 8 + (((m >> 3) + self) & 7) : blk;
        } else if (blk < 248) {
            const int m = blk - 80, r = m % 24; kc = 1 + m / 24; isw = r >= 16;
            tile = isw ? self * 8 + (r & 7) : r;
        } else if (blk < 640) {
            const int m = blk - 248, w = m % 56; isw = 1; kc = 1 + m / 56;
            tile = ((self + 1 + w % 7) & 7) * 8 + ((w / 7 + self) & 7);
        } else {
            const int m = blk - 640; isw = 1; kc = 0;
            tile = ((self + 1 + (m >> 3)) & 7) * 8 + (m & 7);
        }
    };
    const unsigned lw = (lane >> 4) * 8 * (GN * 2) + (lane & 15) * 16, lx = (lane & 15) * (KS * 2) + (lane >> 4) * 16;
    auto load_task = [&](int t, u32x4 (&r)[8]) {
        int isw, kc, tile; pack_decode(t, isw, kc, tile); const int q = t & 3;
        const GAS unsigned char* src = isw ? pw + ((kc * 64 + (q & 1) * 32) * (GN * 2) + (tile * 256 + (q >> 1) * 128) * 2)
                                       : px + ((tile * 256 + q * 64) * (KS * 2) + kc * 128);
        const unsigned lo = isw ? lw : lx, s1 = isw ? GN * 2 : 64, s2 = isw ? GN * 4 : 16 * KS * 2;
#pragma unroll
        for (int i = 0; i < 8; ++i) r[i] = *(const GAS u32x4*)(src + ((i >> 1) * s2 + (i & 1) * s1) + lo);
    };
    auto store_image = [&](unsigned char* blk, int isw, int q, const u32x4 (&c)[8]) {
        unsigned char* d0 = blk + (isw ? (q >> 1) * 8192 + (q & 1) * 1024 : q * 8192) + lane * 16;
        const unsigned s1 = isw ? 2048 : 1024, s4 = isw ? HTB : 4096;
#pragma unroll
        for (int j = 0; j < 8; ++j) st16_sys(d0 + ((j >> 2) * s4 + (j & 3) * s1), c[j]);
    };
    u32x4 cur[8];
    const int task0 = is_pull ? wid * 8 + nt : wid * 144 + (is_a ? g : g - 7) * 8 + nt;
    unsigned peer_lo = 0u, peer_hi = 0u;
    auto peers = [&]() {
#pragma unroll
        for (int r = 0; r < NR; ++r)
            asm("v_writelane_b32 %0, %2, %4\n\tv_writelane_b32 %1, %3, %4" : "+v"(peer_lo), "+v"(peer_hi)
                : "s"((unsigned)(u64)P.ws[r]), "s"((unsigned)((u64)P.ws[r] >> 32)), "i"(r));
    };
    if (is_a || is_c) {
        if (wid >= HEAD_W) {
            if (wid < 2 * HEAD_W) {
                peers();
                int isw, kc, tile; const int t = task0 - HEAD_W * 144; pack_decode(t, isw, kc, tile);
                const size_t unit = (isw ? tile * 8 : 512 + tile) * 4 + (t & 3), head = (isw ? tile & 7 : 64 + tile) * 4 + (t & 3);
                const unsigned char* a = lane < NR ? (unsigned char*)(((u64)peer_hi << 32) | peer_lo) + OFF_HIN + (self * 768 + head) * HS + 64
                                       : lane < 10 ? ws + OFF_RP + ((lane & 1) * 2560 + unit) * UNIT
                                       : ws + (isw ? OFF_WB + (size_t)(self * 64 + (tile & 7)) * BLK : OFF_XA + (size_t)(self * 128 + tile) * BLK) + lane * 128;
                if (lane < 11 && (lane >= NR || !isw || lane == (tile >> 3))) { unsigned z; asm volatile("global_load_dword %0, %1, off sc0 sc1\n\ts_waitcnt vmcnt(0)" : "=&v"(z) : "v"(a) : "memory"); }
            }
            while (__builtin_amdgcn_s_memrealtime() - tstart < PACK_DELAY) __builtin_amdgcn_s_sleep(2); }
        else __builtin_amdgcn_s_setprio(2);
        load_task(task0, cur);
    }
    if (is_b && wid == 7 && lane < 16) {
        const unsigned char* ows = P.ws[owner];
        const unsigned char* a = lane < 2 ? ws + OFF_PP + (size_t)(((2 * owner + lane) * 8 + nt) * 16) * UNIT
                               : lane < 4 ? ows + OFF_PP + (size_t)(((2 * self + (lane & 1)) * 8 + nt) * 16) * UNIT
                               : lane < 5 ? ows + OFF_HP + (size_t)((self * 8 + nt) * 16) * HS + 64
                               : lane < 6 ? ws + OFF_HP + (size_t)((owner * 8 + nt) * 16) * HS + 64
                               : lane < 8 ? ws + OFF_PD + (size_t)(((g - 14) * 2 + (lane & 1)) * 8 + nt) * 131072
                               : P.out + (size_t)(lane - 8) * 64 * (GN * 2) + (owner * 8 + nt) * 512;
        unsigned z; asm volatile("global_load_dword %0, %1, off sc0 sc1\n\ts_waitcnt vmcnt(0)" : "=&v"(z) : "v"(a) : "memory");
    }
    unsigned ord0 = 0u, ord1 = 0u;
    if (is_pull) { ord0 = order_vgpr<0, 64>(0u); ord1 = order_vgpr<64, 118>(0u); } else peers();
    asm volatile("s_waitcnt vmcnt(0)" : "+v"(count), "+v"(ord0), "+v"(ord1) :: "memory");
    const unsigned call = (unsigned)__builtin_amdgcn_readfirstlane(count) + 1u;
    const size_t par = call & 1u;
    const unsigned tag4 = (call & 255u) * 0x01010101u;
    const int wr = wid >> 2, wc = wid & 3;

    if (is_pull) {
        LAS unsigned* const ctl = (LAS unsigned*)(lds + 9 * HTB + 64);
        if (tid < 4) ctl[tid] = 0u;
        __syncthreads();
        const unsigned share = (WIN + 7 - nt) / 8;
        auto lds_draw = [&](int w) {
            unsigned v = 0;
            if (lane == 0) v = __hip_atomic_fetch_add(&ctl[w], 1u, __ATOMIC_RELAXED, __HIP_MEMORY_SCOPE_WORKGROUP);
            return (unsigned)__builtin_amdgcn_readfirstlane(v);
        };
        auto acquire = [&]() {
            const unsigned turn = lds_draw(0);
            while (__hip_atomic_load(&ctl[1], __ATOMIC_RELAXED, __HIP_MEMORY_SCOPE_WORKGROUP) + share <= turn) __builtin_amdgcn_s_sleep(1);
        };
        auto release = [&]() { if (lane == 0) __hip_atomic_fetch_add(&ctl[1], 1u, __ATOMIC_RELAXED, __HIP_MEMORY_SCOPE_WORKGROUP); };
        const int src = (self + 1 + slot) & 7;
        const unsigned char* const sws = P.ws[src];
        {
            unsigned* const tick = (unsigned*)(ws + OFF_TICK + src * 128);
            const unsigned tbase = (call - 1u) * NTASK - PW;
            const unsigned char* const hbase = ws + OFF_HIN + (par * 8 + src) * (768 * HS);
            int task = task0, isw, kc, tile, q;
            decode_unit(ord0, ord1, task, src, self, isw, kc, tile, q);
            Packed pk;
            unsigned touched = 0u;
            {
                const unsigned char* a = lane ? ws + (isw ? OFF_WB + (size_t)((src * 8 + kc) * 8 + tile) * BLK : OFF_XA + (size_t)((src * 8 + kc) * 16 + tile) * BLK) + q * 8192
                    : sws + OFF_RP + (par * 2560 + (isw ? (self * 8 + tile) * 8 + kc : 512 + kc * 16 + tile) * 4 + q) * UNIT + 8320 + self * 128;
                asm volatile("global_load_dword %0, %1, off sc0 sc1" : "=&v"(touched) : "v"(a) : "memory");
            }
            unsigned second = 0u;
            if (task < 4 * PLAIN && lane == 0) second = __hip_atomic_fetch_add(tick, 1u, __ATOMIC_RELAXED, __HIP_MEMORY_SCOPE_AGENT);
            asm volatile("s_waitcnt vmcnt(0)" : "+v"(touched), "+v"(second) :: "memory");
            second = __builtin_amdgcn_readfirstlane(second);
            while (__builtin_amdgcn_s_memrealtime() - tstart < PULL_DELAY) __builtin_amdgcn_s_sleep(2);
            wait_header(hbase + ((isw ? kc * 8 + tile : 64 + kc * 16 + tile) * 4 + q) * HS, tmo, call, pk.mask, pk.meta);
            if (task < 4 * PLAIN) {
                unsigned tw = 0u;
                acquire();
                load_parked(cur, tw, sws + OFF_RP + (par * 2560 + (self * 8 + tile) * 32 + q) * UNIT, tmo, lane);
                PARKED_WAIT(cur, tw);
                release();
                store_image(ws + OFF_WB + (size_t)(src * 64 + tile) * BLK, 1, q, cur);
                asm volatile("s_waitcnt vmcnt(0)" ::: "memory");
                if (lane < RW) st_flag8(ws + OFF_FW + (tile * 64 + src * 8) * 4 + q + lane * FR, call);
                task = (int)(second - tbase);
                decode_unit(ord0, ord1, task < NTASK ? task : 0, src, self, isw, kc, tile, q);
                wait_header(hbase + ((isw ? kc * 8 + tile : 64 + kc * 16 + tile) * 4 + q) * HS, tmo, call, pk.mask, pk.meta);
            }
            while (task < NTASK) {
                const unsigned char* const pl = sws + OFF_RP + (par * 2560 + (isw ? (self * 8 + tile) * 8 + kc : 512 + kc * 16 + tile) * 4 + q) * UNIT;
                acquire();
                fetch(pk, pl, lane);
                FETCH_WAIT(pk, 0);
                release();
                unsigned nn = 1u;
                {
                    u64 keep; const unsigned zero = 0u;
                    asm volatile(SBASE "s_mov_b64 %1, exec\n\ts_mov_b64 exec, 1\n\t"
                                 "global_atomic_add %0, %2, %0, %3 sc0\n\t"
                                 "s_mov_b64 exec, %1" : "+v"(nn), "=&s"(keep) : "v"(zero), "s"(tick) : "memory");
                }
                int ntask, nisw, nkc, ntile, nq; u32x4 h; const unsigned char* hd;
                unpack(pk, cur, pl, scr, lane, [&]() {
                    asm volatile("s_waitcnt vmcnt(0)" : "+v"(nn) :: "memory");
                    ntask = (int)(__builtin_amdgcn_readfirstlane(nn) - tbase);
                    decode_unit(ord0, ord1, ntask < NTASK ? ntask : 0, src, self, nisw, nkc, ntile, nq);
                    hd = hbase + ((nisw ? nkc * 8 + ntile : 64 + nkc * 16 + ntile) * 4 + nq) * HS;
                    const unsigned zero = 0u;
                    asm volatile(SBASE "global_load_dwordx4 %0, %1, %2 sc0 sc1" : "=&v"(h) : "v"(zero), "s"(hd) : "memory");
                });
                const int idx = src * 8 + kc;
                store_image(isw ? ws + OFF_WB + (size_t)(idx * 8 + tile) * BLK : ws + OFF_XA + (size_t)(idx * 16 + tile) * BLK, isw, q, cur);
                asm volatile("s_waitcnt vmcnt(0)" : "+v"(h) :: "memory");
                if (lane < (isw ? RW : RX)) st_flag8(ws + (isw ? OFF_FW : OFF_FX) + (tile * 64 + idx) * 4 + q + lane * FR, call);
                if (ntask >= NTASK) break;
                if (!header_ok(h, call, pk.mask, pk.meta)) wait_header(hd, tmo, call, pk.mask, pk.meta);
                task = ntask; isw = nisw; kc = nkc; tile = ntile; q = nq;
            }
        }
        const int gnt = src * 8 + nt, fr = lane & 15, fq = lane >> 4;
        auto head_of = [&](int u) { return ws + OFF_HP + (size_t)(((((u >> 4) & 1) * 64 + gnt) * 8 + (u & 7)) * 2 + ((u >> 3) & 1)) * HS; };
        u32x4 h = {0u, 0u, 0u, 0u};
        for (int u = (int)lds_draw(2); u < 32;) {
            const int odd = u >> 4, bj = (u >> 3) & 1, uw = u & 7, dt = (slot * 2 + odd) * 8 + nt;
            Packed pk;
            const unsigned char* pl = sws + OFF_PP + (size_t)((((2 * self + odd) * 8 + nt) * 8 + uw) * 2 + bj) * UNIT;
            const unsigned char* const fp = ws + OFF_FP + ((dt * 8 + uw) * 2 + bj) * 4;
            const unsigned char* const dp = ws + OFF_PD + (size_t)((dt * 8 + uw) * 2 + bj) * 8192;
            u32x4 dg[8]; unsigned tw = 0u;
            if ((PLAIN_U >> u) & 1u) {
                wait_word<false>(fp, call, tmo, call);
                load_parked(dg, tw, dp, tmo, lane);
                PARKED_WAIT(dg, tw);
                if (!header_ok(h, call, pk.mask, pk.meta)) wait_header(head_of(u), tmo, call, pk.mask, pk.meta);
                acquire();
                tw = 0u;
                load_parked(cur, tw, pl, tmo, lane);
                PARKED_WAIT(cur, tw);
                release();
                u = (int)lds_draw(2); h = (u32x4){0u, 0u, 0u, 0u};
            } else {
                if (!header_ok(h, call, pk.mask, pk.meta)) wait_header(head_of(u), tmo, call, pk.mask, pk.meta);
                acquire();
                unsigned pf;
                asm volatile("global_load_dword %0, %1, off sc0 sc1" : "=&v"(pf) : "v"(fp) : "memory");
                fetch(pk, pl, lane);
                FETCH_WAIT(pk, 0);
                release();
                asm volatile("" : "+v"(pf));
                if ((unsigned)__builtin_amdgcn_readfirstlane(pf) < call) wait_word<false>(fp, call, tmo, call);
                load_parked(dg, tw, dp, tmo, lane);
                unpack(pk, cur, pl, scr, lane, [&]() {
                    u = (int)lds_draw(2);
                    const unsigned zero = 0u;
                    asm volatile(SBASE "global_load_dwordx4 %0, %1, %2 sc0 sc1" : "=&v"(h) : "v"(zero), "s"(head_of(u)) : "memory");
                });
#pragma unroll
                for (int i = 0; i < 8; ++i) asm volatile("" : "+v"(cur[i]));
                PARKED_WAIT(dg, tw);
                asm volatile("" : "+v"(h));
            }
            const bool bad = (unsigned)__builtin_amdgcn_readfirstlane(tw) == call || ((pk.meta >> 8) & 63u) == 63u;
            unsigned char* dst = P.out + (size_t)(odd * 256 + bj * 128 + (uw & 3) * 32 + fq * 4) * (GN * 2) + (size_t)(gnt * 256 + (uw >> 2) * 128 + fr * 8) * 2;
#pragma unroll
            for (int n = 0; n < 2; ++n)
#pragma unroll
                for (int r = 0; r < 4; ++r) {
                    u32x4 v;
#pragma unroll
                    for (int j = 0; j < 4; ++j) {
                        const unsigned pu = cur[n * 4 + r][j], w = dg[n * 4 + r][j];
                        const f32x2 y = (f32x2){__uint_as_float(w << 16), __uint_as_float(w & 0xffff0000u)}
                                      + (f32x2){__uint_as_float(pu << 16), __uint_as_float(pu & 0xffff0000u)};
                        const f32x2 o = gelu2(y);
                        v[j] = cvt_pk_bf16(o.x, o.y);
                    }
                    if (bad) v = (u32x4){0x7fc07fc0u, 0x7fc07fc0u, 0x7fc07fc0u, 0x7fc07fc0u};
                    st16_sys(dst + (size_t)(n * 16 + r) * (GN * 2), v);
                }
        }
        if (tid == 0) __hip_atomic_store((unsigned*)(ws + OFF_CNT + b * 128), call, __ATOMIC_RELAXED, __HIP_MEMORY_SCOPE_AGENT);
        return;
    }

    if (!is_b) {
        const int hp = lane >> 3;
        unsigned char* const hdst = (unsigned char*)(((u64)(unsigned)__shfl((int)peer_hi, hp) << 32) | (unsigned)__shfl((int)peer_lo, hp))
                                    + OFF_HIN + (par * 8 + self) * (768 * HS) + (lane & 7) * 16;
        for (int t = task0;;) {
#pragma unroll
            for (int i = 0; i < 8; ++i) asm volatile("" : "+v"(cur[i]));
            const int tn = t < (PK_ALL - 1) * 1152 ? t + 1152 : t >= PK_ALL * 1152 ? t + 704 : PK_ALL * 1152 + wid * 88 + (is_a ? g >> 1 : g - 14) * 8 + nt;
            const bool more = tn < 2784 && (tn < PK_ALL * 1152 || is_c || hf);
            int isw, kc, tile; pack_decode(t, isw, kc, tile); const int q = t & 3, idx = self * 8 + kc;
            if (isw) {
                u32x4 tr[8];
#pragma unroll
                for (int j = 0; j < 8; ++j)
#pragma unroll
                    for (int qd = 0; qd < 4; ++qd)
                        tr[j][qd] = perm(cur[2 * qd + 1][j >> 1], cur[2 * qd][j >> 1], (j & 1) ? 0x07060302u : 0x05040100u);
#pragma unroll
                for (int j = 0; j < 8; ++j) cur[j] = tr[j];
            }
            const bool own = (isw ? tile >> 3 : tile >> 1) == self, diag = t >= 992;
            const bool packed = !own && t < 2560, image = !isw || own || diag;
            u32x4 hd = {0u, 0u, 0u, 0u};
            if (packed && t >= 64 && t < 64 + 32 * PLAIN) {
                unsigned char* pl = ws + OFF_RP + (par * 2560 + tile * 32 + q) * UNIT + lane * 16;
#pragma unroll
                for (int i = 0; i < 8; ++i) st16_sys(pl + i * 1024, cur[i]);
                hd = (u32x4){0u, call, 0u, call << 15};
            } else if (packed) {
                Packed pk; pack(cur, pk, scr, lane);
                send(pk, ws + OFF_RP + (par * 2560 + (isw ? tile * 8 + kc : 512 + kc * 16 + tile) * 4 + q) * UNIT, scr, lane);
                hd = header(pk, call);
            }
            if (image)
                store_image(ws + (diag ? OFF_WD + (size_t)(tile * 8 + kc) * BLK : isw ? OFF_WB + (size_t)(idx * 8 + (tile & 7)) * BLK
                                                                                     : OFF_XA + (size_t)(idx * 16 + tile) * BLK), isw, q, cur);
            load_task(more ? tn : t, cur);
            asm volatile("s_waitcnt vmcnt(8)" ::: "memory");
            if (packed && (lane & 7) < HFULL && hp != self && (!isw || hp == (tile >> 3)))
                st16_sys(hdst + ((isw ? kc * 8 + (tile & 7) : 64 + kc * 16 + tile) * 4 + q) * HS, hd);
            if (image && lane < (diag ? 1 : isw ? RW : RX))
                st_flag8(ws + (diag ? OFF_FD + (tile * 8 + kc) * 4 : isw ? OFF_FW + ((tile & 7) * 64 + idx) * 4 : OFF_FX + (tile * 64 + idx) * 4) + q + lane * FR, call);
            if (!isw && tile == 2 * self + 1 && lane == 0) st_flag8(ws + OFF_FQ + kc * 4 + q, call);
            __builtin_amdgcn_s_setprio(0);
            if (!more) break;
            t = tn;
        }
    }

    const unsigned foff = lane * 4;
    const unsigned pickoff = (unsigned)(size_t)(lds + 9 * HTB);
    const int fxc = (nt % RX) * FR, fwc = (g % RW) * FR;
    const unsigned lds4 = (unsigned)(size_t)lds + wid * 4096, lds2 = (unsigned)(size_t)lds + wid * 2048;
    const unsigned voff4 = wid * 4096 + lane * 16, voff2 = wid * 2048 + lane * 16;
#define GL4(bufoff, vo, gbase) asm volatile("s_add_i32 m0, %0, %1\n\ts_nop 3\n\tglobal_load_lds_dwordx4 %2, %3 sc0 sc1\n\t" \
        "global_load_lds_dwordx4 %2, %3 offset:1024 sc0 sc1\n\tglobal_load_lds_dwordx4 %2, %3 offset:2048 sc0 sc1\n\t" \
        "global_load_lds_dwordx4 %2, %3 offset:3072 sc0 sc1" :: "s"(lds4), "i"(bufoff), "v"(vo), "s"(gbase) : "memory", "scc")
#define GL2(bufoff, vo, gbase) asm volatile("s_add_i32 m0, %0, %1\n\ts_nop 3\n\tglobal_load_lds_dwordx4 %2, %3 sc0 sc1\n\t" \
        "global_load_lds_dwordx4 %2, %3 offset:1024 sc0 sc1" :: "s"(lds2), "i"(bufoff), "v"(vo), "s"(gbase) : "memory", "scc")
#define LIKELY(c) __builtin_expect(!!(c), 1)
#define RARE(c) __builtin_expect(!!(c), 0)
#define LDA(dst, base, off) do { _Pragma("unroll") for (int m = 0; m < 4; ++m) _Pragma("unroll") for (int k = 0; k < 2; ++k) \
        dst[m][k] = *(const LAS bf16x8*)(lds + (base) + ((off) + m * 2048 + k * 1024)); } while (0)
#define LDB(dst, base, off) do { _Pragma("unroll") for (int n = 0; n < 2; ++n) _Pragma("unroll") for (int k = 0; k < 2; ++k) \
        dst[n][k] = *(const LAS bf16x8*)(lds + (base) + ((off) + n * 2048 + k * 1024)); } while (0)
#define MMA(ai, bj, At, Bt) do { __builtin_amdgcn_s_setprio(1); _Pragma("unroll") for (int m = 0; m < 4; ++m) _Pragma("unroll") for (int n = 0; n < 2; ++n) _Pragma("unroll") for (int k = 0; k < 2; ++k) \
        acc[ai][bj][m][n] = __builtin_amdgcn_mfma_f32_16x16x32_bf16(Bt[n][k], At[m][k], acc[ai][bj][m][n], 0, 0, 0); __builtin_amdgcn_s_setprio(0); } while (0)
#define MMQ(ai, bj, At, Bt) do { _Pragma("unroll") for (int k = 0; k < 2; ++k) _Pragma("unroll") for (int m = 0; m < 4; ++m) _Pragma("unroll") for (int n = 0; n < 2; ++n) \
        acc[ai][bj][m][n] = __builtin_amdgcn_mfma_f32_16x16x32_bf16(Bt[n][k], At[m][k], acc[ai][bj][m][n], 0, 0, 0); SCHED; } while (0)
#define WAIT_V(n) asm volatile("s_waitcnt vmcnt(" #n ")" ::: "memory")
#define WAIT_L0 do { asm volatile("" ::: "memory"); __builtin_amdgcn_s_waitcnt(0xC07F); asm volatile("" ::: "memory"); } while (0)
#define BAR __builtin_amdgcn_s_barrier()
#define SCHED __builtin_amdgcn_sched_barrier(0)
    typedef unsigned u32x2 __attribute__((ext_vector_type(2)));
#define SNAP() asm volatile(SBASE "global_load_dword %0, %2, %4 sc0 sc1\n\tglobal_load_dword %1, %3, %4 sc0 sc1" : "+v"(sna), "+v"(snb) : "v"(fa), "v"(fb), "s"(ws) : "memory")
#define PUBLISH(m, w) asm volatile("ds_write_b64 %0, %1 offset:%2" :: "v"(pickoff), "v"((u32x2){(unsigned)(m), (unsigned)((m) >> 32)}), "i"(8 * (w)) : "memory")
#define LISTEN(w) asm volatile("ds_read_b64 %0, %1 offset:%2" : "=v"(mv) : "v"(pickoff), "i"(8 * (w)) : "memory"); SCHED
#define N0 asm volatile("s_waitcnt lgkmcnt(0)" : "+v"(mv) :: "memory"); m_ = ((u64)(unsigned)__builtin_amdgcn_readfirstlane(mv.y) << 32) | (unsigned)__builtin_amdgcn_readfirstlane(mv.x); asm volatile("" : "+s"(m_)); SCHED
#define N5 asm volatile("s_andn2_b64 %0, %0, %2\n\ts_andn2_b64 %1, %1, %2" : "+s"(todo), "+s"(avail) : "s"(m_) : "scc"); SCHED
#define MF(ai, bj, At, Bt, m, k) acc[ai][bj][m][0] = __builtin_amdgcn_mfma_f32_16x16x32_bf16(Bt[0][k], At[m][k], acc[ai][bj][m][0], 0, 0, 0); \
                                 acc[ai][bj][m][1] = __builtin_amdgcn_mfma_f32_16x16x32_bf16(Bt[1][k], At[m][k], acc[ai][bj][m][1], 0, 0, 0); SCHED
    const int w0 = wid == 0;
    f32x4 acc[2][2][4][2];
    auto zero_acc = [&]() {
#pragma unroll
        for (int a = 0; a < 2; ++a)
#pragma unroll
            for (int c = 0; c < 2; ++c)
#pragma unroll
                for (int m = 0; m < 4; ++m)
#pragma unroll
                    for (int n = 0; n < 2; ++n) acc[a][c][m][n] = (f32x4){0.f, 0.f, 0.f, 0.f};
    };
    auto to_bf16 = [&](int c, u32x4 (&v)[8]) {
#pragma unroll
        for (int n = 0; n < 2; ++n)
#pragma unroll
            for (int r = 0; r < 4; ++r) {
                u32x4& o = v[n * 4 + r];
                o.x = cvt_pk_bf16(acc[0][c][0][n][r], acc[0][c][1][n][r]); o.y = cvt_pk_bf16(acc[0][c][2][n][r], acc[0][c][3][n][r]);
                o.z = cvt_pk_bf16(acc[1][c][0][n][r], acc[1][c][1][n][r]); o.w = cvt_pk_bf16(acc[1][c][2][n][r], acc[1][c][3][n][r]);
            }
    };
    auto emit_unit = [&](int c, int bj) {
        const int lz = lane_id();
        const unsigned char* const started = ws + OFF_FW + fwc + nt * 256 + owner * 32;
        unsigned og, ot;
        asm volatile("global_load_dword %0, %2, off sc0 sc1\n\tglobal_load_dword %1, %3, off sc1" : "=&v"(og), "=&v"(ot) : "v"(started), "v"(tmo) : "memory");
        u32x4 v[8];
        to_bf16(c, v);
        Packed pk;
        if (wid < 4) __builtin_amdgcn_s_setprio(2);
        const bool plain = (PLAIN_U >> ((mt & 1) * 16 + bj * 8 + wid)) & 1u;
        if (!plain) pack(v, pk, scr, lz);
        __builtin_amdgcn_s_setprio(0);
        asm volatile("s_waitcnt vmcnt(0)" : "+v"(og), "+v"(ot) :: "memory");
        bool bad = (unsigned)__builtin_amdgcn_readfirstlane(ot) == call;
        if ((unsigned)__builtin_amdgcn_readfirstlane(og) != tag4) bad = wait_word<true>(started, tag4, tmo, call) || bad;
        unsigned char* const pl = ws + OFF_PP + (size_t)(((mt * 8 + nt) * 8 + wid) * 2 + bj) * UNIT;
        if (plain) {
#pragma unroll
            for (int i = 0; i < 8; ++i) st16_sys(pl + i * 1024 + lz * 16, v[i]);
            pk.mask = 0ull; pk.meta = 0u;
        } else send(pk, pl, scr, lz);
        asm volatile("s_waitcnt vmcnt(0)" ::: "memory");
        if (bad) pk.meta |= 63u << 8;
        if (lz < HFULL) st16_sys(P.ws[owner] + OFF_HP + (size_t)((((mt & 1) * 64 + self * 8 + nt) * 8 + wid) * 2 + bj) * HS + lz * 16, header(pk, call));
    };
    auto finish = [&]() {
        if (threadIdx.x == 0) __hip_atomic_store((unsigned*)(ws + OFF_CNT + b * 128), call, __ATOMIC_RELAXED, __HIP_MEMORY_SCOPE_AGENT);
    };
    auto park = [&](int c, int pt, int bj) {
        unsigned char* dp = ws + OFF_PD + (size_t)((pt * 8 + wid) * 2 + bj) * 8192 + lane_id() * 16;
        u32x4 v[8];
        to_bf16(c, v);
#pragma unroll
        for (int i = 0; i < 8; ++i) st16_sys(dp + i * 1024, v[i]);
    };
    const u64 help = (HELP_KC * 0x0101010101010101ull) & ~(0xffull << (self * 8));
    if (!is_a) {
      const int nsteps = is_b ? 1 : 1 + (7 + C_ODD + 24 - g) / 4;
#pragma unroll 1
      for (int step = 0;; ++step) {
        const bool diag = step < nsteps - 1;
        const int dj = step * 4 + g - 21, dodd = dj >= 7, link = dj - 7 * dodd, dg = ((self + 1 + link) & 7) * 8 + nt, jm = diag ? 2 * self + dodd : mt;
        u64 todo = diag ? 0xffull << (self * 8) : is_b ? ~(0xffull << (owner * 8)) : (hf ? 0x55aa55aa55aa55aaull : 0xaa55aa55aa55aa55ull) & ~help, avail = 0ull;
        const int nkt = __builtin_popcountll(todo);
        const unsigned char* const abase = diag ? ws + OFF_WD + (size_t)(dg * 8 - self * 8) * BLK : ws + OFF_WB + (size_t)nt * BLK;
        const int ashift = diag ? 15 : 18;
        const unsigned fa = (unsigned)(diag ? OFF_FD + (dg * 8 - self * 8) * 4 : OFF_FW + fwc + nt * 256) + foff;
        const int x0 = (is_c && !diag && hf) ? HTB : 0;
        const unsigned voffx = voff4 ^ x0, voffh = voff2 + HTB;
        const unsigned char* const bbase = ws + OFF_XA + (size_t)jm * BLK;
        const unsigned fb = (unsigned)OFF_FX + fxc + jm * 256 + foff;
        const int aoff = wr * 8192 + lane * 16;
        int boff = 4 * HTB + wc * 4096 + lane * 16;
        asm volatile("" : "+v"(boff));
        zero_acc();
        bf16x8 At[4][2], B0[2][2], B1[2][2];
        const unsigned char *a2 = abase, *b2 = bbase;
        unsigned sna = ~tag4, snb = ~tag4; u32x2 mv; int look = w0;
        int vm = 0;
        int left = __builtin_amdgcn_readfirstlane(nkt);
        if (tid < 6) *(LAS unsigned*)(lds + 9 * HTB + tid * 4) = 0u;
        __syncthreads();
        if (wr == 1) BAR;
#define SA(bf, h) (((bf) * 2 + (h)) * HTB)
#define SB(bf, h) ((4 + (bf) * 2 + (h)) * HTB)
#define F1 asm volatile("s_and_b64 %1, %1, %2\n\ts_or_b64 %0, %0, %1" : "+s"(avail), "+s"(m_) : "s"(todo) : "scc"); SCHED
#define F2 asm volatile("s_ff1_i32_b64 %0, %2\n\ts_lshl_b64 %1, 1, %0" : "=&s"(i_), "=s"(m_) : "s"(avail) : "scc"); SCHED
#define F3 asm volatile("s_and_b64 %0, %0, %2\n\ts_cselect_b32 %1, 4, 0" : "+s"(m_), "=s"(vb_) : "s"(avail) : "scc"); SCHED
#define F6 a2 = abase + ((unsigned)i_ << ashift); asm volatile("" : "+s"(a2)); SCHED
#define F7 b2 = bbase + ((unsigned)i_ << 19); asm volatile("" : "+s"(b2)); SCHED
#define F8 asm volatile("s_bcnt1_i32_b64 %0, %1\n\ts_cmp_lt_u32 %0, %2\n\ts_cselect_b32 %0, %3, 0" : "=&s"(look) : "s"(avail), "i"(SNAP_AT), "s"(w0) : "scc"); \
           vm = (vm | vb_) >> 1; asm volatile("" : "+s"(vm)); SCHED
#define KTILE(bf) do { \
        LDB(B0, boff, SA(bf, 0)); LDB(B1, boff, SA(bf, 1)); LDA(At, aoff, SA(bf, 0)); SCHED; \
        if (LIKELY(vm & 2)) { GL2(SA(1 - bf, 1), voffh, a2); WAIT_V(8); } else WAIT_V(0); \
        if (RARE(look)) { asm volatile("" : "+v"(sna), "+v"(snb)); \
            u64 m = __ballot(sna == tag4 && snb == tag4); \
            if (vm == 0 && ((m | avail) & todo) == 0ull) m = poll_ready(todo, ws, fa, fb, tmo, call, tag4, 0xffffffffu); \
            PUBLISH(m, bf); } \
        int cur_ = vm & 1; asm volatile("" : "+s"(cur_)); __builtin_amdgcn_s_setprio(1); \
        WAIT_L0; BAR; SCHED; LISTEN(bf); \
        u64 m_; int i_, vb_; \
        if (LIKELY(cur_)) { \
            MF(0, 0, At, B0, 0, 0); MF(0, 0, At, B0, 1, 0); MF(0, 0, At, B0, 2, 0); MF(0, 0, At, B0, 3, 0); N0; \
            MF(0, 0, At, B0, 0, 1); F1; MF(0, 0, At, B0, 1, 1); F2; MF(0, 0, At, B0, 2, 1); F3; MF(0, 0, At, B0, 3, 1); N5; \
            MF(0, 1, At, B1, 0, 0); F6; MF(0, 1, At, B1, 1, 0); F7; MF(0, 1, At, B1, 2, 0); F8; MF(0, 1, At, B1, 3, 0); \
            MF(0, 1, At, B1, 0, 1); MF(0, 1, At, B1, 1, 1); MF(0, 1, At, B1, 2, 1); MF(0, 1, At, B1, 3, 1); } \
        asm volatile("" : "+s"(cur_)); \
        if (RARE(!cur_)) { N0; F1; F2; F3; N5; F6; F7; F8; } \
        BAR; __builtin_amdgcn_s_setprio(0); SCHED; \
        LDA(At, aoff, SA(bf, 1)); SCHED; \
        if (RARE(look)) SNAP(); \
        if (LIKELY(vm & 2)) { GL4(SB(bf, 0), voffx, b2); GL2(SA(bf, 0), voff2, a2); WAIT_V(8); } else WAIT_V(2); \
        left -= cur_; \
        asm volatile("" : "+s"(cur_), "+s"(left)); __builtin_amdgcn_s_setprio(1); \
        WAIT_L0; BAR; SCHED; if (LIKELY(cur_)) { MMQ(1, 0, At, B0); MMQ(1, 1, At, B1); } BAR; __builtin_amdgcn_s_setprio(0); SCHED; } while (0)
        for (;;) {
            KTILE(0);
            if (left == 0) break;
            KTILE(1);
            if (left == 0) break;
        }
        asm volatile("s_waitcnt vmcnt(0)\n\ts_nop 15\n\ts_nop 7" : "+v"(sna), "+v"(snb) :: "memory");
        if (wr == 0) BAR;
        if (!diag) break;
        park(0, link * 16 + dodd * 8 + nt, 0); park(1, link * 16 + dodd * 8 + nt, 1);
        asm volatile("s_waitcnt vmcnt(0)" ::: "memory");
        if (lane_id() < 2) st_flag(ws + OFF_FP + (((link * 16 + dodd * 8 + nt) * 8 + wid) * 2 + lane_id()) * 4, call);
      }
      {
        if (is_b) {
            emit_unit(0, 0); emit_unit(1, 1);
        } else {
            const int lz = lane_id(), pt = 112 + (mt & 1) * 8 + nt;
            park(1, pt, 1 - hf);
            asm volatile("s_waitcnt vmcnt(0)" ::: "memory");
            if (lz == 0) st_flag(ws + OFF_FP + ((pt * 8 + wid) * 2 + 1 - hf) * 4, call);
            wait_word<false>(ws + OFF_FP + ((pt * 8 + wid) * 2 + hf) * 4, call, tmo, call);
            wait_word<false>(ws + OFF_FP + (((pt + 16) * 8 + wid) * 2 + hf) * 4, call, tmo, call);
            u32x4 dg[8], dh[8]; unsigned tw = 0u, tv = 0u;
            load_parked(dg, tw, ws + OFF_PD + (size_t)((pt * 8 + wid) * 2 + hf) * 8192, tmo, lz);
            load_parked(dh, tv, ws + OFF_PD + (size_t)(((pt + 16) * 8 + wid) * 2 + hf) * 8192, tmo, lz);
            PARKED_WAIT(dg, tw); PARKED_WAIT(dh, tv);
            const bool bad = (unsigned)__builtin_amdgcn_readfirstlane(tw) == call;
            unsigned char* dst = P.out + (size_t)((mt & 1) * 256 + hf * 128 + wc * 32 + (lz >> 4) * 4) * (GN * 2) + (size_t)((self * 8 + nt) * 256 + wr * 128 + (lz & 15) * 8) * 2;
#pragma unroll
            for (int n = 0; n < 2; ++n)
#pragma unroll
                for (int r = 0; r < 4; ++r) {
                    u32x4 v;
#pragma unroll
                    for (int j = 0; j < 4; ++j) {
                        const unsigned w = dg[n * 4 + r][j], x = dh[n * 4 + r][j];
                        const f32x2 o = gelu2((f32x2){acc[j >> 1][0][(j & 1) * 2][n][r], acc[j >> 1][0][(j & 1) * 2 + 1][n][r]}
                                              + (f32x2){__uint_as_float(w << 16), __uint_as_float(w & 0xffff0000u)}
                                              + (f32x2){__uint_as_float(x << 16), __uint_as_float(x & 0xffff0000u)});
                        v[j] = cvt_pk_bf16(o.x, o.y);
                    }
                    if (bad) v = (u32x4){0x7fc07fc0u, 0x7fc07fc0u, 0x7fc07fc0u, 0x7fc07fc0u};
                    st16_sys(dst + (size_t)(n * 16 + r) * (GN * 2), v);
                }
        }
      }
    } else {
      const int dg = owner * 8 + nt;
      const u64 dmask = 0xffull << (owner * 8);
      const bool mine = (g >> 1) >= C_ODD;
      u64 todo = mine ? ~0ull : ~dmask, avail = 0ull;
      const unsigned char* const abase = ws + OFF_WB + (size_t)nt * BLK;
      const unsigned char* const bbase = ws + OFF_XA + (size_t)mt * BLK + hf * HTB;
      const unsigned char* const adiag = ws + OFF_WD + (size_t)(dg * 8 - owner * 8) * BLK;
      const unsigned char* const bdiag = ws + OFF_XA + (size_t)(2 * self + 1) * BLK + hf * HTB + (size_t)(self - owner) * (8 * 16 * (size_t)BLK);
      const bool dl = (lane >> 3) == owner;
      unsigned fa = (unsigned)(dl ? OFF_FD + (dg * 8 - owner * 8) * 4 : OFF_FW + fwc + nt * 256) + foff;
      unsigned fb = (unsigned)(dl ? OFF_FQ - owner * 32 : OFF_FX + fxc + mt * 256) + foff;
      unsigned xm = hf ? 0xffff0000u : 0x0000ffffu;
      const int cm = 2 * self + (g >> 2), ch = (g >> 1) & 1;
      zero_acc();
#pragma unroll 1
      for (int step = 0;; ++step) {
        int aoff[3], boff[3];
#pragma unroll
        for (int i = 0; i < 3; ++i) {
            aoff[i] = i * 3 * HTB + wr * 8192 + lane * 16; boff[i] = i * 3 * HTB + 2 * HTB + wc * 4096 + lane * 16;
            asm volatile("" : "+v"(aoff[i]), "+v"(boff[i]));
        }
        bf16x8 A0[4][2], A1[4][2], B0[2][2];
        unsigned sna = ~tag4, snb = ~tag4; u32x2 mv;
        int vm = 0;
        int look = w0, more = 1;
        const unsigned char *a2 = ws, *b2 = ws;
        const u64 low = step ? 0ull : dmask;
        const unsigned char* const am = step == 1 ? adiag : abase;
        const unsigned char* const bm = step == 1 ? bdiag : step ? ws + OFF_XA + (size_t)cm * BLK + ch * HTB : bbase;
        const int shm = step == 1 ? 15 : 18, vbm = step == 1 ? 0x44 : 4;
        int mleft = __builtin_popcountll(todo & ~low);
        u64 fill = ~0ull;
        if (tid < 6) *(LAS unsigned*)(lds + 9 * HTB + tid * 4) = 0u;
        __syncthreads();
        if (wr == 1) BAR;
#define N1 asm volatile("s_and_b64 %1, %1, %2\n\ts_or_b64 %0, %0, %1\n\ts_and_b64 %1, %0, %3" : "+s"(avail), "+s"(m_) : "s"(todo), "s"(fill) : "scc"); SCHED
#define N2 asm volatile("s_andn2_b64 %0, %7, %8\n\ts_cselect_b64 %0, %0, %6\n\ts_cselect_b64 %1, %9, %10\n\ts_cselect_b64 %2, %11, %12\n\t" \
                        "s_cselect_b32 %3, %13, 15\n\ts_cselect_b32 %4, %14, 0x44\n\ts_subb_u32 %5, %5, 0" \
                        : "=&s"(c_), "=&s"(a2), "=&s"(b2), "=&s"(sh_), "=&s"(vb_), "+s"(mleft) \
                        : "s"(m_), "s"(avail), "s"(low), "s"(am), "s"(adiag), "s"(bm), "s"(bdiag), "s"(shm), "s"(vbm) : "scc"); SCHED
#define N3 asm volatile("s_ff1_i32_b64 %0, %2\n\ts_lshl_b64 %1, 1, %0" : "=&s"(i_), "=s"(m_) : "s"(c_) : "scc"); SCHED
#define N4 asm volatile("s_and_b64 %0, %0, %2\n\ts_cselect_b32 %1, %1, 0" : "+s"(m_), "+s"(vb_) : "s"(c_) : "scc"); SCHED
#define N6 vm = ((vm >> 1) & 0x33) | vb_; more = vm | mleft; asm volatile("" : "+s"(vm), "+s"(more)); SCHED
#define N7 a2 += (unsigned)i_ << sh_; asm volatile("" : "+s"(a2)); SCHED
#define N8 b2 += (unsigned)i_ << 19; asm volatile("" : "+s"(b2)); SCHED
#define N9 asm volatile("s_cmp_gt_i32 %3, %4\n\ts_cselect_b64 %0, -1, 0\n\ts_andn2_b64 %2, %5, %6\n\ts_bcnt1_i32_b64 %1, %2\n\ts_cmp_lt_u32 %1, %7\n\ts_cselect_b32 %1, %8, 0" \
                        : "=&s"(fill), "=&s"(look), "=&s"(c_) : "s"(mleft), "i"(FILL_STOP), "s"(avail), "s"(low), "i"(SNAP_H), "s"(w0) : "scc"); SCHED
#define MF2(ai, At, m, k) MF(ai, 0, At, B0, m, k)
#define HTILE(s) do { \
        constexpr int nb = ((s) + 2) % 3 * 3 * HTB; \
        if (LIKELY(vm & 4)) { GL2(nb + 2 * HTB, voff2, b2); GL4(nb, voff4, a2); } \
        SCHED; LDB(B0, boff[s], 0); LDA(A0, aoff[s], 0); LDA(A1, aoff[s], HTB); SCHED; \
        asm volatile("s_bitcmp1_b32 %2, 2\n\ts_cbranch_scc1 1f\n\ts_waitcnt vmcnt(0)\n1:\n\ts_waitcnt vmcnt(6)" : "+v"(sna), "+v"(snb) : "s"(vm) : "memory", "scc"); \
        if (RARE(look)) {   \
            u64 m = __ballot(sna == tag4 && ((snb ^ tag4) & xm) == 0u); \
            const u64 want = todo & (fill | ~low); \
            if (vm == 0 && ((m | avail) & want) == 0ull) m = poll_ready(want, ws, fa, fb, tmo, call, tag4, xm); \
            PUBLISH(m, s); \
            if ((m & todo & ~avail & ~low) == 0ull) SNAP(); } \
        WAIT_L0; BAR; SCHED; \
        LISTEN(s); \
        int kind = vm & 17; asm volatile("" : "+s"(kind)); \
        u64 m_, c_; int i_, sh_, vb_; \
        if (LIKELY(kind == 1)) { \
            __builtin_amdgcn_s_setprio(1); \
            MF2(0, A0, 0, 0); MF2(0, A0, 1, 0); MF2(0, A0, 2, 0); MF2(0, A0, 3, 0); N0; \
            MF2(0, A0, 0, 1); N1; MF2(0, A0, 1, 1); N2; MF2(0, A0, 2, 1); MF2(0, A0, 3, 1); N3; \
            MF2(1, A1, 0, 0); N4; MF2(1, A1, 1, 0); N5; MF2(1, A1, 2, 0); N6; MF2(1, A1, 3, 0); N7; \
            MF2(1, A1, 0, 1); N8; MF2(1, A1, 1, 1); N9; MF2(1, A1, 2, 1); MF2(1, A1, 3, 1); \
            __builtin_amdgcn_s_setprio(0); } \
        asm volatile("" : "+s"(kind)); \
        if (RARE(kind == 17)) { MMA(0, 1, A0, B0); MMA(1, 1, A1, B0); } \
        asm volatile("" : "+s"(kind)); \
        if (RARE(kind != 1)) { N0; N1; N2; N3; N4; N5; N6; N7; N8; N9; } \
        BAR; SCHED; } while (0)
        for (;;) {
            HTILE(0);
            if (more == 0) break;
            HTILE(1);
            if (more == 0) break;
            HTILE(2);
            if (more == 0) break;
        }
        asm volatile("s_waitcnt vmcnt(0)\n\ts_nop 15\n\ts_nop 7" : "+v"(sna), "+v"(snb) :: "memory");
        if (wr == 0) BAR;
        if (step == 2) break;
        if (step == 0) {
            emit_unit(0, hf);
        }
#pragma unroll
        for (int i = 0; i < 16; ++i) acc[i >> 3][0][(i >> 1) & 3][i & 1] = (f32x4){0.f, 0.f, 0.f, 0.f};
        if (todo == 0ull) {
            const int pt = (g | 1) * 8 + nt;
            if (mine) {
                park(1, pt, hf);
                asm volatile("s_waitcnt vmcnt(0)" ::: "memory");
                if (lane_id() == 0) st_flag(ws + OFF_FP + ((pt * 8 + wid) * 2 + hf) * 4, call);
            }
            if (hf || g >= 8) { finish(); return; }
            step = 1; todo = help; avail = 0ull;
            fa = (unsigned)OFF_FW + fwc + nt * 256 + foff; fb = (unsigned)OFF_FX + fxc + cm * 256 + foff;
            xm = ch ? 0xffff0000u : 0x0000ffffu;
        }
      }
      const int pt = 128 + (cm & 1) * 8 + nt;
      park(0, pt, ch);
      asm volatile("s_waitcnt vmcnt(0)" ::: "memory");
      if (lane_id() == 0) st_flag(ws + OFF_FP + ((pt * 8 + wid) * 2 + ch) * 4, call);
    }
    finish();
}

}

extern "C" void kernel_launch(int rank, int nranks, ncclComm_t comm, hipStream_t stream,
                              void* const* d_in, const int* in_sizes, int n_in,
                              void* d_out, int out_size, const int* in_dtypes, int out_dtype,
                              void* const* d_out_all, void* const* d_ws_all, size_t ws_bytes) {
    (void)comm; (void)in_dtypes; (void)out_dtype; (void)d_out_all;
    if (nranks != NR || (unsigned)rank >= (unsigned)NR || n_in < 2 || ws_bytes < WS_NEED
        || in_sizes[0] != GM * KS || in_sizes[1] != KS * GN || out_size != (GM / NR) * GN) {
        fprintf(stderr, "kernel_launch: unexpected problem shape\n");
        return;
    }
    Params p;
    p.x = (const unsigned char*)d_in[0];
    p.w = (const unsigned char*)d_in[1];
    p.out = (unsigned char*)d_out;
    for (int i = 0; i < NR; ++i) p.ws[i] = (unsigned char*)d_ws_all[i];
    p.own = p.ws[rank]; p.rank = rank; p.pad = 0;
    hipLaunchKernelGGL(fused_gemm_rs, dim3(256), dim3(512), 0, stream, p);
    HIP_CHECK(hipGetLastError());
}
```

```cpp
#include <hip/hip_runtime.h>
#include <rccl/rccl.h>
#include <cstdio>
#include <cstdint>

#define HIP_CHECK(expr) do { hipError_t _e = (expr); if (_e != hipSuccess) { \
    fprintf(stderr, "HIP error %s at %s:%d\n", hipGetErrorString(_e), __FILE__, __LINE__); \
    return; } } while (0)

namespace {

#define LAS __attribute__((address_space(3)))
#define GAS __attribute__((address_space(1)))
typedef short bf16x8 __attribute__((ext_vector_type(8)));
typedef float f32x4 __attribute__((ext_vector_type(4)));
typedef unsigned u32x4 __attribute__((ext_vector_type(4)));
typedef unsigned long long u64;

constexpr int NR = 8;
constexpr int GM = 4096, GK = 4096, GN = 16384;
constexpr int KS = GK / NR;
constexpr int BLK = 32768;
constexpr int HTB = 16384;
constexpr int WIN = 16;
constexpr int PW = 64;
constexpr int HEAD_W = 2, NEXT_W = 5;
constexpr int PULL_DELAY = 300, PACK_DELAY = 100, PACK_DELAY2 = 250;
constexpr int PK_ALL = 2;
constexpr int SNAP_AT = 2, SNAP_H = 1;
constexpr int FILL_STOP = 6;
constexpr int C_ODD = 1;
constexpr int PLAIN = 2, PKP = 3;
constexpr unsigned PLAIN_U = 0xc0000000u;
constexpr u64 HELP_KC = 0xe0;
constexpr int HS = 128;
constexpr int HFULL = 2;
constexpr int RX = 8, RW = 8;
constexpr u64 DEADLINE = 50000000ull;
constexpr int NTASK = 704;
constexpr int UNIT = 9344;

constexpr size_t OFF_CNT = 0;
constexpr size_t OFF_TMO = 32768;
constexpr size_t OFF_TICK = 33792;
constexpr size_t OFF_FX = 131072;
constexpr size_t OFF_FW = OFF_FX + 4096;
constexpr int FR = 8192;
constexpr size_t OFF_FD = 98304;
constexpr size_t OFF_FQ = 104448;
constexpr size_t OFF_FP = 106496;
constexpr size_t OFF_XA = 1u << 20;
constexpr size_t OFF_WB = OFF_XA + (size_t)64 * 16 * BLK;
constexpr size_t OFF_WD = OFF_WB + (size_t)64 * 8 * BLK;
constexpr size_t OFF_RP = OFF_WD + (size_t)64 * 8 * BLK;
constexpr size_t OFF_PP = OFF_RP + (size_t)5120 * UNIT;
constexpr size_t OFF_PD = OFF_PP + (size_t)2048 * UNIT;
constexpr size_t OFF_HIN = OFF_PD + (size_t)144 * 131072;
constexpr size_t OFF_HP = OFF_HIN + (size_t)16 * 768 * HS;
constexpr size_t OFF_LB = OFF_HP + (size_t)2048 * HS;
constexpr size_t WS_NEED = OFF_LB + (size_t)512 * UNIT + 4096;
static_assert(WS_NEED <= (size_t)161 * 1048576, "workspace");

struct Params {
    const unsigned char* x;
    const unsigned char* w;
    unsigned char* out;
    unsigned char* own;
    unsigned char* ws[NR];
    int rank, pad;
};

#define SBASE "s_nop 4\n\t"
__device__ __forceinline__ void st16_sys(unsigned char* p, u32x4 v) {
    asm volatile("global_store_dwordx4 %0, %1, off sc0 sc1\n\ts_nop 1" :: "v"(p), "v"(v) : "memory");
}
__device__ __forceinline__ void st_flag(unsigned char* p, unsigned v) {
    __hip_atomic_store((unsigned*)p, v, __ATOMIC_RELAXED, __HIP_MEMORY_SCOPE_SYSTEM);
}
__device__ __forceinline__ void st_flag8(unsigned char* p, unsigned v) {
    __hip_atomic_store(p, (unsigned char)v, __ATOMIC_RELAXED, __HIP_MEMORY_SCOPE_SYSTEM);
}
__device__ __forceinline__ unsigned lane_id() {
    return __builtin_amdgcn_mbcnt_hi(~0u, __builtin_amdgcn_mbcnt_lo(~0u, 0u));
}
__device__ __forceinline__ unsigned cvt_pk_bf16(float lo, float hi) {
    unsigned r; asm volatile("v_cvt_pk_bf16_f32 %0, %1, %2" : "=v"(r) : "v"(lo), "v"(hi)); return r;
}
typedef float f32x2 __attribute__((ext_vector_type(2)));
__device__ __forceinline__ f32x2 gelu2(f32x2 y) {
    const float k1 = -2.0f * 0.7978845608028654f * 1.4426950408889634f;
    const float k3 = k1 * 0.044715f;
    const f32x2 z = y * ((y * y) * k3 + k1);
    const f32x2 d = (f32x2){__builtin_amdgcn_exp2f(z.x), __builtin_amdgcn_exp2f(z.y)} + 1.0f;
    return y * (f32x2){__builtin_amdgcn_rcpf(d.x), __builtin_amdgcn_rcpf(d.y)};
}

template <bool EQ>
__device__ __forceinline__ bool wait_word(const unsigned char* p, unsigned want, unsigned char* tmo, unsigned call) {
    const u64 t0 = __builtin_amdgcn_s_memrealtime();
    for (;;) {
        unsigned g, t;
        asm volatile("global_load_dword %0, %2, off sc0 sc1\n\t"
                     "global_load_dword %1, %3, off sc1\n\t"
                     "s_waitcnt vmcnt(0)" : "=&v"(g), "=&v"(t) : "v"(p), "v"(tmo) : "memory");
        if ((unsigned)__builtin_amdgcn_readfirstlane(t) == call || __builtin_amdgcn_s_memrealtime() - t0 > DEADLINE) {
            __hip_atomic_store((unsigned*)tmo, call, __ATOMIC_RELAXED, __HIP_MEMORY_SCOPE_AGENT);
            return true;
        }
        const unsigned v = __builtin_amdgcn_readfirstlane(g);
        if (EQ ? v == want : v >= want) return false;
        __builtin_amdgcn_s_sleep(2);
    }
}

__device__ __forceinline__ u64 now() {
    u64 t; asm volatile("s_memrealtime %0\n\ts_waitcnt lgkmcnt(0)" : "=s"(t) :: "memory"); return t;
}
__device__ __forceinline__ u64 poll_ready(u64 todo, const unsigned char* ws, unsigned fa, unsigned fb,
                                          unsigned char* tmo, unsigned call, unsigned tag4, unsigned xm) {
    const u64 t0 = now();
    for (;;) {
        unsigned a, b, t; const unsigned zero = 0u;
        asm volatile(SBASE "global_load_dword %0, %3, %6 sc0 sc1\n\t"
                     "global_load_dword %1, %4, %6 sc0 sc1\n\t"
                     "global_load_dword %2, %5, %7 sc1\n\t"
                     "s_waitcnt vmcnt(0)"
                     : "=&v"(a), "=&v"(b), "=&v"(t) : "v"(fa), "v"(fb), "v"(zero), "s"(ws), "s"(tmo) : "memory");
        const u64 ready = __ballot(a == tag4 && ((b ^ tag4) & xm) == 0u);
        if ((ready & todo) != 0ull) return ready;
        if ((unsigned)__builtin_amdgcn_readfirstlane(t) == call || now() - t0 > DEADLINE) {
            __hip_atomic_store((unsigned*)tmo, call, __ATOMIC_RELAXED, __HIP_MEMORY_SCOPE_AGENT);
            return ~0ull;
        }
        __builtin_amdgcn_s_sleep(1);
    }
}

__device__ __forceinline__ unsigned bfi(unsigned m, unsigned a, unsigned b) { return (a & m) | (b & ~m); }
__device__ __forceinline__ unsigned perm(unsigned hi, unsigned lo, unsigned sel) { return __builtin_amdgcn_perm(hi, lo, sel); }
#define SDWA2(k) " dst_sel:DWORD dst_unused:UNUSED_PAD src0_sel:BYTE_" #k " src1_sel:BYTE_" #k "\n\t"
#define SDWA1(k) " dst_sel:DWORD dst_unused:UNUSED_PAD src0_sel:DWORD src1_sel:BYTE_" #k "\n\t"
__device__ __forceinline__ unsigned spread_bits(unsigned f, unsigned s) {
    unsigned r, a, b;
    asm("v_lshlrev_b32_sdwa %0, %4, %3" SDWA2(0) "v_lshlrev_b32_sdwa %1, %4, %3" SDWA2(1) "v_lshlrev_b32_sdwa %2, %4, %3" SDWA2(2)
        "v_or3_b32 %0, %0, %1, %2\n\t"
        "v_lshlrev_b32_sdwa %1, %4, %3" SDWA2(3)
        "v_or_b32 %0, %0, %1" : "=&v"(r), "=&v"(a), "=&v"(b) : "v"(f), "v"(s));
    return r;
}
__device__ __forceinline__ unsigned take_codes(unsigned& w, unsigned f) {
    unsigned ln, t, l;
    asm("v_xor_b32_sdwa %2, %0, %4" SDWA1(0) "v_ffbl_b32 %1, %2\n\tv_lshrrev_b32 %0, %1, %0\n\t"
        "v_xor_b32_sdwa %2, %0, %4" SDWA1(1) "v_ffbl_b32 %3, %2\n\tv_lshrrev_b32 %0, %3, %0\n\t"
        "v_xor_b32_sdwa %2, %0, %4" SDWA1(2) "v_lshl_or_b32 %1, %3, 8, %1\n\tv_ffbl_b32 %3, %2\n\tv_lshrrev_b32 %0, %3, %0\n\t"
        "v_xor_b32_sdwa %2, %0, %4" SDWA1(3) "v_lshl_or_b32 %1, %3, 16, %1\n\tv_ffbl_b32 %3, %2\n\tv_lshrrev_b32 %0, %3, %0\n\t"
        "v_lshl_or_b32 %1, %3, 24, %1" : "+v"(w), "=&v"(ln), "=&v"(t), "=&v"(l) : "v"(f));
    return ln;
}
__device__ __forceinline__ unsigned wave_scan(unsigned v) {
    v += (unsigned)__builtin_amdgcn_update_dpp(0, (int)v, 0x111, 0xf, 0xf, false);
    v += (unsigned)__builtin_amdgcn_update_dpp(0, (int)v, 0x112, 0xf, 0xf, false);
    v += (unsigned)__builtin_amdgcn_update_dpp(0, (int)v, 0x114, 0xf, 0xf, false);
    v += (unsigned)__builtin_amdgcn_update_dpp(0, (int)v, 0x118, 0xf, 0xf, false);
    v += (unsigned)__builtin_amdgcn_update_dpp(0, (int)v, 0x142, 0xa, 0xf, false);
    v += (unsigned)__builtin_amdgcn_update_dpp(0, (int)v, 0x143, 0xc, 0xf, false);
    return v;
}
__device__ __forceinline__ unsigned rank_in(u64 mask) {
    return __builtin_amdgcn_mbcnt_hi((unsigned)(mask >> 32), __builtin_amdgcn_mbcnt_lo((unsigned)mask, 0u));
}
constexpr int SCR = 8192;
#define LDS_ORDER() asm volatile("" ::: "memory")

struct Packed { u32x4 sm[4], l1, st; u64 mask; unsigned meta; };

__device__ __forceinline__ void pack(const u32x4 (&d)[8], Packed& o, LAS unsigned char* scr, unsigned lane) {
    *(LAS u32x4*)(scr + lane * 16) = (u32x4){0u, 0u, 0u, 0u};
    unsigned sum = 0; u32x4 er[4];
#pragma unroll
    for (int g = 0; g < 16; ++g) {
        const unsigned d0 = d[g >> 1][(g & 1) * 2], d1 = d[g >> 1][(g & 1) * 2 + 1];
        const unsigned lb = perm(d1, d0, 0x06040200u), hb = perm(d1, d0, 0x07050301u);
        o.sm[g >> 2][g & 3] = bfi(0x7f7f7f7fu, lb, hb);
        const unsigned e = bfi(0x01010101u, lb >> 7, hb << 1);
        er[g >> 2][g & 3] = e;
        if (g % 5 == 0) sum = __builtin_amdgcn_sad_u8(e, 0u, sum);
    }
    const unsigned r2 = min(max(((unsigned)__builtin_amdgcn_readlane(wave_scan(sum), 63) + 4009u) >> 10, 31u), 255u);
    const unsigned kk = r2 * 0x01010101u;
    unsigned bad = 0, top = ~0u, longest = 0, total = 0, piece[8], plen[8], us[16], cs[16];
#pragma unroll
    for (int g = 0; g < 16; ++g) {
        const unsigned u = kk - er[g >> 2][g & 3];
        bad |= u;
        us[g] = u; cs[g] = perm(0x0f0f0f0cu, 0x0a080f07u, u | ((u << 1) & 0x10101010u));
        top &= cs[g];
    }
    const bool hole = __ballot((top & 0x08080808u) != 0x08080808u) != 0ull;
    const unsigned tx = hole ? 0xfdfdfdfdu : 0xfefefefeu, thi = 0x01000000u ^ tx, tlo = 0x01020609u ^ tx;
#pragma unroll
    for (int q = 0; q < 4; ++q) {
        unsigned l1 = 0, acc = 0, base = 0;
#pragma unroll
        for (int gg = 0; gg < 4; ++gg) {
            const unsigned u = us[q * 4 + gg], c = cs[q * 4 + gg];
            l1 |= gg ? (c & 0x06060606u) << (2 * gg - 1) : (c & 0x06060606u) >> 1;
            const unsigned len4 = (perm(thi, tlo, u | (c & 0x10101010u)) ^ tx) + u;
            const unsigned f = c & 0x01010101u;
            unsigned ps = len4 + (len4 << 8); asm("" : "+v"(ps)); ps += ps << 16; ps -= 0x100c0804u;
            const unsigned bits = spread_bits(f, ps);
            if ((gg & 1) == 0) { acc = bits; base = ps >> 24; }
            else {
                acc |= bits << (base & 31u); base += ps >> 24;
                piece[q * 2 + (gg >> 1)] = acc >> 1; plen[q * 2 + (gg >> 1)] = base;
                longest = max(longest, base); total += base;
            }
        }
        o.l1[q] = l1;
    }
    const bool raw = (bad & 0xe0e0e0e0u) != 0u || longest > 31u || total > 112u;
    o.mask = __ballot(raw);
    if (raw) total = 0;
    const unsigned incl = wave_scan(total);
    const unsigned rawoff = (((unsigned)__builtin_amdgcn_readlane(incl, 63) + 512u + 127u) >> 7) << 4;
    o.meta = r2 | ((rawoff + 64u * (unsigned)__builtin_popcountll(o.mask) + 127u) >> 7) << 8 | (hole ? 0x4000u : 0u);
    LDS_ORDER();
    scr[lane] = (unsigned char)total;
    if (!raw) {
        unsigned off = incl - total + 512u;
#pragma unroll
        for (int i = 0; i < 8; ++i) {
            const u64 w = (u64)piece[i] << (off & 31u);
            LAS unsigned* p = (LAS unsigned*)(scr + ((off >> 5) << 2));
            __hip_atomic_fetch_or(p, (unsigned)w, __ATOMIC_RELAXED, __HIP_MEMORY_SCOPE_WORKGROUP);
            __hip_atomic_fetch_or(p + 1, (unsigned)(w >> 32), __ATOMIC_RELAXED, __HIP_MEMORY_SCOPE_WORKGROUP);
            off += plen[i];
        }
    } else {
        LAS u32x4* p = (LAS u32x4*)(scr + rawoff + rank_in(o.mask) * 64u);
#pragma unroll
        for (int i = 0; i < 4; ++i) p[i] = er[i];
    }
    LDS_ORDER();
    o.st = *(const LAS u32x4*)(scr + lane * 16);
    LDS_ORDER();
}
__device__ __forceinline__ void send(const Packed& o, unsigned char* pl, const LAS unsigned char* scr, unsigned lane) {
    unsigned char* p = pl + lane * 16;
    for (unsigned k = 1; k * 8u < ((o.meta >> 8) & 63u); ++k)
        if (k * 64u + lane < ((o.meta >> 8) & 63u) * 8u) st16_sys(p + 5120 + k * 1024, *(const LAS u32x4*)(scr + k * 1024 + lane * 16));
#pragma unroll
    for (int i = 0; i < 4; ++i) st16_sys(p + i * 1024, o.sm[i]);
    st16_sys(p + 4096, o.l1);
    if (lane < ((o.meta >> 8) & 63u) * 8u) st16_sys(p + 5120, o.st);
}
__device__ __forceinline__ u32x4 header(const Packed& o, unsigned call) {
    return (u32x4){(unsigned)o.mask, call, (unsigned)(o.mask >> 32), (call << 15) | o.meta};
}
__device__ __forceinline__ bool header_ok(u32x4 h, unsigned call, u64& mask, unsigned& meta) {
    const unsigned t1 = __builtin_amdgcn_readfirstlane(h.y), t3 = __builtin_amdgcn_readfirstlane(h.w);
    mask = ((u64)(unsigned)__builtin_amdgcn_readfirstlane(h.z) << 32) | (unsigned)__builtin_amdgcn_readfirstlane(h.x);
    meta = t3 & 0x7fffu;
    return t1 == call && (t3 >> 15) == (call & 0x1ffffu);
}
__device__ __forceinline__ void wait_header(const unsigned char* hd, unsigned char* tmo, unsigned call, u64& mask, unsigned& meta) {
    const u64 t0 = __builtin_amdgcn_s_memrealtime();
    for (;;) {
        u32x4 h; unsigned t;
        asm volatile("global_load_dwordx4 %0, %2, off sc0 sc1\n\t"
                     "global_load_dword %1, %3, off sc1\n\t"
                     "s_waitcnt vmcnt(0)" : "=&v"(h), "=&v"(t) : "v"(hd), "v"(tmo) : "memory");
        if (header_ok(h, call, mask, meta)) break;
        if ((unsigned)__builtin_amdgcn_readfirstlane(t) == call || __builtin_amdgcn_s_memrealtime() - t0 > DEADLINE) {
            __hip_atomic_store((unsigned*)tmo, call, __ATOMIC_RELAXED, __HIP_MEMORY_SCOPE_AGENT);
            mask = 0ull; meta = 0u;
            break;
        }
        __builtin_amdgcn_s_sleep(1);
    }
}
__device__ __forceinline__ void fetch(Packed& o, const unsigned char* pl, unsigned lane) {
    const unsigned n = min(max((o.meta >> 8) & 63u, 1u), 8u) * 8u;
    const u64 smask = n >= 64u ? ~0ull : (1ull << n) - 1ull;
    u64 keep;
    asm volatile(SBASE "s_mov_b64 %6, exec\n\t"
                 "global_load_dwordx4 %0, %7, %8 sc0 sc1\n\t"
                 "global_load_dwordx4 %1, %7, %8 offset:1024 sc0 sc1\n\t"
                 "global_load_dwordx4 %2, %7, %8 offset:2048 sc0 sc1\n\t"
                 "global_load_dwordx4 %3, %7, %8 offset:3072 sc0 sc1\n\t"
                 "global_load_dwordx4 %4, %7, %9 sc0 sc1\n\t"
                 "s_mov_b64 exec, %10\n\t"
                 "global_load_dwordx4 %5, %7, %9 offset:1024 sc0 sc1\n\t"
                 "s_mov_b64 exec, %6"
                 : "=&v"(o.sm[0]), "=&v"(o.sm[1]), "=&v"(o.sm[2]), "=&v"(o.sm[3]), "=&v"(o.l1), "=&v"(o.st), "=&s"(keep)
                 : "v"(lane * 16u), "s"(pl), "s"(pl + 4096), "s"(smask) : "memory");
}
#define FETCH_WAIT(o, n) asm volatile("s_waitcnt vmcnt(" #n ")" : "+v"(o.sm[0]), "+v"(o.sm[1]), "+v"(o.sm[2]), "+v"(o.sm[3]), "+v"(o.l1), "+v"(o.st) :: "memory")
template <class Mid>
__device__ __forceinline__ void unpack(const Packed& o, u32x4 (&d)[8], const unsigned char* pl, LAS unsigned char* scr, unsigned lane, Mid mid) {
    *(LAS u32x4*)(scr + lane * 16) = o.st;
    if (((o.meta >> 8) & 63u) > 8u) {
        u32x4 m[4];
        asm volatile("global_load_dwordx4 %0, %4, off sc0 sc1\n\t"
                     "global_load_dwordx4 %1, %4, off offset:1024 sc0 sc1\n\t"
                     "global_load_dwordx4 %2, %4, off offset:2048 sc0 sc1\n\t"
                     "global_load_dwordx4 %3, %4, off offset:3072 sc0 sc1\n\t"
                     "s_waitcnt vmcnt(0)" : "=&v"(m[0]), "=&v"(m[1]), "=&v"(m[2]), "=&v"(m[3]) : "v"(pl + 6144 + lane * 16) : "memory");
#pragma unroll
        for (int k = 0; k < 4; ++k) *(LAS u32x4*)(scr + (k + 1) * 1024 + lane * 16) = m[k];
    }
    LDS_ORDER();
    const unsigned len = scr[lane] & 127u;
    const unsigned incl = wave_scan(len), off = incl - len + 512u;
    const LAS unsigned* p = (const LAS unsigned*)(scr + ((off >> 5) << 2));
    const unsigned w0 = p[0], w1 = p[1], w2 = p[2], w3 = p[3], w4 = p[4];
    unsigned s0 = __builtin_amdgcn_alignbit(w1, w0, off), s1 = __builtin_amdgcn_alignbit(w2, w1, off);
    unsigned s2 = __builtin_amdgcn_alignbit(w3, w2, off), s3 = __builtin_amdgcn_alignbit(w4, w3, off);
    const bool hole = (o.meta & 0x4000u) != 0u;
    const unsigned tx = hole ? 0xf8f8f8f8u : 0xf7f7f7f7u, thi = (hole ? 0x07070008u : 0x08080808u) ^ tx, tlo = 0x0309090au ^ tx;
    const unsigned kk = ((o.meta & 255u) + 8u) * 0x01010101u;
    u32x4 er[4];
#pragma unroll
    for (int q = 0; q < 4; ++q) {
        if (q == 2) { asm volatile("" : "+v"(s0)); mid(); asm volatile("" : "+v"(s0)); }
        const unsigned l1 = o.l1[q], x = l1 & (l1 >> 1);
        unsigned w = 0, used = 0;
#pragma unroll
        for (int gg = 0; gg < 4; ++gg) {
            if ((gg & 1) == 0) { w = (s0 << 1) | 1u; used = 0; }
            const unsigned f = (x >> (2 * gg)) & 0x01010101u;
            const unsigned ln = take_codes(w, f);
            used = __builtin_amdgcn_sad_u8(ln, 0u, used);
            const unsigned y = (perm(thi, tlo, ln | ((ln << 1) & 0x10101010u)) ^ tx) + ln + ((l1 >> (2 * gg)) & 0x03030303u);
            er[q][gg] = kk - y;
            if (gg & 1) {
                s0 = __builtin_amdgcn_alignbit(s1, s0, used); s1 = __builtin_amdgcn_alignbit(s2, s1, used);
                s2 = __builtin_amdgcn_alignbit(s3, s2, used); s3 >>= used & 31u;
            }
        }
    }
    if (o.mask != 0ull) {
        const unsigned rawoff = (((unsigned)__builtin_amdgcn_readlane(incl, 63) + 512u + 127u) >> 7) << 4;
        if ((o.mask >> lane) & 1ull) {
            const LAS u32x4* r = (const LAS u32x4*)(scr + rawoff + rank_in(o.mask) * 64u);
#pragma unroll
            for (int i = 0; i < 4; ++i) er[i] = r[i];
        }
    }
    LDS_ORDER();
#pragma unroll
    for (int g = 0; g < 16; ++g) {
        const unsigned e = er[g >> 2][g & 3], sm = o.sm[g >> 2][g & 3];
        const unsigned hb = bfi(0x80808080u, sm, e >> 1), lb = bfi(0x7f7f7f7fu, sm, e << 7);
        d[g >> 1][(g & 1) * 2] = perm(hb, lb, 0x05010400u); d[g >> 1][(g & 1) * 2 + 1] = perm(hb, lb, 0x07030602u);
    }
}

__device__ __forceinline__ void load_parked(u32x4 (&dg)[8], unsigned& tw, const unsigned char* dp, const unsigned char* tmo, unsigned lane) {
    asm volatile(SBASE "global_load_dwordx4 %0, %9, %10 sc0 sc1\n\t"
                 "global_load_dwordx4 %1, %9, %10 offset:1024 sc0 sc1\n\t"
                 "global_load_dwordx4 %2, %9, %10 offset:2048 sc0 sc1\n\t"
                 "global_load_dwordx4 %3, %9, %10 offset:3072 sc0 sc1\n\t"
                 "global_load_dwordx4 %4, %9, %11 sc0 sc1\n\t"
                 "global_load_dwordx4 %5, %9, %11 offset:1024 sc0 sc1\n\t"
                 "global_load_dwordx4 %6, %9, %11 offset:2048 sc0 sc1\n\t"
                 "global_load_dwordx4 %7, %9, %11 offset:3072 sc0 sc1\n\t"
                 "global_load_dword %8, %8, %12 sc1"
                 : "=&v"(dg[0]), "=&v"(dg[1]), "=&v"(dg[2]), "=&v"(dg[3]), "=&v"(dg[4]), "=&v"(dg[5]), "=&v"(dg[6]), "=&v"(dg[7]), "+v"(tw)
                 : "v"(lane * 16u), "s"(dp), "s"(dp + 4096), "s"(tmo) : "memory");
}
#define PARKED_WAIT(dg, tw) asm volatile("s_waitcnt vmcnt(0)" : "+v"(dg[0]), "+v"(dg[1]), "+v"(dg[2]), "+v"(dg[3]), "+v"(dg[4]), "+v"(dg[5]), \
                                         "+v"(dg[6]), "+v"(dg[7]), "+v"(tw) :: "memory")


constexpr double ORD_W = 0.3, ORD_A1 = 1.17, ORD_B0 = 0.4, ORD_BS = 1.22, ORD_C0 = 10.5, ORD_CS = 0.3;
struct Order { unsigned short v[354]; };
constexpr Order make_order() {
    double key[352] = {}; unsigned short id[352] = {}; int n = 0;
    for (int kc = 0; kc < 8; ++kc)
        for (int h = 0; h < 2; ++h) {
            for (int e = 0; e < 8; ++e) { key[n] = kc - ORD_W + (e + 0.4) / 8 + h * 1e-3 - (kc == 0 && e < PLAIN + PKP ? 9 : 0); id[n++] = (unsigned short)(h << 8 | 0 << 6 | kc << 3 | e); }
            for (int e = 0; e < 6; ++e) { key[n] = (kc + (e + 0.5) / 6) * (h ? ORD_A1 : 1.0); id[n++] = (unsigned short)(h << 8 | 1 << 6 | kc << 3 | e); }
            for (int e = 0; e < 6; ++e) { key[n] = ORD_B0 + (kc + (e + 0.6) / 6) * ORD_BS + h * 1e-3; id[n++] = (unsigned short)(h << 8 | 2 << 6 | kc << 3 | e); }
            for (int e = 0; e < 2; ++e) { key[n] = ORD_C0 + (2 * kc + e + 0.5 * h) * ORD_CS; id[n++] = (unsigned short)(h << 8 | 3 << 6 | kc << 3 | e); }
        }
    Order o = {};
    for (int i = 0; i < 352; ++i) { int r = 0; for (int j = 0; j < 352; ++j) r += (key[j] < key[i]) || (key[j] == key[i] && j < i); o.v[r] = id[i]; }
    return o;
}
constexpr Order ORDER = make_order();
constexpr unsigned order_word(int i) { return ORDER.v[3 * i] | ORDER.v[3 * i + 1] << 10 | (unsigned)ORDER.v[3 * i + 2] << 20; }
template <int I, int N> __device__ __forceinline__ unsigned order_vgpr(unsigned v) {
    if constexpr (I == N) return v;
    else {
        unsigned t;
        asm("s_mov_b32 %1, %2\n\tv_writelane_b32 %0, %1, %3" : "+v"(v), "=&s"(t) : "i"(order_word(I)), "i"(I & 63));
        return order_vgpr<I + 1, N>(v);
    }
}
__device__ __forceinline__ void decode_unit(unsigned ord0, unsigned ord1, int t, int src, int dest, int& isw, int& kc, int& tile, int& q) {
    const int n = t >> 1, dw = n / 3;
    const unsigned w0 = __builtin_amdgcn_readlane(ord0, dw & 63), w1 = __builtin_amdgcn_readlane(ord1, dw & 63);
    const unsigned v = (dw < 64 ? w0 : w1) >> ((n - dw * 3) * 10);
    const int kind = (v >> 6) & 3, e = v & 7;
    kc = (v >> 3) & 7; isw = kind == 0; q = ((v >> 7) & 2) | (t & 1);
    const int d = (dest - src - 1) & 7;
    tile = kind == 0 ? ((e + src) & 7) : kind == 3 ? 2 * dest + e : 2 * ((src + 1 + e + (e >= d)) & 7) + (kind - 1);
}

__global__ __launch_bounds__(512, 2) void fused_gemm_rs(const Params P) {
    __shared__ __attribute__((aligned(16))) unsigned char lds_raw[9 * HTB + 256];
    LAS unsigned char* lds = (LAS unsigned char*)lds_raw;
    const int tid = threadIdx.x, wid = __builtin_amdgcn_readfirstlane(tid >> 6), lane = tid & 63;
    const int b = blockIdx.x, self = P.rank;
    unsigned char* const ws = P.own;
    const GAS unsigned char *px = (const GAS unsigned char*)P.x, *pw = (const GAS unsigned char*)P.w;
    asm volatile("" : "+s"(px), "+s"(pw));
    unsigned count; const unsigned vzero = 0u;
    asm volatile(SBASE "global_load_dword %0, %1, %2 sc1" : "=&v"(count) : "v"(vzero), "s"(ws + OFF_CNT + b * 128) : "memory");
    unsigned char* const tmo = ws + OFF_TMO;
    LAS unsigned char* const scr = lds + wid * SCR;
    const u64 tstart = __builtin_amdgcn_s_memrealtime();
    const int nt = b & 7, slot = b >> 3, r3 = slot - 7;
    const int g = r3 < 0 ? -1 : r3 >= 21 ? r3 : r3 % 3 == 2 ? 14 + r3 / 3 : r3 / 3 * 2 + r3 % 3;
    const bool is_pull = g < 0, is_a = !is_pull && g < 14, is_b = g >= 14 && g < 21, is_c = g >= 21;
    const int owner = is_c ? self : (self + 1 + (is_a ? g >> 1 : g - 14)) & 7;
    const int mt = 2 * owner + (is_c ? (g - 21) >> 1 : is_b), hf = is_c ? (g - 21) & 1 : g & 1;
    auto pack_decode = [&](int t, int& isw, int& kc, int& tile) {
        const int blk = t >> 2;
        if (blk < 160) {
            kc = blk >= 80; const int r = blk - 80 * kc, m = r - 16; isw = r >= 16;
            tile = isw ? ((self + 1 + m) & 7) * 8 + (((m >> 3) + self) & 7) : r;
        } else if (blk < 304) {
            const int m = blk - 160, r = m % 24; kc = 2 + m / 24; isw = r >= 16;
            tile = isw ? self * 8 + (r & 7) : r;
        } else if (blk < 640) {
            const int m = blk - 304, w = m % 56; isw = 1; kc = 2 + m / 56;
            tile = ((self + 1 + w % 7) & 7) * 8 + ((w / 7 + self) & 7);
        } else {
            const int m = blk - 640; isw = 1; kc = m & 1;
            tile = ((self + 1 + (m >> 4)) & 7) * 8 + ((m >> 1) & 7);
        }
    };
    const unsigned lw = (lane >> 4) * 8 * (GN * 2) + (lane & 15) * 16, lx = (lane & 15) * (KS * 2) + (lane >> 4) * 16;
    auto load_task = [&](int t, u32x4 (&r)[8]) {
        int isw, kc, tile; pack_decode(t, isw, kc, tile); const int q = t & 3;
        const GAS unsigned char* src = isw ? pw + ((kc * 64 + (q & 1) * 32) * (GN * 2) + (tile * 256 + (q >> 1) * 128) * 2)
                                       : px + ((tile * 256 + q * 64) * (KS * 2) + kc * 128);
        const unsigned lo = isw ? lw : lx, s1 = isw ? GN * 2 : 64, s2 = isw ? GN * 4 : 16 * KS * 2;
#pragma unroll
        for (int i = 0; i < 8; ++i) r[i] = *(const GAS u32x4*)(src + ((i >> 1) * s2 + (i & 1) * s1) + lo);
    };
    auto store_image = [&](unsigned char* blk, int isw, int q, const u32x4 (&c)[8]) {
        unsigned char* d0 = blk + (isw ? (q >> 1) * 8192 + (q & 1) * 1024 : q * 8192) + lane * 16;
        const unsigned s1 = isw ? 2048 : 1024, s4 = isw ? HTB : 4096;
#pragma unroll
        for (int j = 0; j < 8; ++j) st16_sys(d0 + ((j >> 2) * s4 + (j & 3) * s1), c[j]);
    };
    u32x4 cur[8];
    const int task0 = is_pull ? wid * 8 + nt : wid * 144 + (is_a ? g : g - 7) * 8 + nt;
    unsigned peer_lo = 0u, peer_hi = 0u;
    auto peers = [&]() {
#pragma unroll
        for (int r = 0; r < NR; ++r)
            asm("v_writelane_b32 %0, %2, %4\n\tv_writelane_b32 %1, %3, %4" : "+v"(peer_lo), "+v"(peer_hi)
                : "s"((unsigned)(u64)P.ws[r]), "s"((unsigned)((u64)P.ws[r] >> 32)), "i"(r));
    };
    if (is_a || is_c) {
        if (wid >= HEAD_W) {
            if (wid < 2 * HEAD_W) {
                peers();
                int isw, kc, tile; const int t = task0 - HEAD_W * 144; pack_decode(t, isw, kc, tile);
                const size_t unit = (isw ? tile * 8 : 512 + tile) * 4 + (t & 3), head = (isw ? tile & 7 : 64 + tile) * 4 + (t & 3);
                const unsigned char* a = lane < NR ? (unsigned char*)(((u64)peer_hi << 32) | peer_lo) + OFF_HIN + (self * 768 + head) * HS + 64
                                       : lane < 10 ? ws + OFF_RP + ((lane & 1) * 2560 + unit) * UNIT
                                       : ws + (isw ? OFF_WB + (size_t)(self * 64 + (tile & 7)) * BLK : OFF_XA + (size_t)(self * 128 + tile) * BLK) + lane * 128;
                if (lane < 11 && (lane >= NR || !isw || lane == (tile >> 3))) { unsigned z; asm volatile("global_load_dword %0, %1, off sc0 sc1\n\ts_waitcnt vmcnt(0)" : "=&v"(z) : "v"(a) : "memory"); }
            }
            while (__builtin_amdgcn_s_memrealtime() - tstart < (wid < NEXT_W ? PACK_DELAY : PACK_DELAY2)) __builtin_amdgcn_s_sleep(2);
            if (wid < NEXT_W) __builtin_amdgcn_s_setprio(1); }
        else __builtin_amdgcn_s_setprio(2);
        load_task(task0, cur);
    }
    if (is_b && wid == 7 && lane < 16) {
        const unsigned char* ows = P.ws[owner];
        const unsigned char* a = lane < 2 ? ws + OFF_PP + (size_t)(((2 * owner + lane) * 8 + nt) * 16) * UNIT
                               : lane < 4 ? ows + OFF_PP + (size_t)(((2 * self + (lane & 1)) * 8 + nt) * 16) * UNIT
                               : lane < 5 ? ows + OFF_HP + (size_t)((self * 8 + nt) * 16) * HS + 64
                               : lane < 6 ? ws + OFF_HP + (size_t)((owner * 8 + nt) * 16) * HS + 64
                               : lane < 8 ? ws + OFF_PD + (size_t)(((g - 14) * 2 + (lane & 1)) * 8 + nt) * 131072
                               : P.out + (size_t)(lane - 8) * 64 * (GN * 2) + (owner * 8 + nt) * 512;
        unsigned z; asm volatile("global_load_dword %0, %1, off sc0 sc1\n\ts_waitcnt vmcnt(0)" : "=&v"(z) : "v"(a) : "memory");
    }
    unsigned ord0 = 0u, ord1 = 0u;
    if (is_pull) { ord0 = order_vgpr<0, 64>(0u); ord1 = order_vgpr<64, 118>(0u); } else peers();
    asm volatile("s_waitcnt vmcnt(0)" : "+v"(count), "+v"(ord0), "+v"(ord1) :: "memory");
    const unsigned call = (unsigned)__builtin_amdgcn_readfirstlane(count) + 1u;
    const size_t par = call & 1u;
    const unsigned tag4 = (call & 255u) * 0x01010101u;
    const int wr = wid >> 2, wc = wid & 3;

    if (is_pull) {
        LAS unsigned* const ctl = (LAS unsigned*)(lds + 9 * HTB + 64);
        if (tid < 4) ctl[tid] = 0u;
        __syncthreads();
        const unsigned share = (WIN + 7 - nt) / 8;
        auto lds_draw = [&](int w) {
            unsigned v = 0;
            if (lane == 0) v = __hip_atomic_fetch_add(&ctl[w], 1u, __ATOMIC_RELAXED, __HIP_MEMORY_SCOPE_WORKGROUP);
            return (unsigned)__builtin_amdgcn_readfirstlane(v);
        };
        auto acquire = [&]() {
            const unsigned turn = lds_draw(0);
            while (__hip_atomic_load(&ctl[1], __ATOMIC_RELAXED, __HIP_MEMORY_SCOPE_WORKGROUP) + share <= turn) __builtin_amdgcn_s_sleep(1);
        };
        auto release = [&]() { if (lane == 0) __hip_atomic_fetch_add(&ctl[1], 1u, __ATOMIC_RELAXED, __HIP_MEMORY_SCOPE_WORKGROUP); };
        const int src = (self + 1 + slot) & 7;
        const unsigned char* const sws = P.ws[src];
        {
            unsigned* const tick = (unsigned*)(ws + OFF_TICK + src * 128);
            const unsigned tbase = (call - 1u) * NTASK - PW;
            const unsigned char* const hbase = ws + OFF_HIN + (par * 8 + src) * (768 * HS);
            int task = task0, isw, kc, tile, q;
            decode_unit(ord0, ord1, task, src, self, isw, kc, tile, q);
            Packed pk;
            unsigned touched = 0u;
            {
                const unsigned char* a = lane ? ws + (isw ? OFF_WB + (size_t)((src * 8 + kc) * 8 + tile) * BLK : OFF_XA + (size_t)((src * 8 + kc) * 16 + tile) * BLK) + q * 8192
                    : sws + OFF_RP + (par * 2560 + (isw ? (self * 8 + tile) * 8 + kc : 512 + kc * 16 + tile) * 4 + q) * UNIT + 8320 + self * 128;
                asm volatile("global_load_dword %0, %1, off sc0 sc1" : "=&v"(touched) : "v"(a) : "memory");
            }
            unsigned second = 0u;
            if (task < 4 * PLAIN && lane == 0) second = __hip_atomic_fetch_add(tick, 1u, __ATOMIC_RELAXED, __HIP_MEMORY_SCOPE_AGENT);
            asm volatile("s_waitcnt vmcnt(0)" : "+v"(touched), "+v"(second) :: "memory");
            second = __builtin_amdgcn_readfirstlane(second);
            while (__builtin_amdgcn_s_memrealtime() - tstart < PULL_DELAY) __builtin_amdgcn_s_sleep(2);
            wait_header(hbase + ((isw ? kc * 8 + tile : 64 + kc * 16 + tile) * 4 + q) * HS, tmo, call, pk.mask, pk.meta);
            if (task < 4 * PLAIN) {
                unsigned tw = 0u;
                load_parked(cur, tw, ws + OFF_LB + (((par * 8 + src) * 8 + tile) * 4 + q) * UNIT, tmo, lane);
                PARKED_WAIT(cur, tw);
                store_image(ws + OFF_WB + (size_t)(src * 64 + tile) * BLK, 1, q, cur);
                asm volatile("s_waitcnt vmcnt(0)" ::: "memory");
                if (lane < RW) st_flag8(ws + OFF_FW + (tile * 64 + src * 8) * 4 + q + lane * FR, call);
                task = (int)(second - tbase);
                decode_unit(ord0, ord1, task < NTASK ? task : 0, src, self, isw, kc, tile, q);
                wait_header(hbase + ((isw ? kc * 8 + tile : 64 + kc * 16 + tile) * 4 + q) * HS, tmo, call, pk.mask, pk.meta);
            }
            while (task < NTASK) {
                const bool here = task < 4 * (PLAIN + PKP);
                const unsigned char* const pl = here ? ws + OFF_LB + (((par * 8 + src) * 8 + tile) * 4 + q) * UNIT
                    : sws + OFF_RP + (par * 2560 + (isw ? (self * 8 + tile) * 8 + kc : 512 + kc * 16 + tile) * 4 + q) * UNIT;
                if (!here) acquire();
                fetch(pk, pl, lane);
                FETCH_WAIT(pk, 0);
                if (!here) release();
                unsigned nn = 1u;
                {
                    u64 keep; const unsigned zero = 0u;
                    asm volatile(SBASE "s_mov_b64 %1, exec\n\ts_mov_b64 exec, 1\n\t"
                                 "global_atomic_add %0, %2, %0, %3 sc0\n\t"
                                 "s_mov_b64 exec, %1" : "+v"(nn), "=&s"(keep) : "v"(zero), "s"(tick) : "memory");
                }
                int ntask, nisw, nkc, ntile, nq; u32x4 h; const unsigned char* hd;
                unpack(pk, cur, pl, scr, lane, [&]() {
                    asm volatile("s_waitcnt vmcnt(0)" : "+v"(nn) :: "memory");
                    ntask = (int)(__builtin_amdgcn_readfirstlane(nn) - tbase);
                    decode_unit(ord0, ord1, ntask < NTASK ? ntask : 0, src, self, nisw, nkc, ntile, nq);
                    hd = hbase + ((nisw ? nkc * 8 + ntile : 64 + nkc * 16 + ntile) * 4 + nq) * HS;
                    const unsigned zero = 0u;
                    asm volatile(SBASE "global_load_dwordx4 %0, %1, %2 sc0 sc1" : "=&v"(h) : "v"(zero), "s"(hd) : "memory");
                });
                const int idx = src * 8 + kc;
                store_image(isw ? ws + OFF_WB + (size_t)(idx * 8 + tile) * BLK : ws + OFF_XA + (size_t)(idx * 16 + tile) * BLK, isw, q, cur);
                asm volatile("s_waitcnt vmcnt(0)" : "+v"(h) :: "memory");
                if (lane < (isw ? RW : RX)) st_flag8(ws + (isw ? OFF_FW : OFF_FX) + (tile * 64 + idx) * 4 + q + lane * FR, call);
                if (ntask >= NTASK) break;
                if (!header_ok(h, call, pk.mask, pk.meta)) wait_header(hd, tmo, call, pk.mask, pk.meta);
                task = ntask; isw = nisw; kc = nkc; tile = ntile; q = nq;
            }
        }
        const int gnt = src * 8 + nt, fr = lane & 15, fq = lane >> 4;
        auto head_of = [&](int u) { return ws + OFF_HP + (size_t)(((((u >> 4) & 1) * 64 + gnt) * 8 + (u & 7)) * 2 + ((u >> 3) & 1)) * HS; };
        u32x4 h = {0u, 0u, 0u, 0u};
        for (int u = (int)lds_draw(2); u < 32;) {
            const int odd = u >> 4, bj = (u >> 3) & 1, uw = u & 7, dt = (slot * 2 + odd) * 8 + nt;
            Packed pk;
            const unsigned char* pl = sws + OFF_PP + (size_t)((((2 * self + odd) * 8 + nt) * 8 + uw) * 2 + bj) * UNIT;
            const unsigned char* const fp = ws + OFF_FP + ((dt * 8 + uw) * 2 + bj) * 4;
            const unsigned char* const dp = ws + OFF_PD + (size_t)((dt * 8 + uw) * 2 + bj) * 8192;
            u32x4 dg[8]; unsigned tw = 0u;
            if ((PLAIN_U >> u) & 1u) {
                wait_word<false>(fp, call, tmo, call);
                load_parked(dg, tw, dp, tmo, lane);
                PARKED_WAIT(dg, tw);
                if (!header_ok(h, call, pk.mask, pk.meta)) wait_header(head_of(u), tmo, call, pk.mask, pk.meta);
                acquire();
                tw = 0u;
                load_parked(cur, tw, pl, tmo, lane);
                PARKED_WAIT(cur, tw);
                release();
                u = (int)lds_draw(2); h = (u32x4){0u, 0u, 0u, 0u};
            } else {
                if (!header_ok(h, call, pk.mask, pk.meta)) wait_header(head_of(u), tmo, call, pk.mask, pk.meta);
                acquire();
                unsigned pf;
                asm volatile("global_load_dword %0, %1, off sc0 sc1" : "=&v"(pf) : "v"(fp) : "memory");
                fetch(pk, pl, lane);
                FETCH_WAIT(pk, 0);
                release();
                asm volatile("" : "+v"(pf));
                if ((unsigned)__builtin_amdgcn_readfirstlane(pf) < call) wait_word<false>(fp, call, tmo, call);
                load_parked(dg, tw, dp, tmo, lane);
                unpack(pk, cur, pl, scr, lane, [&]() {
                    u = (int)lds_draw(2);
                    const unsigned zero = 0u;
                    asm volatile(SBASE "global_load_dwordx4 %0, %1, %2 sc0 sc1" : "=&v"(h) : "v"(zero), "s"(head_of(u)) : "memory");
                });
#pragma unroll
                for (int i = 0; i < 8; ++i) asm volatile("" : "+v"(cur[i]));
                PARKED_WAIT(dg, tw);
                asm volatile("" : "+v"(h));
            }
            const bool bad = (unsigned)__builtin_amdgcn_readfirstlane(tw) == call || ((pk.meta >> 8) & 63u) == 63u;
            unsigned char* dst = P.out + (size_t)(odd * 256 + bj * 128 + (uw & 3) * 32 + fq * 4) * (GN * 2) + (size_t)(gnt * 256 + (uw >> 2) * 128 + fr * 8) * 2;
#pragma unroll
            for (int n = 0; n < 2; ++n)
#pragma unroll
                for (int r = 0; r < 4; ++r) {
                    u32x4 v;
#pragma unroll
                    for (int j = 0; j < 4; ++j) {
                        const unsigned pu = cur[n * 4 + r][j], w = dg[n * 4 + r][j];
                        const f32x2 y = (f32x2){__uint_as_float(w << 16), __uint_as_float(w & 0xffff0000u)}
                                      + (f32x2){__uint_as_float(pu << 16), __uint_as_float(pu & 0xffff0000u)};
                        const f32x2 o = gelu2(y);
                        v[j] = cvt_pk_bf16(o.x, o.y);
                    }
                    if (bad) v = (u32x4){0x7fc07fc0u, 0x7fc07fc0u, 0x7fc07fc0u, 0x7fc07fc0u};
                    st16_sys(dst + (size_t)(n * 16 + r) * (GN * 2), v);
                }
        }
        if (tid == 0) __hip_atomic_store((unsigned*)(ws + OFF_CNT + b * 128), call, __ATOMIC_RELAXED, __HIP_MEMORY_SCOPE_AGENT);
        return;
    }

    if (!is_b) {
        const int hp = lane >> 3;
        unsigned char* const hdst = (unsigned char*)(((u64)(unsigned)__shfl((int)peer_hi, hp) << 32) | (unsigned)__shfl((int)peer_lo, hp))
                                    + OFF_HIN + (par * 8 + self) * (768 * HS) + (lane & 7) * 16;
        for (int t = task0;;) {
#pragma unroll
            for (int i = 0; i < 8; ++i) asm volatile("" : "+v"(cur[i]));
            const int tn = t < (PK_ALL - 1) * 1152 ? t + 1152 : t >= PK_ALL * 1152 ? t + 704 : PK_ALL * 1152 + wid * 88 + (is_a ? g >> 1 : g - 14) * 8 + nt;
            const bool more = tn < 3008 && (tn < PK_ALL * 1152 || is_c || hf);
            int isw, kc, tile; pack_decode(t, isw, kc, tile); const int q = t & 3, idx = self * 8 + kc;
            if (isw) {
                u32x4 tr[8];
#pragma unroll
                for (int j = 0; j < 8; ++j)
#pragma unroll
                    for (int qd = 0; qd < 4; ++qd)
                        tr[j][qd] = perm(cur[2 * qd + 1][j >> 1], cur[2 * qd][j >> 1], (j & 1) ? 0x07060302u : 0x05040100u);
#pragma unroll
                for (int j = 0; j < 8; ++j) cur[j] = tr[j];
            }
            const bool own = (isw ? tile >> 3 : tile >> 1) == self, diag = t >= 1216;
            const bool packed = !own && t < 2560, image = !isw || own || diag;
            u32x4 hd = {0u, 0u, 0u, 0u};
            if (packed && t >= 64 && t < 64 + 32 * PLAIN) {
                unsigned char* pl = P.ws[tile >> 3] + OFF_LB + (((par * 8 + self) * 8 + (tile & 7)) * 4 + q) * UNIT + lane * 16;
#pragma unroll
                for (int i = 0; i < 8; ++i) st16_sys(pl + i * 1024, cur[i]);
                hd = (u32x4){0u, call, 0u, call << 15};
            } else if (packed) {
                Packed pk; pack(cur, pk, scr, lane);
                const bool push = t >= 64 && t < 64 + 32 * (PLAIN + PKP);
                if (push) __builtin_amdgcn_s_sleep(12);
                send(pk, push ? P.ws[tile >> 3] + OFF_LB + (((par * 8 + self) * 8 + (tile & 7)) * 4 + q) * UNIT
                              : ws + OFF_RP + (par * 2560 + (isw ? tile * 8 + kc : 512 + kc * 16 + tile) * 4 + q) * UNIT, scr, lane);
                hd = header(pk, call);
            }
            if (image)
                store_image(ws + (diag ? OFF_WD + (size_t)(tile * 8 + kc) * BLK : isw ? OFF_WB + (size_t)(idx * 8 + (tile & 7)) * BLK
                                                                                     : OFF_XA + (size_t)(idx * 16 + tile) * BLK), isw, q, cur);
            load_task(more ? tn : t, cur);
            asm volatile("s_waitcnt vmcnt(8)" ::: "memory");
            if (packed && (lane & 7) < HFULL && hp != self && (!isw || hp == (tile >> 3)))
                st16_sys(hdst + ((isw ? kc * 8 + (tile & 7) : 64 + kc * 16 + tile) * 4 + q) * HS, hd);
            if (image && lane < (diag ? 1 : isw ? RW : RX))
                st_flag8(ws + (diag ? OFF_FD + (tile * 8 + kc) * 4 : isw ? OFF_FW + ((tile & 7) * 64 + idx) * 4 : OFF_FX + (tile * 64 + idx) * 4) + q + lane * FR, call);
            if (!isw && tile == 2 * self + 1 && lane == 0) st_flag8(ws + OFF_FQ + kc * 4 + q, call);
            __builtin_amdgcn_s_setprio(0);
            if (!more) break;
            t = tn;
        }
    }

    const unsigned foff = lane * 4;
    const unsigned pickoff = (unsigned)(size_t)(lds + 9 * HTB);
    const int fxc = (nt % RX) * FR, fwc = (g % RW) * FR;
    const unsigned lds4 = (unsigned)(size_t)lds + wid * 4096, lds2 = (unsigned)(size_t)lds + wid * 2048;
    const unsigned voff4 = wid * 4096 + lane * 16, voff2 = wid * 2048 + lane * 16;
#define GL4(bufoff, vo, gbase) asm volatile("s_add_i32 m0, %0, %1\n\ts_nop 3\n\tglobal_load_lds_dwordx4 %2, %3 sc0 sc1\n\t" \
        "global_load_lds_dwordx4 %2, %3 offset:1024 sc0 sc1\n\tglobal_load_lds_dwordx4 %2, %3 offset:2048 sc0 sc1\n\t" \
        "global_load_lds_dwordx4 %2, %3 offset:3072 sc0 sc1" :: "s"(lds4), "i"(bufoff), "v"(vo), "s"(gbase) : "memory", "scc")
#define GL2(bufoff, vo, gbase) asm volatile("s_add_i32 m0, %0, %1\n\ts_nop 3\n\tglobal_load_lds_dwordx4 %2, %3 sc0 sc1\n\t" \
        "global_load_lds_dwordx4 %2, %3 offset:1024 sc0 sc1" :: "s"(lds2), "i"(bufoff), "v"(vo), "s"(gbase) : "memory", "scc")
#define LIKELY(c) __builtin_expect(!!(c), 1)
#define RARE(c) __builtin_expect(!!(c), 0)
#define LDA(dst, base, off) do { _Pragma("unroll") for (int m = 0; m < 4; ++m) _Pragma("unroll") for (int k = 0; k < 2; ++k) \
        dst[m][k] = *(const LAS bf16x8*)(lds + (base) + ((off) + m * 2048 + k * 1024)); } while (0)
#define LDB(dst, base, off) do { _Pragma("unroll") for (int n = 0; n < 2; ++n) _Pragma("unroll") for (int k = 0; k < 2; ++k) \
        dst[n][k] = *(const LAS bf16x8*)(lds + (base) + ((off) + n * 2048 + k * 1024)); } while (0)
#define MMA(ai, bj, At, Bt) do { __builtin_amdgcn_s_setprio(1); _Pragma("unroll") for (int m = 0; m < 4; ++m) _Pragma("unroll") for (int n = 0; n < 2; ++n) _Pragma("unroll") for (int k = 0; k < 2; ++k) \
        acc[ai][bj][m][n] = __builtin_amdgcn_mfma_f32_16x16x32_bf16(Bt[n][k], At[m][k], acc[ai][bj][m][n], 0, 0, 0); __builtin_amdgcn_s_setprio(0); } while (0)
#define MMQ(ai, bj, At, Bt) do { _Pragma("unroll") for (int k = 0; k < 2; ++k) _Pragma("unroll") for (int m = 0; m < 4; ++m) _Pragma("unroll") for (int n = 0; n < 2; ++n) \
        acc[ai][bj][m][n] = __builtin_amdgcn_mfma_f32_16x16x32_bf16(Bt[n][k], At[m][k], acc[ai][bj][m][n], 0, 0, 0); SCHED; } while (0)
#define WAIT_V(n) asm volatile("s_waitcnt vmcnt(" #n ")" ::: "memory")
#define WAIT_L0 do { asm volatile("" ::: "memory"); __builtin_amdgcn_s_waitcnt(0xC07F); asm volatile("" ::: "memory"); } while (0)
#define BAR __builtin_amdgcn_s_barrier()
#define SCHED __builtin_amdgcn_sched_barrier(0)
    typedef unsigned u32x2 __attribute__((ext_vector_type(2)));
#define SNAP() asm volatile(SBASE "global_load_dword %0, %2, %4 sc0 sc1\n\tglobal_load_dword %1, %3, %4 sc0 sc1" : "+v"(sna), "+v"(snb) : "v"(fa), "v"(fb), "s"(ws) : "memory")
#define PUBLISH(m, w) asm volatile("ds_write_b64 %0, %1 offset:%2" :: "v"(pickoff), "v"((u32x2){(unsigned)(m), (unsigned)((m) >> 32)}), "i"(8 * (w)) : "memory")
#define LISTEN(w) asm volatile("ds_read_b64 %0, %1 offset:%2" : "=v"(mv) : "v"(pickoff), "i"(8 * (w)) : "memory"); SCHED
#define N0 asm volatile("s_waitcnt lgkmcnt(0)" : "+v"(mv) :: "memory"); m_ = ((u64)(unsigned)__builtin_amdgcn_readfirstlane(mv.y) << 32) | (unsigned)__builtin_amdgcn_readfirstlane(mv.x); asm volatile("" : "+s"(m_)); SCHED
#define N5 asm volatile("s_andn2_b64 %0, %0, %2\n\ts_andn2_b64 %1, %1, %2" : "+s"(todo), "+s"(avail) : "s"(m_) : "scc"); SCHED
#define MF(ai, bj, At, Bt, m, k) acc[ai][bj][m][0] = __builtin_amdgcn_mfma_f32_16x16x32_bf16(Bt[0][k], At[m][k], acc[ai][bj][m][0], 0, 0, 0); \
                                 acc[ai][bj][m][1] = __builtin_amdgcn_mfma_f32_16x16x32_bf16(Bt[1][k], At[m][k], acc[ai][bj][m][1], 0, 0, 0); SCHED
    const int w0 = wid == 0;
    f32x4 acc[2][2][4][2];
    auto zero_acc = [&]() {
#pragma unroll
        for (int a = 0; a < 2; ++a)
#pragma unroll
            for (int c = 0; c < 2; ++c)
#pragma unroll
                for (int m = 0; m < 4; ++m)
#pragma unroll
                    for (int n = 0; n < 2; ++n) acc[a][c][m][n] = (f32x4){0.f, 0.f, 0.f, 0.f};
    };
    auto to_bf16 = [&](int c, u32x4 (&v)[8]) {
#pragma unroll
        for (int n = 0; n < 2; ++n)
#pragma unroll
            for (int r = 0; r < 4; ++r) {
                u32x4& o = v[n * 4 + r];
                o.x = cvt_pk_bf16(acc[0][c][0][n][r], acc[0][c][1][n][r]); o.y = cvt_pk_bf16(acc[0][c][2][n][r], acc[0][c][3][n][r]);
                o.z = cvt_pk_bf16(acc[1][c][0][n][r], acc[1][c][1][n][r]); o.w = cvt_pk_bf16(acc[1][c][2][n][r], acc[1][c][3][n][r]);
            }
    };
    auto emit_unit = [&](int c, int bj) {
        const int lz = lane_id();
        const unsigned char* const started = ws + OFF_FW + fwc + nt * 256 + owner * 32;
        unsigned og, ot;
        asm volatile("global_load_dword %0, %2, off sc0 sc1\n\tglobal_load_dword %1, %3, off sc1" : "=&v"(og), "=&v"(ot) : "v"(started), "v"(tmo) : "memory");
        u32x4 v[8];
        to_bf16(c, v);
        Packed pk;
        if (wid < 4) __builtin_amdgcn_s_setprio(2);
        const bool plain = (PLAIN_U >> ((mt & 1) * 16 + bj * 8 + wid)) & 1u;
        if (!plain) pack(v, pk, scr, lz);
        __builtin_amdgcn_s_setprio(0);
        asm volatile("s_waitcnt vmcnt(0)" : "+v"(og), "+v"(ot) :: "memory");
        bool bad = (unsigned)__builtin_amdgcn_readfirstlane(ot) == call;
        if ((unsigned)__builtin_amdgcn_readfirstlane(og) != tag4) bad = wait_word<true>(started, tag4, tmo, call) || bad;
        unsigned char* const pl = ws + OFF_PP + (size_t)(((mt * 8 + nt) * 8 + wid) * 2 + bj) * UNIT;
        if (plain) {
#pragma unroll
            for (int i = 0; i < 8; ++i) st16_sys(pl + i * 1024 + lz * 16, v[i]);
            pk.mask = 0ull; pk.meta = 0u;
        } else send(pk, pl, scr, lz);
        asm volatile("s_waitcnt vmcnt(0)" ::: "memory");
        if (bad) pk.meta |= 63u << 8;
        if (lz < HFULL) st16_sys(P.ws[owner] + OFF_HP + (size_t)((((mt & 1) * 64 + self * 8 + nt) * 8 + wid) * 2 + bj) * HS + lz * 16, header(pk, call));
    };
    auto finish = [&]() {
        if (threadIdx.x == 0) __hip_atomic_store((unsigned*)(ws + OFF_CNT + b * 128), call, __ATOMIC_RELAXED, __HIP_MEMORY_SCOPE_AGENT);
    };
    auto park = [&](int c, int pt, int bj) {
        unsigned char* dp = ws + OFF_PD + (size_t)((pt * 8 + wid) * 2 + bj) * 8192 + lane_id() * 16;
        u32x4 v[8];
        to_bf16(c, v);
#pragma unroll
        for (int i = 0; i < 8; ++i) st16_sys(dp + i * 1024, v[i]);
    };
    const u64 help = (HELP_KC * 0x0101010101010101ull) & ~(0xffull << (self * 8));
    if (!is_a) {
      const int nsteps = is_b ? 1 : 1 + (7 + C_ODD + 24 - g) / 4;
#pragma unroll 1
      for (int step = 0;; ++step) {
        const bool diag = step < nsteps - 1;
        const int dj = step * 4 + g - 21, dodd = dj >= 7, link = dj - 7 * dodd, dg = ((self + 1 + link) & 7) * 8 + nt, jm = diag ? 2 * self + dodd : mt;
        u64 todo = diag ? 0xffull << (self * 8) : is_b ? ~(0xffull << (owner * 8)) : (hf ? 0x55aa55aa55aa55aaull : 0xaa55aa55aa55aa55ull) & ~help, avail = 0ull;
        const int nkt = __builtin_popcountll(todo);
        const unsigned char* const abase = diag ? ws + OFF_WD + (size_t)(dg * 8 - self * 8) * BLK : ws + OFF_WB + (size_t)nt * BLK;
        const int ashift = diag ? 15 : 18;
        const unsigned fa = (unsigned)(diag ? OFF_FD + (dg * 8 - self * 8) * 4 : OFF_FW + fwc + nt * 256) + foff;
        const int x0 = (is_c && !diag && hf) ? HTB : 0;
        const unsigned voffx = voff4 ^ x0, voffh = voff2 + HTB;
        const unsigned char* const bbase = ws + OFF_XA + (size_t)jm * BLK;
        const unsigned fb = (unsigned)OFF_FX + fxc + jm * 256 + foff;
        const int aoff = wr * 8192 + lane * 16;
        int boff = 4 * HTB + wc * 4096 + lane * 16;
        asm volatile("" : "+v"(boff));
        zero_acc();
        bf16x8 At[4][2], B0[2][2], B1[2][2];
        const unsigned char *a2 = abase, *b2 = bbase;
        unsigned sna = ~tag4, snb = ~tag4; u32x2 mv; int look = w0;
        int vm = 0;
        int left = __builtin_amdgcn_readfirstlane(nkt);
        if (tid < 6) *(LAS unsigned*)(lds + 9 * HTB + tid * 4) = 0u;
        __syncthreads();
        if (wr == 1) BAR;
#define SA(bf, h) (((bf) * 2 + (h)) * HTB)
#define SB(bf, h) ((4 + (bf) * 2 + (h)) * HTB)
#define F1 asm volatile("s_and_b64 %1, %1, %2\n\ts_or_b64 %0, %0, %1" : "+s"(avail), "+s"(m_) : "s"(todo) : "scc"); SCHED
#define F2 asm volatile("s_ff1_i32_b64 %0, %2\n\ts_lshl_b64 %1, 1, %0" : "=&s"(i_), "=s"(m_) : "s"(avail) : "scc"); SCHED
#define F3 asm volatile("s_and_b64 %0, %0, %2\n\ts_cselect_b32 %1, 4, 0" : "+s"(m_), "=s"(vb_) : "s"(avail) : "scc"); SCHED
#define F6 a2 = abase + ((unsigned)i_ << ashift); asm volatile("" : "+s"(a2)); SCHED
#define F7 b2 = bbase + ((unsigned)i_ << 19); asm volatile("" : "+s"(b2)); SCHED
#define F8 asm volatile("s_bcnt1_i32_b64 %0, %1\n\ts_cmp_lt_u32 %0, %2\n\ts_cselect_b32 %0, %3, 0" : "=&s"(look) : "s"(avail), "i"(SNAP_AT), "s"(w0) : "scc"); \
           vm = (vm | vb_) >> 1; asm volatile("" : "+s"(vm)); SCHED
#define KTILE(bf) do { \
        LDB(B0, boff, SA(bf, 0)); LDB(B1, boff, SA(bf, 1)); LDA(At, aoff, SA(bf, 0)); SCHED; \
        if (LIKELY(vm & 2)) { GL2(SA(1 - bf, 1), voffh, a2); WAIT_V(8); } else WAIT_V(0); \
        if (RARE(look)) { asm volatile("" : "+v"(sna), "+v"(snb)); \
            u64 m = __ballot(sna == tag4 && snb == tag4); \
            if (vm == 0 && ((m | avail) & todo) == 0ull) m = poll_ready(todo, ws, fa, fb, tmo, call, tag4, 0xffffffffu); \
            PUBLISH(m, bf); } \
        int cur_ = vm & 1; asm volatile("" : "+s"(cur_)); __builtin_amdgcn_s_setprio(1); \
        WAIT_L0; BAR; SCHED; LISTEN(bf); \
        u64 m_; int i_, vb_; \
        if (LIKELY(cur_)) { \
            MF(0, 0, At, B0, 0, 0); MF(0, 0, At, B0, 1, 0); MF(0, 0, At, B0, 2, 0); MF(0, 0, At, B0, 3, 0); N0; \
            MF(0, 0, At, B0, 0, 1); F1; MF(0, 0, At, B0, 1, 1); F2; MF(0, 0, At, B0, 2, 1); F3; MF(0, 0, At, B0, 3, 1); N5; \
            MF(0, 1, At, B1, 0, 0); F6; MF(0, 1, At, B1, 1, 0); F7; MF(0, 1, At, B1, 2, 0); F8; MF(0, 1, At, B1, 3, 0); \
            MF(0, 1, At, B1, 0, 1); MF(0, 1, At, B1, 1, 1); MF(0, 1, At, B1, 2, 1); MF(0, 1, At, B1, 3, 1); } \
        asm volatile("" : "+s"(cur_)); \
        if (RARE(!cur_)) { N0; F1; F2; F3; N5; F6; F7; F8; } \
        BAR; __builtin_amdgcn_s_setprio(0); SCHED; \
        LDA(At, aoff, SA(bf, 1)); SCHED; \
        if (RARE(look)) SNAP(); \
        if (LIKELY(vm & 2)) { GL4(SB(bf, 0), voffx, b2); GL2(SA(bf, 0), voff2, a2); WAIT_V(8); } else WAIT_V(2); \
        left -= cur_; \
        asm volatile("" : "+s"(cur_), "+s"(left)); __builtin_amdgcn_s_setprio(1); \
        WAIT_L0; BAR; SCHED; if (LIKELY(cur_)) { MMQ(1, 0, At, B0); MMQ(1, 1, At, B1); } BAR; __builtin_amdgcn_s_setprio(0); SCHED; } while (0)
        for (;;) {
            KTILE(0);
            if (left == 0) break;
            KTILE(1);
            if (left == 0) break;
        }
        asm volatile("s_waitcnt vmcnt(0)\n\ts_nop 15\n\ts_nop 7" : "+v"(sna), "+v"(snb) :: "memory");
        if (wr == 0) BAR;
        if (!diag) break;
        park(0, link * 16 + dodd * 8 + nt, 0); park(1, link * 16 + dodd * 8 + nt, 1);
        asm volatile("s_waitcnt vmcnt(0)" ::: "memory");
        if (lane_id() < 2) st_flag(ws + OFF_FP + (((link * 16 + dodd * 8 + nt) * 8 + wid) * 2 + lane_id()) * 4, call);
      }
      {
        if (is_b) {
            emit_unit(0, 0); emit_unit(1, 1);
        } else {
            const int lz = lane_id(), pt = 112 + (mt & 1) * 8 + nt;
            park(1, pt, 1 - hf);
            asm volatile("s_waitcnt vmcnt(0)" ::: "memory");
            if (lz == 0) st_flag(ws + OFF_FP + ((pt * 8 + wid) * 2 + 1 - hf) * 4, call);
            wait_word<false>(ws + OFF_FP + ((pt * 8 + wid) * 2 + hf) * 4, call, tmo, call);
            wait_word<false>(ws + OFF_FP + (((pt + 16) * 8 + wid) * 2 + hf) * 4, call, tmo, call);
            u32x4 dg[8], dh[8]; unsigned tw = 0u, tv = 0u;
            load_parked(dg, tw, ws + OFF_PD + (size_t)((pt * 8 + wid) * 2 + hf) * 8192, tmo, lz);
            load_parked(dh, tv, ws + OFF_PD + (size_t)(((pt + 16) * 8 + wid) * 2 + hf) * 8192, tmo, lz);
            PARKED_WAIT(dg, tw); PARKED_WAIT(dh, tv);
            const bool bad = (unsigned)__builtin_amdgcn_readfirstlane(tw) == call;
            unsigned char* dst = P.out + (size_t)((mt & 1) * 256 + hf * 128 + wc * 32 + (lz >> 4) * 4) * (GN * 2) + (size_t)((self * 8 + nt) * 256 + wr * 128 + (lz & 15) * 8) * 2;
#pragma unroll
            for (int n = 0; n < 2; ++n)
#pragma unroll
                for (int r = 0; r < 4; ++r) {
                    u32x4 v;
#pragma unroll
                    for (int j = 0; j < 4; ++j) {
                        const unsigned w = dg[n * 4 + r][j], x = dh[n * 4 + r][j];
                        const f32x2 o = gelu2((f32x2){acc[j >> 1][0][(j & 1) * 2][n][r], acc[j >> 1][0][(j & 1) * 2 + 1][n][r]}
                                              + (f32x2){__uint_as_float(w << 16), __uint_as_float(w & 0xffff0000u)}
                                              + (f32x2){__uint_as_float(x << 16), __uint_as_float(x & 0xffff0000u)});
                        v[j] = cvt_pk_bf16(o.x, o.y);
                    }
                    if (bad) v = (u32x4){0x7fc07fc0u, 0x7fc07fc0u, 0x7fc07fc0u, 0x7fc07fc0u};
                    st16_sys(dst + (size_t)(n * 16 + r) * (GN * 2), v);
                }
        }
      }
    } else {
      const int dg = owner * 8 + nt;
      const u64 dmask = 0xffull << (owner * 8);
      const bool mine = (g >> 1) >= C_ODD;
      u64 todo = mine ? ~0ull : ~dmask, avail = 0ull;
      const unsigned char* const abase = ws + OFF_WB + (size_t)nt * BLK;
      const unsigned char* const bbase = ws + OFF_XA + (size_t)mt * BLK + hf * HTB;
      const unsigned char* const adiag = ws + OFF_WD + (size_t)(dg * 8 - owner * 8) * BLK;
      const unsigned char* const bdiag = ws + OFF_XA + (size_t)(2 * self + 1) * BLK + hf * HTB + (size_t)(self - owner) * (8 * 16 * (size_t)BLK);
      const bool dl = (lane >> 3) == owner;
      unsigned fa = (unsigned)(dl ? OFF_FD + (dg * 8 - owner * 8) * 4 : OFF_FW + fwc + nt * 256) + foff;
      unsigned fb = (unsigned)(dl ? OFF_FQ - owner * 32 : OFF_FX + fxc + mt * 256) + foff;
      unsigned xm = hf ? 0xffff0000u : 0x0000ffffu;
      const int cm = 2 * self + (g >> 2), ch = (g >> 1) & 1;
      zero_acc();
#pragma unroll 1
      for (int step = 0;; ++step) {
        int aoff[3], boff[3];
#pragma unroll
        for (int i = 0; i < 3; ++i) {
            aoff[i] = i * 3 * HTB + wr * 8192 + lane * 16; boff[i] = i * 3 * HTB + 2 * HTB + wc * 4096 + lane * 16;
            asm volatile("" : "+v"(aoff[i]), "+v"(boff[i]));
        }
        bf16x8 A0[4][2], A1[4][2], B0[2][2];
        unsigned sna = ~tag4, snb = ~tag4; u32x2 mv;
        int vm = 0;
        int look = w0, more = 1;
        const unsigned char *a2 = ws, *b2 = ws;
        const u64 low = step ? 0ull : dmask;
        const unsigned char* const am = step == 1 ? adiag : abase;
        const unsigned char* const bm = step == 1 ? bdiag : step ? ws + OFF_XA + (size_t)cm * BLK + ch * HTB : bbase;
        const int shm = step == 1 ? 15 : 18, vbm = step == 1 ? 0x44 : 4;
        int mleft = __builtin_popcountll(todo & ~low);
        u64 fill = ~0ull;
        if (tid < 6) *(LAS unsigned*)(lds + 9 * HTB + tid * 4) = 0u;
        __syncthreads();
        if (wr == 1) BAR;
#define N1 asm volatile("s_and_b64 %1, %1, %2\n\ts_or_b64 %0, %0, %1\n\ts_and_b64 %1, %0, %3" : "+s"(avail), "+s"(m_) : "s"(todo), "s"(fill) : "scc"); SCHED
#define N2 asm volatile("s_andn2_b64 %0, %7, %8\n\ts_cselect_b64 %0, %0, %6\n\ts_cselect_b64 %1, %9, %10\n\ts_cselect_b64 %2, %11, %12\n\t" \
                        "s_cselect_b32 %3, %13, 15\n\ts_cselect_b32 %4, %14, 0x44\n\ts_subb_u32 %5, %5, 0" \
                        : "=&s"(c_), "=&s"(a2), "=&s"(b2), "=&s"(sh_), "=&s"(vb_), "+s"(mleft) \
                        : "s"(m_), "s"(avail), "s"(low), "s"(am), "s"(adiag), "s"(bm), "s"(bdiag), "s"(shm), "s"(vbm) : "scc"); SCHED
#define N3 asm volatile("s_ff1_i32_b64 %0, %2\n\ts_lshl_b64 %1, 1, %0" : "=&s"(i_), "=s"(m_) : "s"(c_) : "scc"); SCHED
#define N4 asm volatile("s_and_b64 %0, %0, %2\n\ts_cselect_b32 %1, %1, 0" : "+s"(m_), "+s"(vb_) : "s"(c_) : "scc"); SCHED
#define N6 vm = ((vm >> 1) & 0x33) | vb_; more = vm | mleft; asm volatile("" : "+s"(vm), "+s"(more)); SCHED
#define N7 a2 += (unsigned)i_ << sh_; asm volatile("" : "+s"(a2)); SCHED
#define N8 b2 += (unsigned)i_ << 19; asm volatile("" : "+s"(b2)); SCHED
#define N9 asm volatile("s_cmp_gt_i32 %3, %4\n\ts_cselect_b64 %0, -1, 0\n\ts_andn2_b64 %2, %5, %6\n\ts_bcnt1_i32_b64 %1, %2\n\ts_cmp_lt_u32 %1, %7\n\ts_cselect_b32 %1, %8, 0" \
                        : "=&s"(fill), "=&s"(look), "=&s"(c_) : "s"(mleft), "i"(FILL_STOP), "s"(avail), "s"(low), "i"(SNAP_H), "s"(w0) : "scc"); SCHED
#define MF2(ai, At, m, k) MF(ai, 0, At, B0, m, k)
#define HTILE(s) do { \
        constexpr int nb = ((s) + 2) % 3 * 3 * HTB; \
        if (LIKELY(vm & 4)) { GL2(nb + 2 * HTB, voff2, b2); GL4(nb, voff4, a2); } \
        SCHED; LDB(B0, boff[s], 0); LDA(A0, aoff[s], 0); LDA(A1, aoff[s], HTB); SCHED; \
        asm volatile("s_bitcmp1_b32 %2, 2\n\ts_cbranch_scc1 1f\n\ts_waitcnt vmcnt(0)\n1:\n\ts_waitcnt vmcnt(6)" : "+v"(sna), "+v"(snb) : "s"(vm) : "memory", "scc"); \
        if (RARE(look)) {   \
            u64 m = __ballot(sna == tag4 && ((snb ^ tag4) & xm) == 0u); \
            const u64 want = todo & (fill | ~low); \
            if (vm == 0 && ((m | avail) & want) == 0ull) m = poll_ready(want, ws, fa, fb, tmo, call, tag4, xm); \
            PUBLISH(m, s); \
            if ((m & todo & ~avail & ~low) == 0ull) SNAP(); } \
        WAIT_L0; BAR; SCHED; \
        LISTEN(s); \
        int kind = vm & 17; asm volatile("" : "+s"(kind)); \
        u64 m_, c_; int i_, sh_, vb_; \
        if (LIKELY(kind == 1)) { \
            __builtin_amdgcn_s_setprio(1); \
            MF2(0, A0, 0, 0); MF2(0, A0, 1, 0); MF2(0, A0, 2, 0); MF2(0, A0, 3, 0); N0; \
            MF2(0, A0, 0, 1); N1; MF2(0, A0, 1, 1); N2; MF2(0, A0, 2, 1); MF2(0, A0, 3, 1); N3; \
            MF2(1, A1, 0, 0); N4; MF2(1, A1, 1, 0); N5; MF2(1, A1, 2, 0); N6; MF2(1, A1, 3, 0); N7; \
            MF2(1, A1, 0, 1); N8; MF2(1, A1, 1, 1); N9; MF2(1, A1, 2, 1); MF2(1, A1, 3, 1); \
            __builtin_amdgcn_s_setprio(0); } \
        asm volatile("" : "+s"(kind)); \
        if (RARE(kind == 17)) { MMA(0, 1, A0, B0); MMA(1, 1, A1, B0); } \
        asm volatile("" : "+s"(kind)); \
        if (RARE(kind != 1)) { N0; N1; N2; N3; N4; N5; N6; N7; N8; N9; } \
        BAR; SCHED; } while (0)
        for (;;) {
            HTILE(0);
            if (more == 0) break;
            HTILE(1);
            if (more == 0) break;
            HTILE(2);
            if (more == 0) break;
        }
        asm volatile("s_waitcnt vmcnt(0)\n\ts_nop 15\n\ts_nop 7" : "+v"(sna), "+v"(snb) :: "memory");
        if (wr == 0) BAR;
        if (step == 2) break;
        if (step == 0) {
            emit_unit(0, hf);
        }
#pragma unroll
        for (int i = 0; i < 16; ++i) acc[i >> 3][0][(i >> 1) & 3][i & 1] = (f32x4){0.f, 0.f, 0.f, 0.f};
        if (todo == 0ull) {
            const int pt = (g | 1) * 8 + nt;
            if (mine) {
                park(1, pt, hf);
                asm volatile("s_waitcnt vmcnt(0)" ::: "memory");
                if (lane_id() == 0) st_flag(ws + OFF_FP + ((pt * 8 + wid) * 2 + hf) * 4, call);
            }
            if (hf || g >= 8) { finish(); return; }
            step = 1; todo = help; avail = 0ull;
            fa = (unsigned)OFF_FW + fwc + nt * 256 + foff; fb = (unsigned)OFF_FX + fxc + cm * 256 + foff;
            xm = ch ? 0xffff0000u : 0x0000ffffu;
        }
      }
      const int pt = 128 + (cm & 1) * 8 + nt;
      park(0, pt, ch);
      asm volatile("s_waitcnt vmcnt(0)" ::: "memory");
      if (lane_id() == 0) st_flag(ws + OFF_FP + ((pt * 8 + wid) * 2 + ch) * 4, call);
    }
    finish();
}

}

extern "C" void kernel_launch(int rank, int nranks, ncclComm_t comm, hipStream_t stream,
                              void* const* d_in, const int* in_sizes, int n_in,
                              void* d_out, int out_size, const int* in_dtypes, int out_dtype,
                              void* const* d_out_all, void* const* d_ws_all, size_t ws_bytes) {
    (void)comm; (void)in_dtypes; (void)out_dtype; (void)d_out_all;
    if (nranks != NR || (unsigned)rank >= (unsigned)NR || n_in < 2 || ws_bytes < WS_NEED
        || in_sizes[0] != GM * KS || in_sizes[1] != KS * GN || out_size != (GM / NR) * GN) {
        fprintf(stderr, "kernel_launch: unexpected problem shape\n");
        return;
    }
    Params p;
    p.x = (const unsigned char*)d_in[0];
    p.w = (const unsigned char*)d_in[1];
    p.out = (unsigned char*)d_out;
    for (int i = 0; i < NR; ++i) p.ws[i] = (unsigned char*)d_ws_all[i];
    p.own = p.ws[rank]; p.rank = rank; p.pad = 0;
    hipLaunchKernelGGL(fused_gemm_rs, dim3(256), dim3(512), 0, stream, p);
    HIP_CHECK(hipGetLastError());
}
```
